```python
import numpy as np

kernel_name = "gpu_ag_gemm_m4096_k4096_n16384_bf16_none"


ABSMAX_RTOL = 0.008

COMPUTE_DTYPE = "bfloat16"
EPILOGUE = "none"
IN_QUANT = None
OUT_QUANT = None
IN_DTYPES = {"x": "bfloat16", "w_mat": "bfloat16"}
WORLD_SIZE = 8

def _to_bf16(x):
    u = np.ascontiguousarray(x, dtype=np.float32).view(np.uint32)
    return ((u + 0x7FFF + ((u >> 16) & 1)) & 0xFFFF0000).view(np.float32)

def setup_inputs():

    x = _to_bf16(np.random.randn(4096, 4096).astype(np.float32))
    w_mat = _to_bf16((np.random.randn(4096, 16384) / np.sqrt(4096)).astype(np.float32))
    return {"x": x, "w_mat": w_mat}

def _epilogue(y):
    return y

def reference(x, w_mat):
    y = (x @ w_mat).astype(np.float32)
    return _epilogue(y)
```

```cpp
#include <hip/hip_runtime.h>
#include <rccl/rccl.h>
#include <cstdio>
#include <cstdint>

#define HIP_CHECK(expr) do { hipError_t _e = (expr); if (_e != hipSuccess) { \
    fprintf(stderr, "HIP error %s at %s:%d\n", hipGetErrorString(_e), __FILE__, __LINE__); \
    return; } } while (0)

#define LAS __attribute__((address_space(3)))
#define GAS __attribute__((address_space(1)))
typedef unsigned u32x4 __attribute__((ext_vector_type(4)));
typedef unsigned u32x2 __attribute__((ext_vector_type(2)));
typedef short s16x4 __attribute__((ext_vector_type(4)));
typedef short s16x8 __attribute__((ext_vector_type(8)));
typedef __bf16 bf16x8 __attribute__((ext_vector_type(8)));
typedef float f32x8 __attribute__((ext_vector_type(8)));
typedef float f32x16 __attribute__((ext_vector_type(16)));

#define AG_T 4
#define AG_AUXA 17

namespace ag {
constexpr int NR = 8;
constexpr int M = 4096, K = 4096, NL = 2048;
constexpr int MS = M / NR;
constexpr int BM = 256, BN = 128, BK = 64;
constexpr int NKT = K / BK;
constexpr int NG = MS / 32;
constexpr int BLK = 32 * BK * 2;
constexpr int NSLOT = 8, SLOT = BK * BN * 2;
constexpr int NWG = (M / BM) * (NL / BN);
constexpr int NWAVE = 12, NTHR = NWAVE * 64;
constexpr int T = AG_T, NSEND = NG * T, STEPS = NKT / T;
static_assert(NKT % T == 0 && 8 * T <= 64 && NSEND <= NWG, "sender geometry");
constexpr int L_CTL = NSLOT * SLOT;
constexpr int L_PROG = L_CTL + 64;
constexpr int LDS_BYTES = L_CTL + 128;
constexpr size_t W_CNT = 0;
constexpr size_t W_TMO = W_CNT + (size_t)NWG * 128;
constexpr size_t W_FLAG = W_TMO + 4096;
constexpr size_t W_DBG = 512 * 1024;
constexpr size_t W_XT = 1u << 20;
constexpr size_t XT_BYTES = (size_t)M * K * 2;
constexpr size_t W_END = W_XT + 2 * XT_BYTES;
static_assert(W_FLAG + (size_t)NR * NG * T * 4 <= W_DBG, "flags");
constexpr unsigned long long DEADLINE = 30000000ull;

struct Args {
    const char* x;
    const char* w;
    char* out;
    char* ws[NR];
    int rank;
    int pad;
};

__device__ __forceinline__ unsigned long long rt() { return __builtin_amdgcn_s_memrealtime(); }
__device__ __forceinline__ __amdgpu_buffer_rsrc_t rsrc(const void* p, unsigned bytes) {
    return __builtin_amdgcn_make_buffer_rsrc(const_cast<void*>(p), 0, bytes, 0x00020000);
}
#define STAMP(i) do {} while (0)

__global__ void __launch_bounds__(NTHR, 3) ag_gemm(const Args a) {
    __shared__ __attribute__((aligned(1024))) unsigned char lds_raw[LDS_BYTES];
    LAS unsigned char* lds = (LAS unsigned char*)lds_raw;
    volatile LAS unsigned* ctl = (volatile LAS unsigned*)(lds + L_CTL);
    volatile LAS unsigned* prog = (volatile LAS unsigned*)(lds + L_PROG);
    const int lane = threadIdx.x & 63;
    const int wave = __builtin_amdgcn_readfirstlane(threadIdx.x >> 6);
    const int bid = blockIdx.x;
    char* const myws = a.ws[NR - 1];
    const int xcd = bid & 7, loc = bid >> 3;
    const int pm = (xcd >> 1) * 4 + (loc & 3), pn = (xcd & 1) * 8 + (loc >> 2);

    if (wave == 0) STAMP(0);
    GAS unsigned* cnt = (GAS unsigned*)(myws + W_CNT + (size_t)bid * 128);
    const unsigned call = __builtin_amdgcn_readfirstlane(__hip_atomic_load(cnt, __ATOMIC_RELAXED, __HIP_MEMORY_SCOPE_AGENT)) + 1u;
    if (threadIdx.x < 32) ((LAS unsigned*)(lds + L_CTL))[threadIdx.x] = 0u;
    __syncthreads();
    if (threadIdx.x == 0) __hip_atomic_store(cnt, call, __ATOMIC_RELAXED, __HIP_MEMORY_SCOPE_AGENT);
    GAS unsigned* tmo = (GAS unsigned*)(myws + W_TMO);
    const unsigned fbase = call * (unsigned)NKT;
    const size_t xt_off = W_XT + (size_t)(call & 1u) * XT_BYTES;

    if (wave < 8) {
        const __amdgpu_buffer_rsrc_t ra = rsrc(myws + xt_off + (size_t)(pm * 8 + wave) * NKT * BLK, NKT * BLK);
        const unsigned va = lane * 16;
        const int h = lane >> 5, blk = (lane >> 4) & 1, q = (lane >> 2) & 3, p = lane & 3;
        const unsigned vb0 = h * 2048 + q * 128 + 64 * (q >> 1) + 32 * blk + 16 * (p & 1) + 8 * (p >> 1);
        f32x16 acc[4];
#pragma unroll
        for (int c = 0; c < 4; ++c)
#pragma unroll
            for (int i = 0; i < 16; ++i) acc[c][i] = 0.f;
        u32x4 af[8];
        s16x4 bf[2][4][2];
        unsigned have_b = 0, have_a = 0;
        auto need = [&](unsigned nb, unsigned na) {
            if (have_b < nb || have_a < na) {
                for (;;) {
                    const u32x2 v = *(volatile LAS u32x2*)(lds + L_CTL);
                    have_b = __builtin_amdgcn_readfirstlane(v.x);
                    have_a = __builtin_amdgcn_readfirstlane(v.y);
                    if (have_b >= nb && have_a >= na) break;
                    __builtin_amdgcn_s_sleep(1);
                }
            }
            asm volatile("" ::: "memory");
        };
        auto read_b = [&](s16x4 (&d)[4][2], unsigned base, int s) {
#pragma unroll
            for (int c = 0; c < 4; ++c)
#pragma unroll
                for (int t = 0; t < 2; ++t)
                    d[c][t] = __builtin_amdgcn_ds_read_tr16_b64_v4i16(
                        (LAS s16x4*)(lds + (base ^ ((c & 1) * 64)) + s * 4096 + t * 1024 + (c >> 1) * 512));
        };
        auto load_a = [&](int kt, int s) {
            return __builtin_bit_cast(u32x4, __builtin_amdgcn_raw_buffer_load_b128(ra, va + s * 1024, kt * BLK, AG_AUXA));
        };
        need(1, 2);
        if (wave == 0) STAMP(1);
#pragma unroll
        for (int s = 0; s < 4; ++s) af[s] = load_a(0, s);
#pragma unroll
        for (int s = 0; s < 4; ++s) af[4 + s] = load_a(1, s);
        read_b(bf[0], vb0, 0);
        auto tile = [&](int kt, auto PAR, auto LOAD) {
            constexpr int par = PAR.value;
            constexpr bool load = LOAD.value;
            if constexpr (load) need(kt + 2, kt + 3); else need(NKT, NKT);
            const unsigned cur = vb0 + (kt & (NSLOT - 1)) * SLOT, nxt = vb0 + ((kt + 1) & (NSLOT - 1)) * SLOT;
#pragma unroll
            for (int s = 0; s < 4; ++s) {
                if (s < 3) read_b(bf[(s + 1) & 1], cur, s + 1); else read_b(bf[0], nxt, 0);
                const bf16x8 xa = __builtin_bit_cast(bf16x8, af[par * 4 + s]);
#pragma unroll
                for (int c = 0; c < 4; ++c) {
                    const s16x8 wb = __builtin_shufflevector(bf[s & 1][c][0], bf[s & 1][c][1], 0, 1, 2, 3, 4, 5, 6, 7);
                    acc[c] = __builtin_amdgcn_mfma_f32_32x32x16_bf16(__builtin_bit_cast(bf16x8, wb), xa, acc[c], 0, 0, 0);
                }
                if constexpr (load) af[par * 4 + s] = load_a(kt + 2, s);
#pragma unroll
                for (int c = 0; c < 4; ++c) {
                    __builtin_amdgcn_sched_group_barrier(0x100, 2, 0);
                    __builtin_amdgcn_sched_group_barrier(0x008, 1, 0);
                }
                __builtin_amdgcn_sched_group_barrier(0x020, 1, 0);
                __builtin_amdgcn_sched_barrier(0);
            }
            prog[wave] = kt + 1;
        };
        using I0 = std::integral_constant<int, 0>;
        using I1 = std::integral_constant<int, 1>;
        for (int kt = 0; kt < NKT - 2; kt += 2) {
            tile(kt, I0{}, std::true_type{});
            tile(kt + 1, I1{}, std::true_type{});
        }
        tile(NKT - 2, I0{}, std::false_type{});
        tile(NKT - 1, I1{}, std::false_type{});
        if (wave == 0) STAMP(2);
        const bool bad = ctl[2] != 0u || __hip_atomic_load(tmo, __ATOMIC_RELAXED, __HIP_MEMORY_SCOPE_AGENT) == call;
        GAS char* orow = (GAS char*)a.out + ((size_t)(pm * BM + wave * 32 + (lane & 31)) * NL + pn * BN + 8 * h) * 2;
#pragma unroll
        for (int c = 0; c < 4; ++c)
#pragma unroll
            for (int b2 = 0; b2 < 2; ++b2) {
                f32x8 v;
#pragma unroll
                for (int i = 0; i < 8; ++i) v[i] = acc[c][8 * b2 + i];
                u32x4 o = __builtin_bit_cast(u32x4, __builtin_convertvector(v, bf16x8));
                if (bad) o = (u32x4){0x7fc07fc0u, 0x7fc07fc0u, 0x7fc07fc0u, 0x7fc07fc0u};
                *(GAS u32x4*)(orow + (32 * c + 16 * b2) * 2) = o;
            }
        if (wave == 0) STAMP(3);
    } else if (wave == 8) {
        __builtin_amdgcn_s_setprio(2);
        const int q = (lane >> 3) & 3;
        const unsigned voff = q * (NL * 2) + (lane >> 5) * 128 + ((((lane >> 2) & 1) ^ (q >> 1)) * 64) + (lane & 3) * 16;
        const GAS char* wb = (const GAS char*)a.w + pn * BN * 2 + voff;
        int pub = 0;
        for (int kt = 0; kt < NKT; ++kt) {
            if (kt >= NSLOT) {
                const unsigned nd = kt - NSLOT + 1;
                if (!__all(prog[lane & 7] >= nd)) {
                    if (pub < kt) {
                        asm volatile("s_waitcnt vmcnt(0)" ::: "memory");
                        pub = kt;
                        ctl[0] = pub;
                    }
                    while (!__all(prog[lane & 7] >= nd)) __builtin_amdgcn_s_sleep(2);
                }
                asm volatile("" ::: "memory");
            }
            const GAS char* src = wb + (size_t)kt * BK * (NL * 2);
            LAS unsigned char* dst = lds + (kt & (NSLOT - 1)) * SLOT;
#pragma unroll
            for (int i = 0; i < 16; ++i)
                __builtin_amdgcn_global_load_lds((const GAS unsigned*)(src + i * 4 * (NL * 2)), (LAS unsigned*)(dst + i * 1024), 16, 0, 0);
            if (kt + 1 - pub > 2) {
                asm volatile("s_waitcnt vmcnt(32)" ::: "memory");
                ++pub;
                ctl[0] = pub;
            }
        }
        asm volatile("s_waitcnt vmcnt(0)" ::: "memory");
        ctl[0] = NKT;
    } else if (wave == 9) {
        __builtin_amdgcn_s_setprio(2);
        const int fl = lane < 8 * T ? lane : 0;
        GAS unsigned* flag = (GAS unsigned*)(myws + W_FLAG) + pm * 8 * T + fl;
        const unsigned long long t0 = rt();
        int cur = 0;
        while (cur < NKT) {
            const int pr = (int)(__hip_atomic_load(flag, __ATOMIC_RELAXED, __HIP_MEMORY_SCOPE_SYSTEM) - fbase);
            const int first_missing = (pr < 0 ? 0 : (pr > STEPS ? STEPS : pr)) * T + fl % T;
            int c2 = cur;
            while (c2 < NKT && __all(first_missing > c2)) ++c2;
            if (c2 > cur) {
                if (cur == 0) STAMP(4);
                cur = c2;
                asm volatile("" ::: "memory");
                ctl[1] = cur;
            } else {
                if (__builtin_amdgcn_readfirstlane(__hip_atomic_load(tmo, __ATOMIC_RELAXED, __HIP_MEMORY_SCOPE_AGENT)) == call
                    || rt() - t0 > DEADLINE) {
                    __hip_atomic_store(tmo, call, __ATOMIC_RELAXED, __HIP_MEMORY_SCOPE_AGENT);
                    ctl[2] = 1u;
                    ctl[1] = NKT;
                    break;
                }
                __builtin_amdgcn_s_sleep(4);
            }
        }
        STAMP(5);
    } else if (wave == 10) {
        if (bid < NSEND) {
            __builtin_amdgcn_s_setprio(3);
            const int g = bid % NG, t = bid / NG;
            const GAS char* src = (const GAS char*)a.x + (size_t)(g * 32 + (lane & 31)) * (K * 2) + (lane >> 5) * 16;
            const size_t dst_off = xt_off + (size_t)(a.rank * NG + g) * NKT * BLK;
            __amdgpu_buffer_rsrc_t rd[NR];
#pragma unroll
            for (int i = 0; i < NR; ++i) rd[i] = rsrc(a.ws[i] + dst_off, NKT * BLK);
            char* fw = a.ws[0];
#pragma unroll
            for (int i = 1; i < NR; ++i) fw = (lane & 7) == i ? a.ws[i] : fw;
            GAS unsigned* flag = (GAS unsigned*)(fw + W_FLAG) + (a.rank * NG + g) * T + t;
            u32x4 d[4], nx[4];
#pragma unroll
            for (int s = 0; s < 4; ++s) nx[s] = *(const GAS u32x4*)(src + t * (BK * 2) + s * 32);
            STAMP(6);
            for (int j = 0; j < STEPS; ++j) {
                const int kt = j * T + t;
#pragma unroll
                for (int s = 0; s < 4; ++s) d[s] = nx[s];
                const int kn = j + 1 < STEPS ? kt + T : kt;
#pragma unroll
                for (int s = 0; s < 4; ++s) nx[s] = *(const GAS u32x4*)(src + kn * (BK * 2) + s * 32);
#pragma unroll
                for (int i = 0; i < NR; ++i)
#pragma unroll
                    for (int s = 0; s < 4; ++s)
                        __builtin_amdgcn_raw_buffer_store_b128(d[s], rd[i], lane * 16 + s * 1024, kt * BLK, 17);
                asm volatile("s_waitcnt vmcnt(0)" ::: "memory");
                if (lane < NR) __hip_atomic_store(flag, fbase + (unsigned)(j + 1), __ATOMIC_RELAXED, __HIP_MEMORY_SCOPE_SYSTEM);
            }
            STAMP(7);
        }
    }
}
}

extern "C" void kernel_launch(int rank, int nranks, ncclComm_t comm,
                              hipStream_t stream,
                              void* const* d_in, const int* in_sizes, int n_in,
                              void* d_out, int out_size,
                              const int* in_dtypes, int out_dtype,
                              void* const* d_out_all,
                              void* const* d_ws_all, size_t ws_bytes) {
    if (nranks != ag::NR || (unsigned)rank >= (unsigned)ag::NR || n_in != 2 || ws_bytes < ag::W_END ||
        in_sizes[0] != ag::MS * ag::K || in_sizes[1] != ag::K * ag::NL || out_size != ag::M * ag::NL) {
        fprintf(stderr, "ag_gemm: unexpected problem shape\n");
        return;
    }
    ag::Args a;
    a.x = (const char*)d_in[0];
    a.w = (const char*)d_in[1];
    a.out = (char*)d_out;
    for (int i = 0; i < ag::NR; ++i) a.ws[i] = (char*)d_ws_all[(rank + 1 + i) % ag::NR];
    a.rank = rank;
    a.pad = 0;
    hipLaunchKernelGGL(ag::ag_gemm, dim3(ag::NWG), dim3(ag::NTHR), 0, stream, a);
    HIP_CHECK(hipGetLastError());
}
```

```cpp
#include <hip/hip_runtime.h>
#include <rccl/rccl.h>
#include <cstdio>
#include <cstdint>
#include <type_traits>

#define HIP_CHECK(expr) do { hipError_t _e = (expr); if (_e != hipSuccess) { \
    fprintf(stderr, "HIP error %s at %s:%d\n", hipGetErrorString(_e), __FILE__, __LINE__); \
    return; } } while (0)

#define LAS __attribute__((address_space(3)))
#define GAS __attribute__((address_space(1)))
typedef unsigned u32x4 __attribute__((ext_vector_type(4)));
typedef unsigned u32x2 __attribute__((ext_vector_type(2)));
typedef unsigned short u16x2 __attribute__((ext_vector_type(2)));
typedef short s16x4 __attribute__((ext_vector_type(4)));
typedef short s16x8 __attribute__((ext_vector_type(8)));
typedef __bf16 bf16x4 __attribute__((ext_vector_type(4)));
typedef __bf16 bf16x8 __attribute__((ext_vector_type(8)));
typedef float f32x4 __attribute__((ext_vector_type(4)));

#define AG_NB 10
#define AG_NSC 24
#define AG_SW 2
#define AG_SCHED 2
#define AG_DLOC 0
#define AG_PSLEEP 2
#define AG_ROT 0
#define AG_UAUX 17
#define AG_FS 32
#define AG_PW 2
#define AG_RH 0
#define AG_RT 0
#define AG_AUXA 17

namespace ag {
constexpr int NR = 8;
constexpr int M = 4096, K = 4096, NL = 2048;
constexpr int MS = M / NR;
constexpr int NB = AG_NB;
constexpr int BM = 256, BN = NB * 16, BK = 64;
constexpr int NKT = K / BK;
constexpr int NG = MS / 32;
constexpr int BLK = 32 * BK * 2;
constexpr int NBLK = NG * NKT;
constexpr int SLOT = BK * BN * 2, NSLOT = NB == 10 ? 7 : 8, NPIECE = SLOT / 1024;
constexpr int NTN = (NL + BN - 1) / BN;
constexpr int NWG = 256, CPX = 32 - 2 * NTN;
constexpr int NWAVE = 8, NTHR = NWAVE * 64;
constexpr int NSC = AG_NSC < 8 * CPX ? AG_NSC : 8 * CPX, SW = AG_SW, NSEND = NSC * SW;
static_assert(NB == 8 || NB == 10, "tile width");
static_assert(2 * NSEND <= NBLK && SW <= NWAVE, "sender geometry");
constexpr int PBLK = BLK * 3 / 4;
constexpr int RH = AG_RH, RT = AG_RT;
constexpr int NPK = (NKT - RH - RT) * NG * (NR - 1);
constexpr int NDW = AG_DLOC == 0 ? (8 * CPX - NSC) * NWAVE : 16 * NTN * 2;
constexpr int DSLOTS = (NPK + NDW - 1) / NDW, DSTRIDE = (DSLOTS * 16 + 127) / 128 * 128;
static_assert(NDW > 0 && DSLOTS <= 64, "unpacker geometry");
constexpr int L_CTL = NSLOT * SLOT;
constexpr int L_PROG = L_CTL + 64;
constexpr int LDS_BYTES = L_CTL + 128;
static_assert(LDS_BYTES > 80 * 1024 && LDS_BYTES <= 160 * 1024, "one workgroup per CU");
constexpr size_t W_CNT = 0;
constexpr size_t W_TMO = W_CNT + (size_t)NWG * 128;
constexpr size_t W_TICKET = W_TMO + 1024;
constexpr size_t W_PF = 128 * 1024;
constexpr size_t PF_BYTES = (size_t)NDW * DSTRIDE;
constexpr size_t W_DBG = 768 * 1024;
constexpr size_t W_XT = 1u << 20;
constexpr size_t XT_BYTES = (size_t)M * K * 2;
constexpr size_t W_PK = W_XT + 2 * XT_BYTES;
constexpr size_t PK_BYTES = (size_t)NR * NBLK * PBLK;
constexpr size_t W_FLAG = W_PK + 2 * PK_BYTES;
constexpr int FS = AG_FS;
constexpr size_t W_END = W_FLAG + (size_t)NR * NBLK * FS * 4;
static_assert(W_TICKET + 128 <= W_PF && W_PF + 2 * PF_BYTES <= W_DBG, "flags");
constexpr unsigned OOB = 0x40000000u;
constexpr unsigned long long DEADLINE = 30000000ull;

struct Args {
    const char* x;
    const char* w;
    char* out;
    char* ws[NR];
    int rank;
    int pad;
};

__device__ __forceinline__ unsigned long long rt() { return __builtin_amdgcn_s_memrealtime(); }
__device__ __forceinline__ __amdgpu_buffer_rsrc_t rsrc(const void* p, unsigned bytes) {
    return __builtin_amdgcn_make_buffer_rsrc(const_cast<void*>(p), 0, bytes, 0x00020000);
}
__device__ __forceinline__ void store16(u32x4 d, __amdgpu_buffer_rsrc_t r, unsigned off) {
    __builtin_amdgcn_raw_buffer_store_b128(d, r, off, 0, 17);
}
__device__ __forceinline__ void wait_vm0() {
    __builtin_amdgcn_s_waitcnt(0x0F70);
    asm volatile("s_waitcnt vmcnt(0)" ::: "memory");
}
__device__ __forceinline__ bool give_up(GAS unsigned* tmo, unsigned call, unsigned long long t0) {
    const unsigned long long dt = rt() - t0;
    if (dt < 50000ull) return false;
    return dt > DEADLINE || __builtin_amdgcn_readfirstlane(__hip_atomic_load(tmo, __ATOMIC_RELAXED, __HIP_MEMORY_SCOPE_AGENT)) == call;
}
#define STAMP(i) do {} while (0)
#define STAMPC(i) do {} while (0)

__global__ void __launch_bounds__(NTHR, 2) ag_gemm(const Args a) {
    __shared__ __attribute__((aligned(1024))) unsigned char lds_raw[LDS_BYTES];
    LAS unsigned char* lds = (LAS unsigned char*)lds_raw;
    volatile LAS unsigned* ctl = (volatile LAS unsigned*)(lds + L_CTL);
    volatile LAS unsigned* prog = (volatile LAS unsigned*)(lds + L_PROG);
    const int lane = threadIdx.x & 63;
    const int wave = __builtin_amdgcn_readfirstlane(threadIdx.x >> 6);
    const int bid = blockIdx.x;
    char* const myws = a.ws[NR - 1];
    const int xcd = bid & 7, loc = bid >> 3;
    const bool is_comm = loc < CPX;
    const int pm = ((xcd + AG_ROT) & 7) * 2 + ((loc - CPX) & 1), pn = (loc - CPX) >> 1;

    if (wave == 0) STAMP(0);
    GAS unsigned* cnt = (GAS unsigned*)(myws + W_CNT + (size_t)bid * 128);
    const unsigned call = __builtin_amdgcn_readfirstlane(__hip_atomic_load(cnt, __ATOMIC_RELAXED, __HIP_MEMORY_SCOPE_AGENT)) + 1u;
    if (threadIdx.x < 32) ((LAS unsigned*)(lds + L_CTL))[threadIdx.x] = 0u;
    __syncthreads();
    if (threadIdx.x == 0) __hip_atomic_store(cnt, call, __ATOMIC_RELAXED, __HIP_MEMORY_SCOPE_AGENT);
    GAS unsigned* tmo = (GAS unsigned*)(myws + W_TMO);
    const size_t xt_off = W_XT + (size_t)(call & 1u) * XT_BYTES;
    const bool has_half = NB == 10 && (pn + 1) * BN <= NL;
    const int oS = (NB == 10 && (pn & 1)) ? 64 : 0, oH = (NB == 10 && !(pn & 1) && has_half) ? 256 : 0;

    const int cid = loc * 8 + xcd;
    const size_t pk_off = W_PK + (size_t)(call & 1u) * PK_BYTES, pf_off = W_PF + (size_t)(call & 1u) * PF_BYTES;
    if (is_comm && cid < NSC) {
        if (wave < SW) {
            const int sid = wave * NSC + cid;
            const GAS char* src = (const GAS char*)a.x + (size_t)(lane & 15) * (K * 2) + (lane >> 4) * 16;
            __amdgpu_buffer_rsrc_t rd[NR];
#pragma unroll
            for (int i = 0; i < NR; ++i) rd[i] = rsrc(a.ws[i], (unsigned)W_END);
            const unsigned xt0 = (unsigned)xt_off + a.rank * NG * BLK, pk0 = (unsigned)pk_off + a.rank * NG * PBLK;
            char* fw = a.ws[0];
#pragma unroll
            for (int i = 1; i < NR; ++i) fw = (lane & 7) == i ? a.ws[i] : fw;
            GAS unsigned* flag = (GAS unsigned*)(fw + W_FLAG) + a.rank * NBLK * FS;
            GAS char* pflag = (GAS char*)fw + pf_off;
            const unsigned tbase = (call - 1u) * (unsigned)NBLK - 2u * NSEND;
            auto fetch = [&](u32x4 (&d)[4], unsigned b) {
                const unsigned bb = b < (unsigned)NBLK ? b : 0u;
                const GAS char* p = src + (size_t)(bb % NG) * 32 * (K * 2) + (bb / NG) * (BK * 2);
#pragma unroll
                for (int s = 0; s < 4; ++s) d[s] = *(const GAS u32x4*)(p + (s >> 1) * 16 * (K * 2) + (s & 1) * 64);
            };
            auto pack = [&](const u32x4 (&d)[4], u32x4 (&pk)[3], unsigned& base, unsigned long long& miss) {
                unsigned h[8], hm[8];
                u16x2 mx = (u16x2){0, 0};
#pragma unroll
                for (int j = 0; j < 8; ++j) {
                    const unsigned d0 = d[j >> 1][(j & 1) * 2], d1 = d[j >> 1][(j & 1) * 2 + 1];
                    pk[j >> 2][j & 3] = __builtin_amdgcn_perm(d1, d0, 0x06040200u);
                    h[j] = __builtin_amdgcn_perm(d1, d0, 0x07050301u);
                    hm[j] = h[j] | 0x80808080u;
                    mx = __builtin_elementwise_max(mx, __builtin_bit_cast(u16x2, hm[j]));
                    mx = __builtin_elementwise_max(mx, __builtin_bit_cast(u16x2, hm[j] << 8));
                }
                unsigned m = (mx.x > mx.y ? mx.x : mx.y) >> 8;
                m = max(m, (unsigned)__builtin_amdgcn_update_dpp(0, (int)m, 0x111, 0xf, 0xf, true));
                m = max(m, (unsigned)__builtin_amdgcn_update_dpp(0, (int)m, 0x112, 0xf, 0xf, true));
                m = max(m, (unsigned)__builtin_amdgcn_update_dpp(0, (int)m, 0x114, 0xf, 0xf, true));
                m = max(m, (unsigned)__builtin_amdgcn_update_dpp(0, (int)m, 0x118, 0xf, 0xf, true));
                const unsigned top = max(max((unsigned)__builtin_amdgcn_readlane(m, 15), (unsigned)__builtin_amdgcn_readlane(m, 31)),
                                         max((unsigned)__builtin_amdgcn_readlane(m, 47), (unsigned)__builtin_amdgcn_readlane(m, 63))) & 127u;
                base = top > 7u ? top - 7u : 0u;
                const unsigned b4 = base * 0x01010101u;
                unsigned all = 0x80808080u;
#pragma unroll
                for (int j = 0; j < 4; ++j) {
                    const unsigned ra = hm[2 * j] - b4, rb = hm[2 * j + 1] - b4;
                    all &= ra & rb;
                    const unsigned na = (h[2 * j] & 0x80808080u) | (ra & 0x07070707u);
                    const unsigned nb = (h[2 * j + 1] & 0x80808080u) | (rb & 0x07070707u);
                    pk[2][j] = na | __builtin_amdgcn_alignbit(nb, nb, 28);
                }
                miss = __ballot(all != 0x80808080u);
            };
            unsigned b0 = sid, b1 = sid + NSEND;
            u32x4 d[4], nx[4], pk[3];
            unsigned base;
            unsigned long long miss;
            fetch(d, b0);
            pack(d, pk, base, miss);
            if (wave == 0) STAMP(6);
            unsigned long long nsent = 0;
            while (b0 < (unsigned)NBLK) {
                fetch(nx, b1);
                const unsigned tk = __builtin_amdgcn_raw_ptr_buffer_atomic_add_i32(1, rd[NR - 1], lane == 0 ? (unsigned)W_TICKET : OOB, 0, 0);
                const unsigned g = b0 % NG, kt = b0 / NG;
                const unsigned so = xt0 + (kt * (NR * NG) + g) * BLK + lane * 16, sp = pk0 + (kt * (NR * NG) + g) * PBLK + lane * 16;
                const bool plain = kt - RH >= (unsigned)(NKT - RH - RT);
                if (plain || __builtin_popcountll(miss) > 16) miss = ~0ull;
                if (miss != 0ull) {
                    const unsigned vo = (miss >> lane) & 1ull ? so : OOB;
#pragma unroll
                    for (int i = 0; i < NR - 1; ++i)
#pragma unroll
                        for (int s = 0; s < 4; ++s)
                            store16(d[s], rd[i], vo + s * 1024);
                }
                const unsigned vp = miss != ~0ull ? sp : OOB;
#pragma unroll
                for (int i = 0; i < NR - 1; ++i)
#pragma unroll
                    for (int s = 0; s < 3; ++s)
                        store16(pk[s], rd[i], vp + s * 1024);
#pragma unroll
                for (int s = 0; s < 4; ++s)
                    store16(d[s], rd[NR - 1], so + s * 1024);
                const u32x4 rec = (u32x4){(call << 8) | base, (unsigned)miss, call, (unsigned)(miss >> 32)};
#pragma unroll
                for (int s = 0; s < 4; ++s) d[s] = nx[s];
                pack(d, pk, base, miss);
                wait_vm0();
                if (lane < NR) {
                    if (plain || lane == NR - 1) {
                        __hip_atomic_store(flag + (((g >> 3) * NKT + kt) * 8 + (g & 7)) * FS, call, __ATOMIC_RELAXED, __HIP_MEMORY_SCOPE_SYSTEM);
                    } else {
                        const unsigned sq = (b0 - RH * NG) * (NR - 1) + lane;
                        GAS char* q = pflag + (sq % NDW) * DSTRIDE + (sq / NDW) * 16;
                        asm volatile("global_store_dwordx4 %0, %1, off sc0 sc1\n\ts_nop 1" :: "v"(q), "v"(rec) : "memory");
                    }
                }
                b0 = b1;
                ++nsent;
                b1 = __builtin_amdgcn_readfirstlane(tk) - tbase;
            }
            if (wave == 0) STAMP(7);
        }
    } else if (AG_DLOC == 0 ? is_comm : (!is_comm && wave >= 6)) {
        const int dw = AG_DLOC == 0 ? wave * (NDW / NWAVE) + (cid - NSC) : (wave - 6) * (NDW / 2) + (cid - 8 * CPX);
        const unsigned sq = lane * NDW + dw, sv = sq < (unsigned)NPK ? sq : 0u;
        const unsigned bq = sv / (NR - 1) + RH * NG, from = (a.rank + 2 * NR - 1 - sv % (NR - 1)) % NR;
        const unsigned g = bq % NG, kt = bq / NG, bi = kt * (NR * NG) + from * NG + g;
        GAS unsigned* flag = (GAS unsigned*)(myws + W_FLAG) + (from * NBLK + ((g >> 3) * NKT + kt) * 8 + (g & 7)) * FS;
        const __amdgpu_buffer_rsrc_t rf = rsrc(myws + pf_off + (size_t)dw * DSTRIDE, DSTRIDE);
        const __amdgpu_buffer_rsrc_t rp = rsrc(myws + pk_off, (unsigned)PK_BYTES);
        const __amdgpu_buffer_rsrc_t rx = rsrc(myws + xt_off, (unsigned)XT_BYTES);
        unsigned long long todo = __ballot(sq < (unsigned)NPK), pend = 0;
        const unsigned long long t0 = rt();
        auto get = [&](u32x4 (&p)[3], int j, bool on) {
            const unsigned o = __builtin_amdgcn_readlane(bi, j) * PBLK, vo = on ? lane * 16u : OOB;
#pragma unroll
            for (int s = 0; s < 3; ++s)
                p[s] = __builtin_bit_cast(u32x4, __builtin_amdgcn_raw_buffer_load_b128(rp, vo + s * 1024, o, 17));
        };
        auto put = [&](const u32x4 (&p)[3], const u32x4& rec, int j, bool on) {
            const unsigned o = __builtin_amdgcn_readlane(bi, j) * BLK;
            const unsigned b4 = (__builtin_amdgcn_readlane(rec.x, j) & 127u) * 0x01010101u;
            const unsigned long long miss = (unsigned)__builtin_amdgcn_readlane(rec.y, j)
                                          | ((unsigned long long)(unsigned)__builtin_amdgcn_readlane(rec.w, j) << 32);
            const unsigned vo = !on || ((miss >> lane) & 1ull) ? OOB : o + lane * 16u;
#pragma unroll
            for (int s = 0; s < 4; ++s) {
                const unsigned n = p[2][s];
                const unsigned ha = (n & 0x87878787u) + b4, hb = (__builtin_amdgcn_alignbit(n, n, 4) & 0x87878787u) + b4;
                const unsigned la = p[s >> 1][(s & 1) * 2], lb = p[s >> 1][(s & 1) * 2 + 1];
                const u32x4 d = (u32x4){__builtin_amdgcn_perm(ha, la, 0x05010400u), __builtin_amdgcn_perm(ha, la, 0x07030602u),
                                        __builtin_amdgcn_perm(hb, lb, 0x05010400u), __builtin_amdgcn_perm(hb, lb, 0x07030602u)};
                __builtin_amdgcn_raw_buffer_store_b128(d, rx, vo + s * 1024, 0, AG_UAUX);
            }
        };
        if (wave == 7) STAMP(8);
        unsigned long long lat_r = 0, lat_w = 0, lat_n = 0;
        for (;;) {
            const unsigned head = todo ? __builtin_ctzll(todo) : 0u;
            const u32x4 rec = __builtin_bit_cast(u32x4, __builtin_amdgcn_raw_buffer_load_b128(
                rf, (unsigned)lane - head < 8u ? lane * 16u : OOB, 0, 17));
            wait_vm0();
            if (pend) {
                if ((pend >> lane) & 1ull) __hip_atomic_store(flag, call, __ATOMIC_RELAXED, __HIP_MEMORY_SCOPE_SYSTEM);
                pend = 0;
            }
            if (!todo) break;
            const unsigned long long rdy = todo & __ballot((int)(rec.z - call) >= 0 && (rec.x >> 8) == (rec.z & 0xffffffu));
            if (rdy) {
                const int j0 = __builtin_ctzll(rdy);
                const unsigned long long r1 = rdy & (rdy - 1);
                const int j1 = r1 ? __builtin_ctzll(r1) : j0;
                u32x4 p0[3], p1[3];
                get(p0, j0, true);
                get(p1, j1, r1 != 0);
                put(p0, rec, j0, true);
                put(p1, rec, j1, r1 != 0);
                pend = (1ull << j0) | (1ull << j1);
                todo &= ~pend;
            } else {
                if (give_up(tmo, call, t0)) {
                    __hip_atomic_store(tmo, call, __ATOMIC_RELAXED, __HIP_MEMORY_SCOPE_AGENT);
                    break;
                }
                __builtin_amdgcn_s_sleep(1);
            }
        }
        if (wave == 7) STAMP(9);
    } else if (is_comm) {
    }
    else if (wave < 4) {
        const __amdgpu_buffer_rsrc_t ra = rsrc(myws + xt_off + (size_t)(pm * 8 + wave * 2) * BLK, (unsigned)XT_BYTES - (pm * 8 + wave * 2) * BLK);
        const unsigned va = lane * 16;
        const int g = lane >> 4, q = (lane >> 2) & 3, p = lane & 3;
        const unsigned vbS = g * 1024 + q * 128 + 32 * ((q >> 1) ^ (2 * (g & 1))) + 8 * p;
        const unsigned vbH = 16384 + (g >> 1) * 1024 + 64 * (8 * (g & 1) + q) + 32 * (g & 1) + 8 * p;
        f32x4 acc[4][NB];
#pragma unroll
        for (int m = 0; m < 4; ++m)
#pragma unroll
            for (int n = 0; n < NB; ++n) acc[m][n] = (f32x4){0.f, 0.f, 0.f, 0.f};
        u32x4 af[4][4];
        constexpr int PB = 3;
        s16x8 bf[PB + 1];
        unsigned have_b = 0, have_a = 0;
        auto need = [&](unsigned nb, unsigned na) {
            if (have_b < nb || have_a < na) {
                for (;;) {
                    const u32x2 v = *(volatile LAS u32x2*)(lds + L_CTL);
                    have_b = __builtin_amdgcn_readfirstlane(v.x);
                    have_a = __builtin_amdgcn_readfirstlane(v.y);
                    if (have_b >= nb && have_a >= na) break;
                    __builtin_amdgcn_s_sleep(1);
                }
            }
            asm volatile("" ::: "memory");
        };
        auto read_h = [&](unsigned slot_off, int ks, int n, int t) {
            if (n < 8)
                return __builtin_amdgcn_ds_read_tr16_b64_v4i16(
                    (LAS s16x4*)(lds + ((vbS + slot_off) ^ (32 * ((n & 3) ^ (2 * t)))) + (n >> 2) * 8192 + ks * 4096 + t * 512));
            return __builtin_amdgcn_ds_read_tr16_b64_v4i16(
                (LAS s16x4*)(lds + ((vbH + slot_off) ^ (32 * (n & 1))) + ks * 2048 + t * 256));
        };
        auto load_a = [&](int kt, int ks, int m) {
            return __builtin_bit_cast(u32x4, __builtin_amdgcn_raw_buffer_load_b128(
                ra, va, kt * (NR * NG * BLK) + m * 2048 + ks * 1024, AG_AUXA));
        };
#define SB() __builtin_amdgcn_sched_barrier(0)
#define MFMA(C, B, A) asm volatile("v_mfma_f32_16x16x32_bf16 %0, %1, %2, %0" : "+v"(C) : "v"(B), "v"(A))
        need(1, 2);
        if (wave == 0) { STAMP(1); STAMPC(32); __builtin_amdgcn_s_waitcnt(0); }
#pragma unroll
        for (int j = 0; j < 3; ++j)
#pragma unroll
            for (int m = 0; m < 4; ++m) af[j][m] = load_a(j >> 1, j & 1, m);
#pragma unroll
        for (int i = 0; i < PB; ++i) bf[i] = __builtin_shufflevector(read_h(0, 0, i, 0), read_h(0, 0, i, 1), 0, 1, 2, 3, 4, 5, 6, 7);
        unsigned so_cur = 0;
        auto tile = [&](int kt, auto PAR, auto LAST) {
            constexpr int par = PAR.value;
            constexpr int last = LAST.value;
            if constexpr (last == 0) need(kt + 2, kt + 3); else need(NKT, NKT);
            const unsigned so_nxt = so_cur + SLOT == NSLOT * SLOT ? 0u : so_cur + SLOT;
#pragma unroll
            for (int ks = 0; ks < 2; ++ks) {
                const int j = par * 2 + ks;
#pragma unroll
                for (int n = 0; n < NB; ++n) {
                    const int i = ks * NB + n + PB;
                    const int r = (par * 2 * NB + i) % (PB + 1), u = (par * 2 * NB + ks * NB + n) % (PB + 1);
#define RD(t) (i < 2 * NB ? read_h(so_cur, i / NB, i % NB, t) : read_h(so_nxt, 0, i - 2 * NB, t))
#define LDA() do { if (n < 4 && (last == 0 || (last == 1 && ks == 0))) af[(j + 3) & 3][n] = ks == 0 ? load_a(kt + 1, 1, n) : load_a(kt + 2, 0, n); } while (0)
                    SB(); MFMA(acc[0][n], bf[u], af[j][0]); SB();
                    const s16x4 lo = RD(0);
                    SB(); MFMA(acc[1][n], bf[u], af[j][1]); SB();
                    const s16x4 hi = RD(1);
                    bf[r] = __builtin_shufflevector(lo, hi, 0, 1, 2, 3, 4, 5, 6, 7);
                    SB(); MFMA(acc[2][n], bf[u], af[j][2]); SB();
                    LDA();
                    SB(); MFMA(acc[3][n], bf[u], af[j][3]); SB();
                }
            }
            prog[wave] = kt + 1;
            so_cur = so_nxt;
        };
        static_assert((4 * NB) % (PB + 1) == 0, "fragment ring closes over two K-tiles");
        using I0 = std::integral_constant<int, 0>;
        using I1 = std::integral_constant<int, 1>;
        using I2 = std::integral_constant<int, 2>;
        for (int kt = 0; kt < NKT - 2; kt += 2) {
            tile(kt, I0{}, I0{});
            tile(kt + 1, I1{}, I0{});
        }
        tile(NKT - 2, I0{}, I1{});
        tile(NKT - 1, I1{}, I2{});
        asm volatile("s_nop 15" ::: "memory");
        SB();
        if (wave == 0) { STAMP(2); STAMPC(33); }
        const bool bad = ctl[2] != 0u || __hip_atomic_load(tmo, __ATOMIC_RELAXED, __HIP_MEMORY_SCOPE_AGENT) == call;
        int l2;
        asm volatile("v_mbcnt_lo_u32_b32 %0, -1, 0\n\tv_mbcnt_hi_u32_b32 %0, -1, %0" : "=v"(l2));
        GAS char* orow = (GAS char*)a.out + ((size_t)(pm * BM + wave * 64 + (l2 & 15)) * NL + pn * BN) * 2 + ((l2 >> 4) & 1) * 32 + (l2 >> 5) * 16;
#pragma unroll
        for (int m = 0; m < 4; ++m)
#pragma unroll
            for (int n = 0; n < NB; n += 2) {
                const u32x2 X = __builtin_bit_cast(u32x2, __builtin_convertvector(acc[m][n], bf16x4));
                const u32x2 Y = __builtin_bit_cast(u32x2, __builtin_convertvector(acc[m][n + 1], bf16x4));
                const auto s0 = __builtin_amdgcn_permlane16_swap(X.x, Y.x, false, false);
                const auto s1 = __builtin_amdgcn_permlane16_swap(X.y, Y.y, false, false);
                u32x4 o = (u32x4){s0[0], s1[0], s0[1], s1[1]};
                if (bad) o = (u32x4){0x7fc07fc0u, 0x7fc07fc0u, 0x7fc07fc0u, 0x7fc07fc0u};
                if (n < 8) *(GAS u32x4*)(orow + (size_t)m * 16 * NL * 2 + oS + n * 32) = o;
                else if (has_half) *(GAS u32x4*)(orow + (size_t)m * 16 * NL * 2 + oH) = o;
            }
        if (wave == 0) STAMP(3);
    } else if (wave == 4) {
        __builtin_amdgcn_s_setprio(2);
        const int r8 = lane >> 3, r16 = lane >> 2;
        const GAS char* wb = (const GAS char*)a.w + pn * BN * 2;
        const GAS char* w_ev = wb + oS + r8 * (NL * 2) + ((((lane >> 1) & 3) ^ (r8 >> 1)) * 32) + (lane & 1) * 16;
        const GAS char* w_od = wb + oS + r8 * (NL * 2) + ((((lane >> 1) & 3) ^ (r8 >> 1) ^ 2) * 32) + (lane & 1) * 16;
        const GAS char* w_hf = wb + oH + r16 * (NL * 2) + ((((lane >> 1) & 1) ^ (r16 >> 3)) * 32) + (lane & 1) * 16;
        int pub = 0;
        unsigned so = 0;
        for (int kt = 0; kt < NKT; ++kt) {
            if (kt >= NSLOT) {
                const unsigned nd = kt - NSLOT + 1;
                if (!__all(prog[lane & 3] >= nd)) {
                    if (pub < kt) {
                        asm volatile("s_waitcnt vmcnt(0)" ::: "memory");
                        pub = kt;
                        ctl[0] = pub;
                    }
                    while (!__all(prog[lane & 3] >= nd)) __builtin_amdgcn_s_sleep(2);
                }
                asm volatile("" ::: "memory");
                if ((nd & 7) == 0) STAMP(16 + (nd >> 3));
            }
            const size_t ko = (size_t)kt * BK * (NL * 2);
            LAS unsigned char* dst = lds + so;
            {
#pragma unroll
                for (int i = 0; i < 8; ++i)
#pragma unroll
                    for (int s = 0; s < 2; ++s)
                        __builtin_amdgcn_global_load_lds((const GAS unsigned*)(((i & 1) ? w_od : w_ev) + ko + i * 8 * (NL * 2) + s * 128),
                                                         (LAS unsigned*)(dst + s * 8192 + i * 1024), 16, 0, 0);
                if constexpr (NB == 10) {
#pragma unroll
                    for (int i = 0; i < 4; ++i)
                        __builtin_amdgcn_global_load_lds((const GAS unsigned*)(w_hf + ko + i * 16 * (NL * 2)),
                                                         (LAS unsigned*)(dst + 16384 + i * 1024), 16, 0, 0);
                }
            }
            so = so + SLOT == NSLOT * SLOT ? 0u : so + SLOT;
            if (kt + 1 - pub > 2) {
                asm volatile("s_waitcnt vmcnt(%0)" :: "n"(2 * NPIECE) : "memory");
                ++pub;
                ctl[0] = pub;
            }
        }
        asm volatile("s_waitcnt vmcnt(0)" ::: "memory");
        ctl[0] = NKT;
    }
    else if (wave == 5) {
        __builtin_amdgcn_s_setprio(2);
        GAS unsigned* flag = (GAS unsigned*)(myws + W_FLAG) + pm * NKT * 8 * FS;
        const unsigned long long t0 = rt();
        int cur = 0;
        unsigned long long npoll = 0;
        while (cur < NKT) {
            ++npoll;
            const int idx = cur * 8 + lane;
            unsigned v = call;
            if (lane < AG_PW * 8 && idx < NKT * 8) v = __hip_atomic_load(flag + idx * FS, __ATOMIC_RELAXED, __HIP_MEMORY_SCOPE_SYSTEM);
            const unsigned long long miss = __ballot((int)(v - call) < 0);
            const int adv = miss ? (__builtin_ctzll(miss) >> 3) : AG_PW;
            if (adv > 0) {
                if (cur == 0) STAMP(4);
                if (((cur + adv) >> 3) != (cur >> 3)) STAMP(24 + (((cur + adv) >> 3) & 7));
                cur = cur + adv < NKT ? cur + adv : NKT;
                asm volatile("" ::: "memory");
                ctl[1] = cur;
            } else {
                if (give_up(tmo, call, t0)) {
                    __hip_atomic_store(tmo, call, __ATOMIC_RELAXED, __HIP_MEMORY_SCOPE_AGENT);
                    ctl[2] = 1u;
                    ctl[1] = NKT;
                    break;
                }
                __builtin_amdgcn_s_sleep(AG_PSLEEP);
            }
        }
        STAMP(5);
    }
}
}

extern "C" void kernel_launch(int rank, int nranks, ncclComm_t comm,
                              hipStream_t stream,
                              void* const* d_in, const int* in_sizes, int n_in,
                              void* d_out, int out_size,
                              const int* in_dtypes, int out_dtype,
                              void* const* d_out_all,
                              void* const* d_ws_all, size_t ws_bytes) {
    if (nranks != ag::NR || (unsigned)rank >= (unsigned)ag::NR || n_in != 2 || ws_bytes < ag::W_END ||
        in_sizes[0] != ag::MS * ag::K || in_sizes[1] != ag::K * ag::NL || out_size != ag::M * ag::NL) {
        fprintf(stderr, "ag_gemm: unexpected problem shape\n");
        return;
    }
    ag::Args a;
    a.x = (const char*)d_in[0];
    a.w = (const char*)d_in[1];
    a.out = (char*)d_out;
    for (int i = 0; i < ag::NR; ++i) a.ws[i] = (char*)d_ws_all[(rank + 1 + i) % ag::NR];
    a.rank = rank;
    a.pad = 0;
    hipLaunchKernelGGL(ag::ag_gemm, dim3(ag::NWG), dim3(ag::NTHR), 0, stream, a);
    HIP_CHECK(hipGetLastError());
}
```

```cpp
#include <hip/hip_runtime.h>
#include <rccl/rccl.h>
#include <cstdio>
#include <cstdint>
#include <type_traits>

#define HIP_CHECK(expr) do { hipError_t _e = (expr); if (_e != hipSuccess) { \
    fprintf(stderr, "HIP error %s at %s:%d\n", hipGetErrorString(_e), __FILE__, __LINE__); \
    return; } } while (0)

#define LAS __attribute__((address_space(3)))
#define GAS __attribute__((address_space(1)))
typedef unsigned u32x4 __attribute__((ext_vector_type(4)));
typedef unsigned u32x2 __attribute__((ext_vector_type(2)));
typedef unsigned short u16x2 __attribute__((ext_vector_type(2)));
typedef short s16x4 __attribute__((ext_vector_type(4)));
typedef short s16x8 __attribute__((ext_vector_type(8)));
typedef __bf16 bf16x4 __attribute__((ext_vector_type(4)));
typedef __bf16 bf16x8 __attribute__((ext_vector_type(8)));
typedef float f32x4 __attribute__((ext_vector_type(4)));

#define AG_NB 10
#define AG_NSC 24
#define AG_SW 2
#define AG_SCHED 2
#define AG_DLOC 0
#define AG_PSLEEP 2
#define AG_ROT 0
#define AG_UAUX 17
#define AG_FS 32
#define AG_PW 2
#define AG_SB 52
#define AG_WARM 1
#define AG_OAUX 17
#define AG_RH 0
#define AG_RT 0
#define AG_AUXA 17

namespace ag {
constexpr int NR = 8;
constexpr int M = 4096, K = 4096, NL = 2048;
constexpr int MS = M / NR;
constexpr int NB = AG_NB;
constexpr int BM = 256, BN = NB * 16, BK = 64;
constexpr int NKT = K / BK;
constexpr int NG = MS / 32;
constexpr int BLK = 32 * BK * 2;
constexpr int NBLK = NG * NKT;
constexpr int SLOT = BK * BN * 2, NSLOT = NB == 10 ? 7 : 8, NPIECE = SLOT / 1024;
constexpr int NTN = (NL + BN - 1) / BN;
constexpr int NWG = 256, CPX = 32 - 2 * NTN;
constexpr int NWAVE = 8, NTHR = NWAVE * 64;
constexpr int NSC = AG_NSC < 8 * CPX ? AG_NSC : 8 * CPX, SW = AG_SW, NSEND = NSC * SW;
static_assert(NB == 8 || NB == 10, "tile width");
static_assert(2 * NSEND <= NBLK && SW <= NWAVE, "sender geometry");
constexpr int PBLK = BLK * 3 / 4;
constexpr int RH = AG_RH, RT = AG_RT;
constexpr int NPK = (NKT - RH - RT) * NG * (NR - 1);
constexpr int NDW = AG_DLOC == 0 ? (8 * CPX - NSC) * NWAVE : 16 * NTN * 2;
constexpr int DSLOTS = (NPK + NDW - 1) / NDW, DSTRIDE = (DSLOTS * 16 + 127) / 128 * 128;
static_assert(NDW > 0 && DSLOTS <= 64, "unpacker geometry");
constexpr int L_CTL = NSLOT * SLOT;
constexpr int L_PROG = L_CTL + 64;
constexpr int LDS_BYTES = L_CTL + 128;
static_assert(LDS_BYTES > 80 * 1024 && LDS_BYTES <= 160 * 1024, "one workgroup per CU");
constexpr size_t W_CNT = 0;
constexpr size_t W_TMO = W_CNT + (size_t)NWG * 128;
constexpr size_t W_TICKET = W_TMO + 1024;
constexpr size_t W_PF = 128 * 1024;
constexpr size_t PF_BYTES = (size_t)NDW * DSTRIDE;
constexpr size_t W_DBG = 768 * 1024;
constexpr size_t W_XT = 1u << 20;
constexpr size_t XT_BYTES = (size_t)M * K * 2;
constexpr size_t W_PK = W_XT + 2 * XT_BYTES;
constexpr size_t PK_BYTES = (size_t)NR * NBLK * PBLK;
constexpr size_t W_FLAG = W_PK + 2 * PK_BYTES;
constexpr int FS = AG_FS;
constexpr size_t W_END = W_FLAG + (size_t)NR * NBLK * FS * 4;
static_assert(W_TICKET + 128 <= W_PF && W_PF + 2 * PF_BYTES <= W_DBG, "flags");
constexpr unsigned OOB = 0x40000000u;
constexpr unsigned long long DEADLINE = 30000000ull;

struct Args {
    const char* x;
    const char* w;
    char* out;
    char* ws[NR];
    int rank;
    int pad;
};

__device__ __forceinline__ unsigned long long rt() { return __builtin_amdgcn_s_memrealtime(); }
__device__ __forceinline__ __amdgpu_buffer_rsrc_t rsrc(const void* p, unsigned bytes) {
    return __builtin_amdgcn_make_buffer_rsrc(const_cast<void*>(p), 0, bytes, 0x00020000);
}
__device__ __forceinline__ void store16(u32x4 d, __amdgpu_buffer_rsrc_t r, unsigned off) {
    __builtin_amdgcn_raw_buffer_store_b128(d, r, off, 0, 17);
}
__device__ __forceinline__ void wait_vm0() {
    __builtin_amdgcn_s_waitcnt(0x0F70);
    asm volatile("s_waitcnt vmcnt(0)" ::: "memory");
}
__device__ __forceinline__ bool give_up(GAS unsigned* tmo, unsigned call, unsigned long long t0) {
    const unsigned long long dt = rt() - t0;
    if (dt < 50000ull) return false;
    return dt > DEADLINE || __builtin_amdgcn_readfirstlane(__hip_atomic_load(tmo, __ATOMIC_RELAXED, __HIP_MEMORY_SCOPE_AGENT)) == call;
}
#define STAMP(i) do {} while (0)
#define STAMPC(i) do {} while (0)

__global__ void __launch_bounds__(NTHR, 2) ag_gemm(const Args a) {
    __shared__ __attribute__((aligned(1024))) unsigned char lds_raw[LDS_BYTES];
    LAS unsigned char* lds = (LAS unsigned char*)lds_raw;
    volatile LAS unsigned* ctl = (volatile LAS unsigned*)(lds + L_CTL);
    volatile LAS unsigned* prog = (volatile LAS unsigned*)(lds + L_PROG);
    const int lane = threadIdx.x & 63;
    const int wave = __builtin_amdgcn_readfirstlane(threadIdx.x >> 6);
    const int bid = blockIdx.x;
    char* const myws = a.ws[NR - 1];
    const int xcd = bid & 7, loc = bid >> 3;
    const bool is_comm = loc < CPX;
    const int pm = ((xcd + AG_ROT) & 7) * 2 + ((loc - CPX) & 1), pn = (loc - CPX) >> 1;

    if (wave == 0) STAMP(0);
    const int cid = loc * 8 + xcd;
    auto fetch = [&](u32x4 (&d)[4], unsigned b) {
        const unsigned bb = b < (unsigned)NBLK ? b : 0u;
        const GAS char* p = (const GAS char*)a.x + (size_t)((bb % NG) * 32 + (lane & 15)) * (K * 2) + (bb / NG) * (BK * 2) + (lane >> 4) * 16;
#pragma unroll
        for (int s = 0; s < 4; ++s) d[s] = *(const GAS u32x4*)(p + (s >> 1) * 16 * (K * 2) + (s & 1) * 64);
    };
    auto begin = [&]() {
        GAS unsigned* cnt = (GAS unsigned*)(myws + W_CNT + (size_t)bid * 128);
        const unsigned c = __builtin_amdgcn_readfirstlane(__hip_atomic_load(cnt, __ATOMIC_RELAXED, __HIP_MEMORY_SCOPE_AGENT)) + 1u;
        if (threadIdx.x < 32) ((LAS unsigned*)(lds + L_CTL))[threadIdx.x] = 0u;
        __syncthreads();
        if (threadIdx.x == 0) __hip_atomic_store(cnt, c, __ATOMIC_RELAXED, __HIP_MEMORY_SCOPE_AGENT);
        return c;
    };
    if (is_comm && cid < NSC) {
        u32x4 d[4];
        if (wave < SW) fetch(d, wave * NSC + cid);
        if (wave == 0) STAMP(10);
        const unsigned call = begin();
        if (wave == 0) STAMP(11);
        const size_t xt_off = W_XT + (size_t)(call & 1u) * XT_BYTES;
        const size_t pk_off = W_PK + (size_t)(call & 1u) * PK_BYTES, pf_off = W_PF + (size_t)(call & 1u) * PF_BYTES;
        if (wave < SW) {
            const int sid = wave * NSC + cid;
            __amdgpu_buffer_rsrc_t rd[NR];
#pragma unroll
            for (int i = 0; i < NR; ++i) rd[i] = rsrc(a.ws[i], (unsigned)W_END);
            const unsigned xt0 = (unsigned)xt_off + a.rank * NBLK * BLK, pk0 = (unsigned)pk_off + a.rank * NBLK * PBLK;
            char* fw = a.ws[0];
#pragma unroll
            for (int i = 1; i < NR; ++i) fw = (lane & 7) == i ? a.ws[i] : fw;
            GAS unsigned* flag = (GAS unsigned*)(fw + W_FLAG) + a.rank * NBLK * FS;
            GAS char* pflag = (GAS char*)fw + pf_off;
            const unsigned tbase = (call - 1u) * (unsigned)NBLK - 2u * NSEND;
            auto pack = [&](const u32x4 (&d)[4], u32x4 (&pk)[3], unsigned& base, unsigned long long& miss) {
                unsigned h[8], hm[8];
                u16x2 mx = (u16x2){0, 0};
#pragma unroll
                for (int j = 0; j < 8; ++j) {
                    const unsigned d0 = d[j >> 1][(j & 1) * 2], d1 = d[j >> 1][(j & 1) * 2 + 1];
                    pk[j >> 2][j & 3] = __builtin_amdgcn_perm(d1, d0, 0x06040200u);
                    h[j] = __builtin_amdgcn_perm(d1, d0, 0x07050301u);
                    hm[j] = h[j] | 0x80808080u;
                    mx = __builtin_elementwise_max(mx, __builtin_bit_cast(u16x2, hm[j]));
                    mx = __builtin_elementwise_max(mx, __builtin_bit_cast(u16x2, hm[j] << 8));
                }
                unsigned m = (mx.x > mx.y ? mx.x : mx.y) >> 8;
                m = max(m, (unsigned)__builtin_amdgcn_update_dpp(0, (int)m, 0x111, 0xf, 0xf, true));
                m = max(m, (unsigned)__builtin_amdgcn_update_dpp(0, (int)m, 0x112, 0xf, 0xf, true));
                m = max(m, (unsigned)__builtin_amdgcn_update_dpp(0, (int)m, 0x114, 0xf, 0xf, true));
                m = max(m, (unsigned)__builtin_amdgcn_update_dpp(0, (int)m, 0x118, 0xf, 0xf, true));
                const unsigned top = max(max((unsigned)__builtin_amdgcn_readlane(m, 15), (unsigned)__builtin_amdgcn_readlane(m, 31)),
                                         max((unsigned)__builtin_amdgcn_readlane(m, 47), (unsigned)__builtin_amdgcn_readlane(m, 63))) & 127u;
                base = top > 7u ? top - 7u : 0u;
                const unsigned b4 = base * 0x01010101u;
                unsigned all = 0x80808080u;
#pragma unroll
                for (int j = 0; j < 4; ++j) {
                    const unsigned ra = hm[2 * j] - b4, rb = hm[2 * j + 1] - b4;
                    all &= ra & rb;
                    const unsigned na = (h[2 * j] & 0x80808080u) | (ra & 0x07070707u);
                    const unsigned nb = (h[2 * j + 1] & 0x80808080u) | (rb & 0x07070707u);
                    pk[2][j] = na | __builtin_amdgcn_alignbit(nb, nb, 28);
                }
                miss = __ballot(all != 0x80808080u);
            };
            unsigned b0 = sid, b1 = sid + NSEND;
            u32x4 nx[4], pk[3];
            unsigned base;
            unsigned long long miss;
            pack(d, pk, base, miss);
            if (wave == 0) STAMP(6);
            unsigned long long nsent = 0;
            while (b0 < (unsigned)NBLK) {
                fetch(nx, b1);
                const unsigned tk = __builtin_amdgcn_raw_ptr_buffer_atomic_add_i32(1, rd[NR - 1], lane == 0 ? (unsigned)W_TICKET : OOB, 0, 0);
                const unsigned g = b0 % NG, kt = b0 / NG;
                const unsigned so = xt0 + b0 * BLK + lane * 16, sp = pk0 + b0 * PBLK + lane * 16;
                const bool plain = kt - RH >= (unsigned)(NKT - RH - RT);
                if (plain || __builtin_popcountll(miss) > 16) miss = ~0ull;
                if (miss != 0ull) {
                    const unsigned vo = (miss >> lane) & 1ull ? so : OOB;
#pragma unroll
                    for (int i = 0; i < NR - 1; ++i)
#pragma unroll
                        for (int s = 0; s < 4; ++s)
                            store16(d[s], rd[i], vo + s * 1024);
                }
                const unsigned vp = miss != ~0ull ? sp : OOB;
#pragma unroll
                for (int i = 0; i < NR - 1; ++i)
#pragma unroll
                    for (int s = 0; s < 3; ++s)
                        store16(pk[s], rd[i], vp + s * 1024);
#pragma unroll
                for (int s = 0; s < 4; ++s)
                    store16(d[s], rd[NR - 1], so + s * 1024);
                const u32x4 rec = (u32x4){(call << 8) | base, (unsigned)miss, call, (unsigned)(miss >> 32)};
#pragma unroll
                for (int s = 0; s < 4; ++s) d[s] = nx[s];
                pack(d, pk, base, miss);
                wait_vm0();
                if (lane < NR) {
                    if (plain || lane == NR - 1) {
                        __hip_atomic_store(flag + (((g >> 3) * NKT + kt) * 8 + (g & 7)) * FS, call, __ATOMIC_RELAXED, __HIP_MEMORY_SCOPE_SYSTEM);
                    } else {
                        const unsigned sq = (b0 - RH * NG) * (NR - 1) + lane;
                        GAS char* q = pflag + (sq % NDW) * DSTRIDE + (sq / NDW) * 16;
                        asm volatile("global_store_dwordx4 %0, %1, off sc0 sc1\n\ts_nop 1" :: "v"(q), "v"(rec) : "memory");
                    }
                }
                b0 = b1;
                ++nsent;
                b1 = __builtin_amdgcn_readfirstlane(tk) - tbase;
            }
            if (wave == 0) STAMP(7);
        }
        return;
    }
    const unsigned call = begin();
    GAS unsigned* tmo = (GAS unsigned*)(myws + W_TMO);
    const size_t xt_off = W_XT + (size_t)(call & 1u) * XT_BYTES;
    const bool has_half = NB == 10 && (pn + 1) * BN <= NL;
    const int oS = (NB == 10 && (pn & 1)) ? 64 : 0, oH = (NB == 10 && !(pn & 1) && has_half) ? 256 : 0;

    const size_t pk_off = W_PK + (size_t)(call & 1u) * PK_BYTES, pf_off = W_PF + (size_t)(call & 1u) * PF_BYTES;
    if (AG_DLOC == 0 ? is_comm : (!is_comm && wave >= 6)) {
        const int dw = AG_DLOC == 0 ? wave * (NDW / NWAVE) + (cid - NSC) : (wave - 6) * (NDW / 2) + (cid - 8 * CPX);
        const unsigned sq = lane * NDW + dw, sv = sq < (unsigned)NPK ? sq : 0u;
        const unsigned bq = sv / (NR - 1) + RH * NG, from = (a.rank + 2 * NR - 1 - sv % (NR - 1)) % NR;
        const unsigned g = bq % NG, kt = bq / NG, bi = from * NBLK + bq;
        GAS unsigned* flag = (GAS unsigned*)(myws + W_FLAG) + (from * NBLK + ((g >> 3) * NKT + kt) * 8 + (g & 7)) * FS;
        const __amdgpu_buffer_rsrc_t rf = rsrc(myws + pf_off + (size_t)dw * DSTRIDE, DSTRIDE);
        const __amdgpu_buffer_rsrc_t rp = rsrc(myws + pk_off, (unsigned)PK_BYTES);
        const __amdgpu_buffer_rsrc_t rx = rsrc(myws + xt_off, (unsigned)XT_BYTES);
        unsigned long long todo = __ballot(sq < (unsigned)NPK), pend = 0;
        const unsigned long long t0 = rt();
        auto get = [&](u32x4 (&p)[3], int j, bool on) {
            const unsigned o = __builtin_amdgcn_readlane(bi, j) * PBLK, vo = on ? lane * 16u : OOB;
#pragma unroll
            for (int s = 0; s < 3; ++s)
                p[s] = __builtin_bit_cast(u32x4, __builtin_amdgcn_raw_buffer_load_b128(rp, vo + s * 1024, o, 17));
        };
        auto put = [&](const u32x4 (&p)[3], const u32x4& rec, int j, bool on) {
            const unsigned o = __builtin_amdgcn_readlane(bi, j) * BLK;
            const unsigned b4 = (__builtin_amdgcn_readlane(rec.x, j) & 127u) * 0x01010101u;
            const unsigned long long miss = (unsigned)__builtin_amdgcn_readlane(rec.y, j)
                                          | ((unsigned long long)(unsigned)__builtin_amdgcn_readlane(rec.w, j) << 32);
            const unsigned vo = !on || ((miss >> lane) & 1ull) ? OOB : o + lane * 16u;
#pragma unroll
            for (int s = 0; s < 4; ++s) {
                const unsigned n = p[2][s];
                const unsigned ha = (n & 0x87878787u) + b4, hb = (__builtin_amdgcn_alignbit(n, n, 4) & 0x87878787u) + b4;
                const unsigned la = p[s >> 1][(s & 1) * 2], lb = p[s >> 1][(s & 1) * 2 + 1];
                const u32x4 d = (u32x4){__builtin_amdgcn_perm(ha, la, 0x05010400u), __builtin_amdgcn_perm(ha, la, 0x07030602u),
                                        __builtin_amdgcn_perm(hb, lb, 0x05010400u), __builtin_amdgcn_perm(hb, lb, 0x07030602u)};
                __builtin_amdgcn_raw_buffer_store_b128(d, rx, vo + s * 1024, 0, AG_UAUX);
            }
        };
        if (wave == 7) STAMP(8);
        unsigned long long lat_r = 0, lat_w = 0, lat_n = 0;
        for (;;) {
            const unsigned head = todo ? __builtin_ctzll(todo) : 0u;
            const u32x4 rec = __builtin_bit_cast(u32x4, __builtin_amdgcn_raw_buffer_load_b128(
                rf, (unsigned)lane - head < 8u ? lane * 16u : OOB, 0, 17));
            wait_vm0();
            if (pend) {
                if ((pend >> lane) & 1ull) __hip_atomic_store(flag, call, __ATOMIC_RELAXED, __HIP_MEMORY_SCOPE_SYSTEM);
                pend = 0;
            }
            if (!todo) break;
            const unsigned long long rdy = todo & __ballot((int)(rec.z - call) >= 0 && (rec.x >> 8) == (rec.z & 0xffffffu));
            if (rdy) {
                const int j0 = __builtin_ctzll(rdy);
                const unsigned long long r1 = rdy & (rdy - 1);
                const int j1 = r1 ? __builtin_ctzll(r1) : j0;
                u32x4 p0[3], p1[3];
                get(p0, j0, true);
                get(p1, j1, r1 != 0);
                put(p0, rec, j0, true);
                put(p1, rec, j1, r1 != 0);
                pend = (1ull << j0) | (1ull << j1);
                todo &= ~pend;
            } else {
                if (give_up(tmo, call, t0)) {
                    __hip_atomic_store(tmo, call, __ATOMIC_RELAXED, __HIP_MEMORY_SCOPE_AGENT);
                    break;
                }
                __builtin_amdgcn_s_sleep(1);
            }
        }
        if (wave == 7) STAMP(9);
    } else if (is_comm) {
    }
    else if (wave < 4 || (AG_WARM && wave == 7)) {
        const int g0 = (pm & 1) * 8 + (wave & 3) * 2;
        const __amdgpu_buffer_rsrc_t ra = rsrc(myws + xt_off + ((size_t)(pm >> 1) * NBLK + g0) * BLK, (NBLK - g0) * BLK);
        const unsigned va = lane * 16;
        const int g = lane >> 4, q = (lane >> 2) & 3, p = lane & 3;
        const unsigned vbS = g * 1024 + q * 128 + 32 * ((q >> 1) ^ (2 * (g & 1))) + 8 * p;
        const unsigned vbH = 16384 + (g >> 1) * 1024 + 64 * (8 * (g & 1) + q) + 32 * (g & 1) + 8 * p;
        f32x4 acc[4][NB];
#pragma unroll
        for (int m = 0; m < 4; ++m)
#pragma unroll
            for (int n = 0; n < NB; ++n) acc[m][n] = (f32x4){0.f, 0.f, 0.f, 0.f};
        u32x4 af[4][4];
        constexpr int PB = 3;
        s16x8 bf[PB + 1];
        unsigned have_b = 0, have_a = 0;
        auto need = [&](unsigned nb, unsigned na) {
            if (have_b < nb || have_a < na) {
                for (;;) {
                    const u32x2 v = *(volatile LAS u32x2*)(lds + L_CTL);
                    have_b = __builtin_amdgcn_readfirstlane(v.x);
                    have_a = __builtin_amdgcn_readfirstlane(v.y);
                    if (have_b >= nb && have_a >= na) break;
                    __builtin_amdgcn_s_sleep(1);
                }
            }
            asm volatile("" ::: "memory");
        };
        auto read_h = [&](unsigned slot_off, int ks, int n, int t) {
            if (n < 8)
                return __builtin_amdgcn_ds_read_tr16_b64_v4i16(
                    (LAS s16x4*)(lds + ((vbS + slot_off) ^ (32 * ((n & 3) ^ (2 * t)))) + (n >> 2) * 8192 + ks * 4096 + t * 512));
            return __builtin_amdgcn_ds_read_tr16_b64_v4i16(
                (LAS s16x4*)(lds + ((vbH + slot_off) ^ (32 * (n & 1))) + ks * 2048 + t * 256));
        };
        auto load_a = [&](int kt, int ks, int m) {
            return __builtin_bit_cast(u32x4, __builtin_amdgcn_raw_buffer_load_b128(
                ra, va, kt * (NG * BLK) + m * 2048 + ks * 1024, AG_AUXA));
        };
#define SB() __builtin_amdgcn_sched_barrier(0)
#define MFMA(C, B, A) asm volatile("v_mfma_f32_16x16x32_bf16 %0, %1, %2, %0" : "+v"(C) : "v"(B), "v"(A))
        if (wave < 4) {
        need(1, 2);
        if (wave == 0) { STAMP(1); STAMPC(32); __builtin_amdgcn_s_waitcnt(0); }
#pragma unroll
        for (int j = 0; j < 3; ++j)
#pragma unroll
            for (int m = 0; m < 4; ++m) af[j][m] = load_a(j >> 1, j & 1, m);
#pragma unroll
        for (int i = 0; i < PB; ++i) bf[i] = __builtin_shufflevector(read_h(0, 0, i, 0), read_h(0, 0, i, 1), 0, 1, 2, 3, 4, 5, 6, 7);
        unsigned so_cur = 0;
        auto tile = [&](int kt, auto PAR, auto LAST) {
            constexpr int par = PAR.value;
            constexpr int last = LAST.value;
            const int k1 = last && kt + 1 >= NKT ? NKT - 1 : kt + 1;
            if constexpr (last == 0) need(kt + 2, kt + 3); else need(k1 + 1, k1 + 1);
            const unsigned so_nxt = so_cur + SLOT == NSLOT * SLOT ? 0u : so_cur + SLOT;
#pragma unroll
            for (int ks = 0; ks < 2; ++ks) {
                const int j = par * 2 + ks;
#pragma unroll
                for (int n = 0; n < NB; ++n) {
                    const int i = ks * NB + n + PB;
                    const int r = (par * 2 * NB + i) % (PB + 1), u = (par * 2 * NB + ks * NB + n) % (PB + 1);
#define RD(t) (i < 2 * NB ? read_h(so_cur, i / NB, i % NB, t) : read_h(so_nxt, 0, i - 2 * NB, t))
#define LDA() do { if (last == 0) { if (n < 4) af[(j + 3) & 3][n] = ks == 0 ? load_a(kt + 1, 1, n) : load_a(kt + 2, 0, n); } \
                   else if (ks == 0 && n < 8) af[(par ^ 1) * 2 + (n >> 2)][n & 3] = load_a(k1, n >> 2, n & 3); } while (0)
                    SB(); MFMA(acc[0][n], bf[u], af[j][0]); SB();
                    const s16x4 lo = RD(0);
                    SB(); MFMA(acc[1][n], bf[u], af[j][1]); SB();
                    const s16x4 hi = RD(1);
                    bf[r] = __builtin_shufflevector(lo, hi, 0, 1, 2, 3, 4, 5, 6, 7);
                    SB(); MFMA(acc[2][n], bf[u], af[j][2]); SB();
                    LDA();
                    SB(); MFMA(acc[3][n], bf[u], af[j][3]); SB();
                }
            }
            prog[wave] = kt + 1;
            so_cur = so_nxt;
        };
        static_assert((4 * NB) % (PB + 1) == 0, "fragment ring closes over two K-tiles");
        using I0 = std::integral_constant<int, 0>;
        using I1 = std::integral_constant<int, 1>;
        for (int kt = 0; kt < AG_SB; kt += 2) {
            tile(kt, I0{}, I0{});
            tile(kt + 1, I1{}, I0{});
        }
#pragma nounroll
        for (int kt = AG_SB; kt < NKT; kt += 2) {
            tile(kt, I0{}, I1{});
            tile(kt + 1, I1{}, I1{});
        }
        }
        asm volatile("s_nop 15" ::: "memory");
        SB();
        if (wave == 0) { STAMP(2); STAMPC(33); }
        const bool bad = ctl[2] != 0u || __hip_atomic_load(tmo, __ATOMIC_RELAXED, __HIP_MEMORY_SCOPE_AGENT) == call;
        int l2;
        asm volatile("v_mbcnt_lo_u32_b32 %0, -1, 0\n\tv_mbcnt_hi_u32_b32 %0, -1, %0" : "=v"(l2));
        const __amdgpu_buffer_rsrc_t ro = rsrc(a.out + ((size_t)(pm * BM + (wave & 3) * 64) * NL + pn * BN) * 2,
                                               wave < 4 ? (64 * NL - pn * BN) * 2 : 0);
        const unsigned vo = (l2 & 15) * (NL * 2) + ((l2 >> 4) & 1) * 32 + (l2 >> 5) * 16;
        const unsigned voS = vo + oS, voH = has_half ? vo + oH : OOB;
#pragma unroll
        for (int m = 0; m < 4; ++m)
#pragma unroll
            for (int n = 0; n < NB; n += 2) {
                const u32x2 X = __builtin_bit_cast(u32x2, __builtin_convertvector(acc[m][n], bf16x4));
                const u32x2 Y = __builtin_bit_cast(u32x2, __builtin_convertvector(acc[m][n + 1], bf16x4));
                const auto s0 = __builtin_amdgcn_permlane16_swap(X.x, Y.x, false, false);
                const auto s1 = __builtin_amdgcn_permlane16_swap(X.y, Y.y, false, false);
                u32x4 o = (u32x4){s0[0], s1[0], s0[1], s1[1]};
                if (bad) o = (u32x4){0x7fc07fc0u, 0x7fc07fc0u, 0x7fc07fc0u, 0x7fc07fc0u};
                __builtin_amdgcn_raw_buffer_store_b128(o, ro, (n < 8 ? voS + n * 32 : voH) + m * 16 * NL * 2, 0, AG_OAUX);
            }
        if (wave == 0) STAMP(3);
    } else if (wave == 4) {
        __builtin_amdgcn_s_setprio(2);
        const int r8 = lane >> 3, r16 = lane >> 2;
        const GAS char* wb = (const GAS char*)a.w + pn * BN * 2;
        const GAS char* w_ev = wb + oS + r8 * (NL * 2) + ((((lane >> 1) & 3) ^ (r8 >> 1)) * 32) + (lane & 1) * 16;
        const GAS char* w_od = wb + oS + r8 * (NL * 2) + ((((lane >> 1) & 3) ^ (r8 >> 1) ^ 2) * 32) + (lane & 1) * 16;
        const GAS char* w_hf = wb + oH + r16 * (NL * 2) + ((((lane >> 1) & 1) ^ (r16 >> 3)) * 32) + (lane & 1) * 16;
        int pub = 0;
        unsigned so = 0;
        for (int kt = 0; kt < NKT; ++kt) {
            if (kt >= NSLOT) {
                const unsigned nd = kt - NSLOT + 1;
                if (!__all(prog[lane & 3] >= nd)) {
                    if (pub < kt) {
                        asm volatile("s_waitcnt vmcnt(0)" ::: "memory");
                        pub = kt;
                        ctl[0] = pub;
                    }
                    while (!__all(prog[lane & 3] >= nd)) __builtin_amdgcn_s_sleep(2);
                }
                asm volatile("" ::: "memory");
                if ((nd & 7) == 0) STAMP(16 + (nd >> 3));
            }
            const size_t ko = (size_t)kt * BK * (NL * 2);
            LAS unsigned char* dst = lds + so;
            {
#pragma unroll
                for (int i = 0; i < 8; ++i)
#pragma unroll
                    for (int s = 0; s < 2; ++s)
                        __builtin_amdgcn_global_load_lds((const GAS unsigned*)(((i & 1) ? w_od : w_ev) + ko + i * 8 * (NL * 2) + s * 128),
                                                         (LAS unsigned*)(dst + s * 8192 + i * 1024), 16, 0, 0);
                if constexpr (NB == 10) {
#pragma unroll
                    for (int i = 0; i < 4; ++i)
                        __builtin_amdgcn_global_load_lds((const GAS unsigned*)(w_hf + ko + i * 16 * (NL * 2)),
                                                         (LAS unsigned*)(dst + 16384 + i * 1024), 16, 0, 0);
                }
            }
            so = so + SLOT == NSLOT * SLOT ? 0u : so + SLOT;
            if (kt + 1 - pub > 2) {
                asm volatile("s_waitcnt vmcnt(%0)" :: "n"(2 * NPIECE) : "memory");
                ++pub;
                ctl[0] = pub;
            }
        }
        asm volatile("s_waitcnt vmcnt(0)" ::: "memory");
        ctl[0] = NKT;
    }
    else if (wave == 5) {
        __builtin_amdgcn_s_setprio(2);
        GAS unsigned* flag = (GAS unsigned*)(myws + W_FLAG) + pm * NKT * 8 * FS;
        const unsigned long long t0 = rt();
        int cur = 0;
        unsigned long long npoll = 0;
        while (cur < NKT) {
            ++npoll;
            const int idx = cur * 8 + lane;
            unsigned v = call;
            if (lane < AG_PW * 8 && idx < NKT * 8) v = __hip_atomic_load(flag + idx * FS, __ATOMIC_RELAXED, __HIP_MEMORY_SCOPE_SYSTEM);
            const unsigned long long miss = __ballot((int)(v - call) < 0);
            const int adv = miss ? (__builtin_ctzll(miss) >> 3) : AG_PW;
            if (adv > 0) {
                if (cur == 0) STAMP(4);
                if (((cur + adv) >> 3) != (cur >> 3)) STAMP(24 + (((cur + adv) >> 3) & 7));
                cur = cur + adv < NKT ? cur + adv : NKT;
                asm volatile("" ::: "memory");
                ctl[1] = cur;
            } else {
                if (give_up(tmo, call, t0)) {
                    __hip_atomic_store(tmo, call, __ATOMIC_RELAXED, __HIP_MEMORY_SCOPE_AGENT);
                    ctl[2] = 1u;
                    ctl[1] = NKT;
                    break;
                }
                __builtin_amdgcn_s_sleep(AG_PSLEEP);
            }
        }
        STAMP(5);
    }
}
}

extern "C" void kernel_launch(int rank, int nranks, ncclComm_t comm,
                              hipStream_t stream,
                              void* const* d_in, const int* in_sizes, int n_in,
                              void* d_out, int out_size,
                              const int* in_dtypes, int out_dtype,
                              void* const* d_out_all,
                              void* const* d_ws_all, size_t ws_bytes) {
    if (nranks != ag::NR || (unsigned)rank >= (unsigned)ag::NR || n_in != 2 || ws_bytes < ag::W_END ||
        in_sizes[0] != ag::MS * ag::K || in_sizes[1] != ag::K * ag::NL || out_size != ag::M * ag::NL) {
        fprintf(stderr, "ag_gemm: unexpected problem shape\n");
        return;
    }
    ag::Args a;
    a.x = (const char*)d_in[0];
    a.w = (const char*)d_in[1];
    a.out = (char*)d_out;
    for (int i = 0; i < ag::NR; ++i) a.ws[i] = (char*)d_ws_all[(rank + 1 + i) % ag::NR];
    a.rank = rank;
    a.pad = 0;
    hipLaunchKernelGGL(ag::ag_gemm, dim3(ag::NWG), dim3(ag::NTHR), 0, stream, a);
    HIP_CHECK(hipGetLastError());
}
```

```cpp
#include <hip/hip_runtime.h>
#include <rccl/rccl.h>
#include <cstdio>
#include <cstdint>
#include <type_traits>

#define HIP_CHECK(expr) do { hipError_t _e = (expr); if (_e != hipSuccess) { \
    fprintf(stderr, "HIP error %s at %s:%d\n", hipGetErrorString(_e), __FILE__, __LINE__); \
    return; } } while (0)

#define LAS __attribute__((address_space(3)))
#define GAS __attribute__((address_space(1)))
typedef unsigned u32x4 __attribute__((ext_vector_type(4)));
typedef unsigned u32x2 __attribute__((ext_vector_type(2)));
typedef unsigned short u16x2 __attribute__((ext_vector_type(2)));
typedef short s16x4 __attribute__((ext_vector_type(4)));
typedef short s16x8 __attribute__((ext_vector_type(8)));
typedef __bf16 bf16x4 __attribute__((ext_vector_type(4)));
typedef __bf16 bf16x8 __attribute__((ext_vector_type(8)));
typedef float f32x4 __attribute__((ext_vector_type(4)));
typedef float f32x16 __attribute__((ext_vector_type(16)));
typedef float f32x32 __attribute__((ext_vector_type(32)));

#define AG_SW 4
#define AG_PSLEEP 2
#define AG_TURN 1
#define AG_UPRIO 0
#define AG_PF 2
#define AG_PF0 3
#define AG_PFAUX 0
#define AG_PFD 0
#define AG_CPRIO 0
#define AG_WSLEEP 2
#define AG_WPRIO 0
#define AG_T1 1
#define AG_A32 1
#define AG_PQ 0
#define AG_PPRIO 2
#define AG_HOLE 0
#define AG_EARLY 2
#define AG_SD 0
#define AG_PP 0
#define AG_MAP 1
#define AG_FAUX 17
#define AG_USLEEP 1
#define AG_HAUX 17
#define AG_UAUX 17
#define AG_FS 32
#define AG_PW 2
#define AG_OAUX 17

namespace ag {
constexpr int NR = 8;
constexpr int M = 4096, K = 4096, NL = 2048;
constexpr int MS = M / NR;
constexpr int NB = 9;
constexpr int BM = 256, BN = 128, BK = 64;
constexpr int NKT = K / BK;
constexpr int NG = MS / 32;
constexpr int BLK = 32 * BK * 2;
constexpr int NBLK = NG * NKT;
constexpr int RS = NB * 32;
constexpr int SLOT = BK * RS, NSLOT = 8, NPIECE = SLOT / 1024;
constexpr int NTN = NL / BN - 1;
constexpr int NWG = 256, CPX = 4;
constexpr int NWAVE = 8, NTHR = NWAVE * 64;
constexpr int NSC = NWG - 16 * NTN, SW = AG_SW, NSEND = NSC * SW;
static_assert(2 * NSEND <= NBLK && SW <= NWAVE, "sender geometry");
constexpr int PBLK = 3072, NPATCH = 24, NTAIL = 88;
constexpr int NU = 4 * NTN;
constexpr int L_CTL = NSLOT * SLOT;
constexpr unsigned ALL = 1000;
constexpr int L_UNP = L_CTL + 128, UNP = 2048 + 64 + 448;
constexpr int L_SEL = L_UNP + 2 * UNP, LDS_BYTES = L_SEL + 2048;
static_assert(LDS_BYTES > 80 * 1024 && LDS_BYTES <= 160 * 1024, "one workgroup per CU");
constexpr size_t W_CNT = 0;
constexpr size_t W_TMO = W_CNT + (size_t)NWG * 128;
constexpr size_t W_TICKET = W_TMO + 1024;
constexpr size_t W_DBG = 768 * 1024;
constexpr size_t W_XT = 1u << 20;
constexpr size_t XT_BYTES = (size_t)M * K * 2;
constexpr size_t W_PK = W_XT + 2 * XT_BYTES;
constexpr size_t PK_BYTES = (size_t)NR * NBLK * PBLK;
constexpr size_t W_FLAG = W_PK + 2 * PK_BYTES;
constexpr int FS = AG_FS;
constexpr unsigned FL_BYTES = NR * NBLK * FS * 4;
constexpr size_t W_END = W_FLAG + 2 * (size_t)FL_BYTES;
static_assert(W_TICKET + 128 <= W_DBG, "flags");
constexpr unsigned OOB = 0x40000000u;
constexpr unsigned long long DEADLINE = 30000000ull;

struct Args {
    const char* x;
    const char* w;
    char* out;
    char* ws[NR];
    int rank;
    int pad;
};

__device__ __forceinline__ unsigned long long rt() { return __builtin_amdgcn_s_memrealtime(); }
__device__ __forceinline__ __amdgpu_buffer_rsrc_t rsrc(const void* p, unsigned bytes) {
    return __builtin_amdgcn_make_buffer_rsrc(const_cast<void*>(p), 0, bytes, 0x00020000);
}
__device__ __forceinline__ void store16(u32x4 d, __amdgpu_buffer_rsrc_t r, unsigned off) {
    __builtin_amdgcn_raw_buffer_store_b128(d, r, off, 0, 17);
}
__device__ __forceinline__ void wait_vm0() {
    __builtin_amdgcn_s_waitcnt(0x0F70);
    asm volatile("s_waitcnt vmcnt(0)" ::: "memory");
}
__device__ __forceinline__ bool give_up(GAS unsigned* tmo, unsigned call, unsigned long long t0) {
    const unsigned long long dt = rt() - t0;
    if (dt < 50000ull) return false;
    return dt > DEADLINE || __builtin_amdgcn_readfirstlane(__hip_atomic_load(tmo, __ATOMIC_RELAXED, __HIP_MEMORY_SCOPE_AGENT)) == call;
}
constexpr unsigned B1 = 0x01010101u;
__device__ __forceinline__ unsigned tag_of(unsigned call) { return ((call + 1u) << 30) & 0x80000000u; }
__device__ __forceinline__ unsigned bfi(unsigned m, unsigned a, unsigned b) { return (a & m) | (b & ~m); }
__device__ __forceinline__ unsigned perm(unsigned hi, unsigned lo, unsigned sel) { return __builtin_amdgcn_perm(hi, lo, sel); }
__device__ __forceinline__ unsigned key2(unsigned e0) { return (((e0 + 3u) & 3u) ^ 3u) * B1; }
__device__ __forceinline__ unsigned table2(unsigned e0) {
    return e0 * B1 + (((0x03020100u ^ key2(e0)) + (4u - (e0 & 3u)) * B1) & 3u * B1);
}
__device__ __forceinline__ unsigned table3(unsigned e0, unsigned hi) {
    const unsigned r = (e0 & 7u) * B1;
    return e0 * B1 - ((8u * B1 + r - ((0x03020100u + hi * 4u * B1) ^ r)) & 7u * B1) + (hi ? 0u : 3u);
}
__device__ __forceinline__ unsigned scan(unsigned v) {
    v += (unsigned)__builtin_amdgcn_update_dpp(0, (int)v, 0x111, 0xf, 0xf, true);
    v += (unsigned)__builtin_amdgcn_update_dpp(0, (int)v, 0x112, 0xf, 0xf, true);
    v += (unsigned)__builtin_amdgcn_update_dpp(0, (int)v, 0x114, 0xf, 0xf, true);
    v += (unsigned)__builtin_amdgcn_update_dpp(0, (int)v, 0x118, 0xf, 0xf, true);
    v += (unsigned)__builtin_amdgcn_update_dpp(0, (int)v, 0x142, 0xa, 0xf, false);
    v += (unsigned)__builtin_amdgcn_update_dpp(0, (int)v, 0x143, 0xc, 0xf, false);
    return v;
}
__device__ __forceinline__ u32x2 mover(unsigned m, bool gather) {
    unsigned sel = gather ? 12u * B1 : 0u, n = 0;
#pragma unroll
    for (unsigned b = 0; b < 4; ++b) {
        const unsigned c = gather ? ((m >> b) & 1u) * 3u : (m >> (2u * b)) & 3u;
        if (gather) sel ^= c ? (12u ^ b) << (8u * n) : 0u;
        else sel |= (c == 3u ? n : 4u + c) << (8u * b);
        n += c == 3u;
    }
    return (u32x2){sel, 3u * n};
}
#define STAMP(i) do {} while (0)
#define STAMPC(i) do {} while (0)
#undef TRACE
#define TRACE(s, b, i) do {} while (0)

#define SB() __builtin_amdgcn_sched_barrier(0)
#define FENCE() asm volatile("" ::: "memory")
#define AG_T0 ""
#define AG_KLOOP \
    "v_mov_b64 v[0:1], 0\n" \
    "v_mov_b64 v[2:3], 0\n" \
    "v_mov_b64 v[4:5], 0\n" \
    "v_mov_b64 v[6:7], 0\n" \
    "v_mov_b64 v[8:9], 0\n" \
    "v_mov_b64 v[10:11], 0\n" \
    "v_mov_b64 v[12:13], 0\n" \
    "v_mov_b64 v[14:15], 0\n" \
    "v_mov_b64 v[16:17], 0\n" \
    "v_mov_b64 v[18:19], 0\n" \
    "v_mov_b64 v[20:21], 0\n" \
    "v_mov_b64 v[22:23], 0\n" \
    "v_mov_b64 v[24:25], 0\n" \
    "v_mov_b64 v[26:27], 0\n" \
    "v_mov_b64 v[28:29], 0\n" \
    "v_mov_b64 v[30:31], 0\n" \
    "v_mov_b64 v[32:33], 0\n" \
    "v_mov_b64 v[34:35], 0\n" \
    "v_mov_b64 v[36:37], 0\n" \
    "v_mov_b64 v[38:39], 0\n" \
    "v_mov_b64 v[40:41], 0\n" \
    "v_mov_b64 v[42:43], 0\n" \
    "v_mov_b64 v[44:45], 0\n" \
    "v_mov_b64 v[46:47], 0\n" \
    "v_mov_b64 v[48:49], 0\n" \
    "v_mov_b64 v[50:51], 0\n" \
    "v_mov_b64 v[52:53], 0\n" \
    "v_mov_b64 v[54:55], 0\n" \
    "v_mov_b64 v[56:57], 0\n" \
    "v_mov_b64 v[58:59], 0\n" \
    "v_mov_b64 v[60:61], 0\n" \
    "v_mov_b64 v[62:63], 0\n" \
    "v_mov_b64 v[64:65], 0\n" \
    "v_mov_b64 v[66:67], 0\n" \
    "v_mov_b64 v[68:69], 0\n" \
    "v_mov_b64 v[70:71], 0\n" \
    "v_mov_b64 v[72:73], 0\n" \
    "v_mov_b64 v[74:75], 0\n" \
    "v_mov_b64 v[76:77], 0\n" \
    "v_mov_b64 v[78:79], 0\n" \
    "v_mov_b64 v[80:81], 0\n" \
    "v_mov_b64 v[82:83], 0\n" \
    "v_mov_b64 v[84:85], 0\n" \
    "v_mov_b64 v[86:87], 0\n" \
    "v_mov_b64 v[88:89], 0\n" \
    "v_mov_b64 v[90:91], 0\n" \
    "v_mov_b64 v[92:93], 0\n" \
    "v_mov_b64 v[94:95], 0\n" \
    "v_mov_b64 v[96:97], 0\n" \
    "v_mov_b64 v[98:99], 0\n" \
    "v_mov_b64 v[100:101], 0\n" \
    "v_mov_b64 v[102:103], 0\n" \
    "v_mov_b64 v[104:105], 0\n" \
    "v_mov_b64 v[106:107], 0\n" \
    "v_mov_b64 v[108:109], 0\n" \
    "v_mov_b64 v[110:111], 0\n" \
    "v_mov_b64 v[112:113], 0\n" \
    "v_mov_b64 v[114:115], 0\n" \
    "v_mov_b64 v[116:117], 0\n" \
    "v_mov_b64 v[118:119], 0\n" \
    "v_mov_b64 v[120:121], 0\n" \
    "v_mov_b64 v[122:123], 0\n" \
    "v_mov_b64 v[124:125], 0\n" \
    "v_mov_b64 v[126:127], 0\n" \
    "v_mov_b64 v[128:129], 0\n" \
    "v_mov_b64 v[130:131], 0\n" \
    "v_mov_b64 v[132:133], 0\n" \
    "v_mov_b64 v[134:135], 0\n" \
    "v_mov_b64 v[136:137], 0\n" \
    "v_mov_b64 v[138:139], 0\n" \
    "v_mov_b64 v[140:141], 0\n" \
    "v_mov_b64 v[142:143], 0\n" \
    "v_mov_b32 v240, v241\n" \
    "v_add_u32 v243, 0x1000, v242\n" \
    "v_mov_b32 v246, 0\n" \
    "s_mov_b32 %[nd], 1\n" \
    "s_mov_b32 %[ps], 0\n" \
    "P0%=:\n" \
    "ds_read2_b32 v[244:245], v247 offset1:4\n" \
    "s_waitcnt lgkmcnt(0)\n" \
    "v_min_u32 v244, v244, v245\n" \
    "s_nop 0\n" \
    "v_readfirstlane_b32 %[hv], v244\n" \
    "s_cmp_ge_u32 %[hv], %[nd]\n" \
    "s_cbranch_scc1 R0%=\n" \
    "s_sleep 1\n" \
    "s_branch P0%=\n" \
    "R0%=:\n" \
    "buffer_load_dwordx4 v[144:147], v242, %[rs], 0 offen offset:0 sc0 sc1\n" \
    "buffer_load_dwordx4 v[148:151], v242, %[rs], 0 offen offset:1024 sc0 sc1\n" \
    "buffer_load_dwordx4 v[152:155], v242, %[rs], 0 offen offset:2048 sc0 sc1\n" \
    "buffer_load_dwordx4 v[156:159], v242, %[rs], 0 offen offset:3072 sc0 sc1\n" \
    "buffer_load_dwordx4 v[160:163], v243, %[rs], 0 offen offset:0 sc0 sc1\n" \
    "buffer_load_dwordx4 v[164:167], v243, %[rs], 0 offen offset:1024 sc0 sc1\n" \
    "buffer_load_dwordx4 v[168:171], v243, %[rs], 0 offen offset:2048 sc0 sc1\n" \
    "buffer_load_dwordx4 v[172:175], v243, %[rs], 0 offen offset:3072 sc0 sc1\n" \
    "v_add_u32 v242, 0x10000, v242\n" \
    "v_add_u32 v243, 0x10000, v243\n" \
    "s_mov_b32 %[nd], 2\n" \
    "P1%=:\n" \
    "ds_read2_b32 v[244:245], v247 offset1:4\n" \
    "s_waitcnt lgkmcnt(0)\n" \
    "v_min_u32 v244, v244, v245\n" \
    "s_nop 0\n" \
    "v_readfirstlane_b32 %[hv], v244\n" \
    "s_cmp_ge_u32 %[hv], %[nd]\n" \
    "s_cbranch_scc1 R1%=\n" \
    "s_sleep 1\n" \
    "s_branch P1%=\n" \
    "R1%=:\n" \
    AG_T0 \
    "s_cmp_eq_u32 %[n8], 0\n" \
    "s_cbranch_scc0 E8%=\n" \
    "ds_read_b64_tr_b16 v[208:209], v240 offset:0\n" \
    "ds_read_b64_tr_b16 v[210:211], v240 offset:4608\n" \
    "ds_read_b64_tr_b16 v[212:213], v240 offset:9216\n" \
    "ds_read_b64_tr_b16 v[214:215], v240 offset:13824\n" \
    "ds_read_b64_tr_b16 v[216:217], v240 offset:32\n" \
    "ds_read_b64_tr_b16 v[218:219], v240 offset:4640\n" \
    "ds_read_b64_tr_b16 v[220:221], v240 offset:9248\n" \
    "ds_read_b64_tr_b16 v[222:223], v240 offset:13856\n" \
    "s_waitcnt lgkmcnt(4) vmcnt(0)\n" \
    "L9_%=:\n" \
    "v_mfma_f32_16x16x32_bf16 v[0:3], v[208:211], v[144:147], v[0:3]\n" \
    "ds_read_b64_tr_b16 v[224:225], v240 offset:64\n" \
    "v_mfma_f32_16x16x32_bf16 v[0:3], v[212:215], v[148:151], v[0:3]\n" \
    "ds_read_b64_tr_b16 v[226:227], v240 offset:4672\n" \
    "v_mfma_f32_16x16x32_bf16 v[36:39], v[212:215], v[156:159], v[36:39]\n" \
    "ds_read_b64_tr_b16 v[228:229], v240 offset:9280\n" \
    "v_mfma_f32_16x16x32_bf16 v[36:39], v[208:211], v[152:155], v[36:39]\n" \
    "ds_read_b64_tr_b16 v[230:231], v240 offset:13888\n" \
    "v_mfma_f32_16x16x32_bf16 v[72:75], v[208:211], v[160:163], v[72:75]\n" \
    "buffer_load_dwordx4 v[176:179], v242, %[rs], 0 offen offset:0 sc0 sc1\n" \
    "v_mfma_f32_16x16x32_bf16 v[72:75], v[212:215], v[164:167], v[72:75]\n" \
    "buffer_load_dwordx4 v[180:183], v242, %[rs], 0 offen offset:1024 sc0 sc1\n" \
    "v_mfma_f32_16x16x32_bf16 v[108:111], v[212:215], v[172:175], v[108:111]\n" \
    "buffer_load_dwordx4 v[184:187], v242, %[rs], 0 offen offset:2048 sc0 sc1\n" \
    "v_mfma_f32_16x16x32_bf16 v[108:111], v[208:211], v[168:171], v[108:111]\n" \
    "s_waitcnt lgkmcnt(4)\n" \
    "v_mfma_f32_16x16x32_bf16 v[112:115], v[216:219], v[168:171], v[112:115]\n" \
    "ds_read_b64_tr_b16 v[208:209], v240 offset:96\n" \
    "s_add_u32 %[nd], %[nd], 1\n" \
    "v_mfma_f32_16x16x32_bf16 v[112:115], v[220:223], v[172:175], v[112:115]\n" \
    "ds_read_b64_tr_b16 v[210:211], v240 offset:4704\n" \
    "s_add_u32 %[ps], %[ps], 18432\n" \
    "v_mfma_f32_16x16x32_bf16 v[76:79], v[220:223], v[164:167], v[76:79]\n" \
    "ds_read_b64_tr_b16 v[212:213], v240 offset:9312\n" \
    "s_cmp_eq_u32 %[ps], 147456\n" \
    "v_mfma_f32_16x16x32_bf16 v[76:79], v[216:219], v[160:163], v[76:79]\n" \
    "ds_read_b64_tr_b16 v[214:215], v240 offset:13920\n" \
    "s_cselect_b32 %[ps], 0, %[ps]\n" \
    "v_mfma_f32_16x16x32_bf16 v[40:43], v[216:219], v[152:155], v[40:43]\n" \
    "buffer_load_dwordx4 v[188:191], v242, %[rs], 0 offen offset:3072 sc0 sc1\n" \
    "v_mfma_f32_16x16x32_bf16 v[40:43], v[220:223], v[156:159], v[40:43]\n" \
    "buffer_load_dwordx4 v[192:195], v243, %[rs], 0 offen offset:0 sc0 sc1\n" \
    "v_mfma_f32_16x16x32_bf16 v[4:7], v[220:223], v[148:151], v[4:7]\n" \
    "buffer_load_dwordx4 v[196:199], v243, %[rs], 0 offen offset:1024 sc0 sc1\n" \
    "v_mfma_f32_16x16x32_bf16 v[4:7], v[216:219], v[144:147], v[4:7]\n" \
    "s_waitcnt lgkmcnt(4)\n" \
    "v_mfma_f32_16x16x32_bf16 v[8:11], v[224:227], v[144:147], v[8:11]\n" \
    "ds_read_b64_tr_b16 v[216:217], v240 offset:128\n" \
    "v_mfma_f32_16x16x32_bf16 v[8:11], v[228:231], v[148:151], v[8:11]\n" \
    "ds_read_b64_tr_b16 v[218:219], v240 offset:4736\n" \
    "v_mfma_f32_16x16x32_bf16 v[44:47], v[228:231], v[156:159], v[44:47]\n" \
    "ds_read_b64_tr_b16 v[220:221], v240 offset:9344\n" \
    "v_mfma_f32_16x16x32_bf16 v[44:47], v[224:227], v[152:155], v[44:47]\n" \
    "ds_read_b64_tr_b16 v[222:223], v240 offset:13952\n" \
    "v_mfma_f32_16x16x32_bf16 v[80:83], v[224:227], v[160:163], v[80:83]\n" \
    "buffer_load_dwordx4 v[200:203], v243, %[rs], 0 offen offset:2048 sc0 sc1\n" \
    "v_mfma_f32_16x16x32_bf16 v[80:83], v[228:231], v[164:167], v[80:83]\n" \
    "buffer_load_dwordx4 v[204:207], v243, %[rs], 0 offen offset:3072 sc0 sc1\n" \
    "v_mfma_f32_16x16x32_bf16 v[116:119], v[228:231], v[172:175], v[116:119]\n" \
    "v_add_u32 v242, 0x10000, v242\n" \
    "v_mfma_f32_16x16x32_bf16 v[116:119], v[224:227], v[168:171], v[116:119]\n" \
    "s_waitcnt lgkmcnt(4)\n" \
    "v_mfma_f32_16x16x32_bf16 v[120:123], v[208:211], v[168:171], v[120:123]\n" \
    "ds_read_b64_tr_b16 v[224:225], v240 offset:160\n" \
    "v_mfma_f32_16x16x32_bf16 v[120:123], v[212:215], v[172:175], v[120:123]\n" \
    "ds_read_b64_tr_b16 v[226:227], v240 offset:4768\n" \
    "v_mfma_f32_16x16x32_bf16 v[84:87], v[212:215], v[164:167], v[84:87]\n" \
    "ds_read_b64_tr_b16 v[228:229], v240 offset:9376\n" \
    "v_mfma_f32_16x16x32_bf16 v[84:87], v[208:211], v[160:163], v[84:87]\n" \
    "ds_read_b64_tr_b16 v[230:231], v240 offset:13984\n" \
    "v_mfma_f32_16x16x32_bf16 v[48:51], v[208:211], v[152:155], v[48:51]\n" \
    "v_add_u32 v243, 0x10000, v243\n" \
    "v_mfma_f32_16x16x32_bf16 v[48:51], v[212:215], v[156:159], v[48:51]\n" \
    "v_add_u32 v246, 1, v246\n" \
    "v_mfma_f32_16x16x32_bf16 v[12:15], v[212:215], v[148:151], v[12:15]\n" \
    "v_mfma_f32_16x16x32_bf16 v[12:15], v[208:211], v[144:147], v[12:15]\n" \
    "s_waitcnt lgkmcnt(4)\n" \
    "v_mfma_f32_16x16x32_bf16 v[16:19], v[216:219], v[144:147], v[16:19]\n" \
    "ds_read_b64_tr_b16 v[208:209], v240 offset:192\n" \
    "v_mfma_f32_16x16x32_bf16 v[16:19], v[220:223], v[148:151], v[16:19]\n" \
    "ds_read_b64_tr_b16 v[210:211], v240 offset:4800\n" \
    "v_mfma_f32_16x16x32_bf16 v[52:55], v[220:223], v[156:159], v[52:55]\n" \
    "ds_read_b64_tr_b16 v[212:213], v240 offset:9408\n" \
    "v_mfma_f32_16x16x32_bf16 v[52:55], v[216:219], v[152:155], v[52:55]\n" \
    "ds_read_b64_tr_b16 v[214:215], v240 offset:14016\n" \
    "v_mfma_f32_16x16x32_bf16 v[88:91], v[216:219], v[160:163], v[88:91]\n" \
    "v_mfma_f32_16x16x32_bf16 v[88:91], v[220:223], v[164:167], v[88:91]\n" \
    "v_mfma_f32_16x16x32_bf16 v[124:127], v[220:223], v[172:175], v[124:127]\n" \
    "v_mfma_f32_16x16x32_bf16 v[124:127], v[216:219], v[168:171], v[124:127]\n" \
    "s_waitcnt lgkmcnt(4)\n" \
    "v_mfma_f32_16x16x32_bf16 v[128:131], v[224:227], v[168:171], v[128:131]\n" \
    "ds_read_b64_tr_b16 v[216:217], v240 offset:224\n" \
    "v_mfma_f32_16x16x32_bf16 v[128:131], v[228:231], v[172:175], v[128:131]\n" \
    "ds_read_b64_tr_b16 v[218:219], v240 offset:4832\n" \
    "v_mfma_f32_16x16x32_bf16 v[92:95], v[228:231], v[164:167], v[92:95]\n" \
    "ds_read_b64_tr_b16 v[220:221], v240 offset:9440\n" \
    "v_mfma_f32_16x16x32_bf16 v[92:95], v[224:227], v[160:163], v[92:95]\n" \
    "ds_read_b64_tr_b16 v[222:223], v240 offset:14048\n" \
    "v_mfma_f32_16x16x32_bf16 v[56:59], v[224:227], v[152:155], v[56:59]\n" \
    "v_mfma_f32_16x16x32_bf16 v[56:59], v[228:231], v[156:159], v[56:59]\n" \
    "v_mfma_f32_16x16x32_bf16 v[20:23], v[228:231], v[148:151], v[20:23]\n" \
    "v_mfma_f32_16x16x32_bf16 v[20:23], v[224:227], v[144:147], v[20:23]\n" \
    "s_waitcnt lgkmcnt(4)\n" \
    "v_mfma_f32_16x16x32_bf16 v[24:27], v[208:211], v[144:147], v[24:27]\n" \
    "ds_read_b64_tr_b16 v[224:225], v240 offset:256\n" \
    "v_mfma_f32_16x16x32_bf16 v[24:27], v[212:215], v[148:151], v[24:27]\n" \
    "ds_read_b64_tr_b16 v[226:227], v240 offset:4864\n" \
    "v_mfma_f32_16x16x32_bf16 v[60:63], v[212:215], v[156:159], v[60:63]\n" \
    "ds_read_b64_tr_b16 v[228:229], v240 offset:9472\n" \
    "v_mfma_f32_16x16x32_bf16 v[60:63], v[208:211], v[152:155], v[60:63]\n" \
    "ds_read_b64_tr_b16 v[230:231], v240 offset:14080\n" \
    "v_mfma_f32_16x16x32_bf16 v[96:99], v[208:211], v[160:163], v[96:99]\n" \
    "v_add_u32 v240, %[ps], v241\n" \
    "v_mfma_f32_16x16x32_bf16 v[96:99], v[212:215], v[164:167], v[96:99]\n" \
    "ds_write_b32 v247, v246 offset:32\n" \
    "v_mfma_f32_16x16x32_bf16 v[132:135], v[212:215], v[172:175], v[132:135]\n" \
    "v_mfma_f32_16x16x32_bf16 v[132:135], v[208:211], v[168:171], v[132:135]\n" \
    "s_waitcnt lgkmcnt(5)\n" \
    "v_mfma_f32_16x16x32_bf16 v[136:139], v[216:219], v[168:171], v[136:139]\n" \
    "ds_read2_b32 v[244:245], v247 offset1:4\n" \
    "ds_read_b64_tr_b16 v[208:209], v240 offset:0\n" \
    "v_mfma_f32_16x16x32_bf16 v[136:139], v[220:223], v[172:175], v[136:139]\n" \
    "ds_read_b64_tr_b16 v[210:211], v240 offset:4608\n" \
    "v_mfma_f32_16x16x32_bf16 v[100:103], v[220:223], v[164:167], v[100:103]\n" \
    "ds_read_b64_tr_b16 v[212:213], v240 offset:9216\n" \
    "v_mfma_f32_16x16x32_bf16 v[100:103], v[216:219], v[160:163], v[100:103]\n" \
    "ds_read_b64_tr_b16 v[214:215], v240 offset:13824\n" \
    "v_mfma_f32_16x16x32_bf16 v[64:67], v[216:219], v[152:155], v[64:67]\n" \
    "v_mfma_f32_16x16x32_bf16 v[64:67], v[220:223], v[156:159], v[64:67]\n" \
    "v_mfma_f32_16x16x32_bf16 v[28:31], v[220:223], v[148:151], v[28:31]\n" \
    "v_mfma_f32_16x16x32_bf16 v[28:31], v[216:219], v[144:147], v[28:31]\n" \
    "s_waitcnt lgkmcnt(4)\n" \
    "v_mfma_f32_16x16x32_bf16 v[32:35], v[224:227], v[144:147], v[32:35]\n" \
    "ds_read_b64_tr_b16 v[216:217], v240 offset:32\n" \
    "v_mfma_f32_16x16x32_bf16 v[32:35], v[228:231], v[148:151], v[32:35]\n" \
    "ds_read_b64_tr_b16 v[218:219], v240 offset:4640\n" \
    "v_mfma_f32_16x16x32_bf16 v[68:71], v[228:231], v[156:159], v[68:71]\n" \
    "ds_read_b64_tr_b16 v[220:221], v240 offset:9248\n" \
    "v_mfma_f32_16x16x32_bf16 v[68:71], v[224:227], v[152:155], v[68:71]\n" \
    "ds_read_b64_tr_b16 v[222:223], v240 offset:13856\n" \
    "v_mfma_f32_16x16x32_bf16 v[104:107], v[224:227], v[160:163], v[104:107]\n" \
    "v_min_u32 v244, v244, v245\n" \
    "v_mfma_f32_16x16x32_bf16 v[104:107], v[228:231], v[164:167], v[104:107]\n" \
    "v_readfirstlane_b32 %[hv], v244\n" \
    "v_mfma_f32_16x16x32_bf16 v[140:143], v[228:231], v[172:175], v[140:143]\n" \
    "s_cmp_ge_u32 %[hv], %[nd]\n" \
    "v_mfma_f32_16x16x32_bf16 v[140:143], v[224:227], v[168:171], v[140:143]\n" \
    "s_waitcnt lgkmcnt(4) vmcnt(0)\n" \
    "s_cbranch_scc0 P9_%=\n" \
    "R9_%=:\n" \
    "v_mfma_f32_16x16x32_bf16 v[0:3], v[208:211], v[176:179], v[0:3]\n" \
    "ds_read_b64_tr_b16 v[224:225], v240 offset:64\n" \
    "v_mfma_f32_16x16x32_bf16 v[0:3], v[212:215], v[180:183], v[0:3]\n" \
    "ds_read_b64_tr_b16 v[226:227], v240 offset:4672\n" \
    "v_mfma_f32_16x16x32_bf16 v[36:39], v[212:215], v[188:191], v[36:39]\n" \
    "ds_read_b64_tr_b16 v[228:229], v240 offset:9280\n" \
    "v_mfma_f32_16x16x32_bf16 v[36:39], v[208:211], v[184:187], v[36:39]\n" \
    "ds_read_b64_tr_b16 v[230:231], v240 offset:13888\n" \
    "v_mfma_f32_16x16x32_bf16 v[72:75], v[208:211], v[192:195], v[72:75]\n" \
    "buffer_load_dwordx4 v[144:147], v242, %[rs], 0 offen offset:0 sc0 sc1\n" \
    "v_mfma_f32_16x16x32_bf16 v[72:75], v[212:215], v[196:199], v[72:75]\n" \
    "buffer_load_dwordx4 v[148:151], v242, %[rs], 0 offen offset:1024 sc0 sc1\n" \
    "v_mfma_f32_16x16x32_bf16 v[108:111], v[212:215], v[204:207], v[108:111]\n" \
    "buffer_load_dwordx4 v[152:155], v242, %[rs], 0 offen offset:2048 sc0 sc1\n" \
    "v_mfma_f32_16x16x32_bf16 v[108:111], v[208:211], v[200:203], v[108:111]\n" \
    "s_waitcnt lgkmcnt(4)\n" \
    "v_mfma_f32_16x16x32_bf16 v[112:115], v[216:219], v[200:203], v[112:115]\n" \
    "ds_read_b64_tr_b16 v[208:209], v240 offset:96\n" \
    "s_add_u32 %[nd], %[nd], 1\n" \
    "v_mfma_f32_16x16x32_bf16 v[112:115], v[220:223], v[204:207], v[112:115]\n" \
    "ds_read_b64_tr_b16 v[210:211], v240 offset:4704\n" \
    "s_add_u32 %[ps], %[ps], 18432\n" \
    "v_mfma_f32_16x16x32_bf16 v[76:79], v[220:223], v[196:199], v[76:79]\n" \
    "ds_read_b64_tr_b16 v[212:213], v240 offset:9312\n" \
    "s_cmp_eq_u32 %[ps], 147456\n" \
    "v_mfma_f32_16x16x32_bf16 v[76:79], v[216:219], v[192:195], v[76:79]\n" \
    "ds_read_b64_tr_b16 v[214:215], v240 offset:13920\n" \
    "s_cselect_b32 %[ps], 0, %[ps]\n" \
    "v_mfma_f32_16x16x32_bf16 v[40:43], v[216:219], v[184:187], v[40:43]\n" \
    "buffer_load_dwordx4 v[156:159], v242, %[rs], 0 offen offset:3072 sc0 sc1\n" \
    "v_mfma_f32_16x16x32_bf16 v[40:43], v[220:223], v[188:191], v[40:43]\n" \
    "buffer_load_dwordx4 v[160:163], v243, %[rs], 0 offen offset:0 sc0 sc1\n" \
    "v_mfma_f32_16x16x32_bf16 v[4:7], v[220:223], v[180:183], v[4:7]\n" \
    "buffer_load_dwordx4 v[164:167], v243, %[rs], 0 offen offset:1024 sc0 sc1\n" \
    "v_mfma_f32_16x16x32_bf16 v[4:7], v[216:219], v[176:179], v[4:7]\n" \
    "s_waitcnt lgkmcnt(4)\n" \
    "v_mfma_f32_16x16x32_bf16 v[8:11], v[224:227], v[176:179], v[8:11]\n" \
    "ds_read_b64_tr_b16 v[216:217], v240 offset:128\n" \
    "s_cmp_eq_u32 %[nd], 66\n" \
    "v_mfma_f32_16x16x32_bf16 v[8:11], v[228:231], v[180:183], v[8:11]\n" \
    "ds_read_b64_tr_b16 v[218:219], v240 offset:4736\n" \
    "s_cselect_b32 %[ck], -1, %[nd]\n" \
    "v_mfma_f32_16x16x32_bf16 v[44:47], v[228:231], v[188:191], v[44:47]\n" \
    "ds_read_b64_tr_b16 v[220:221], v240 offset:9344\n" \
    "v_mfma_f32_16x16x32_bf16 v[44:47], v[224:227], v[184:187], v[44:47]\n" \
    "ds_read_b64_tr_b16 v[222:223], v240 offset:13952\n" \
    "v_mfma_f32_16x16x32_bf16 v[80:83], v[224:227], v[192:195], v[80:83]\n" \
    "buffer_load_dwordx4 v[168:171], v243, %[rs], 0 offen offset:2048 sc0 sc1\n" \
    "v_mfma_f32_16x16x32_bf16 v[80:83], v[228:231], v[196:199], v[80:83]\n" \
    "buffer_load_dwordx4 v[172:175], v243, %[rs], 0 offen offset:3072 sc0 sc1\n" \
    "v_mfma_f32_16x16x32_bf16 v[116:119], v[228:231], v[204:207], v[116:119]\n" \
    "v_add_u32 v242, 0x10000, v242\n" \
    "v_mfma_f32_16x16x32_bf16 v[116:119], v[224:227], v[200:203], v[116:119]\n" \
    "s_waitcnt lgkmcnt(4)\n" \
    "v_mfma_f32_16x16x32_bf16 v[120:123], v[208:211], v[200:203], v[120:123]\n" \
    "ds_read_b64_tr_b16 v[224:225], v240 offset:160\n" \
    "v_mfma_f32_16x16x32_bf16 v[120:123], v[212:215], v[204:207], v[120:123]\n" \
    "ds_read_b64_tr_b16 v[226:227], v240 offset:4768\n" \
    "v_mfma_f32_16x16x32_bf16 v[84:87], v[212:215], v[196:199], v[84:87]\n" \
    "ds_read_b64_tr_b16 v[228:229], v240 offset:9376\n" \
    "v_mfma_f32_16x16x32_bf16 v[84:87], v[208:211], v[192:195], v[84:87]\n" \
    "ds_read_b64_tr_b16 v[230:231], v240 offset:13984\n" \
    "v_mfma_f32_16x16x32_bf16 v[48:51], v[208:211], v[184:187], v[48:51]\n" \
    "v_add_u32 v243, 0x10000, v243\n" \
    "v_mfma_f32_16x16x32_bf16 v[48:51], v[212:215], v[188:191], v[48:51]\n" \
    "v_add_u32 v246, 1, v246\n" \
    "v_mfma_f32_16x16x32_bf16 v[12:15], v[212:215], v[180:183], v[12:15]\n" \
    "v_mfma_f32_16x16x32_bf16 v[12:15], v[208:211], v[176:179], v[12:15]\n" \
    "s_waitcnt lgkmcnt(4)\n" \
    "v_mfma_f32_16x16x32_bf16 v[16:19], v[216:219], v[176:179], v[16:19]\n" \
    "ds_read_b64_tr_b16 v[208:209], v240 offset:192\n" \
    "v_mfma_f32_16x16x32_bf16 v[16:19], v[220:223], v[180:183], v[16:19]\n" \
    "ds_read_b64_tr_b16 v[210:211], v240 offset:4800\n" \
    "v_mfma_f32_16x16x32_bf16 v[52:55], v[220:223], v[188:191], v[52:55]\n" \
    "ds_read_b64_tr_b16 v[212:213], v240 offset:9408\n" \
    "v_mfma_f32_16x16x32_bf16 v[52:55], v[216:219], v[184:187], v[52:55]\n" \
    "ds_read_b64_tr_b16 v[214:215], v240 offset:14016\n" \
    "v_mfma_f32_16x16x32_bf16 v[88:91], v[216:219], v[192:195], v[88:91]\n" \
    "v_mfma_f32_16x16x32_bf16 v[88:91], v[220:223], v[196:199], v[88:91]\n" \
    "v_mfma_f32_16x16x32_bf16 v[124:127], v[220:223], v[204:207], v[124:127]\n" \
    "v_mfma_f32_16x16x32_bf16 v[124:127], v[216:219], v[200:203], v[124:127]\n" \
    "s_waitcnt lgkmcnt(4)\n" \
    "v_mfma_f32_16x16x32_bf16 v[128:131], v[224:227], v[200:203], v[128:131]\n" \
    "ds_read_b64_tr_b16 v[216:217], v240 offset:224\n" \
    "v_mfma_f32_16x16x32_bf16 v[128:131], v[228:231], v[204:207], v[128:131]\n" \
    "ds_read_b64_tr_b16 v[218:219], v240 offset:4832\n" \
    "v_mfma_f32_16x16x32_bf16 v[92:95], v[228:231], v[196:199], v[92:95]\n" \
    "ds_read_b64_tr_b16 v[220:221], v240 offset:9440\n" \
    "v_mfma_f32_16x16x32_bf16 v[92:95], v[224:227], v[192:195], v[92:95]\n" \
    "ds_read_b64_tr_b16 v[222:223], v240 offset:14048\n" \
    "v_mfma_f32_16x16x32_bf16 v[56:59], v[224:227], v[184:187], v[56:59]\n" \
    "v_mfma_f32_16x16x32_bf16 v[56:59], v[228:231], v[188:191], v[56:59]\n" \
    "v_mfma_f32_16x16x32_bf16 v[20:23], v[228:231], v[180:183], v[20:23]\n" \
    "v_mfma_f32_16x16x32_bf16 v[20:23], v[224:227], v[176:179], v[20:23]\n" \
    "s_waitcnt lgkmcnt(4)\n" \
    "v_mfma_f32_16x16x32_bf16 v[24:27], v[208:211], v[176:179], v[24:27]\n" \
    "ds_read_b64_tr_b16 v[224:225], v240 offset:256\n" \
    "v_mfma_f32_16x16x32_bf16 v[24:27], v[212:215], v[180:183], v[24:27]\n" \
    "ds_read_b64_tr_b16 v[226:227], v240 offset:4864\n" \
    "v_mfma_f32_16x16x32_bf16 v[60:63], v[212:215], v[188:191], v[60:63]\n" \
    "ds_read_b64_tr_b16 v[228:229], v240 offset:9472\n" \
    "v_mfma_f32_16x16x32_bf16 v[60:63], v[208:211], v[184:187], v[60:63]\n" \
    "ds_read_b64_tr_b16 v[230:231], v240 offset:14080\n" \
    "v_mfma_f32_16x16x32_bf16 v[96:99], v[208:211], v[192:195], v[96:99]\n" \
    "v_add_u32 v240, %[ps], v241\n" \
    "v_mfma_f32_16x16x32_bf16 v[96:99], v[212:215], v[196:199], v[96:99]\n" \
    "ds_write_b32 v247, v246 offset:32\n" \
    "v_mfma_f32_16x16x32_bf16 v[132:135], v[212:215], v[204:207], v[132:135]\n" \
    "v_mfma_f32_16x16x32_bf16 v[132:135], v[208:211], v[200:203], v[132:135]\n" \
    "s_waitcnt lgkmcnt(5)\n" \
    "v_mfma_f32_16x16x32_bf16 v[136:139], v[216:219], v[200:203], v[136:139]\n" \
    "ds_read2_b32 v[244:245], v247 offset1:4\n" \
    "ds_read_b64_tr_b16 v[208:209], v240 offset:0\n" \
    "v_mfma_f32_16x16x32_bf16 v[136:139], v[220:223], v[204:207], v[136:139]\n" \
    "ds_read_b64_tr_b16 v[210:211], v240 offset:4608\n" \
    "v_mfma_f32_16x16x32_bf16 v[100:103], v[220:223], v[196:199], v[100:103]\n" \
    "ds_read_b64_tr_b16 v[212:213], v240 offset:9216\n" \
    "v_mfma_f32_16x16x32_bf16 v[100:103], v[216:219], v[192:195], v[100:103]\n" \
    "ds_read_b64_tr_b16 v[214:215], v240 offset:13824\n" \
    "v_mfma_f32_16x16x32_bf16 v[64:67], v[216:219], v[184:187], v[64:67]\n" \
    "v_mfma_f32_16x16x32_bf16 v[64:67], v[220:223], v[188:191], v[64:67]\n" \
    "v_mfma_f32_16x16x32_bf16 v[28:31], v[220:223], v[180:183], v[28:31]\n" \
    "v_mfma_f32_16x16x32_bf16 v[28:31], v[216:219], v[176:179], v[28:31]\n" \
    "s_waitcnt lgkmcnt(4)\n" \
    "v_mfma_f32_16x16x32_bf16 v[32:35], v[224:227], v[176:179], v[32:35]\n" \
    "ds_read_b64_tr_b16 v[216:217], v240 offset:32\n" \
    "v_mfma_f32_16x16x32_bf16 v[32:35], v[228:231], v[180:183], v[32:35]\n" \
    "ds_read_b64_tr_b16 v[218:219], v240 offset:4640\n" \
    "v_mfma_f32_16x16x32_bf16 v[68:71], v[228:231], v[188:191], v[68:71]\n" \
    "ds_read_b64_tr_b16 v[220:221], v240 offset:9248\n" \
    "v_mfma_f32_16x16x32_bf16 v[68:71], v[224:227], v[184:187], v[68:71]\n" \
    "ds_read_b64_tr_b16 v[222:223], v240 offset:13856\n" \
    "v_mfma_f32_16x16x32_bf16 v[104:107], v[224:227], v[192:195], v[104:107]\n" \
    "v_min_u32 v244, v244, v245\n" \
    "v_mfma_f32_16x16x32_bf16 v[104:107], v[228:231], v[196:199], v[104:107]\n" \
    "v_readfirstlane_b32 %[hv], v244\n" \
    "v_mfma_f32_16x16x32_bf16 v[140:143], v[228:231], v[204:207], v[140:143]\n" \
    "s_cmp_ge_u32 %[hv], %[ck]\n" \
    "v_mfma_f32_16x16x32_bf16 v[140:143], v[224:227], v[200:203], v[140:143]\n" \
    "s_waitcnt lgkmcnt(4) vmcnt(0)\n" \
    "s_cbranch_scc1 L9_%=\n" \
    "s_cmp_eq_u32 %[nd], 66\n" \
    "s_cbranch_scc1 LEND%=\n" \
    "Q9_%=:\n" \
    "ds_read2_b32 v[244:245], v247 offset1:4\n" \
    "s_waitcnt lgkmcnt(0)\n" \
    "v_min_u32 v244, v244, v245\n" \
    "s_nop 0\n" \
    "v_readfirstlane_b32 %[hv], v244\n" \
    "s_cmp_ge_u32 %[hv], %[nd]\n" \
    "s_cbranch_scc1 L9_%=\n" \
    "s_sleep 1\n" \
    "s_branch Q9_%=\n" \
    "E8%=:\n" \
    "ds_read_b64_tr_b16 v[208:209], v240 offset:0\n" \
    "ds_read_b64_tr_b16 v[210:211], v240 offset:4608\n" \
    "ds_read_b64_tr_b16 v[212:213], v240 offset:9216\n" \
    "ds_read_b64_tr_b16 v[214:215], v240 offset:13824\n" \
    "ds_read_b64_tr_b16 v[216:217], v240 offset:32\n" \
    "ds_read_b64_tr_b16 v[218:219], v240 offset:4640\n" \
    "ds_read_b64_tr_b16 v[220:221], v240 offset:9248\n" \
    "ds_read_b64_tr_b16 v[222:223], v240 offset:13856\n" \
    "ds_read_b64_tr_b16 v[224:225], v240 offset:64\n" \
    "ds_read_b64_tr_b16 v[226:227], v240 offset:4672\n" \
    "ds_read_b64_tr_b16 v[228:229], v240 offset:9280\n" \
    "ds_read_b64_tr_b16 v[230:231], v240 offset:13888\n" \
    "s_waitcnt lgkmcnt(8) vmcnt(0)\n" \
    "L8_%=:\n" \
    "v_mfma_f32_16x16x32_bf16 v[0:3], v[208:211], v[144:147], v[0:3]\n" \
    "ds_read_b64_tr_b16 v[232:233], v240 offset:96\n" \
    "v_mfma_f32_16x16x32_bf16 v[0:3], v[212:215], v[148:151], v[0:3]\n" \
    "ds_read_b64_tr_b16 v[234:235], v240 offset:4704\n" \
    "v_mfma_f32_16x16x32_bf16 v[36:39], v[212:215], v[156:159], v[36:39]\n" \
    "ds_read_b64_tr_b16 v[236:237], v240 offset:9312\n" \
    "v_mfma_f32_16x16x32_bf16 v[36:39], v[208:211], v[152:155], v[36:39]\n" \
    "ds_read_b64_tr_b16 v[238:239], v240 offset:13920\n" \
    "v_mfma_f32_16x16x32_bf16 v[72:75], v[208:211], v[160:163], v[72:75]\n" \
    "buffer_load_dwordx4 v[176:179], v242, %[rs], 0 offen offset:0 sc0 sc1\n" \
    "v_mfma_f32_16x16x32_bf16 v[72:75], v[212:215], v[164:167], v[72:75]\n" \
    "buffer_load_dwordx4 v[180:183], v242, %[rs], 0 offen offset:1024 sc0 sc1\n" \
    "v_mfma_f32_16x16x32_bf16 v[108:111], v[212:215], v[172:175], v[108:111]\n" \
    "buffer_load_dwordx4 v[184:187], v242, %[rs], 0 offen offset:2048 sc0 sc1\n" \
    "v_mfma_f32_16x16x32_bf16 v[108:111], v[208:211], v[168:171], v[108:111]\n" \
    "s_waitcnt lgkmcnt(8)\n" \
    "v_mfma_f32_16x16x32_bf16 v[112:115], v[216:219], v[168:171], v[112:115]\n" \
    "ds_read_b64_tr_b16 v[208:209], v240 offset:128\n" \
    "s_add_u32 %[nd], %[nd], 1\n" \
    "v_mfma_f32_16x16x32_bf16 v[112:115], v[220:223], v[172:175], v[112:115]\n" \
    "ds_read_b64_tr_b16 v[210:211], v240 offset:4736\n" \
    "s_add_u32 %[ps], %[ps], 18432\n" \
    "v_mfma_f32_16x16x32_bf16 v[76:79], v[220:223], v[164:167], v[76:79]\n" \
    "ds_read_b64_tr_b16 v[212:213], v240 offset:9344\n" \
    "s_cmp_eq_u32 %[ps], 147456\n" \
    "v_mfma_f32_16x16x32_bf16 v[76:79], v[216:219], v[160:163], v[76:79]\n" \
    "ds_read_b64_tr_b16 v[214:215], v240 offset:13952\n" \
    "s_cselect_b32 %[ps], 0, %[ps]\n" \
    "v_mfma_f32_16x16x32_bf16 v[40:43], v[216:219], v[152:155], v[40:43]\n" \
    "buffer_load_dwordx4 v[188:191], v242, %[rs], 0 offen offset:3072 sc0 sc1\n" \
    "v_mfma_f32_16x16x32_bf16 v[40:43], v[220:223], v[156:159], v[40:43]\n" \
    "buffer_load_dwordx4 v[192:195], v243, %[rs], 0 offen offset:0 sc0 sc1\n" \
    "v_mfma_f32_16x16x32_bf16 v[4:7], v[220:223], v[148:151], v[4:7]\n" \
    "buffer_load_dwordx4 v[196:199], v243, %[rs], 0 offen offset:1024 sc0 sc1\n" \
    "v_mfma_f32_16x16x32_bf16 v[4:7], v[216:219], v[144:147], v[4:7]\n" \
    "s_waitcnt lgkmcnt(8)\n" \
    "v_mfma_f32_16x16x32_bf16 v[8:11], v[224:227], v[144:147], v[8:11]\n" \
    "ds_read_b64_tr_b16 v[216:217], v240 offset:160\n" \
    "v_mfma_f32_16x16x32_bf16 v[8:11], v[228:231], v[148:151], v[8:11]\n" \
    "ds_read_b64_tr_b16 v[218:219], v240 offset:4768\n" \
    "v_mfma_f32_16x16x32_bf16 v[44:47], v[228:231], v[156:159], v[44:47]\n" \
    "ds_read_b64_tr_b16 v[220:221], v240 offset:9376\n" \
    "v_mfma_f32_16x16x32_bf16 v[44:47], v[224:227], v[152:155], v[44:47]\n" \
    "ds_read_b64_tr_b16 v[222:223], v240 offset:13984\n" \
    "v_mfma_f32_16x16x32_bf16 v[80:83], v[224:227], v[160:163], v[80:83]\n" \
    "buffer_load_dwordx4 v[200:203], v243, %[rs], 0 offen offset:2048 sc0 sc1\n" \
    "v_mfma_f32_16x16x32_bf16 v[80:83], v[228:231], v[164:167], v[80:83]\n" \
    "buffer_load_dwordx4 v[204:207], v243, %[rs], 0 offen offset:3072 sc0 sc1\n" \
    "v_mfma_f32_16x16x32_bf16 v[116:119], v[228:231], v[172:175], v[116:119]\n" \
    "v_add_u32 v242, 0x10000, v242\n" \
    "v_mfma_f32_16x16x32_bf16 v[116:119], v[224:227], v[168:171], v[116:119]\n" \
    "s_waitcnt lgkmcnt(8)\n" \
    "v_mfma_f32_16x16x32_bf16 v[120:123], v[232:235], v[168:171], v[120:123]\n" \
    "ds_read_b64_tr_b16 v[224:225], v240 offset:192\n" \
    "v_mfma_f32_16x16x32_bf16 v[120:123], v[236:239], v[172:175], v[120:123]\n" \
    "ds_read_b64_tr_b16 v[226:227], v240 offset:4800\n" \
    "v_mfma_f32_16x16x32_bf16 v[84:87], v[236:239], v[164:167], v[84:87]\n" \
    "ds_read_b64_tr_b16 v[228:229], v240 offset:9408\n" \
    "v_mfma_f32_16x16x32_bf16 v[84:87], v[232:235], v[160:163], v[84:87]\n" \
    "ds_read_b64_tr_b16 v[230:231], v240 offset:14016\n" \
    "v_mfma_f32_16x16x32_bf16 v[48:51], v[232:235], v[152:155], v[48:51]\n" \
    "v_add_u32 v243, 0x10000, v243\n" \
    "v_mfma_f32_16x16x32_bf16 v[48:51], v[236:239], v[156:159], v[48:51]\n" \
    "v_add_u32 v246, 1, v246\n" \
    "v_mfma_f32_16x16x32_bf16 v[12:15], v[236:239], v[148:151], v[12:15]\n" \
    "v_mfma_f32_16x16x32_bf16 v[12:15], v[232:235], v[144:147], v[12:15]\n" \
    "s_waitcnt lgkmcnt(8)\n" \
    "v_mfma_f32_16x16x32_bf16 v[16:19], v[208:211], v[144:147], v[16:19]\n" \
    "ds_read_b64_tr_b16 v[232:233], v240 offset:224\n" \
    "v_mfma_f32_16x16x32_bf16 v[16:19], v[212:215], v[148:151], v[16:19]\n" \
    "ds_read_b64_tr_b16 v[234:235], v240 offset:4832\n" \
    "v_mfma_f32_16x16x32_bf16 v[52:55], v[212:215], v[156:159], v[52:55]\n" \
    "ds_read_b64_tr_b16 v[236:237], v240 offset:9440\n" \
    "v_mfma_f32_16x16x32_bf16 v[52:55], v[208:211], v[152:155], v[52:55]\n" \
    "ds_read_b64_tr_b16 v[238:239], v240 offset:14048\n" \
    "v_mfma_f32_16x16x32_bf16 v[88:91], v[208:211], v[160:163], v[88:91]\n" \
    "v_add_u32 v240, %[ps], v241\n" \
    "v_mfma_f32_16x16x32_bf16 v[88:91], v[212:215], v[164:167], v[88:91]\n" \
    "ds_write_b32 v247, v246 offset:32\n" \
    "v_mfma_f32_16x16x32_bf16 v[124:127], v[212:215], v[172:175], v[124:127]\n" \
    "v_mfma_f32_16x16x32_bf16 v[124:127], v[208:211], v[168:171], v[124:127]\n" \
    "s_waitcnt lgkmcnt(9)\n" \
    "v_mfma_f32_16x16x32_bf16 v[128:131], v[216:219], v[168:171], v[128:131]\n" \
    "ds_read_b64_tr_b16 v[208:209], v240 offset:0\n" \
    "v_mfma_f32_16x16x32_bf16 v[128:131], v[220:223], v[172:175], v[128:131]\n" \
    "ds_read_b64_tr_b16 v[210:211], v240 offset:4608\n" \
    "v_mfma_f32_16x16x32_bf16 v[92:95], v[220:223], v[164:167], v[92:95]\n" \
    "ds_read_b64_tr_b16 v[212:213], v240 offset:9216\n" \
    "v_mfma_f32_16x16x32_bf16 v[92:95], v[216:219], v[160:163], v[92:95]\n" \
    "ds_read_b64_tr_b16 v[214:215], v240 offset:13824\n" \
    "v_mfma_f32_16x16x32_bf16 v[56:59], v[216:219], v[152:155], v[56:59]\n" \
    "v_mfma_f32_16x16x32_bf16 v[56:59], v[220:223], v[156:159], v[56:59]\n" \
    "v_mfma_f32_16x16x32_bf16 v[20:23], v[220:223], v[148:151], v[20:23]\n" \
    "v_mfma_f32_16x16x32_bf16 v[20:23], v[216:219], v[144:147], v[20:23]\n" \
    "s_waitcnt lgkmcnt(9)\n" \
    "v_mfma_f32_16x16x32_bf16 v[24:27], v[224:227], v[144:147], v[24:27]\n" \
    "ds_read2_b32 v[244:245], v247 offset1:4\n" \
    "ds_read_b64_tr_b16 v[216:217], v240 offset:32\n" \
    "v_mfma_f32_16x16x32_bf16 v[24:27], v[228:231], v[148:151], v[24:27]\n" \
    "ds_read_b64_tr_b16 v[218:219], v240 offset:4640\n" \
    "v_mfma_f32_16x16x32_bf16 v[60:63], v[228:231], v[156:159], v[60:63]\n" \
    "ds_read_b64_tr_b16 v[220:221], v240 offset:9248\n" \
    "v_mfma_f32_16x16x32_bf16 v[60:63], v[224:227], v[152:155], v[60:63]\n" \
    "ds_read_b64_tr_b16 v[222:223], v240 offset:13856\n" \
    "v_mfma_f32_16x16x32_bf16 v[96:99], v[224:227], v[160:163], v[96:99]\n" \
    "v_mfma_f32_16x16x32_bf16 v[96:99], v[228:231], v[164:167], v[96:99]\n" \
    "v_mfma_f32_16x16x32_bf16 v[132:135], v[228:231], v[172:175], v[132:135]\n" \
    "v_mfma_f32_16x16x32_bf16 v[132:135], v[224:227], v[168:171], v[132:135]\n" \
    "s_waitcnt lgkmcnt(4)\n" \
    "v_mfma_f32_16x16x32_bf16 v[136:139], v[232:235], v[168:171], v[136:139]\n" \
    "ds_read_b64_tr_b16 v[224:225], v240 offset:64\n" \
    "v_mfma_f32_16x16x32_bf16 v[136:139], v[236:239], v[172:175], v[136:139]\n" \
    "ds_read_b64_tr_b16 v[226:227], v240 offset:4672\n" \
    "v_mfma_f32_16x16x32_bf16 v[100:103], v[236:239], v[164:167], v[100:103]\n" \
    "ds_read_b64_tr_b16 v[228:229], v240 offset:9280\n" \
    "v_mfma_f32_16x16x32_bf16 v[100:103], v[232:235], v[160:163], v[100:103]\n" \
    "ds_read_b64_tr_b16 v[230:231], v240 offset:13888\n" \
    "v_mfma_f32_16x16x32_bf16 v[64:67], v[232:235], v[152:155], v[64:67]\n" \
    "v_min_u32 v244, v244, v245\n" \
    "v_mfma_f32_16x16x32_bf16 v[64:67], v[236:239], v[156:159], v[64:67]\n" \
    "v_readfirstlane_b32 %[hv], v244\n" \
    "v_mfma_f32_16x16x32_bf16 v[28:31], v[236:239], v[148:151], v[28:31]\n" \
    "s_cmp_ge_u32 %[hv], %[nd]\n" \
    "v_mfma_f32_16x16x32_bf16 v[28:31], v[232:235], v[144:147], v[28:31]\n" \
    "s_waitcnt lgkmcnt(8) vmcnt(0)\n" \
    "s_cbranch_scc0 P8_%=\n" \
    "R8_%=:\n" \
    "v_mfma_f32_16x16x32_bf16 v[0:3], v[208:211], v[176:179], v[0:3]\n" \
    "ds_read_b64_tr_b16 v[232:233], v240 offset:96\n" \
    "v_mfma_f32_16x16x32_bf16 v[0:3], v[212:215], v[180:183], v[0:3]\n" \
    "ds_read_b64_tr_b16 v[234:235], v240 offset:4704\n" \
    "v_mfma_f32_16x16x32_bf16 v[36:39], v[212:215], v[188:191], v[36:39]\n" \
    "ds_read_b64_tr_b16 v[236:237], v240 offset:9312\n" \
    "v_mfma_f32_16x16x32_bf16 v[36:39], v[208:211], v[184:187], v[36:39]\n" \
    "ds_read_b64_tr_b16 v[238:239], v240 offset:13920\n" \
    "v_mfma_f32_16x16x32_bf16 v[72:75], v[208:211], v[192:195], v[72:75]\n" \
    "buffer_load_dwordx4 v[144:147], v242, %[rs], 0 offen offset:0 sc0 sc1\n" \
    "v_mfma_f32_16x16x32_bf16 v[72:75], v[212:215], v[196:199], v[72:75]\n" \
    "buffer_load_dwordx4 v[148:151], v242, %[rs], 0 offen offset:1024 sc0 sc1\n" \
    "v_mfma_f32_16x16x32_bf16 v[108:111], v[212:215], v[204:207], v[108:111]\n" \
    "buffer_load_dwordx4 v[152:155], v242, %[rs], 0 offen offset:2048 sc0 sc1\n" \
    "v_mfma_f32_16x16x32_bf16 v[108:111], v[208:211], v[200:203], v[108:111]\n" \
    "s_waitcnt lgkmcnt(8)\n" \
    "v_mfma_f32_16x16x32_bf16 v[112:115], v[216:219], v[200:203], v[112:115]\n" \
    "ds_read_b64_tr_b16 v[208:209], v240 offset:128\n" \
    "s_add_u32 %[nd], %[nd], 1\n" \
    "v_mfma_f32_16x16x32_bf16 v[112:115], v[220:223], v[204:207], v[112:115]\n" \
    "ds_read_b64_tr_b16 v[210:211], v240 offset:4736\n" \
    "s_add_u32 %[ps], %[ps], 18432\n" \
    "v_mfma_f32_16x16x32_bf16 v[76:79], v[220:223], v[196:199], v[76:79]\n" \
    "ds_read_b64_tr_b16 v[212:213], v240 offset:9344\n" \
    "s_cmp_eq_u32 %[ps], 147456\n" \
    "v_mfma_f32_16x16x32_bf16 v[76:79], v[216:219], v[192:195], v[76:79]\n" \
    "ds_read_b64_tr_b16 v[214:215], v240 offset:13952\n" \
    "s_cselect_b32 %[ps], 0, %[ps]\n" \
    "v_mfma_f32_16x16x32_bf16 v[40:43], v[216:219], v[184:187], v[40:43]\n" \
    "buffer_load_dwordx4 v[156:159], v242, %[rs], 0 offen offset:3072 sc0 sc1\n" \
    "v_mfma_f32_16x16x32_bf16 v[40:43], v[220:223], v[188:191], v[40:43]\n" \
    "buffer_load_dwordx4 v[160:163], v243, %[rs], 0 offen offset:0 sc0 sc1\n" \
    "v_mfma_f32_16x16x32_bf16 v[4:7], v[220:223], v[180:183], v[4:7]\n" \
    "buffer_load_dwordx4 v[164:167], v243, %[rs], 0 offen offset:1024 sc0 sc1\n" \
    "v_mfma_f32_16x16x32_bf16 v[4:7], v[216:219], v[176:179], v[4:7]\n" \
    "s_waitcnt lgkmcnt(8)\n" \
    "v_mfma_f32_16x16x32_bf16 v[8:11], v[224:227], v[176:179], v[8:11]\n" \
    "ds_read_b64_tr_b16 v[216:217], v240 offset:160\n" \
    "s_cmp_eq_u32 %[nd], 66\n" \
    "v_mfma_f32_16x16x32_bf16 v[8:11], v[228:231], v[180:183], v[8:11]\n" \
    "ds_read_b64_tr_b16 v[218:219], v240 offset:4768\n" \
    "s_cselect_b32 %[ck], -1, %[nd]\n" \
    "v_mfma_f32_16x16x32_bf16 v[44:47], v[228:231], v[188:191], v[44:47]\n" \
    "ds_read_b64_tr_b16 v[220:221], v240 offset:9376\n" \
    "v_mfma_f32_16x16x32_bf16 v[44:47], v[224:227], v[184:187], v[44:47]\n" \
    "ds_read_b64_tr_b16 v[222:223], v240 offset:13984\n" \
    "v_mfma_f32_16x16x32_bf16 v[80:83], v[224:227], v[192:195], v[80:83]\n" \
    "buffer_load_dwordx4 v[168:171], v243, %[rs], 0 offen offset:2048 sc0 sc1\n" \
    "v_mfma_f32_16x16x32_bf16 v[80:83], v[228:231], v[196:199], v[80:83]\n" \
    "buffer_load_dwordx4 v[172:175], v243, %[rs], 0 offen offset:3072 sc0 sc1\n" \
    "v_mfma_f32_16x16x32_bf16 v[116:119], v[228:231], v[204:207], v[116:119]\n" \
    "v_add_u32 v242, 0x10000, v242\n" \
    "v_mfma_f32_16x16x32_bf16 v[116:119], v[224:227], v[200:203], v[116:119]\n" \
    "s_waitcnt lgkmcnt(8)\n" \
    "v_mfma_f32_16x16x32_bf16 v[120:123], v[232:235], v[200:203], v[120:123]\n" \
    "ds_read_b64_tr_b16 v[224:225], v240 offset:192\n" \
    "v_mfma_f32_16x16x32_bf16 v[120:123], v[236:239], v[204:207], v[120:123]\n" \
    "ds_read_b64_tr_b16 v[226:227], v240 offset:4800\n" \
    "v_mfma_f32_16x16x32_bf16 v[84:87], v[236:239], v[196:199], v[84:87]\n" \
    "ds_read_b64_tr_b16 v[228:229], v240 offset:9408\n" \
    "v_mfma_f32_16x16x32_bf16 v[84:87], v[232:235], v[192:195], v[84:87]\n" \
    "ds_read_b64_tr_b16 v[230:231], v240 offset:14016\n" \
    "v_mfma_f32_16x16x32_bf16 v[48:51], v[232:235], v[184:187], v[48:51]\n" \
    "v_add_u32 v243, 0x10000, v243\n" \
    "v_mfma_f32_16x16x32_bf16 v[48:51], v[236:239], v[188:191], v[48:51]\n" \
    "v_add_u32 v246, 1, v246\n" \
    "v_mfma_f32_16x16x32_bf16 v[12:15], v[236:239], v[180:183], v[12:15]\n" \
    "v_mfma_f32_16x16x32_bf16 v[12:15], v[232:235], v[176:179], v[12:15]\n" \
    "s_waitcnt lgkmcnt(8)\n" \
    "v_mfma_f32_16x16x32_bf16 v[16:19], v[208:211], v[176:179], v[16:19]\n" \
    "ds_read_b64_tr_b16 v[232:233], v240 offset:224\n" \
    "v_mfma_f32_16x16x32_bf16 v[16:19], v[212:215], v[180:183], v[16:19]\n" \
    "ds_read_b64_tr_b16 v[234:235], v240 offset:4832\n" \
    "v_mfma_f32_16x16x32_bf16 v[52:55], v[212:215], v[188:191], v[52:55]\n" \
    "ds_read_b64_tr_b16 v[236:237], v240 offset:9440\n" \
    "v_mfma_f32_16x16x32_bf16 v[52:55], v[208:211], v[184:187], v[52:55]\n" \
    "ds_read_b64_tr_b16 v[238:239], v240 offset:14048\n" \
    "v_mfma_f32_16x16x32_bf16 v[88:91], v[208:211], v[192:195], v[88:91]\n" \
    "v_add_u32 v240, %[ps], v241\n" \
    "v_mfma_f32_16x16x32_bf16 v[88:91], v[212:215], v[196:199], v[88:91]\n" \
    "ds_write_b32 v247, v246 offset:32\n" \
    "v_mfma_f32_16x16x32_bf16 v[124:127], v[212:215], v[204:207], v[124:127]\n" \
    "v_mfma_f32_16x16x32_bf16 v[124:127], v[208:211], v[200:203], v[124:127]\n" \
    "s_waitcnt lgkmcnt(9)\n" \
    "v_mfma_f32_16x16x32_bf16 v[128:131], v[216:219], v[200:203], v[128:131]\n" \
    "ds_read_b64_tr_b16 v[208:209], v240 offset:0\n" \
    "v_mfma_f32_16x16x32_bf16 v[128:131], v[220:223], v[204:207], v[128:131]\n" \
    "ds_read_b64_tr_b16 v[210:211], v240 offset:4608\n" \
    "v_mfma_f32_16x16x32_bf16 v[92:95], v[220:223], v[196:199], v[92:95]\n" \
    "ds_read_b64_tr_b16 v[212:213], v240 offset:9216\n" \
    "v_mfma_f32_16x16x32_bf16 v[92:95], v[216:219], v[192:195], v[92:95]\n" \
    "ds_read_b64_tr_b16 v[214:215], v240 offset:13824\n" \
    "v_mfma_f32_16x16x32_bf16 v[56:59], v[216:219], v[184:187], v[56:59]\n" \
    "v_mfma_f32_16x16x32_bf16 v[56:59], v[220:223], v[188:191], v[56:59]\n" \
    "v_mfma_f32_16x16x32_bf16 v[20:23], v[220:223], v[180:183], v[20:23]\n" \
    "v_mfma_f32_16x16x32_bf16 v[20:23], v[216:219], v[176:179], v[20:23]\n" \
    "s_waitcnt lgkmcnt(9)\n" \
    "v_mfma_f32_16x16x32_bf16 v[24:27], v[224:227], v[176:179], v[24:27]\n" \
    "ds_read2_b32 v[244:245], v247 offset1:4\n" \
    "ds_read_b64_tr_b16 v[216:217], v240 offset:32\n" \
    "v_mfma_f32_16x16x32_bf16 v[24:27], v[228:231], v[180:183], v[24:27]\n" \
    "ds_read_b64_tr_b16 v[218:219], v240 offset:4640\n" \
    "v_mfma_f32_16x16x32_bf16 v[60:63], v[228:231], v[188:191], v[60:63]\n" \
    "ds_read_b64_tr_b16 v[220:221], v240 offset:9248\n" \
    "v_mfma_f32_16x16x32_bf16 v[60:63], v[224:227], v[184:187], v[60:63]\n" \
    "ds_read_b64_tr_b16 v[222:223], v240 offset:13856\n" \
    "v_mfma_f32_16x16x32_bf16 v[96:99], v[224:227], v[192:195], v[96:99]\n" \
    "v_mfma_f32_16x16x32_bf16 v[96:99], v[228:231], v[196:199], v[96:99]\n" \
    "v_mfma_f32_16x16x32_bf16 v[132:135], v[228:231], v[204:207], v[132:135]\n" \
    "v_mfma_f32_16x16x32_bf16 v[132:135], v[224:227], v[200:203], v[132:135]\n" \
    "s_waitcnt lgkmcnt(4)\n" \
    "v_mfma_f32_16x16x32_bf16 v[136:139], v[232:235], v[200:203], v[136:139]\n" \
    "ds_read_b64_tr_b16 v[224:225], v240 offset:64\n" \
    "v_mfma_f32_16x16x32_bf16 v[136:139], v[236:239], v[204:207], v[136:139]\n" \
    "ds_read_b64_tr_b16 v[226:227], v240 offset:4672\n" \
    "v_mfma_f32_16x16x32_bf16 v[100:103], v[236:239], v[196:199], v[100:103]\n" \
    "ds_read_b64_tr_b16 v[228:229], v240 offset:9280\n" \
    "v_mfma_f32_16x16x32_bf16 v[100:103], v[232:235], v[192:195], v[100:103]\n" \
    "ds_read_b64_tr_b16 v[230:231], v240 offset:13888\n" \
    "v_mfma_f32_16x16x32_bf16 v[64:67], v[232:235], v[184:187], v[64:67]\n" \
    "v_min_u32 v244, v244, v245\n" \
    "v_mfma_f32_16x16x32_bf16 v[64:67], v[236:239], v[188:191], v[64:67]\n" \
    "v_readfirstlane_b32 %[hv], v244\n" \
    "v_mfma_f32_16x16x32_bf16 v[28:31], v[236:239], v[180:183], v[28:31]\n" \
    "s_cmp_ge_u32 %[hv], %[ck]\n" \
    "v_mfma_f32_16x16x32_bf16 v[28:31], v[232:235], v[176:179], v[28:31]\n" \
    "s_waitcnt lgkmcnt(8) vmcnt(0)\n" \
    "s_cbranch_scc1 L8_%=\n" \
    "s_cmp_eq_u32 %[nd], 66\n" \
    "s_cbranch_scc1 LEND%=\n" \
    "Q8_%=:\n" \
    "ds_read2_b32 v[244:245], v247 offset1:4\n" \
    "s_waitcnt lgkmcnt(0)\n" \
    "v_min_u32 v244, v244, v245\n" \
    "s_nop 0\n" \
    "v_readfirstlane_b32 %[hv], v244\n" \
    "s_cmp_ge_u32 %[hv], %[nd]\n" \
    "s_cbranch_scc1 L8_%=\n" \
    "s_sleep 1\n" \
    "s_branch Q8_%=\n" \
    "LEND%=:\n" \
    "s_waitcnt vmcnt(0) lgkmcnt(0)\n" \
    "s_nop 15\n" \
    ""

__global__ void __launch_bounds__(NTHR, 2) ag_gemm(const Args a) {
    __shared__ __attribute__((aligned(1024))) unsigned char lds_raw[LDS_BYTES];
    LAS unsigned char* lds = (LAS unsigned char*)lds_raw;
    volatile LAS unsigned* ctl = (volatile LAS unsigned*)(lds + L_CTL);
    volatile LAS unsigned* prog = ctl + 8;
    const int lane = threadIdx.x & 63;
    const int wave = __builtin_amdgcn_readfirstlane(threadIdx.x >> 6);
    const int bid = blockIdx.x;
    char* const myws = a.ws[NR - 1];
    const int xcd = bid & 7, loc = bid >> 3;
    const bool is_comm = loc < 2;
    const int cid = loc * 8 + xcd, tl = (xcd & 1) * (2 * NTN) + loc - 2, pm = (xcd >> 1) * 4 + (tl & 3), pn = tl >> 2;
    const int pq = AG_PQ == 1 ? (pn < 8 ? pn : -1) : AG_PQ == 2 ? pn - 7 : AG_PQ == 3 ? (pn < 6 ? pn : pn - 13 < 0 ? -1 : pn - 7) : (pn & 1) ? -1 : pn >> 1;

    if (wave == 0) STAMP(0);
    auto fetch = [&](u32x4 (&d)[4], unsigned b) {
        const unsigned bb = b < (unsigned)NBLK ? b : 0u;
        const GAS char* p = (const GAS char*)a.x + (size_t)((bb % NG) * 32 + (lane & 15)) * (K * 2) + (bb / NG) * (BK * 2) + (lane >> 4) * 16;
#pragma unroll
        for (int s = 0; s < 4; ++s) d[s] = *(const GAS u32x4*)(p + (s >> 1) * 16 * (K * 2) + (s & 1) * 64);
    };
    auto begin = [&]() {
        GAS unsigned* cnt = (GAS unsigned*)(myws + W_CNT + (size_t)bid * 128);
        const unsigned c = __builtin_amdgcn_readfirstlane(__hip_atomic_load(cnt, __ATOMIC_RELAXED, __HIP_MEMORY_SCOPE_AGENT)) + 1u;
        if (threadIdx.x < 32) ((LAS unsigned*)(lds + L_CTL))[threadIdx.x] = 0u;
        __syncthreads();
        if (threadIdx.x == 0) __hip_atomic_store(cnt, c, __ATOMIC_RELAXED, __HIP_MEMORY_SCOPE_AGENT);
        return c;
    };
    if (is_comm) {
        u32x4 d[4];
        if (wave < SW) fetch(d, wave * NSC + cid);
        if (wave == 0) STAMP(10);
        const unsigned call = begin();
        if (wave == 0) STAMP(11);
        const size_t xt_off = W_XT + (size_t)(call & 1u) * XT_BYTES, pk_off = W_PK + (size_t)(call & 1u) * PK_BYTES;
        if (wave < SW) {
            const int sid = wave * NSC + cid;
            __amdgpu_buffer_rsrc_t rd[NR];
#pragma unroll
            for (int i = 0; i < NR; ++i) rd[i] = rsrc(a.ws[i], (unsigned)W_END);
            const unsigned xt0 = (unsigned)xt_off + a.rank * NBLK * BLK, pk0 = (unsigned)pk_off + a.rank * NBLK * PBLK;
            const unsigned tbase = (call - 1u) * (unsigned)NBLK - 2u * NSEND;
            const unsigned tag = tag_of(call);
            LAS unsigned char* lx = lds + wave * 8192;
            LAS unsigned char* ls = lx + 2048;
            LAS unsigned short* lp = (LAS unsigned short*)(lx + 4096);
            LAS unsigned char* lm = lx + 4352;
            *(LAS u32x2*)(lm + (lane & 15) * 8) = mover(lane & 15, true);
            auto low = [&](const u32x4 (&d)[4], int j, unsigned& e) {
                const unsigned d0 = d[j >> 1][(j & 1) * 2], d1 = d[j >> 1][(j & 1) * 2 + 1];
                const unsigned lo = perm(d1, d0, 0x06040200u), hi = perm(d1, d0, 0x07050301u);
                e = bfi(B1, lo >> 7, hi << 1);
                return bfi(0x7f7f7f7fu, lo, hi);
            };
            auto pack = [&](const u32x4 (&d)[4], u32x4 (&pk)[3], bool& raw, unsigned& nch) {
                unsigned e[8], sum = 0;
                *(LAS u32x2*)(ls + 512 + lane * 8) = (u32x2){0u, 0u};
#pragma unroll
                for (int j = 0; j < 8; ++j) {
                    pk[j >> 2][j & 3] = low(d, j, e[j]);
                    sum = __builtin_amdgcn_sad_u8(e[j], 0u, sum);
                }
                const unsigned mean = (unsigned)__builtin_amdgcn_readlane(scan(sum), 63) >> 11;
                const unsigned e0 = mean < 7u ? 7u : mean > 252u ? 252u : mean;
                const unsigned k2 = key2(e0), t2 = table2(e0), t3l = table3(e0, 0), t3h = table3(e0, 1);
                u32x4 x[2];
                u32x2 c = {0u, 0u}, mv[8];
                unsigned mask = 0, i3[8];
#pragma unroll
                for (int j = 0; j < 8; ++j) {
                    const unsigned sel = (e[j] ^ k2) & 3u * B1;
                    const unsigned t = ((perm(t2, t2, sel) ^ e[j]) >> 1) + 0x7f7f7f7fu;
                    const unsigned m = perm(t, t << 8, 0x0b090a08u);
                    const unsigned code = sel | (m & 3u * B1);
                    c[j >> 2] |= __builtin_amdgcn_udot4(code, 0x40100401u, 0u, false) << (8 * (j & 3));
                    mv[j] = *(LAS u32x2*)(lm + __builtin_amdgcn_udot4(code & (code >> 1) & B1, 0x40201008u, 0u, false));
                    i3[j] = (e[j] ^ e0 * B1) & m & 7u * B1;
                    const unsigned y = (perm(t3h, t3l, i3[j]) ^ e[j]) & m;
                    x[j >> 2][j & 3] = y;
                    mask = ((((y & 0x7f7f7f7fu) + 0x7f7f7f7fu) | y) & 0x80808080u) | (mask >> 1);
                }
                *(LAS u32x4*)(lx + lane * 32) = x[0];
                *(LAS u32x4*)(lx + lane * 32 + 16) = x[1];
                *(LAS u32x2*)(ls + lane * 8) = c;
                unsigned cnt = 0;
                for (;;) {
                    const unsigned long long act = __ballot(mask != 0u);
                    if (act == 0ull || cnt > (unsigned)NPATCH) break;
                    const unsigned p = __builtin_ctz(mask | 0x80000000u);
                    const unsigned at = __builtin_amdgcn_mbcnt_hi((unsigned)(act >> 32), __builtin_amdgcn_mbcnt_lo((unsigned)act, cnt));
                    lp[mask != 0u ? at : 127u] = (unsigned short)(lane * 32 + (p & 7u) * 4 + (p >> 3));
                    mask &= mask - 1u;
                    cnt += __builtin_popcountll(act);
                }
                FENCE();
                const unsigned pos = lp[lane] & 0x7ffu;
                unsigned long long acc = 0;
                unsigned nb = 0;
#pragma unroll
                for (int j = 7; j >= 0; --j) {
                    const unsigned g = perm(0u, i3[j], mv[j].x);
                    acc = (acc << mv[j].y) | __builtin_amdgcn_udot4(g, 0x003e0801u, g >> 15, false);
                    nb += mv[j].y;
                }
                const unsigned xb = lx[pos];
                const unsigned in = scan(nb), off = in - nb + cnt * 32u;
                const unsigned nd = cnt + (((unsigned)__builtin_amdgcn_readlane(in, 63) + 31u) >> 5);
                raw = cnt > (unsigned)NPATCH || nd > (unsigned)NTAIL || __ballot(nb > 63u) != 0ull;
                nch = raw || nd < 24u ? 40u : (34u * 4u + 15u + nd) >> 4 << 2;
                LAS unsigned* lt = (LAS unsigned*)(ls + 544);
                const unsigned long long sa = acc << (off & 31u);
                __hip_atomic_fetch_or(lt + (off >> 5), (unsigned)sa, __ATOMIC_RELAXED, __HIP_MEMORY_SCOPE_WAVEFRONT);
                __hip_atomic_fetch_or(lt + (off >> 5) + 1, (unsigned)(sa >> 32), __ATOMIC_RELAXED, __HIP_MEMORY_SCOPE_WAVEFRONT);
                __hip_atomic_fetch_or(lt + (off >> 5) + 2, (unsigned)(acc >> 33) >> (~off & 31u), __ATOMIC_RELAXED, __HIP_MEMORY_SCOPE_WAVEFRONT);
                lt[(unsigned)lane < cnt ? lane : 300] = pos | (xb << 16);
                FENCE();
                pk[2] = *(LAS u32x4*)(ls + lane * 16);
                FENCE();
                const unsigned long long m0 = __ballot((int)pk[0].x < 0), m1 = __ballot((int)pk[1].x < 0), m2 = __ballot((int)pk[2].x < 0);
                if (lane == 32) pk[2] = (u32x4){e0 | (raw ? 256u : cnt << 16) | nch << 24, (unsigned)m0, (unsigned)(m0 >> 32), (unsigned)m2};
                if (lane == 33) pk[2] = (u32x4){(unsigned)(m2 >> 32), (unsigned)m1, (unsigned)(m1 >> 32), 0u};
#pragma unroll
                for (int s = 0; s < 3; ++s) pk[s].x = (pk[s].x & 0x7fffffffu) | tag;
            };
            unsigned b0 = sid, b1 = sid + NSEND;
            u32x4 nx[4], pk[3];
            bool raw;
            unsigned skip = 0, nch;
            if (AG_EARLY == 1 || wave == 0) {
#pragma unroll
                for (int j = 0; j < 8; ++j) pk[j >> 2][j & 3] = low(d, j, nch);
#pragma unroll
                for (int s = 0; s < 2; ++s) pk[s].x = (pk[s].x & 0x7fffffffu) | tag;
#pragma unroll
                for (int i = 0; i < NR - 1; ++i)
#pragma unroll
                    for (int s = 0; s < 2; ++s)
                        store16(pk[s], rd[i], pk0 + b0 * PBLK + lane * 16 + s * 1024);
                SB();
                skip = OOB;
            }
            pack(d, pk, raw, nch);
            if (wave == 0) STAMP(6);
            unsigned long long nsent = 0;
            unsigned seq = wave;
            while (b0 < (unsigned)NBLK) {
                while (ctl[0] < seq) __builtin_amdgcn_s_sleep(1);
                FENCE();
                if (raw) {
                    const unsigned so = xt0 + b0 * BLK + lane * 16;
#pragma unroll
                    for (int i = 0; i < NR - 1; ++i)
#pragma unroll
                        for (int s = 0; s < 4; ++s)
                            store16(d[s], rd[i], so + s * 1024);
                    wait_vm0();
                }
                TRACE(a.rank, b0, 0);
                fetch(nx, b1);
                const unsigned tk = __builtin_amdgcn_raw_ptr_buffer_atomic_add_i32(1, rd[NR - 1], lane == 0 ? (unsigned)W_TICKET : OOB, 0, 0);
                const unsigned sp = pk0 + b0 * PBLK + lane * 16, sp2 = (unsigned)lane < nch ? sp + 2048 : OOB;
#pragma unroll
                for (int i = 0; i < NR - 1; ++i)
#pragma unroll
                    for (int s = 0; s < 2; ++s)
                        store16(pk[s], rd[i], (sp | skip) + s * 1024);
                skip = 0;
#pragma unroll
                for (int i = 0; i < NR - 1; ++i)
                    store16(pk[2], rd[i], sp2);
                FENCE();
                if (lane == 0) __hip_atomic_fetch_max((LAS unsigned*)(lds + L_CTL), seq + 1, __ATOMIC_RELAXED, __HIP_MEMORY_SCOPE_WORKGROUP);
                seq = AG_TURN == 2 ? 0 : seq + SW;
                TRACE(a.rank, b0, 1);
#pragma unroll
                for (int s = 0; s < 4; ++s) d[s] = nx[s];
                pack(d, pk, raw, nch);
                b0 = b1;
                ++nsent;
                b1 = __builtin_amdgcn_readfirstlane(tk) - tbase;
            }
            ctl[0] = ~0u;
            if (wave == 0) STAMP(7);
        }
        return;
    }
    const unsigned call = begin();
    GAS unsigned* tmo = (GAS unsigned*)(myws + W_TMO);
    const size_t xt_off = W_XT + (size_t)(call & 1u) * XT_BYTES;
    const unsigned oQ = pq < 0 ? OOB : ((NTN - pn) * BN + pq * 16) * 2;
    if (wave >= 6) {
        __builtin_amdgcn_s_setprio(AG_UPRIO);
        const int src = ((xcd + 2) & 6) + wave - 6;
        const unsigned u = tl;
        const __amdgpu_buffer_rsrc_t rf = rsrc(myws + W_FLAG + (size_t)src * NBLK * FS * 4, FL_BYTES + NBLK * FS * 4);
        const __amdgpu_buffer_rsrc_t rp = rsrc(myws + W_PK + (size_t)(call & 1u) * PK_BYTES + (size_t)src * NBLK * PBLK, NBLK * PBLK);
        const __amdgpu_buffer_rsrc_t rx = rsrc(myws + xt_off + (size_t)src * NBLK * BLK, NBLK * BLK);
        auto done = [&](unsigned b) {
            const unsigned g = b % NG, kt = b / NG;
            const unsigned fo = lane == 0 ? (((g >> 3) * NKT + kt) * 8 + (g & 7)) * FS * 4 : OOB;
            __builtin_amdgcn_raw_buffer_store_b32(call, rf, fo, 0, 17);
        };
        if (wave == 7) STAMP(8);
        unsigned long long lat_r = 0, lat_w = 0, lat_n = 0;
        if (src == a.rank) {
            u32x4 d[4];
            fetch(d, u);
            for (unsigned b = u; b < (unsigned)NBLK; b += NU) {
#pragma unroll
                for (int s = 0; s < 4; ++s)
                    __builtin_amdgcn_raw_buffer_store_b128(d[s], rx, b * BLK + lane * 16 + s * 1024, 0, AG_UAUX);
                fetch(d, b + NU);
                wait_vm0();
                done(b);
            }
        } else {
            LAS unsigned char* lx = lds + L_UNP + (wave - 6) * UNP;
            LAS unsigned char* ls = lx + 2112;
            LAS unsigned char* lm = lds + L_SEL;
            *(LAS u32x4*)(lx + lane * 32) = (u32x4){0u, 0u, 0u, 0u};
            *(LAS u32x4*)(lx + lane * 32 + 16) = (u32x4){0u, 0u, 0u, 0u};
#pragma unroll
            for (int i = 0; i < 4; ++i) *(LAS u32x2*)(lm + (lane + i * 64) * 8) = mover(lane + i * 64, false);
            const unsigned long long want = tag_of(call) ? ~0ull : 0ull;
            const unsigned keep = (lane & 1) ? ~0u : 0x7fffffffu, sh = 31 - (lane >> 1), lw = (lane < 24 ? lane : 24) * 16;
            const unsigned long long t0 = rt();
            unsigned b = u, pend = ~0u;
            for (;;) {
                const unsigned so = b * PBLK;
                const unsigned hl = __builtin_amdgcn_raw_buffer_load_b32(rp, lane < 32 && b < (unsigned)NBLK ? so + 2560 + lane * 4 : OOB, 0, AG_HAUX);
                wait_vm0();
                if (pend != ~0u) {
                    done(pend);
                    TRACE(src, pend, 2);
                    if (b < (unsigned)NBLK) TRACE(src, b, 0);
                    pend = ~0u;
                }
                if (b >= (unsigned)NBLK) break;
                if (((unsigned)(__ballot((int)hl < 0) ^ want) & 0x11111111u) != 0u) {
                    if (give_up(tmo, call, t0)) {
                        __hip_atomic_store(tmo, call, __ATOMIC_RELAXED, __HIP_MEMORY_SCOPE_AGENT);
                        break;
                    }
                    __builtin_amdgcn_s_sleep(AG_USLEEP);
                    continue;
                }
                u32x4 p[2], t;
                u32x2 c;
                auto get = [&]() {
                    c = __builtin_bit_cast(u32x2, __builtin_amdgcn_raw_buffer_load_b64(rp, so + lane * 8 + 2048, 0, 17));
                    t = __builtin_bit_cast(u32x4, __builtin_amdgcn_raw_buffer_load_b128(rp, lane + 32 < ((__builtin_amdgcn_readlane(hl, 0) >> 24) & 63) ? so + lane * 16 + 2560 : OOB, 0, 17));
#pragma unroll
                    for (int s = 0; s < 2; ++s)
                        p[s] = __builtin_bit_cast(u32x4, __builtin_amdgcn_raw_buffer_load_b128(rp, so + lane * 16 + s * 1024, 0, 17));
                };
                get();
                TRACE(src, b, 1);
                const unsigned h0 = __builtin_amdgcn_readlane(hl, 0), m2l = __builtin_amdgcn_readlane(hl, 3), m2h = __builtin_amdgcn_readlane(hl, 4);
                const unsigned long long m0 = (unsigned)__builtin_amdgcn_readlane(hl, 1) | ((unsigned long long)(unsigned)__builtin_amdgcn_readlane(hl, 2) << 32);
                const unsigned long long m1 = (unsigned)__builtin_amdgcn_readlane(hl, 5) | ((unsigned long long)(unsigned)__builtin_amdgcn_readlane(hl, 6) << 32);
                const unsigned e0 = h0 & 255u, cnt = (h0 >> 16) & 255u, used = (1u << (((h0 >> 24) & 63u) - 32u)) - 1u;
                const unsigned t2 = table2(e0), t3l = table3(e0, 0), t3h = table3(e0, 1);
                const unsigned pa = (unsigned)lane - 8u < cnt ? hl & 0x7ffu : 2048u;
                lx[pa] = (unsigned char)(hl >> 16);
                FENCE();
                u32x4 x[2];
                x[0] = *(LAS u32x4*)(lx + lane * 32);
                x[1] = *(LAS u32x4*)(lx + lane * 32 + 16);
                FENCE();
                lx[pa] = 0;
                auto late = [&]() {
                    return ((__ballot((int)p[0].x < 0) ^ want) | (__ballot((int)p[1].x < 0) ^ want) |
                            ((__ballot((int)c.x < 0) ^ want) & 0x5555555555555555ull) | ((__ballot((int)t.x < 0) ^ want) & used)) != 0ull;
                };
                bool ok = true;
                while (late()) {
                    if (give_up(tmo, call, t0)) { ok = false; break; }
                    ++lat_w;
                    get();
                }
                if (!ok) {
                    __hip_atomic_store(tmo, call, __ATOMIC_RELAXED, __HIP_MEMORY_SCOPE_AGENT);
                    break;
                }
                t.x = bfi(0x7fffffffu, t.x, m2h << (31 - (lane & 31)));
                *(LAS u32x4*)(ls + lw) = t;
                c.x = bfi(keep, c.x, m2l << sh);
                const u32x2 f = c & (c >> 1) & 0x55555555u;
                u32x2 mv[8];
#pragma unroll
                for (int j = 0; j < 8; ++j)
                    mv[j] = *(LAS u32x2*)(lm + ((c[j >> 2] >> (8 * (j & 3))) & 255u) * 8);
                const unsigned nb = 3u * (__builtin_popcount(f.x) + __builtin_popcount(f.y));
                const unsigned off = scan(nb) - nb + cnt * 32u;
                FENCE();
                LAS unsigned* lt = (LAS unsigned*)(ls + 32) + (off >> 5);
                const unsigned w0 = lt[0], w1 = lt[1], w2 = lt[2];
                FENCE();
                unsigned long long acc = __builtin_amdgcn_alignbit(w1, w0, off) | ((unsigned long long)__builtin_amdgcn_alignbit(w2, w1, off) << 32);
                p[0].x = (p[0].x & 0x7fffffffu) | (((m0 >> lane) & 1ull) ? 0x80000000u : 0u);
                p[1].x = (p[1].x & 0x7fffffffu) | (((m1 >> lane) & 1ull) ? 0x80000000u : 0u);
                const unsigned vo = (h0 & 256u) ? OOB : b * BLK + lane * 16;
#pragma unroll
                for (int s = 0; s < 4; ++s) {
                    u32x4 d;
#pragma unroll
                    for (int i = 0; i < 2; ++i) {
                        const int j = 2 * s + i;
                        const unsigned q = bfi(0xffffu, (unsigned)acc, (unsigned)acc << 10);
                        const unsigned g = perm(t3h, t3l, (q & 0x00070007u) | ((q << 5) & 0x07000700u));
                        acc = __builtin_amdgcn_alignbit((unsigned)(acc >> 32), (unsigned)acc, mv[j].y) | ((unsigned long long)((unsigned)(acc >> 32) >> mv[j].y) << 32);
                        const unsigned e = perm(t2, g, mv[j].x) ^ x[j >> 2][j & 3];
                        const unsigned lo = bfi(0x7f7f7f7fu, p[j >> 2][j & 3], e << 7), hi = bfi(0x7f7f7f7fu, e >> 1, p[j >> 2][j & 3]);
                        d[2 * i] = perm(hi, lo, 0x05010400u);
                        d[2 * i + 1] = perm(hi, lo, 0x07030602u);
                    }
                    __builtin_amdgcn_raw_buffer_store_b128(d, rx, vo + s * 1024, 0, AG_UAUX);
                }
                pend = b;
                b += NU;
            }
        }
        if (wave == 7) STAMP(9);
    }
    else if (wave < 4) {
        {
            const unsigned cw = ((xcd >> 1) * (4 * NTN) + tl) * 4 + wave;
            const __amdgpu_buffer_rsrc_t rw = rsrc(a.w, K * NL * 2), rs = rsrc(a.x, MS * K * 2);
            constexpr int W0 = AG_PF0 * BK * NL * 2, ST = 16 * NTN * 4 * 1024, NW = (K * NL * 2 - W0 + ST - 1) / ST, NX = (MS * K * 2 + ST - 1) / ST;
            u32x4 pf[NW], px[NX];
#pragma unroll
            for (int i = 0; i < NW; ++i)
                pf[i] = __builtin_bit_cast(u32x4, __builtin_amdgcn_raw_buffer_load_b128(rw, W0 + i * ST + cw * 1024 + lane * 16, 0, AG_PFAUX));
            if (AG_PF == 2) {
#pragma unroll
                for (int i = 0; i < NX; ++i)
                    px[i] = __builtin_bit_cast(u32x4, __builtin_amdgcn_raw_buffer_load_b128(rs, i * ST + cw * 1024 + lane * 16, 0, AG_PFAUX));
#pragma unroll
                for (int i = 0; i < NX; ++i) asm volatile("" :: "v"(px[i]));
            }
#pragma unroll
            for (int i = 0; i < NW; ++i) asm volatile("" :: "v"(pf[i]));
            if (wave == 0) STAMP(12);
        }
        const int g0 = (pm & 1) * 8 + wave * 2;
        const unsigned long long xb = (unsigned long long)(myws + xt_off + ((size_t)(pm >> 1) * NBLK + g0) * BLK);
        const u32x4 ra = (u32x4){(unsigned)xb, (unsigned)(xb >> 32), (unsigned)((NBLK - g0) * BLK), 0x00020000u};
        const unsigned lb = (unsigned)(unsigned long long)lds;
        unsigned vx = lane * 16, have, need, pos, chk;
        f32x32 c[5];
        f32x16 c4;
        unsigned long long t1 = 0, t2 = 0;
        char* ob = a.out + ((size_t)(pm * BM + wave * 64) * NL + pn * BN) * 2;
        asm volatile(AG_KLOOP
                     : "={v[0:31]}"(c[0]), "={v[32:63]}"(c[1]), "={v[64:95]}"(c[2]), "={v[96:127]}"(c[3]), "={v[128:143]}"(c4),
                       "+{v242}"(vx), [hv] "=&s"(have), [nd] "=&s"(need), [ps] "=&s"(pos), [ck] "=&s"(chk), [t1] "+s"(t1), [t2] "+s"(t2)
                     : "{v241}"(lb + (lane >> 2) * RS + (lane & 3) * 8), "{v247}"(lb + L_CTL + wave * 4), [rs] "s"(ra), [n8] "s"(pq >> 31), "s"(ob)
                     : "v144", "v145", "v146", "v147", "v148", "v149", "v150", "v151", "v152", "v153", "v154", "v155", "v156", "v157", "v158", "v159", "v160", "v161", "v162", "v163", "v164", "v165", "v166", "v167", "v168", "v169", "v170", "v171", "v172", "v173", "v174", "v175", "v176", "v177", "v178", "v179", "v180", "v181", "v182", "v183", "v184", "v185", "v186", "v187", "v188", "v189", "v190", "v191", "v192", "v193", "v194", "v195", "v196", "v197", "v198", "v199", "v200", "v201", "v202", "v203", "v204", "v205", "v206", "v207", "v208", "v209", "v210", "v211", "v212", "v213", "v214", "v215", "v216", "v217", "v218", "v219", "v220", "v221", "v222", "v223", "v224", "v225", "v226", "v227", "v228", "v229", "v230", "v231", "v232", "v233", "v234", "v235", "v236", "v237", "v238", "v239", "v240", "v243", "v244", "v245", "v246", "scc", "memory");
#pragma unroll
        for (int i = 0; i < 16; ++i) c[4][i] = c4[i];
        int l2;
        asm volatile("v_mbcnt_lo_u32_b32 %0, -1, 0\n\tv_mbcnt_hi_u32_b32 %0, -1, %0" : "=v"(l2));
        constexpr int ES = RS + 16, E0 = NKT - NSLOT;
        unsigned pv = ctl[4 + (l2 & 7)];
        while (!__all(pv >= (unsigned)(E0 + 5))) {
            __builtin_amdgcn_s_sleep(1);
            pv = ctl[4 + (l2 & 7)];
        }
        const bool bad = __builtin_amdgcn_readfirstlane(pv) != ALL;
        asm volatile("" ::: "memory");
        static_assert(E0 % NSLOT == 0 && 4 * 64 * ES <= 5 * SLOT, "epilogue staging");
        LAS unsigned char* st = lds + wave * 64 * ES;
        LAS unsigned char* sw = st + (l2 & 15) * ES + (l2 >> 4) * 8;
        LAS unsigned char* srS = st + (l2 >> 4) * ES + (l2 & 15) * 16;
        LAS unsigned char* srH = st + (l2 >> 1) * ES + 256 + (l2 & 1) * 16;
        const __amdgpu_buffer_rsrc_t ro = rsrc(ob, (64 * NL - pn * BN) * 2);
        const unsigned voS = (l2 >> 4) * (NL * 2) + (l2 & 15) * 16, voH = (l2 >> 1) * (NL * 2) + (l2 & 1) * 16 + oQ;
        auto put = [&](int m) {
#pragma unroll
            for (int n = 0; n < NB; ++n) {
                const int e = 4 * (m * NB + n);
                const f32x4 v = (f32x4){c[e >> 5][e & 31], c[e >> 5][(e & 31) + 1], c[e >> 5][(e & 31) + 2], c[e >> 5][(e & 31) + 3]};
                *(LAS u32x2*)(sw + m * 16 * ES + n * 32) = __builtin_bit_cast(u32x2, __builtin_convertvector(v, bf16x4));
            }
        };
        u32x4 o[2][5];
        auto get = [&](int m) {
#pragma unroll
            for (int s = 0; s < 4; ++s) o[m & 1][s] = *(LAS u32x4*)(srS + (m * 16 + s * 4) * ES);
            if (m & 1) o[1][4] = *(LAS u32x4*)(srH + (m - 1) * 16 * ES);
        };
        auto out = [&](int m) {
#pragma unroll
            for (int s = 0; s < 4 + (m & 1); ++s)
                __builtin_amdgcn_raw_buffer_store_b128(o[m & 1][s], ro, s < 4 ? voS + (m * 16 + s * 4) * NL * 2 : voH + (m - 1) * 16 * NL * 2,
                                                       0, AG_OAUX);
        };
        if (!bad) {
            put(0); get(0); put(1); SB(); out(0); get(1); put(2); SB(); out(1); get(2); put(3); SB(); out(2); get(3); out(3);
        } else {
#pragma unroll
            for (int s = 0; s < 10; ++s) o[s / 5][s % 5] = (u32x4){0x7fc07fc0u, 0x7fc07fc0u, 0x7fc07fc0u, 0x7fc07fc0u};
            out(0); out(1); out(2); out(3);
        }
    } else if (wave == 4) {
        __builtin_amdgcn_s_setprio(AG_WPRIO);
        const __amdgpu_buffer_rsrc_t rw = rsrc(a.w + pn * BN * 2, (unsigned)(K * NL - pn * BN) * 2);
        unsigned vo[NPIECE / 2];
#pragma unroll
        for (int i = 0; i < NPIECE / 2; ++i) {
            const unsigned o = i * 1024 + lane * 16, r = o / RS, cb = o % RS;
            const unsigned k = ((r >> 2) & 3) * 8 + (r >> 4) * 4 + (r & 3);
            vo[i] = k * (NL * 2) + cb + (cb < 256 ? 0u : oQ - 256);
        }
        int pub = 0;
        unsigned so = 0;
        for (int kt = 0; kt < NKT; ++kt) {
            if (kt >= NSLOT) {
                const unsigned nd = kt - NSLOT + 1;
                if (!__all(prog[lane & 3] >= nd)) {
                    if (pub < kt) {
                        asm volatile("s_waitcnt vmcnt(0)" ::: "memory");
                        pub = kt;
                        ctl[lane & 3] = pub;
                    }
                    while (!__all(prog[lane & 3] >= nd)) __builtin_amdgcn_s_sleep(AG_WSLEEP);
                }
                asm volatile("" ::: "memory");
                if ((nd & 7) == 0) STAMP(16 + (nd >> 3));
            }
            LAS unsigned char* dst = lds + so;
            {
#pragma unroll
                for (int i = 0; i < NPIECE; ++i)
                    __builtin_amdgcn_raw_ptr_buffer_load_lds(rw, (LAS void*)(dst + i * 1024), 16, vo[i % (NPIECE / 2)],
                                                             (kt * 2 + i / (NPIECE / 2)) * (32 * NL * 2), 0, 0);
            }
            so = so + SLOT == NSLOT * SLOT ? 0u : so + SLOT;
            if (kt + 1 - pub > 2) {
                asm volatile("s_waitcnt vmcnt(%0)" :: "n"(2 * NPIECE) : "memory");
                ++pub;
                ctl[lane & 3] = pub;
            }
        }
        asm volatile("s_waitcnt vmcnt(0)" ::: "memory");
        ctl[lane & 3] = ALL;
    }
    else if (wave == 5) {
        __builtin_amdgcn_s_setprio(AG_PPRIO);
        GAS unsigned* flag = (GAS unsigned*)(myws + W_FLAG) + pm * NKT * 8 * FS;
        const unsigned long long t0 = rt();
        int cur = 0;
        unsigned long long npoll = 0;
        while (cur < NKT) {
            ++npoll;
            const int idx = cur * 8 + lane;
            unsigned v = call;
            if (lane < AG_PW * 8 && idx < NKT * 8) v = __hip_atomic_load(flag + idx * FS, __ATOMIC_RELAXED, __HIP_MEMORY_SCOPE_SYSTEM);
            const unsigned long long miss = __ballot((int)(v - call) < 0);
            const int adv = miss ? (__builtin_ctzll(miss) >> 3) : AG_PW;
            if (adv > 0) {
                if (cur == 0) STAMP(4);
                if (((cur + adv) >> 3) != (cur >> 3)) STAMP(24 + (((cur + adv) >> 3) & 7));
                cur = cur + adv < NKT ? cur + adv : NKT;
                asm volatile("" ::: "memory");
                ctl[4 + (lane & 3)] = cur < NKT ? cur : ALL;
            } else {
                if (give_up(tmo, call, t0)) {
                    __hip_atomic_store(tmo, call, __ATOMIC_RELAXED, __HIP_MEMORY_SCOPE_AGENT);
                    ctl[4 + (lane & 3)] = ALL + 1;
                    break;
                }
                __builtin_amdgcn_s_sleep(AG_PSLEEP);
            }
        }
        STAMP(5);
    }
}
}

extern "C" void kernel_launch(int rank, int nranks, ncclComm_t comm,
                              hipStream_t stream,
                              void* const* d_in, const int* in_sizes, int n_in,
                              void* d_out, int out_size,
                              const int* in_dtypes, int out_dtype,
                              void* const* d_out_all,
                              void* const* d_ws_all, size_t ws_bytes) {
    if (nranks != ag::NR || (unsigned)rank >= (unsigned)ag::NR || n_in != 2 || ws_bytes < ag::W_END ||
        in_sizes[0] != ag::MS * ag::K || in_sizes[1] != ag::K * ag::NL || out_size != ag::M * ag::NL) {
        fprintf(stderr, "ag_gemm: unexpected problem shape\n");
        return;
    }
    ag::Args a;
    a.x = (const char*)d_in[0];
    a.w = (const char*)d_in[1];
    a.out = (char*)d_out;
    for (int i = 0; i < ag::NR; ++i) a.ws[i] = (char*)d_ws_all[(rank + 1 + i) % ag::NR];
    a.rank = rank;
    a.pad = 0;
    hipLaunchKernelGGL(ag::ag_gemm, dim3(ag::NWG), dim3(ag::NTHR), 0, stream, a);
    HIP_CHECK(hipGetLastError());
}
```

```cpp
#include <hip/hip_runtime.h>
#include <rccl/rccl.h>
#include <cstdio>
#include <cstdint>
#include <type_traits>

#define HIP_CHECK(expr) do { hipError_t _e = (expr); if (_e != hipSuccess) { \
    fprintf(stderr, "HIP error %s at %s:%d\n", hipGetErrorString(_e), __FILE__, __LINE__); \
    return; } } while (0)

#define LAS __attribute__((address_space(3)))
#define GAS __attribute__((address_space(1)))
typedef unsigned u32x4 __attribute__((ext_vector_type(4)));
typedef unsigned u32x2 __attribute__((ext_vector_type(2)));
typedef unsigned short u16x2 __attribute__((ext_vector_type(2)));
typedef short s16x4 __attribute__((ext_vector_type(4)));
typedef short s16x8 __attribute__((ext_vector_type(8)));
typedef __bf16 bf16x4 __attribute__((ext_vector_type(4)));
typedef __bf16 bf16x8 __attribute__((ext_vector_type(8)));
typedef float f32x4 __attribute__((ext_vector_type(4)));
typedef float f32x16 __attribute__((ext_vector_type(16)));
typedef float f32x32 __attribute__((ext_vector_type(32)));

#define AG_SW 4
#define AG_PSLEEP 2
#define AG_TURN 1
#define AG_UPRIO 0
#define AG_PF 2
#define AG_PF0 3
#define AG_PFAUX 0
#define AG_PFD 0
#define AG_CPRIO 0
#define AG_WSLEEP 2
#define AG_WPRIO 0
#define AG_T1 1
#define AG_A32 1
#define AG_PQ 0
#define AG_PPRIO 2
#define AG_DRY 1
#define AG_SPEC 1
#define AG_CPF 1
#define AG_PD 0
#define AG_UHI 0
#define AG_BAL 0
#define AG_HOLE 0
#define AG_EARLY 3
#define AG_SD 0
#define AG_PP 0
#define AG_MAP 1
#define AG_FAUX 17
#define AG_USLEEP 1
#define AG_HAUX 17
#define AG_UAUX 17
#define AG_FS 32
#define AG_PW 2
#define AG_OAUX 17

namespace ag {
constexpr int NR = 8;
constexpr int M = 4096, K = 4096, NL = 2048;
constexpr int MS = M / NR;
constexpr int NB = 9;
constexpr int BM = 256, BN = 128, BK = 64;
constexpr int NKT = K / BK;
constexpr int NG = MS / 32;
constexpr int BLK = 32 * BK * 2;
constexpr int NBLK = NG * NKT;
constexpr int RS = NB * 32;
constexpr int SLOT = BK * RS, NSLOT = 8, NPIECE = SLOT / 1024;
constexpr int NTN = NL / BN - 1;
constexpr int NWG = 256, CPX = 4;
constexpr int NWAVE = 8, NTHR = NWAVE * 64;
constexpr int NSC = NWG - 16 * NTN, SW = AG_SW, NSEND = NSC * SW;
static_assert(2 * NSEND <= NBLK && SW <= NWAVE, "sender geometry");
constexpr int PBLK = 3072;
constexpr unsigned HW = 6, NPATCH = 32 - HW, NTAIL = 64 - HW, NLONG = NTAIL + 32;
constexpr int NU = 4 * NTN;
constexpr int L_SEL = 0, L_W = 2048;
constexpr int L_CTL = L_W + NSLOT * SLOT;
constexpr unsigned ALL = 1000;
constexpr int L_UNP = L_CTL + 128, UNP = 2048 + 64 + 448;
constexpr int L_SINK = L_UNP + 2 * UNP, LDS_BYTES = L_SINK + 256;
static_assert(LDS_BYTES > 80 * 1024 && LDS_BYTES <= 160 * 1024, "one workgroup per CU");
constexpr size_t W_CNT = 0;
constexpr size_t W_TMO = W_CNT + (size_t)NWG * 128;
constexpr size_t W_TICKET = W_TMO + 1024;
constexpr size_t W_DBG = 768 * 1024;
constexpr size_t W_XT = 1u << 20;
constexpr size_t XT_BYTES = (size_t)M * K * 2;
constexpr size_t W_PK = W_XT + 2 * XT_BYTES;
constexpr size_t PK_BYTES = (size_t)NR * NBLK * PBLK;
constexpr size_t W_FLAG = W_PK + 2 * PK_BYTES;
constexpr int FS = AG_FS;
constexpr unsigned FL_BYTES = NR * NBLK * FS * 4;
constexpr size_t W_END = W_FLAG + 2 * (size_t)FL_BYTES;
static_assert(W_TICKET + 128 <= W_DBG, "flags");
constexpr unsigned OOB = 0x40000000u;
constexpr unsigned long long DEADLINE = 30000000ull;

struct Args {
    const char* x;
    const char* w;
    char* out;
    char* ws[NR];
    int rank;
    int pad;
};

__device__ __forceinline__ unsigned long long rt() { return __builtin_amdgcn_s_memrealtime(); }
__device__ __forceinline__ __amdgpu_buffer_rsrc_t rsrc(const void* p, unsigned bytes) {
    return __builtin_amdgcn_make_buffer_rsrc(const_cast<void*>(p), 0, bytes, 0x00020000);
}
__device__ __forceinline__ void store16(u32x4 d, __amdgpu_buffer_rsrc_t r, unsigned off) {
    __builtin_amdgcn_raw_buffer_store_b128(d, r, off, 0, 17);
}
__device__ __forceinline__ void wait_vm0() {
    __builtin_amdgcn_s_waitcnt(0x0F70);
    asm volatile("s_waitcnt vmcnt(0)" ::: "memory");
}
__device__ __forceinline__ bool give_up(GAS unsigned* tmo, unsigned call, unsigned long long t0) {
    const unsigned long long dt = rt() - t0;
    if (dt < 50000ull) return false;
    return dt > DEADLINE || __builtin_amdgcn_readfirstlane(__hip_atomic_load(tmo, __ATOMIC_RELAXED, __HIP_MEMORY_SCOPE_AGENT)) == call;
}
constexpr unsigned B1 = 0x01010101u;
__device__ __forceinline__ unsigned tag_of(unsigned call) { return ((call + 1u) << 30) & 0x80000000u; }
__device__ __forceinline__ unsigned bfi(unsigned m, unsigned a, unsigned b) { return (a & m) | (b & ~m); }
__device__ __forceinline__ unsigned perm(unsigned hi, unsigned lo, unsigned sel) { return __builtin_amdgcn_perm(hi, lo, sel); }
__device__ __forceinline__ unsigned key2(unsigned e0) { return (((e0 + 3u) & 3u) ^ 3u) * B1; }
__device__ __forceinline__ unsigned table2(unsigned e0) {
    return e0 * B1 + (((0x03020100u ^ key2(e0)) + (4u - (e0 & 3u)) * B1) & 3u * B1);
}
__device__ __forceinline__ unsigned table3(unsigned e0, unsigned hi) {
    const unsigned r = (e0 & 7u) * B1;
    return e0 * B1 - ((8u * B1 + r - ((0x03020100u + hi * 4u * B1) ^ r)) & 7u * B1) + (hi ? 0u : 3u);
}
__device__ __forceinline__ unsigned scan(unsigned v) {
    v += (unsigned)__builtin_amdgcn_update_dpp(0, (int)v, 0x111, 0xf, 0xf, true);
    v += (unsigned)__builtin_amdgcn_update_dpp(0, (int)v, 0x112, 0xf, 0xf, true);
    v += (unsigned)__builtin_amdgcn_update_dpp(0, (int)v, 0x114, 0xf, 0xf, true);
    v += (unsigned)__builtin_amdgcn_update_dpp(0, (int)v, 0x118, 0xf, 0xf, true);
    v += (unsigned)__builtin_amdgcn_update_dpp(0, (int)v, 0x142, 0xa, 0xf, false);
    v += (unsigned)__builtin_amdgcn_update_dpp(0, (int)v, 0x143, 0xc, 0xf, false);
    return v;
}
__device__ __forceinline__ u32x2 mover(unsigned m, bool gather) {
    unsigned sel = gather ? 12u * B1 : 0u, n = 0;
#pragma unroll
    for (unsigned b = 0; b < 4; ++b) {
        const unsigned c = gather ? ((m >> b) & 1u) * 3u : (m >> (2u * b)) & 3u;
        if (gather) sel ^= c ? (12u ^ b) << (8u * n) : 0u;
        else sel |= (c == 3u ? n : 4u + c) << (8u * b);
        n += c == 3u;
    }
    return (u32x2){sel, 3u * n};
}
#define STAMP(i) do {} while (0)
#define STAMPC(i) do {} while (0)
#undef TRACE
#define TRACE(s, b, i) do {} while (0)

#define SB() __builtin_amdgcn_sched_barrier(0)
#define FENCE() asm volatile("" ::: "memory")
#define AG_T0 ""
#define AG_KLOOP \
    "s_cmp_eq_u32 %[dry], 0\n" \
    "s_cbranch_scc1 GO%=\n" \
    "v_mov_b32 v244, -2\n" \
    "s_mov_b64 exec, 0\n" \
    "s_mov_b32 %[nd], 64\n" \
    "s_mov_b32 %[ps], 0\n" \
    "s_branch R1%=\n" \
    "GO%=:\n" \
    "v_mov_b64 v[0:1], 0\n" \
    "v_mov_b64 v[2:3], 0\n" \
    "v_mov_b64 v[4:5], 0\n" \
    "v_mov_b64 v[6:7], 0\n" \
    "v_mov_b64 v[8:9], 0\n" \
    "v_mov_b64 v[10:11], 0\n" \
    "v_mov_b64 v[12:13], 0\n" \
    "v_mov_b64 v[14:15], 0\n" \
    "v_mov_b64 v[16:17], 0\n" \
    "v_mov_b64 v[18:19], 0\n" \
    "v_mov_b64 v[20:21], 0\n" \
    "v_mov_b64 v[22:23], 0\n" \
    "v_mov_b64 v[24:25], 0\n" \
    "v_mov_b64 v[26:27], 0\n" \
    "v_mov_b64 v[28:29], 0\n" \
    "v_mov_b64 v[30:31], 0\n" \
    "v_mov_b64 v[32:33], 0\n" \
    "v_mov_b64 v[34:35], 0\n" \
    "v_mov_b64 v[36:37], 0\n" \
    "v_mov_b64 v[38:39], 0\n" \
    "v_mov_b64 v[40:41], 0\n" \
    "v_mov_b64 v[42:43], 0\n" \
    "v_mov_b64 v[44:45], 0\n" \
    "v_mov_b64 v[46:47], 0\n" \
    "v_mov_b64 v[48:49], 0\n" \
    "v_mov_b64 v[50:51], 0\n" \
    "v_mov_b64 v[52:53], 0\n" \
    "v_mov_b64 v[54:55], 0\n" \
    "v_mov_b64 v[56:57], 0\n" \
    "v_mov_b64 v[58:59], 0\n" \
    "v_mov_b64 v[60:61], 0\n" \
    "v_mov_b64 v[62:63], 0\n" \
    "v_mov_b64 v[64:65], 0\n" \
    "v_mov_b64 v[66:67], 0\n" \
    "v_mov_b64 v[68:69], 0\n" \
    "v_mov_b64 v[70:71], 0\n" \
    "v_mov_b64 v[72:73], 0\n" \
    "v_mov_b64 v[74:75], 0\n" \
    "v_mov_b64 v[76:77], 0\n" \
    "v_mov_b64 v[78:79], 0\n" \
    "v_mov_b64 v[80:81], 0\n" \
    "v_mov_b64 v[82:83], 0\n" \
    "v_mov_b64 v[84:85], 0\n" \
    "v_mov_b64 v[86:87], 0\n" \
    "v_mov_b64 v[88:89], 0\n" \
    "v_mov_b64 v[90:91], 0\n" \
    "v_mov_b64 v[92:93], 0\n" \
    "v_mov_b64 v[94:95], 0\n" \
    "v_mov_b64 v[96:97], 0\n" \
    "v_mov_b64 v[98:99], 0\n" \
    "v_mov_b64 v[100:101], 0\n" \
    "v_mov_b64 v[102:103], 0\n" \
    "v_mov_b64 v[104:105], 0\n" \
    "v_mov_b64 v[106:107], 0\n" \
    "v_mov_b64 v[108:109], 0\n" \
    "v_mov_b64 v[110:111], 0\n" \
    "v_mov_b64 v[112:113], 0\n" \
    "v_mov_b64 v[114:115], 0\n" \
    "v_mov_b64 v[116:117], 0\n" \
    "v_mov_b64 v[118:119], 0\n" \
    "v_mov_b64 v[120:121], 0\n" \
    "v_mov_b64 v[122:123], 0\n" \
    "v_mov_b64 v[124:125], 0\n" \
    "v_mov_b64 v[126:127], 0\n" \
    "v_mov_b64 v[128:129], 0\n" \
    "v_mov_b64 v[130:131], 0\n" \
    "v_mov_b64 v[132:133], 0\n" \
    "v_mov_b64 v[134:135], 0\n" \
    "v_mov_b64 v[136:137], 0\n" \
    "v_mov_b64 v[138:139], 0\n" \
    "v_mov_b64 v[140:141], 0\n" \
    "v_mov_b64 v[142:143], 0\n" \
    "v_mov_b32 v240, v241\n" \
    "v_xor_b32 v243, 0x1000, v242\n" \
    "v_mov_b32 v246, 0\n" \
    "s_mov_b32 %[nd], 1\n" \
    "s_mov_b32 %[ps], 0\n" \
    "P0%=:\n" \
    "ds_read2_b32 v[244:245], v247 offset1:4\n" \
    "s_waitcnt lgkmcnt(0)\n" \
    "v_min_u32 v244, v244, v245\n" \
    "s_nop 0\n" \
    "v_readfirstlane_b32 %[hv], v244\n" \
    "s_cmp_ge_u32 %[hv], %[nd]\n" \
    "s_cbranch_scc1 R0%=\n" \
    "s_sleep 1\n" \
    "s_branch P0%=\n" \
    "R0%=:\n" \
    "buffer_load_dwordx4 v[144:147], v242, %[rs], 0 offen offset:0 sc0 sc1\n" \
    "buffer_load_dwordx4 v[148:151], v242, %[rs], 0 offen offset:1024 sc0 sc1\n" \
    "buffer_load_dwordx4 v[152:155], v242, %[rs], 0 offen offset:2048 sc0 sc1\n" \
    "buffer_load_dwordx4 v[156:159], v242, %[rs], 0 offen offset:3072 sc0 sc1\n" \
    "buffer_load_dwordx4 v[160:163], v243, %[rs], 0 offen offset:0 sc0 sc1\n" \
    "buffer_load_dwordx4 v[164:167], v243, %[rs], 0 offen offset:1024 sc0 sc1\n" \
    "buffer_load_dwordx4 v[168:171], v243, %[rs], 0 offen offset:2048 sc0 sc1\n" \
    "buffer_load_dwordx4 v[172:175], v243, %[rs], 0 offen offset:3072 sc0 sc1\n" \
    "v_add_u32 v242, 0x10000, v242\n" \
    "v_add_u32 v243, 0x10000, v243\n" \
    "s_mov_b32 %[nd], 2\n" \
    "P1%=:\n" \
    "ds_read2_b32 v[244:245], v247 offset1:4\n" \
    "s_waitcnt lgkmcnt(0)\n" \
    "v_min_u32 v244, v244, v245\n" \
    "s_nop 0\n" \
    "v_readfirstlane_b32 %[hv], v244\n" \
    "s_cmp_ge_u32 %[hv], %[nd]\n" \
    "s_cbranch_scc1 R1%=\n" \
    "s_sleep 1\n" \
    "s_branch P1%=\n" \
    "R1%=:\n" \
    AG_T0 \
    "s_cmp_eq_u32 %[n8], 0\n" \
    "s_cbranch_scc0 E8%=\n" \
    "ds_read_b64_tr_b16 v[208:209], v240 offset:0\n" \
    "ds_read_b64_tr_b16 v[210:211], v240 offset:4608\n" \
    "ds_read_b64_tr_b16 v[212:213], v240 offset:9216\n" \
    "ds_read_b64_tr_b16 v[214:215], v240 offset:13824\n" \
    "ds_read_b64_tr_b16 v[216:217], v240 offset:32\n" \
    "ds_read_b64_tr_b16 v[218:219], v240 offset:4640\n" \
    "ds_read_b64_tr_b16 v[220:221], v240 offset:9248\n" \
    "ds_read_b64_tr_b16 v[222:223], v240 offset:13856\n" \
    "s_waitcnt lgkmcnt(4) vmcnt(0)\n" \
    "L9_%=:\n" \
    "v_mfma_f32_16x16x32_bf16 v[0:3], v[208:211], v[144:147], v[0:3]\n" \
    "ds_read_b64_tr_b16 v[224:225], v240 offset:64\n" \
    "v_mfma_f32_16x16x32_bf16 v[0:3], v[212:215], v[148:151], v[0:3]\n" \
    "ds_read_b64_tr_b16 v[226:227], v240 offset:4672\n" \
    "v_mfma_f32_16x16x32_bf16 v[36:39], v[212:215], v[156:159], v[36:39]\n" \
    "ds_read_b64_tr_b16 v[228:229], v240 offset:9280\n" \
    "v_mfma_f32_16x16x32_bf16 v[36:39], v[208:211], v[152:155], v[36:39]\n" \
    "ds_read_b64_tr_b16 v[230:231], v240 offset:13888\n" \
    "v_mfma_f32_16x16x32_bf16 v[72:75], v[208:211], v[160:163], v[72:75]\n" \
    "buffer_load_dwordx4 v[176:179], v242, %[rs], 0 offen offset:0 sc0 sc1\n" \
    "v_mfma_f32_16x16x32_bf16 v[72:75], v[212:215], v[164:167], v[72:75]\n" \
    "buffer_load_dwordx4 v[180:183], v242, %[rs], 0 offen offset:1024 sc0 sc1\n" \
    "v_mfma_f32_16x16x32_bf16 v[108:111], v[212:215], v[172:175], v[108:111]\n" \
    "buffer_load_dwordx4 v[184:187], v242, %[rs], 0 offen offset:2048 sc0 sc1\n" \
    "v_mfma_f32_16x16x32_bf16 v[108:111], v[208:211], v[168:171], v[108:111]\n" \
    "s_waitcnt lgkmcnt(4)\n" \
    "v_mfma_f32_16x16x32_bf16 v[112:115], v[216:219], v[168:171], v[112:115]\n" \
    "ds_read_b64_tr_b16 v[208:209], v240 offset:96\n" \
    "s_add_u32 %[nd], %[nd], 1\n" \
    "v_mfma_f32_16x16x32_bf16 v[112:115], v[220:223], v[172:175], v[112:115]\n" \
    "ds_read_b64_tr_b16 v[210:211], v240 offset:4704\n" \
    "s_add_u32 %[ps], %[ps], 18432\n" \
    "v_mfma_f32_16x16x32_bf16 v[76:79], v[220:223], v[164:167], v[76:79]\n" \
    "ds_read_b64_tr_b16 v[212:213], v240 offset:9312\n" \
    "s_cmp_eq_u32 %[ps], 147456\n" \
    "v_mfma_f32_16x16x32_bf16 v[76:79], v[216:219], v[160:163], v[76:79]\n" \
    "ds_read_b64_tr_b16 v[214:215], v240 offset:13920\n" \
    "s_cselect_b32 %[ps], 0, %[ps]\n" \
    "v_mfma_f32_16x16x32_bf16 v[40:43], v[216:219], v[152:155], v[40:43]\n" \
    "buffer_load_dwordx4 v[188:191], v242, %[rs], 0 offen offset:3072 sc0 sc1\n" \
    "v_mfma_f32_16x16x32_bf16 v[40:43], v[220:223], v[156:159], v[40:43]\n" \
    "buffer_load_dwordx4 v[192:195], v243, %[rs], 0 offen offset:0 sc0 sc1\n" \
    "v_mfma_f32_16x16x32_bf16 v[4:7], v[220:223], v[148:151], v[4:7]\n" \
    "buffer_load_dwordx4 v[196:199], v243, %[rs], 0 offen offset:1024 sc0 sc1\n" \
    "v_mfma_f32_16x16x32_bf16 v[4:7], v[216:219], v[144:147], v[4:7]\n" \
    "s_waitcnt lgkmcnt(4)\n" \
    "v_mfma_f32_16x16x32_bf16 v[8:11], v[224:227], v[144:147], v[8:11]\n" \
    "ds_read_b64_tr_b16 v[216:217], v240 offset:128\n" \
    "v_mfma_f32_16x16x32_bf16 v[8:11], v[228:231], v[148:151], v[8:11]\n" \
    "ds_read_b64_tr_b16 v[218:219], v240 offset:4736\n" \
    "v_mfma_f32_16x16x32_bf16 v[44:47], v[228:231], v[156:159], v[44:47]\n" \
    "ds_read_b64_tr_b16 v[220:221], v240 offset:9344\n" \
    "v_mfma_f32_16x16x32_bf16 v[44:47], v[224:227], v[152:155], v[44:47]\n" \
    "ds_read_b64_tr_b16 v[222:223], v240 offset:13952\n" \
    "v_mfma_f32_16x16x32_bf16 v[80:83], v[224:227], v[160:163], v[80:83]\n" \
    "buffer_load_dwordx4 v[200:203], v243, %[rs], 0 offen offset:2048 sc0 sc1\n" \
    "v_mfma_f32_16x16x32_bf16 v[80:83], v[228:231], v[164:167], v[80:83]\n" \
    "buffer_load_dwordx4 v[204:207], v243, %[rs], 0 offen offset:3072 sc0 sc1\n" \
    "v_mfma_f32_16x16x32_bf16 v[116:119], v[228:231], v[172:175], v[116:119]\n" \
    "v_add_u32 v242, 0x10000, v242\n" \
    "v_mfma_f32_16x16x32_bf16 v[116:119], v[224:227], v[168:171], v[116:119]\n" \
    "s_waitcnt lgkmcnt(4)\n" \
    "v_mfma_f32_16x16x32_bf16 v[120:123], v[208:211], v[168:171], v[120:123]\n" \
    "ds_read_b64_tr_b16 v[224:225], v240 offset:160\n" \
    "v_mfma_f32_16x16x32_bf16 v[120:123], v[212:215], v[172:175], v[120:123]\n" \
    "ds_read_b64_tr_b16 v[226:227], v240 offset:4768\n" \
    "v_mfma_f32_16x16x32_bf16 v[84:87], v[212:215], v[164:167], v[84:87]\n" \
    "ds_read_b64_tr_b16 v[228:229], v240 offset:9376\n" \
    "v_mfma_f32_16x16x32_bf16 v[84:87], v[208:211], v[160:163], v[84:87]\n" \
    "ds_read_b64_tr_b16 v[230:231], v240 offset:13984\n" \
    "v_mfma_f32_16x16x32_bf16 v[48:51], v[208:211], v[152:155], v[48:51]\n" \
    "v_add_u32 v243, 0x10000, v243\n" \
    "v_mfma_f32_16x16x32_bf16 v[48:51], v[212:215], v[156:159], v[48:51]\n" \
    "v_add_u32 v246, 1, v246\n" \
    "v_mfma_f32_16x16x32_bf16 v[12:15], v[212:215], v[148:151], v[12:15]\n" \
    "v_mfma_f32_16x16x32_bf16 v[12:15], v[208:211], v[144:147], v[12:15]\n" \
    "s_waitcnt lgkmcnt(4)\n" \
    "v_mfma_f32_16x16x32_bf16 v[16:19], v[216:219], v[144:147], v[16:19]\n" \
    "ds_read_b64_tr_b16 v[208:209], v240 offset:192\n" \
    "v_mfma_f32_16x16x32_bf16 v[16:19], v[220:223], v[148:151], v[16:19]\n" \
    "ds_read_b64_tr_b16 v[210:211], v240 offset:4800\n" \
    "v_mfma_f32_16x16x32_bf16 v[52:55], v[220:223], v[156:159], v[52:55]\n" \
    "ds_read_b64_tr_b16 v[212:213], v240 offset:9408\n" \
    "v_mfma_f32_16x16x32_bf16 v[52:55], v[216:219], v[152:155], v[52:55]\n" \
    "ds_read_b64_tr_b16 v[214:215], v240 offset:14016\n" \
    "v_mfma_f32_16x16x32_bf16 v[88:91], v[216:219], v[160:163], v[88:91]\n" \
    "v_mfma_f32_16x16x32_bf16 v[88:91], v[220:223], v[164:167], v[88:91]\n" \
    "v_mfma_f32_16x16x32_bf16 v[124:127], v[220:223], v[172:175], v[124:127]\n" \
    "v_mfma_f32_16x16x32_bf16 v[124:127], v[216:219], v[168:171], v[124:127]\n" \
    "s_waitcnt lgkmcnt(4)\n" \
    "v_mfma_f32_16x16x32_bf16 v[128:131], v[224:227], v[168:171], v[128:131]\n" \
    "ds_read_b64_tr_b16 v[216:217], v240 offset:224\n" \
    "v_mfma_f32_16x16x32_bf16 v[128:131], v[228:231], v[172:175], v[128:131]\n" \
    "ds_read_b64_tr_b16 v[218:219], v240 offset:4832\n" \
    "v_mfma_f32_16x16x32_bf16 v[92:95], v[228:231], v[164:167], v[92:95]\n" \
    "ds_read_b64_tr_b16 v[220:221], v240 offset:9440\n" \
    "v_mfma_f32_16x16x32_bf16 v[92:95], v[224:227], v[160:163], v[92:95]\n" \
    "ds_read_b64_tr_b16 v[222:223], v240 offset:14048\n" \
    "v_mfma_f32_16x16x32_bf16 v[56:59], v[224:227], v[152:155], v[56:59]\n" \
    "v_mfma_f32_16x16x32_bf16 v[56:59], v[228:231], v[156:159], v[56:59]\n" \
    "v_mfma_f32_16x16x32_bf16 v[20:23], v[228:231], v[148:151], v[20:23]\n" \
    "v_mfma_f32_16x16x32_bf16 v[20:23], v[224:227], v[144:147], v[20:23]\n" \
    "s_waitcnt lgkmcnt(4)\n" \
    "v_mfma_f32_16x16x32_bf16 v[24:27], v[208:211], v[144:147], v[24:27]\n" \
    "ds_read_b64_tr_b16 v[224:225], v240 offset:256\n" \
    "v_mfma_f32_16x16x32_bf16 v[24:27], v[212:215], v[148:151], v[24:27]\n" \
    "ds_read_b64_tr_b16 v[226:227], v240 offset:4864\n" \
    "v_mfma_f32_16x16x32_bf16 v[60:63], v[212:215], v[156:159], v[60:63]\n" \
    "ds_read_b64_tr_b16 v[228:229], v240 offset:9472\n" \
    "v_mfma_f32_16x16x32_bf16 v[60:63], v[208:211], v[152:155], v[60:63]\n" \
    "ds_read_b64_tr_b16 v[230:231], v240 offset:14080\n" \
    "v_mfma_f32_16x16x32_bf16 v[96:99], v[208:211], v[160:163], v[96:99]\n" \
    "v_add_u32 v240, %[ps], v241\n" \
    "v_mfma_f32_16x16x32_bf16 v[96:99], v[212:215], v[164:167], v[96:99]\n" \
    "ds_write_b32 v247, v246 offset:32\n" \
    "v_mfma_f32_16x16x32_bf16 v[132:135], v[212:215], v[172:175], v[132:135]\n" \
    "v_mfma_f32_16x16x32_bf16 v[132:135], v[208:211], v[168:171], v[132:135]\n" \
    "s_waitcnt lgkmcnt(5)\n" \
    "v_mfma_f32_16x16x32_bf16 v[136:139], v[216:219], v[168:171], v[136:139]\n" \
    "ds_read2_b32 v[244:245], v247 offset1:4\n" \
    "ds_read_b64_tr_b16 v[208:209], v240 offset:0\n" \
    "v_mfma_f32_16x16x32_bf16 v[136:139], v[220:223], v[172:175], v[136:139]\n" \
    "ds_read_b64_tr_b16 v[210:211], v240 offset:4608\n" \
    "v_mfma_f32_16x16x32_bf16 v[100:103], v[220:223], v[164:167], v[100:103]\n" \
    "ds_read_b64_tr_b16 v[212:213], v240 offset:9216\n" \
    "v_mfma_f32_16x16x32_bf16 v[100:103], v[216:219], v[160:163], v[100:103]\n" \
    "ds_read_b64_tr_b16 v[214:215], v240 offset:13824\n" \
    "v_mfma_f32_16x16x32_bf16 v[64:67], v[216:219], v[152:155], v[64:67]\n" \
    "v_mfma_f32_16x16x32_bf16 v[64:67], v[220:223], v[156:159], v[64:67]\n" \
    "v_mfma_f32_16x16x32_bf16 v[28:31], v[220:223], v[148:151], v[28:31]\n" \
    "v_mfma_f32_16x16x32_bf16 v[28:31], v[216:219], v[144:147], v[28:31]\n" \
    "s_waitcnt lgkmcnt(4)\n" \
    "v_mfma_f32_16x16x32_bf16 v[32:35], v[224:227], v[144:147], v[32:35]\n" \
    "ds_read_b64_tr_b16 v[216:217], v240 offset:32\n" \
    "v_mfma_f32_16x16x32_bf16 v[32:35], v[228:231], v[148:151], v[32:35]\n" \
    "ds_read_b64_tr_b16 v[218:219], v240 offset:4640\n" \
    "v_mfma_f32_16x16x32_bf16 v[68:71], v[228:231], v[156:159], v[68:71]\n" \
    "ds_read_b64_tr_b16 v[220:221], v240 offset:9248\n" \
    "v_mfma_f32_16x16x32_bf16 v[68:71], v[224:227], v[152:155], v[68:71]\n" \
    "ds_read_b64_tr_b16 v[222:223], v240 offset:13856\n" \
    "v_mfma_f32_16x16x32_bf16 v[104:107], v[224:227], v[160:163], v[104:107]\n" \
    "v_min_u32 v244, v244, v245\n" \
    "v_mfma_f32_16x16x32_bf16 v[104:107], v[228:231], v[164:167], v[104:107]\n" \
    "v_readfirstlane_b32 %[hv], v244\n" \
    "v_mfma_f32_16x16x32_bf16 v[140:143], v[228:231], v[172:175], v[140:143]\n" \
    "s_cmp_ge_u32 %[hv], %[nd]\n" \
    "v_mfma_f32_16x16x32_bf16 v[140:143], v[224:227], v[168:171], v[140:143]\n" \
    "s_waitcnt lgkmcnt(4) vmcnt(0)\n" \
    "s_cbranch_scc0 P9_%=\n" \
    "R9_%=:\n" \
    "v_mfma_f32_16x16x32_bf16 v[0:3], v[208:211], v[176:179], v[0:3]\n" \
    "ds_read_b64_tr_b16 v[224:225], v240 offset:64\n" \
    "v_mfma_f32_16x16x32_bf16 v[0:3], v[212:215], v[180:183], v[0:3]\n" \
    "ds_read_b64_tr_b16 v[226:227], v240 offset:4672\n" \
    "v_mfma_f32_16x16x32_bf16 v[36:39], v[212:215], v[188:191], v[36:39]\n" \
    "ds_read_b64_tr_b16 v[228:229], v240 offset:9280\n" \
    "v_mfma_f32_16x16x32_bf16 v[36:39], v[208:211], v[184:187], v[36:39]\n" \
    "ds_read_b64_tr_b16 v[230:231], v240 offset:13888\n" \
    "v_mfma_f32_16x16x32_bf16 v[72:75], v[208:211], v[192:195], v[72:75]\n" \
    "buffer_load_dwordx4 v[144:147], v242, %[rs], 0 offen offset:0 sc0 sc1\n" \
    "v_mfma_f32_16x16x32_bf16 v[72:75], v[212:215], v[196:199], v[72:75]\n" \
    "buffer_load_dwordx4 v[148:151], v242, %[rs], 0 offen offset:1024 sc0 sc1\n" \
    "v_mfma_f32_16x16x32_bf16 v[108:111], v[212:215], v[204:207], v[108:111]\n" \
    "buffer_load_dwordx4 v[152:155], v242, %[rs], 0 offen offset:2048 sc0 sc1\n" \
    "v_mfma_f32_16x16x32_bf16 v[108:111], v[208:211], v[200:203], v[108:111]\n" \
    "s_waitcnt lgkmcnt(4)\n" \
    "v_mfma_f32_16x16x32_bf16 v[112:115], v[216:219], v[200:203], v[112:115]\n" \
    "ds_read_b64_tr_b16 v[208:209], v240 offset:96\n" \
    "s_add_u32 %[nd], %[nd], 1\n" \
    "v_mfma_f32_16x16x32_bf16 v[112:115], v[220:223], v[204:207], v[112:115]\n" \
    "ds_read_b64_tr_b16 v[210:211], v240 offset:4704\n" \
    "s_add_u32 %[ps], %[ps], 18432\n" \
    "v_mfma_f32_16x16x32_bf16 v[76:79], v[220:223], v[196:199], v[76:79]\n" \
    "ds_read_b64_tr_b16 v[212:213], v240 offset:9312\n" \
    "s_cmp_eq_u32 %[ps], 147456\n" \
    "v_mfma_f32_16x16x32_bf16 v[76:79], v[216:219], v[192:195], v[76:79]\n" \
    "ds_read_b64_tr_b16 v[214:215], v240 offset:13920\n" \
    "s_cselect_b32 %[ps], 0, %[ps]\n" \
    "v_mfma_f32_16x16x32_bf16 v[40:43], v[216:219], v[184:187], v[40:43]\n" \
    "buffer_load_dwordx4 v[156:159], v242, %[rs], 0 offen offset:3072 sc0 sc1\n" \
    "v_mfma_f32_16x16x32_bf16 v[40:43], v[220:223], v[188:191], v[40:43]\n" \
    "buffer_load_dwordx4 v[160:163], v243, %[rs], 0 offen offset:0 sc0 sc1\n" \
    "v_mfma_f32_16x16x32_bf16 v[4:7], v[220:223], v[180:183], v[4:7]\n" \
    "buffer_load_dwordx4 v[164:167], v243, %[rs], 0 offen offset:1024 sc0 sc1\n" \
    "v_mfma_f32_16x16x32_bf16 v[4:7], v[216:219], v[176:179], v[4:7]\n" \
    "s_waitcnt lgkmcnt(4)\n" \
    "v_mfma_f32_16x16x32_bf16 v[8:11], v[224:227], v[176:179], v[8:11]\n" \
    "ds_read_b64_tr_b16 v[216:217], v240 offset:128\n" \
    "s_cmp_eq_u32 %[nd], 66\n" \
    "v_mfma_f32_16x16x32_bf16 v[8:11], v[228:231], v[180:183], v[8:11]\n" \
    "ds_read_b64_tr_b16 v[218:219], v240 offset:4736\n" \
    "s_cselect_b32 %[ck], -1, %[nd]\n" \
    "v_mfma_f32_16x16x32_bf16 v[44:47], v[228:231], v[188:191], v[44:47]\n" \
    "ds_read_b64_tr_b16 v[220:221], v240 offset:9344\n" \
    "v_mfma_f32_16x16x32_bf16 v[44:47], v[224:227], v[184:187], v[44:47]\n" \
    "ds_read_b64_tr_b16 v[222:223], v240 offset:13952\n" \
    "v_mfma_f32_16x16x32_bf16 v[80:83], v[224:227], v[192:195], v[80:83]\n" \
    "buffer_load_dwordx4 v[168:171], v243, %[rs], 0 offen offset:2048 sc0 sc1\n" \
    "v_mfma_f32_16x16x32_bf16 v[80:83], v[228:231], v[196:199], v[80:83]\n" \
    "buffer_load_dwordx4 v[172:175], v243, %[rs], 0 offen offset:3072 sc0 sc1\n" \
    "v_mfma_f32_16x16x32_bf16 v[116:119], v[228:231], v[204:207], v[116:119]\n" \
    "v_add_u32 v242, 0x10000, v242\n" \
    "v_mfma_f32_16x16x32_bf16 v[116:119], v[224:227], v[200:203], v[116:119]\n" \
    "s_waitcnt lgkmcnt(4)\n" \
    "v_mfma_f32_16x16x32_bf16 v[120:123], v[208:211], v[200:203], v[120:123]\n" \
    "ds_read_b64_tr_b16 v[224:225], v240 offset:160\n" \
    "v_mfma_f32_16x16x32_bf16 v[120:123], v[212:215], v[204:207], v[120:123]\n" \
    "ds_read_b64_tr_b16 v[226:227], v240 offset:4768\n" \
    "v_mfma_f32_16x16x32_bf16 v[84:87], v[212:215], v[196:199], v[84:87]\n" \
    "ds_read_b64_tr_b16 v[228:229], v240 offset:9376\n" \
    "v_mfma_f32_16x16x32_bf16 v[84:87], v[208:211], v[192:195], v[84:87]\n" \
    "ds_read_b64_tr_b16 v[230:231], v240 offset:13984\n" \
    "v_mfma_f32_16x16x32_bf16 v[48:51], v[208:211], v[184:187], v[48:51]\n" \
    "v_add_u32 v243, 0x10000, v243\n" \
    "v_mfma_f32_16x16x32_bf16 v[48:51], v[212:215], v[188:191], v[48:51]\n" \
    "v_add_u32 v246, 1, v246\n" \
    "v_mfma_f32_16x16x32_bf16 v[12:15], v[212:215], v[180:183], v[12:15]\n" \
    "v_mfma_f32_16x16x32_bf16 v[12:15], v[208:211], v[176:179], v[12:15]\n" \
    "s_waitcnt lgkmcnt(4)\n" \
    "v_mfma_f32_16x16x32_bf16 v[16:19], v[216:219], v[176:179], v[16:19]\n" \
    "ds_read_b64_tr_b16 v[208:209], v240 offset:192\n" \
    "v_mfma_f32_16x16x32_bf16 v[16:19], v[220:223], v[180:183], v[16:19]\n" \
    "ds_read_b64_tr_b16 v[210:211], v240 offset:4800\n" \
    "v_mfma_f32_16x16x32_bf16 v[52:55], v[220:223], v[188:191], v[52:55]\n" \
    "ds_read_b64_tr_b16 v[212:213], v240 offset:9408\n" \
    "v_mfma_f32_16x16x32_bf16 v[52:55], v[216:219], v[184:187], v[52:55]\n" \
    "ds_read_b64_tr_b16 v[214:215], v240 offset:14016\n" \
    "v_mfma_f32_16x16x32_bf16 v[88:91], v[216:219], v[192:195], v[88:91]\n" \
    "v_mfma_f32_16x16x32_bf16 v[88:91], v[220:223], v[196:199], v[88:91]\n" \
    "v_mfma_f32_16x16x32_bf16 v[124:127], v[220:223], v[204:207], v[124:127]\n" \
    "v_mfma_f32_16x16x32_bf16 v[124:127], v[216:219], v[200:203], v[124:127]\n" \
    "s_waitcnt lgkmcnt(4)\n" \
    "v_mfma_f32_16x16x32_bf16 v[128:131], v[224:227], v[200:203], v[128:131]\n" \
    "ds_read_b64_tr_b16 v[216:217], v240 offset:224\n" \
    "v_mfma_f32_16x16x32_bf16 v[128:131], v[228:231], v[204:207], v[128:131]\n" \
    "ds_read_b64_tr_b16 v[218:219], v240 offset:4832\n" \
    "v_mfma_f32_16x16x32_bf16 v[92:95], v[228:231], v[196:199], v[92:95]\n" \
    "ds_read_b64_tr_b16 v[220:221], v240 offset:9440\n" \
    "v_mfma_f32_16x16x32_bf16 v[92:95], v[224:227], v[192:195], v[92:95]\n" \
    "ds_read_b64_tr_b16 v[222:223], v240 offset:14048\n" \
    "v_mfma_f32_16x16x32_bf16 v[56:59], v[224:227], v[184:187], v[56:59]\n" \
    "v_mfma_f32_16x16x32_bf16 v[56:59], v[228:231], v[188:191], v[56:59]\n" \
    "v_mfma_f32_16x16x32_bf16 v[20:23], v[228:231], v[180:183], v[20:23]\n" \
    "v_mfma_f32_16x16x32_bf16 v[20:23], v[224:227], v[176:179], v[20:23]\n" \
    "s_waitcnt lgkmcnt(4)\n" \
    "v_mfma_f32_16x16x32_bf16 v[24:27], v[208:211], v[176:179], v[24:27]\n" \
    "ds_read_b64_tr_b16 v[224:225], v240 offset:256\n" \
    "v_mfma_f32_16x16x32_bf16 v[24:27], v[212:215], v[180:183], v[24:27]\n" \
    "ds_read_b64_tr_b16 v[226:227], v240 offset:4864\n" \
    "v_mfma_f32_16x16x32_bf16 v[60:63], v[212:215], v[188:191], v[60:63]\n" \
    "ds_read_b64_tr_b16 v[228:229], v240 offset:9472\n" \
    "v_mfma_f32_16x16x32_bf16 v[60:63], v[208:211], v[184:187], v[60:63]\n" \
    "ds_read_b64_tr_b16 v[230:231], v240 offset:14080\n" \
    "v_mfma_f32_16x16x32_bf16 v[96:99], v[208:211], v[192:195], v[96:99]\n" \
    "v_add_u32 v240, %[ps], v241\n" \
    "v_mfma_f32_16x16x32_bf16 v[96:99], v[212:215], v[196:199], v[96:99]\n" \
    "ds_write_b32 v247, v246 offset:32\n" \
    "v_mfma_f32_16x16x32_bf16 v[132:135], v[212:215], v[204:207], v[132:135]\n" \
    "v_mfma_f32_16x16x32_bf16 v[132:135], v[208:211], v[200:203], v[132:135]\n" \
    "s_waitcnt lgkmcnt(5)\n" \
    "v_mfma_f32_16x16x32_bf16 v[136:139], v[216:219], v[200:203], v[136:139]\n" \
    "ds_read2_b32 v[244:245], v247 offset1:4\n" \
    "ds_read_b64_tr_b16 v[208:209], v240 offset:0\n" \
    "v_mfma_f32_16x16x32_bf16 v[136:139], v[220:223], v[204:207], v[136:139]\n" \
    "ds_read_b64_tr_b16 v[210:211], v240 offset:4608\n" \
    "v_mfma_f32_16x16x32_bf16 v[100:103], v[220:223], v[196:199], v[100:103]\n" \
    "ds_read_b64_tr_b16 v[212:213], v240 offset:9216\n" \
    "v_mfma_f32_16x16x32_bf16 v[100:103], v[216:219], v[192:195], v[100:103]\n" \
    "ds_read_b64_tr_b16 v[214:215], v240 offset:13824\n" \
    "v_mfma_f32_16x16x32_bf16 v[64:67], v[216:219], v[184:187], v[64:67]\n" \
    "v_mfma_f32_16x16x32_bf16 v[64:67], v[220:223], v[188:191], v[64:67]\n" \
    "v_mfma_f32_16x16x32_bf16 v[28:31], v[220:223], v[180:183], v[28:31]\n" \
    "v_mfma_f32_16x16x32_bf16 v[28:31], v[216:219], v[176:179], v[28:31]\n" \
    "s_waitcnt lgkmcnt(4)\n" \
    "v_mfma_f32_16x16x32_bf16 v[32:35], v[224:227], v[176:179], v[32:35]\n" \
    "ds_read_b64_tr_b16 v[216:217], v240 offset:32\n" \
    "v_mfma_f32_16x16x32_bf16 v[32:35], v[228:231], v[180:183], v[32:35]\n" \
    "ds_read_b64_tr_b16 v[218:219], v240 offset:4640\n" \
    "v_mfma_f32_16x16x32_bf16 v[68:71], v[228:231], v[188:191], v[68:71]\n" \
    "ds_read_b64_tr_b16 v[220:221], v240 offset:9248\n" \
    "v_mfma_f32_16x16x32_bf16 v[68:71], v[224:227], v[184:187], v[68:71]\n" \
    "ds_read_b64_tr_b16 v[222:223], v240 offset:13856\n" \
    "v_mfma_f32_16x16x32_bf16 v[104:107], v[224:227], v[192:195], v[104:107]\n" \
    "v_min_u32 v244, v244, v245\n" \
    "v_mfma_f32_16x16x32_bf16 v[104:107], v[228:231], v[196:199], v[104:107]\n" \
    "v_readfirstlane_b32 %[hv], v244\n" \
    "v_mfma_f32_16x16x32_bf16 v[140:143], v[228:231], v[204:207], v[140:143]\n" \
    "s_cmp_ge_u32 %[hv], %[ck]\n" \
    "v_mfma_f32_16x16x32_bf16 v[140:143], v[224:227], v[200:203], v[140:143]\n" \
    "s_waitcnt lgkmcnt(4) vmcnt(0)\n" \
    "s_cbranch_scc1 L9_%=\n" \
    "s_cmp_eq_u32 %[nd], 66\n" \
    "s_cbranch_scc1 LEND%=\n" \
    "Q9_%=:\n" \
    "ds_read2_b32 v[244:245], v247 offset1:4\n" \
    "s_waitcnt lgkmcnt(0)\n" \
    "v_min_u32 v244, v244, v245\n" \
    "s_nop 0\n" \
    "v_readfirstlane_b32 %[hv], v244\n" \
    "s_cmp_ge_u32 %[hv], %[nd]\n" \
    "s_cbranch_scc1 L9_%=\n" \
    "s_sleep 1\n" \
    "s_branch Q9_%=\n" \
    "E8%=:\n" \
    "ds_read_b64_tr_b16 v[208:209], v240 offset:0\n" \
    "ds_read_b64_tr_b16 v[210:211], v240 offset:4608\n" \
    "ds_read_b64_tr_b16 v[212:213], v240 offset:9216\n" \
    "ds_read_b64_tr_b16 v[214:215], v240 offset:13824\n" \
    "ds_read_b64_tr_b16 v[216:217], v240 offset:32\n" \
    "ds_read_b64_tr_b16 v[218:219], v240 offset:4640\n" \
    "ds_read_b64_tr_b16 v[220:221], v240 offset:9248\n" \
    "ds_read_b64_tr_b16 v[222:223], v240 offset:13856\n" \
    "ds_read_b64_tr_b16 v[224:225], v240 offset:64\n" \
    "ds_read_b64_tr_b16 v[226:227], v240 offset:4672\n" \
    "ds_read_b64_tr_b16 v[228:229], v240 offset:9280\n" \
    "ds_read_b64_tr_b16 v[230:231], v240 offset:13888\n" \
    "s_waitcnt lgkmcnt(8) vmcnt(0)\n" \
    "L8_%=:\n" \
    "v_mfma_f32_16x16x32_bf16 v[0:3], v[208:211], v[144:147], v[0:3]\n" \
    "ds_read_b64_tr_b16 v[232:233], v240 offset:96\n" \
    "v_mfma_f32_16x16x32_bf16 v[0:3], v[212:215], v[148:151], v[0:3]\n" \
    "ds_read_b64_tr_b16 v[234:235], v240 offset:4704\n" \
    "v_mfma_f32_16x16x32_bf16 v[36:39], v[212:215], v[156:159], v[36:39]\n" \
    "ds_read_b64_tr_b16 v[236:237], v240 offset:9312\n" \
    "v_mfma_f32_16x16x32_bf16 v[36:39], v[208:211], v[152:155], v[36:39]\n" \
    "ds_read_b64_tr_b16 v[238:239], v240 offset:13920\n" \
    "v_mfma_f32_16x16x32_bf16 v[72:75], v[208:211], v[160:163], v[72:75]\n" \
    "buffer_load_dwordx4 v[176:179], v242, %[rs], 0 offen offset:0 sc0 sc1\n" \
    "v_mfma_f32_16x16x32_bf16 v[72:75], v[212:215], v[164:167], v[72:75]\n" \
    "buffer_load_dwordx4 v[180:183], v242, %[rs], 0 offen offset:1024 sc0 sc1\n" \
    "v_mfma_f32_16x16x32_bf16 v[108:111], v[212:215], v[172:175], v[108:111]\n" \
    "buffer_load_dwordx4 v[184:187], v242, %[rs], 0 offen offset:2048 sc0 sc1\n" \
    "v_mfma_f32_16x16x32_bf16 v[108:111], v[208:211], v[168:171], v[108:111]\n" \
    "s_waitcnt lgkmcnt(8)\n" \
    "v_mfma_f32_16x16x32_bf16 v[112:115], v[216:219], v[168:171], v[112:115]\n" \
    "ds_read_b64_tr_b16 v[208:209], v240 offset:128\n" \
    "s_add_u32 %[nd], %[nd], 1\n" \
    "v_mfma_f32_16x16x32_bf16 v[112:115], v[220:223], v[172:175], v[112:115]\n" \
    "ds_read_b64_tr_b16 v[210:211], v240 offset:4736\n" \
    "s_add_u32 %[ps], %[ps], 18432\n" \
    "v_mfma_f32_16x16x32_bf16 v[76:79], v[220:223], v[164:167], v[76:79]\n" \
    "ds_read_b64_tr_b16 v[212:213], v240 offset:9344\n" \
    "s_cmp_eq_u32 %[ps], 147456\n" \
    "v_mfma_f32_16x16x32_bf16 v[76:79], v[216:219], v[160:163], v[76:79]\n" \
    "ds_read_b64_tr_b16 v[214:215], v240 offset:13952\n" \
    "s_cselect_b32 %[ps], 0, %[ps]\n" \
    "v_mfma_f32_16x16x32_bf16 v[40:43], v[216:219], v[152:155], v[40:43]\n" \
    "buffer_load_dwordx4 v[188:191], v242, %[rs], 0 offen offset:3072 sc0 sc1\n" \
    "v_mfma_f32_16x16x32_bf16 v[40:43], v[220:223], v[156:159], v[40:43]\n" \
    "buffer_load_dwordx4 v[192:195], v243, %[rs], 0 offen offset:0 sc0 sc1\n" \
    "v_mfma_f32_16x16x32_bf16 v[4:7], v[220:223], v[148:151], v[4:7]\n" \
    "buffer_load_dwordx4 v[196:199], v243, %[rs], 0 offen offset:1024 sc0 sc1\n" \
    "v_mfma_f32_16x16x32_bf16 v[4:7], v[216:219], v[144:147], v[4:7]\n" \
    "s_waitcnt lgkmcnt(8)\n" \
    "v_mfma_f32_16x16x32_bf16 v[8:11], v[224:227], v[144:147], v[8:11]\n" \
    "ds_read_b64_tr_b16 v[216:217], v240 offset:160\n" \
    "v_mfma_f32_16x16x32_bf16 v[8:11], v[228:231], v[148:151], v[8:11]\n" \
    "ds_read_b64_tr_b16 v[218:219], v240 offset:4768\n" \
    "v_mfma_f32_16x16x32_bf16 v[44:47], v[228:231], v[156:159], v[44:47]\n" \
    "ds_read_b64_tr_b16 v[220:221], v240 offset:9376\n" \
    "v_mfma_f32_16x16x32_bf16 v[44:47], v[224:227], v[152:155], v[44:47]\n" \
    "ds_read_b64_tr_b16 v[222:223], v240 offset:13984\n" \
    "v_mfma_f32_16x16x32_bf16 v[80:83], v[224:227], v[160:163], v[80:83]\n" \
    "buffer_load_dwordx4 v[200:203], v243, %[rs], 0 offen offset:2048 sc0 sc1\n" \
    "v_mfma_f32_16x16x32_bf16 v[80:83], v[228:231], v[164:167], v[80:83]\n" \
    "buffer_load_dwordx4 v[204:207], v243, %[rs], 0 offen offset:3072 sc0 sc1\n" \
    "v_mfma_f32_16x16x32_bf16 v[116:119], v[228:231], v[172:175], v[116:119]\n" \
    "v_add_u32 v242, 0x10000, v242\n" \
    "v_mfma_f32_16x16x32_bf16 v[116:119], v[224:227], v[168:171], v[116:119]\n" \
    "s_waitcnt lgkmcnt(8)\n" \
    "v_mfma_f32_16x16x32_bf16 v[120:123], v[232:235], v[168:171], v[120:123]\n" \
    "ds_read_b64_tr_b16 v[224:225], v240 offset:192\n" \
    "v_mfma_f32_16x16x32_bf16 v[120:123], v[236:239], v[172:175], v[120:123]\n" \
    "ds_read_b64_tr_b16 v[226:227], v240 offset:4800\n" \
    "v_mfma_f32_16x16x32_bf16 v[84:87], v[236:239], v[164:167], v[84:87]\n" \
    "ds_read_b64_tr_b16 v[228:229], v240 offset:9408\n" \
    "v_mfma_f32_16x16x32_bf16 v[84:87], v[232:235], v[160:163], v[84:87]\n" \
    "ds_read_b64_tr_b16 v[230:231], v240 offset:14016\n" \
    "v_mfma_f32_16x16x32_bf16 v[48:51], v[232:235], v[152:155], v[48:51]\n" \
    "v_add_u32 v243, 0x10000, v243\n" \
    "v_mfma_f32_16x16x32_bf16 v[48:51], v[236:239], v[156:159], v[48:51]\n" \
    "v_add_u32 v246, 1, v246\n" \
    "v_mfma_f32_16x16x32_bf16 v[12:15], v[236:239], v[148:151], v[12:15]\n" \
    "v_mfma_f32_16x16x32_bf16 v[12:15], v[232:235], v[144:147], v[12:15]\n" \
    "s_waitcnt lgkmcnt(8)\n" \
    "v_mfma_f32_16x16x32_bf16 v[16:19], v[208:211], v[144:147], v[16:19]\n" \
    "ds_read_b64_tr_b16 v[232:233], v240 offset:224\n" \
    "v_mfma_f32_16x16x32_bf16 v[16:19], v[212:215], v[148:151], v[16:19]\n" \
    "ds_read_b64_tr_b16 v[234:235], v240 offset:4832\n" \
    "v_mfma_f32_16x16x32_bf16 v[52:55], v[212:215], v[156:159], v[52:55]\n" \
    "ds_read_b64_tr_b16 v[236:237], v240 offset:9440\n" \
    "v_mfma_f32_16x16x32_bf16 v[52:55], v[208:211], v[152:155], v[52:55]\n" \
    "ds_read_b64_tr_b16 v[238:239], v240 offset:14048\n" \
    "v_mfma_f32_16x16x32_bf16 v[88:91], v[208:211], v[160:163], v[88:91]\n" \
    "v_add_u32 v240, %[ps], v241\n" \
    "v_mfma_f32_16x16x32_bf16 v[88:91], v[212:215], v[164:167], v[88:91]\n" \
    "ds_write_b32 v247, v246 offset:32\n" \
    "v_mfma_f32_16x16x32_bf16 v[124:127], v[212:215], v[172:175], v[124:127]\n" \
    "v_mfma_f32_16x16x32_bf16 v[124:127], v[208:211], v[168:171], v[124:127]\n" \
    "s_waitcnt lgkmcnt(9)\n" \
    "v_mfma_f32_16x16x32_bf16 v[128:131], v[216:219], v[168:171], v[128:131]\n" \
    "ds_read_b64_tr_b16 v[208:209], v240 offset:0\n" \
    "v_mfma_f32_16x16x32_bf16 v[128:131], v[220:223], v[172:175], v[128:131]\n" \
    "ds_read_b64_tr_b16 v[210:211], v240 offset:4608\n" \
    "v_mfma_f32_16x16x32_bf16 v[92:95], v[220:223], v[164:167], v[92:95]\n" \
    "ds_read_b64_tr_b16 v[212:213], v240 offset:9216\n" \
    "v_mfma_f32_16x16x32_bf16 v[92:95], v[216:219], v[160:163], v[92:95]\n" \
    "ds_read_b64_tr_b16 v[214:215], v240 offset:13824\n" \
    "v_mfma_f32_16x16x32_bf16 v[56:59], v[216:219], v[152:155], v[56:59]\n" \
    "v_mfma_f32_16x16x32_bf16 v[56:59], v[220:223], v[156:159], v[56:59]\n" \
    "v_mfma_f32_16x16x32_bf16 v[20:23], v[220:223], v[148:151], v[20:23]\n" \
    "v_mfma_f32_16x16x32_bf16 v[20:23], v[216:219], v[144:147], v[20:23]\n" \
    "s_waitcnt lgkmcnt(9)\n" \
    "v_mfma_f32_16x16x32_bf16 v[24:27], v[224:227], v[144:147], v[24:27]\n" \
    "ds_read2_b32 v[244:245], v247 offset1:4\n" \
    "ds_read_b64_tr_b16 v[216:217], v240 offset:32\n" \
    "v_mfma_f32_16x16x32_bf16 v[24:27], v[228:231], v[148:151], v[24:27]\n" \
    "ds_read_b64_tr_b16 v[218:219], v240 offset:4640\n" \
    "v_mfma_f32_16x16x32_bf16 v[60:63], v[228:231], v[156:159], v[60:63]\n" \
    "ds_read_b64_tr_b16 v[220:221], v240 offset:9248\n" \
    "v_mfma_f32_16x16x32_bf16 v[60:63], v[224:227], v[152:155], v[60:63]\n" \
    "ds_read_b64_tr_b16 v[222:223], v240 offset:13856\n" \
    "v_mfma_f32_16x16x32_bf16 v[96:99], v[224:227], v[160:163], v[96:99]\n" \
    "v_mfma_f32_16x16x32_bf16 v[96:99], v[228:231], v[164:167], v[96:99]\n" \
    "v_mfma_f32_16x16x32_bf16 v[132:135], v[228:231], v[172:175], v[132:135]\n" \
    "v_mfma_f32_16x16x32_bf16 v[132:135], v[224:227], v[168:171], v[132:135]\n" \
    "s_waitcnt lgkmcnt(4)\n" \
    "v_mfma_f32_16x16x32_bf16 v[136:139], v[232:235], v[168:171], v[136:139]\n" \
    "ds_read_b64_tr_b16 v[224:225], v240 offset:64\n" \
    "v_mfma_f32_16x16x32_bf16 v[136:139], v[236:239], v[172:175], v[136:139]\n" \
    "ds_read_b64_tr_b16 v[226:227], v240 offset:4672\n" \
    "v_mfma_f32_16x16x32_bf16 v[100:103], v[236:239], v[164:167], v[100:103]\n" \
    "ds_read_b64_tr_b16 v[228:229], v240 offset:9280\n" \
    "v_mfma_f32_16x16x32_bf16 v[100:103], v[232:235], v[160:163], v[100:103]\n" \
    "ds_read_b64_tr_b16 v[230:231], v240 offset:13888\n" \
    "v_mfma_f32_16x16x32_bf16 v[64:67], v[232:235], v[152:155], v[64:67]\n" \
    "v_min_u32 v244, v244, v245\n" \
    "v_mfma_f32_16x16x32_bf16 v[64:67], v[236:239], v[156:159], v[64:67]\n" \
    "v_readfirstlane_b32 %[hv], v244\n" \
    "v_mfma_f32_16x16x32_bf16 v[28:31], v[236:239], v[148:151], v[28:31]\n" \
    "s_cmp_ge_u32 %[hv], %[nd]\n" \
    "v_mfma_f32_16x16x32_bf16 v[28:31], v[232:235], v[144:147], v[28:31]\n" \
    "s_waitcnt lgkmcnt(8) vmcnt(0)\n" \
    "s_cbranch_scc0 P8_%=\n" \
    "R8_%=:\n" \
    "v_mfma_f32_16x16x32_bf16 v[0:3], v[208:211], v[176:179], v[0:3]\n" \
    "ds_read_b64_tr_b16 v[232:233], v240 offset:96\n" \
    "v_mfma_f32_16x16x32_bf16 v[0:3], v[212:215], v[180:183], v[0:3]\n" \
    "ds_read_b64_tr_b16 v[234:235], v240 offset:4704\n" \
    "v_mfma_f32_16x16x32_bf16 v[36:39], v[212:215], v[188:191], v[36:39]\n" \
    "ds_read_b64_tr_b16 v[236:237], v240 offset:9312\n" \
    "v_mfma_f32_16x16x32_bf16 v[36:39], v[208:211], v[184:187], v[36:39]\n" \
    "ds_read_b64_tr_b16 v[238:239], v240 offset:13920\n" \
    "v_mfma_f32_16x16x32_bf16 v[72:75], v[208:211], v[192:195], v[72:75]\n" \
    "buffer_load_dwordx4 v[144:147], v242, %[rs], 0 offen offset:0 sc0 sc1\n" \
    "v_mfma_f32_16x16x32_bf16 v[72:75], v[212:215], v[196:199], v[72:75]\n" \
    "buffer_load_dwordx4 v[148:151], v242, %[rs], 0 offen offset:1024 sc0 sc1\n" \
    "v_mfma_f32_16x16x32_bf16 v[108:111], v[212:215], v[204:207], v[108:111]\n" \
    "buffer_load_dwordx4 v[152:155], v242, %[rs], 0 offen offset:2048 sc0 sc1\n" \
    "v_mfma_f32_16x16x32_bf16 v[108:111], v[208:211], v[200:203], v[108:111]\n" \
    "s_waitcnt lgkmcnt(8)\n" \
    "v_mfma_f32_16x16x32_bf16 v[112:115], v[216:219], v[200:203], v[112:115]\n" \
    "ds_read_b64_tr_b16 v[208:209], v240 offset:128\n" \
    "s_add_u32 %[nd], %[nd], 1\n" \
    "v_mfma_f32_16x16x32_bf16 v[112:115], v[220:223], v[204:207], v[112:115]\n" \
    "ds_read_b64_tr_b16 v[210:211], v240 offset:4736\n" \
    "s_add_u32 %[ps], %[ps], 18432\n" \
    "v_mfma_f32_16x16x32_bf16 v[76:79], v[220:223], v[196:199], v[76:79]\n" \
    "ds_read_b64_tr_b16 v[212:213], v240 offset:9344\n" \
    "s_cmp_eq_u32 %[ps], 147456\n" \
    "v_mfma_f32_16x16x32_bf16 v[76:79], v[216:219], v[192:195], v[76:79]\n" \
    "ds_read_b64_tr_b16 v[214:215], v240 offset:13952\n" \
    "s_cselect_b32 %[ps], 0, %[ps]\n" \
    "v_mfma_f32_16x16x32_bf16 v[40:43], v[216:219], v[184:187], v[40:43]\n" \
    "buffer_load_dwordx4 v[156:159], v242, %[rs], 0 offen offset:3072 sc0 sc1\n" \
    "v_mfma_f32_16x16x32_bf16 v[40:43], v[220:223], v[188:191], v[40:43]\n" \
    "buffer_load_dwordx4 v[160:163], v243, %[rs], 0 offen offset:0 sc0 sc1\n" \
    "v_mfma_f32_16x16x32_bf16 v[4:7], v[220:223], v[180:183], v[4:7]\n" \
    "buffer_load_dwordx4 v[164:167], v243, %[rs], 0 offen offset:1024 sc0 sc1\n" \
    "v_mfma_f32_16x16x32_bf16 v[4:7], v[216:219], v[176:179], v[4:7]\n" \
    "s_waitcnt lgkmcnt(8)\n" \
    "v_mfma_f32_16x16x32_bf16 v[8:11], v[224:227], v[176:179], v[8:11]\n" \
    "ds_read_b64_tr_b16 v[216:217], v240 offset:160\n" \
    "s_cmp_eq_u32 %[nd], 66\n" \
    "v_mfma_f32_16x16x32_bf16 v[8:11], v[228:231], v[180:183], v[8:11]\n" \
    "ds_read_b64_tr_b16 v[218:219], v240 offset:4768\n" \
    "s_cselect_b32 %[ck], -1, %[nd]\n" \
    "v_mfma_f32_16x16x32_bf16 v[44:47], v[228:231], v[188:191], v[44:47]\n" \
    "ds_read_b64_tr_b16 v[220:221], v240 offset:9376\n" \
    "v_mfma_f32_16x16x32_bf16 v[44:47], v[224:227], v[184:187], v[44:47]\n" \
    "ds_read_b64_tr_b16 v[222:223], v240 offset:13984\n" \
    "v_mfma_f32_16x16x32_bf16 v[80:83], v[224:227], v[192:195], v[80:83]\n" \
    "buffer_load_dwordx4 v[168:171], v243, %[rs], 0 offen offset:2048 sc0 sc1\n" \
    "v_mfma_f32_16x16x32_bf16 v[80:83], v[228:231], v[196:199], v[80:83]\n" \
    "buffer_load_dwordx4 v[172:175], v243, %[rs], 0 offen offset:3072 sc0 sc1\n" \
    "v_mfma_f32_16x16x32_bf16 v[116:119], v[228:231], v[204:207], v[116:119]\n" \
    "v_add_u32 v242, 0x10000, v242\n" \
    "v_mfma_f32_16x16x32_bf16 v[116:119], v[224:227], v[200:203], v[116:119]\n" \
    "s_waitcnt lgkmcnt(8)\n" \
    "v_mfma_f32_16x16x32_bf16 v[120:123], v[232:235], v[200:203], v[120:123]\n" \
    "ds_read_b64_tr_b16 v[224:225], v240 offset:192\n" \
    "v_mfma_f32_16x16x32_bf16 v[120:123], v[236:239], v[204:207], v[120:123]\n" \
    "ds_read_b64_tr_b16 v[226:227], v240 offset:4800\n" \
    "v_mfma_f32_16x16x32_bf16 v[84:87], v[236:239], v[196:199], v[84:87]\n" \
    "ds_read_b64_tr_b16 v[228:229], v240 offset:9408\n" \
    "v_mfma_f32_16x16x32_bf16 v[84:87], v[232:235], v[192:195], v[84:87]\n" \
    "ds_read_b64_tr_b16 v[230:231], v240 offset:14016\n" \
    "v_mfma_f32_16x16x32_bf16 v[48:51], v[232:235], v[184:187], v[48:51]\n" \
    "v_add_u32 v243, 0x10000, v243\n" \
    "v_mfma_f32_16x16x32_bf16 v[48:51], v[236:239], v[188:191], v[48:51]\n" \
    "v_add_u32 v246, 1, v246\n" \
    "v_mfma_f32_16x16x32_bf16 v[12:15], v[236:239], v[180:183], v[12:15]\n" \
    "v_mfma_f32_16x16x32_bf16 v[12:15], v[232:235], v[176:179], v[12:15]\n" \
    "s_waitcnt lgkmcnt(8)\n" \
    "v_mfma_f32_16x16x32_bf16 v[16:19], v[208:211], v[176:179], v[16:19]\n" \
    "ds_read_b64_tr_b16 v[232:233], v240 offset:224\n" \
    "v_mfma_f32_16x16x32_bf16 v[16:19], v[212:215], v[180:183], v[16:19]\n" \
    "ds_read_b64_tr_b16 v[234:235], v240 offset:4832\n" \
    "v_mfma_f32_16x16x32_bf16 v[52:55], v[212:215], v[188:191], v[52:55]\n" \
    "ds_read_b64_tr_b16 v[236:237], v240 offset:9440\n" \
    "v_mfma_f32_16x16x32_bf16 v[52:55], v[208:211], v[184:187], v[52:55]\n" \
    "ds_read_b64_tr_b16 v[238:239], v240 offset:14048\n" \
    "v_mfma_f32_16x16x32_bf16 v[88:91], v[208:211], v[192:195], v[88:91]\n" \
    "v_add_u32 v240, %[ps], v241\n" \
    "v_mfma_f32_16x16x32_bf16 v[88:91], v[212:215], v[196:199], v[88:91]\n" \
    "ds_write_b32 v247, v246 offset:32\n" \
    "v_mfma_f32_16x16x32_bf16 v[124:127], v[212:215], v[204:207], v[124:127]\n" \
    "v_mfma_f32_16x16x32_bf16 v[124:127], v[208:211], v[200:203], v[124:127]\n" \
    "s_waitcnt lgkmcnt(9)\n" \
    "v_mfma_f32_16x16x32_bf16 v[128:131], v[216:219], v[200:203], v[128:131]\n" \
    "ds_read_b64_tr_b16 v[208:209], v240 offset:0\n" \
    "v_mfma_f32_16x16x32_bf16 v[128:131], v[220:223], v[204:207], v[128:131]\n" \
    "ds_read_b64_tr_b16 v[210:211], v240 offset:4608\n" \
    "v_mfma_f32_16x16x32_bf16 v[92:95], v[220:223], v[196:199], v[92:95]\n" \
    "ds_read_b64_tr_b16 v[212:213], v240 offset:9216\n" \
    "v_mfma_f32_16x16x32_bf16 v[92:95], v[216:219], v[192:195], v[92:95]\n" \
    "ds_read_b64_tr_b16 v[214:215], v240 offset:13824\n" \
    "v_mfma_f32_16x16x32_bf16 v[56:59], v[216:219], v[184:187], v[56:59]\n" \
    "v_mfma_f32_16x16x32_bf16 v[56:59], v[220:223], v[188:191], v[56:59]\n" \
    "v_mfma_f32_16x16x32_bf16 v[20:23], v[220:223], v[180:183], v[20:23]\n" \
    "v_mfma_f32_16x16x32_bf16 v[20:23], v[216:219], v[176:179], v[20:23]\n" \
    "s_waitcnt lgkmcnt(9)\n" \
    "v_mfma_f32_16x16x32_bf16 v[24:27], v[224:227], v[176:179], v[24:27]\n" \
    "ds_read2_b32 v[244:245], v247 offset1:4\n" \
    "ds_read_b64_tr_b16 v[216:217], v240 offset:32\n" \
    "v_mfma_f32_16x16x32_bf16 v[24:27], v[228:231], v[180:183], v[24:27]\n" \
    "ds_read_b64_tr_b16 v[218:219], v240 offset:4640\n" \
    "v_mfma_f32_16x16x32_bf16 v[60:63], v[228:231], v[188:191], v[60:63]\n" \
    "ds_read_b64_tr_b16 v[220:221], v240 offset:9248\n" \
    "v_mfma_f32_16x16x32_bf16 v[60:63], v[224:227], v[184:187], v[60:63]\n" \
    "ds_read_b64_tr_b16 v[222:223], v240 offset:13856\n" \
    "v_mfma_f32_16x16x32_bf16 v[96:99], v[224:227], v[192:195], v[96:99]\n" \
    "v_mfma_f32_16x16x32_bf16 v[96:99], v[228:231], v[196:199], v[96:99]\n" \
    "v_mfma_f32_16x16x32_bf16 v[132:135], v[228:231], v[204:207], v[132:135]\n" \
    "v_mfma_f32_16x16x32_bf16 v[132:135], v[224:227], v[200:203], v[132:135]\n" \
    "s_waitcnt lgkmcnt(4)\n" \
    "v_mfma_f32_16x16x32_bf16 v[136:139], v[232:235], v[200:203], v[136:139]\n" \
    "ds_read_b64_tr_b16 v[224:225], v240 offset:64\n" \
    "v_mfma_f32_16x16x32_bf16 v[136:139], v[236:239], v[204:207], v[136:139]\n" \
    "ds_read_b64_tr_b16 v[226:227], v240 offset:4672\n" \
    "v_mfma_f32_16x16x32_bf16 v[100:103], v[236:239], v[196:199], v[100:103]\n" \
    "ds_read_b64_tr_b16 v[228:229], v240 offset:9280\n" \
    "v_mfma_f32_16x16x32_bf16 v[100:103], v[232:235], v[192:195], v[100:103]\n" \
    "ds_read_b64_tr_b16 v[230:231], v240 offset:13888\n" \
    "v_mfma_f32_16x16x32_bf16 v[64:67], v[232:235], v[184:187], v[64:67]\n" \
    "v_min_u32 v244, v244, v245\n" \
    "v_mfma_f32_16x16x32_bf16 v[64:67], v[236:239], v[188:191], v[64:67]\n" \
    "v_readfirstlane_b32 %[hv], v244\n" \
    "v_mfma_f32_16x16x32_bf16 v[28:31], v[236:239], v[180:183], v[28:31]\n" \
    "s_cmp_ge_u32 %[hv], %[ck]\n" \
    "v_mfma_f32_16x16x32_bf16 v[28:31], v[232:235], v[176:179], v[28:31]\n" \
    "s_waitcnt lgkmcnt(8) vmcnt(0)\n" \
    "s_cbranch_scc1 L8_%=\n" \
    "s_cmp_eq_u32 %[nd], 66\n" \
    "s_cbranch_scc1 LEND%=\n" \
    "Q8_%=:\n" \
    "ds_read2_b32 v[244:245], v247 offset1:4\n" \
    "s_waitcnt lgkmcnt(0)\n" \
    "v_min_u32 v244, v244, v245\n" \
    "s_nop 0\n" \
    "v_readfirstlane_b32 %[hv], v244\n" \
    "s_cmp_ge_u32 %[hv], %[nd]\n" \
    "s_cbranch_scc1 L8_%=\n" \
    "s_sleep 1\n" \
    "s_branch Q8_%=\n" \
    "LEND%=:\n" \
    "s_waitcnt vmcnt(0) lgkmcnt(0)\n" \
    "s_nop 15\n" \
    ""

__global__ void __launch_bounds__(NTHR, 2) ag_gemm(const Args a) {
    __shared__ __attribute__((aligned(1024))) unsigned char lds_raw[LDS_BYTES];
    unsigned code;
    {
        unsigned long long pc;
        unsigned len;
        asm volatile("s_getpc_b64 %0\nAG_CODE0:\n\ts_mov_b32 %1, AG_CODE1-AG_CODE0" : "=s"(pc), "=s"(len));
        code = __builtin_amdgcn_raw_buffer_load_b32(rsrc((const void*)pc, len), threadIdx.x * 64, 0, 0);
        SB();
    }
    LAS unsigned char* lds = (LAS unsigned char*)lds_raw;
    volatile LAS unsigned* ctl = (volatile LAS unsigned*)(lds + L_CTL);
    volatile LAS unsigned* prog = ctl + 8;
    const int lane = threadIdx.x & 63;
    const int wave = __builtin_amdgcn_readfirstlane(threadIdx.x >> 6);
    const int bid = blockIdx.x;
    char* const myws = a.ws[NR - 1];
    asm volatile("" :: "s"(a.x), "s"(a.ws[0]), "s"(a.ws[1]), "s"(a.ws[2]), "s"(a.ws[3]), "s"(a.ws[4]), "s"(a.ws[5]), "s"(a.ws[6]), "s"(myws), "s"(a.rank));
    const int xcd = bid & 7, loc = bid >> 3;
    const bool is_comm = loc < 2;
    const int cid = loc * 8 + xcd, tl = (xcd & 1) * (2 * NTN) + loc - 2, pm = (xcd >> 1) * 4 + (tl & 3), pn = tl >> 2;
    const int pq = (pn & 1) ? -1 : pn >> 1, part = pq >> 31, flip = 0;

    if (wave == 0) STAMP(0);
    const __amdgpu_buffer_rsrc_t rin = rsrc(a.x, MS * K * 2);
    auto fetch = [&](u32x4 (&d)[4], unsigned b) {
        const unsigned o = (b < (unsigned)NBLK ? (b % NG) * 32 * (K * 2) + (b / NG) * (BK * 2) : OOB) + (lane & 15) * (K * 2) + (lane >> 4) * 16;
#pragma unroll
        for (int s = 0; s < 4; ++s)
            d[s] = __builtin_bit_cast(u32x4, __builtin_amdgcn_raw_buffer_load_b128(rin, o + (s & 1) * 64, (s >> 1) * 16 * (K * 2), 0));
    };
    u32x4 d[4];
    fetch(d, is_comm && wave < SW ? wave * NSC + cid : NBLK);
    GAS unsigned* cnt = (GAS unsigned*)(myws + W_CNT + (size_t)bid * 128);
    const unsigned cv = __hip_atomic_load(cnt, __ATOMIC_RELAXED, __HIP_MEMORY_SCOPE_AGENT);
    SB();
    if (wave == 0) STAMP(10);
    if (threadIdx.x < 32) ((LAS unsigned*)(lds + L_CTL))[threadIdx.x] = 0u;
    if (threadIdx.x < 256) *(LAS u32x2*)(lds + L_SEL + threadIdx.x * 8) = is_comm ? mover(threadIdx.x, true) : mover(threadIdx.x, false);
    SB();
    asm volatile("" :: "v"(code));
    const unsigned call = __builtin_amdgcn_readfirstlane(cv) + 1u;
    __syncthreads();
    if (threadIdx.x == 0) __hip_atomic_store(cnt, call, __ATOMIC_RELAXED, __HIP_MEMORY_SCOPE_AGENT);
    LAS unsigned char* lm = lds + L_SEL;
    int mul = loc;
    asm volatile("" : "+s"(mul));
    if (mul > 1) {
    GAS unsigned* tmo = (GAS unsigned*)(myws + W_TMO);
    const size_t xt_off = W_XT + (size_t)(call & 1u) * XT_BYTES;
    const unsigned oQ = pq < 0 ? OOB : ((NTN - pn) * BN + pq * 16) * 2;
    if (wave >= 6) {
        __builtin_amdgcn_s_setprio(AG_UPRIO);
        const int src = ((xcd + 2) & 6) + wave - 6;
        const unsigned u = tl;
        const __amdgpu_buffer_rsrc_t rf = rsrc(myws + W_FLAG + (size_t)src * NBLK * FS * 4, FL_BYTES + NBLK * FS * 4);
        const __amdgpu_buffer_rsrc_t rp = rsrc(myws + W_PK + (size_t)(call & 1u) * PK_BYTES + (size_t)src * NBLK * PBLK, NBLK * PBLK);
        const __amdgpu_buffer_rsrc_t rx = rsrc(myws + xt_off + (size_t)src * NBLK * BLK, NBLK * BLK);
        auto done = [&](unsigned b) {
            const unsigned g = b % NG, kt = b / NG;
            const unsigned fo = lane == 0 ? (((g >> 3) * NKT + kt) * 8 + (g & 7)) * FS * 4 : OOB;
            __builtin_amdgcn_raw_buffer_store_b32(call, rf, fo, 0, 17);
        };
        if (wave == 7) STAMP(8);
        unsigned long long lat_r = 0, lat_w = 0, lat_n = 0;
        if (src == a.rank) {
            fetch(d, u);
            for (unsigned b = u; b < (unsigned)NBLK; b += NU) {
#pragma unroll
                for (int s = 0; s < 4; ++s)
                    __builtin_amdgcn_raw_buffer_store_b128(d[s], rx, b * BLK + lane * 16 + s * 1024, 0, AG_UAUX);
                fetch(d, b + NU);
                wait_vm0();
                done(b);
            }
        } else {
            LAS unsigned char* lx = lds + L_UNP + (wave - 6) * UNP;
            LAS unsigned char* ls = lx + 2112;
            *(LAS u32x4*)(lx + lane * 32) = (u32x4){0u, 0u, 0u, 0u};
            *(LAS u32x4*)(lx + lane * 32 + 16) = (u32x4){0u, 0u, 0u, 0u};
            const unsigned long long want = tag_of(call) ? ~0ull : 0ull;
            const unsigned keep = (lane & 1) ? ~0u : 0x7fffffffu, keept = lane < 16 ? 0x7fffffffu : ~0u, sh = 31 - (lane >> 1), lw = (lane < 24 ? lane : 24) * 16;
            const unsigned long long t0 = rt();
            unsigned b = u, pend = ~0u;
            for (;;) {
                const unsigned so = b * PBLK;
                const unsigned hl = __builtin_amdgcn_raw_buffer_load_b32(rp, lane < 32 && b < (unsigned)NBLK ? so + 2560 + lane * 4 : OOB, 0, AG_HAUX);
                wait_vm0();
                const bool first = pend != ~0u;
                if (first) {
                    done(pend);
                    TRACE(src, pend, 2);
                    pend = ~0u;
                }
                if (b >= (unsigned)NBLK) break;
                if (((unsigned)(__ballot((int)hl < 0) ^ want) & 0x11111111u) != 0u) {
                    if (give_up(tmo, call, t0)) {
                        __hip_atomic_store(tmo, call, __ATOMIC_RELAXED, __HIP_MEMORY_SCOPE_AGENT);
                        break;
                    }
                    __builtin_amdgcn_s_sleep(AG_USLEEP);
                    continue;
                }
                if (first) lat_w += 100;
                ++lat_n;
                u32x4 p[2], t, x[2];
                u32x2 c;
                const unsigned h0 = __builtin_amdgcn_readlane(hl, 0);
                c = __builtin_bit_cast(u32x2, __builtin_amdgcn_raw_buffer_load_b64(rp, so + lane * 8 + 2048, 0, 17));
                t = __builtin_bit_cast(u32x4, __builtin_amdgcn_raw_buffer_load_b128(rp, lane < ((h0 & 512u) ? 24 : 16) ? so + lane * 16 + 2560 : OOB, 0, 17));
#pragma unroll
                for (int s = 0; s < 2; ++s)
                    p[s] = __builtin_bit_cast(u32x4, __builtin_amdgcn_raw_buffer_load_b128(rp, so + lane * 16 + s * 1024, 0, 17));
                TRACE(src, b, 1);
                const unsigned m2l = __builtin_amdgcn_readlane(hl, 3), m1l = bfi(0x7fffffffu, __builtin_amdgcn_readlane(hl, 4), h0 << 15);
                const unsigned long long m0 = (unsigned)__builtin_amdgcn_readlane(hl, 1) | ((unsigned long long)(unsigned)__builtin_amdgcn_readlane(hl, 2) << 32);
                const unsigned long long m1 = m1l | ((unsigned long long)(unsigned)__builtin_amdgcn_readlane(hl, 5) << 32);
                const unsigned e0 = h0 & 255u, cnt = (h0 >> 10) & 31u;
                const unsigned pa = (unsigned)lane - HW < cnt ? hl & 0x7ffu : 2048u;
                lx[pa] = (unsigned char)(hl >> 16);
                FENCE();
                x[0] = *(LAS u32x4*)(lx + lane * 32);
                x[1] = *(LAS u32x4*)(lx + lane * 32 + 16);
                FENCE();
                lx[pa] = 0;
                unsigned t2 = table2(e0), t3l = table3(e0, 0), t3h = table3(e0, 1), top0, top1, topc = m2l << sh, topt = h0 << (16 - (lane & 15));
                unsigned vo = (h0 & 256u) ? OOB : b * BLK + lane * 16, lq = (unsigned)(size_t)ls + (HW + cnt) * 4u;
                asm("v_cndmask_b32 %0, 0, -1, %1" : "=v"(top0) : "s"(m0));
                asm("v_cndmask_b32 %0, 0, -1, %1" : "=v"(top1) : "s"(m1));
                asm volatile("" : "+v"(t2), "+v"(t3l), "+v"(top0), "+v"(top1), "+v"(topc), "+v"(topt), "+v"(vo));
                asm volatile("" : "+s"(t3h), "+s"(lq));
                if (((__ballot((int)p[0].x < 0) ^ want) | (__ballot((int)p[1].x < 0) ^ want) |
                     ((__ballot((int)c.x < 0) ^ want) & 0x5555555555555555ull) | ((__ballot((int)t.x < 0) ^ want) & 0xffffull)) != 0ull) {
                    if (!give_up(tmo, call, t0)) continue;
                    __hip_atomic_store(tmo, call, __ATOMIC_RELAXED, __HIP_MEMORY_SCOPE_AGENT);
                    break;
                }
                t.x = bfi(keept, t.x, topt);
                *(LAS u32x4*)(ls + lw) = t;
                c.x = bfi(keep, c.x, topc);
                const u32x2 f = c & (c >> 1) & 0x55555555u;
                const unsigned nb = 3u * (__builtin_popcount(f.x) + __builtin_popcount(f.y));
                const unsigned off = scan(nb) - nb;
                u32x2 mv[8];
                FENCE();
                LAS unsigned* lt = (LAS unsigned*)(size_t)(((off >> 5) << 2) + lq);
                const unsigned w0 = lt[0], w1 = lt[1], w2 = lt[2];
#pragma unroll
                for (int j = 0; j < 8; ++j)
                    mv[j] = *(LAS u32x2*)(lm + ((c[j >> 2] >> (8 * (j & 3))) & 255u) * 8);
                FENCE();
                unsigned al = __builtin_amdgcn_alignbit(w1, w0, off), ah = __builtin_amdgcn_alignbit(w2, w1, off);
                p[0].x = bfi(0x7fffffffu, p[0].x, top0);
                p[1].x = bfi(0x7fffffffu, p[1].x, top1);
#pragma unroll
                for (int s = 0; s < 4; ++s) {
                    u32x4 d;
#pragma unroll
                    for (int i = 0; i < 2; ++i) {
                        const int j = 2 * s + i;
                        unsigned q = al;
                        al = __builtin_amdgcn_alignbit(ah, al, mv[j].y);
                        ah >>= mv[j].y;
                        asm("v_lshrrev_b32_sdwa %0, %1, %0 dst_sel:WORD_1 dst_unused:UNUSED_PRESERVE src0_sel:DWORD src1_sel:DWORD" : "+v"(q) : "v"(6), "v"(al));
                        const unsigned g = perm(t3h, t3l, (q & 0x00070007u) | ((q << 5) & 0x07000700u));
                        const unsigned e = perm(t2, g, mv[j].x) ^ x[j >> 2][j & 3];
                        const unsigned r0 = perm(e, p[j >> 2][j & 3], 0x05010400u), r1 = perm(e, p[j >> 2][j & 3], 0x07030602u);
                        d[2 * i] = __builtin_amdgcn_alignbit(r0, r0, 1);
                        d[2 * i + 1] = __builtin_amdgcn_alignbit(r1, r1, 1);
                    }
                    __builtin_amdgcn_raw_buffer_store_b128(d, rx, vo + s * 1024, 0, AG_UAUX);
                }
                pend = b;
                b += NU;
            }
        }
        if (wave == 7) STAMP(9);
        asm volatile("AG_CODE1:");
    }
    else if (wave < 4) {
        constexpr int PW = AG_DRY ? 3 : 4;
        if (wave < PW) {
            const unsigned cw = ((xcd >> 1) * (4 * NTN) + tl) * PW + wave;
            constexpr int W1 = (AG_PD ? AG_PD : NKT) * BK * NL * 2;
            const __amdgpu_buffer_rsrc_t rw = rsrc(a.w, W1), rs = rsrc(a.x, MS * K * 2);
            constexpr int W0 = AG_PF0 * BK * NL * 2, ST = 16 * NTN * PW * 1024, NW = (W1 - W0 + ST - 1) / ST, NX = (MS * K * 2 + ST - 1) / ST;
            u32x4 pf[NW], px[NX];
#pragma unroll
            for (int i = 0; i < NW; ++i)
                pf[i] = __builtin_bit_cast(u32x4, __builtin_amdgcn_raw_buffer_load_b128(rw, W0 + i * ST + cw * 1024 + lane * 16, 0, AG_PFAUX));
            if (AG_PF == 2) {
#pragma unroll
                for (int i = 0; i < NX; ++i)
                    px[i] = __builtin_bit_cast(u32x4, __builtin_amdgcn_raw_buffer_load_b128(rs, i * ST + cw * 1024 + lane * 16, 0, AG_PFAUX));
#pragma unroll
                for (int i = 0; i < NX; ++i) asm volatile("" :: "v"(px[i]));
            }
#pragma unroll
            for (int i = 0; i < NW; ++i) asm volatile("" :: "v"(pf[i]));
            if (wave == 0) STAMP(12);
        }
        const int g0 = (pm & 1) * 8 + wave * 2;
        const unsigned long long xb = (unsigned long long)(myws + xt_off + ((size_t)(pm >> 1) * NBLK + g0) * BLK);
        const u32x4 ra = (u32x4){(unsigned)xb, (unsigned)(xb >> 32), (unsigned)((NBLK - g0) * BLK), 0x00020000u};
        const unsigned lb = (unsigned)(unsigned long long)lds;
        int dry = AG_DRY ? (wave + 1) >> 2 : 0;
      for (;;) {
        int l1;
        asm volatile("v_mbcnt_lo_u32_b32 %0, -1, 0\n\tv_mbcnt_hi_u32_b32 %0, -1, %0" : "=v"(l1));
        unsigned vx = l1 * 16 + flip * BLK, have, need, pos, chk;
        f32x32 c[5];
        f32x16 c4;
        unsigned long long t1 = 0, t2 = 0;
        char* ob = a.out + ((size_t)(pm * BM + wave * 64) * NL + pn * BN) * 2;
        asm volatile(AG_KLOOP
                     : "={v[0:31]}"(c[0]), "={v[32:63]}"(c[1]), "={v[64:95]}"(c[2]), "={v[96:127]}"(c[3]), "={v[128:143]}"(c4),
                       "+{v242}"(vx), [hv] "=&s"(have), [nd] "=&s"(need), [ps] "=&s"(pos), [ck] "=&s"(chk), [t1] "+s"(t1), [t2] "+s"(t2)
                     : "{v241}"(lb + L_W + (l1 >> 2) * RS + (l1 & 3) * 8), "{v247}"(lb + L_CTL + wave * 4 + (l1 >> 6)), [rs] "s"(ra), [n8] "s"(part), "s"(ob),
                       [dry] "s"(dry)
                     : "v144", "v145", "v146", "v147", "v148", "v149", "v150", "v151", "v152", "v153", "v154", "v155", "v156", "v157", "v158", "v159", "v160", "v161", "v162", "v163", "v164", "v165", "v166", "v167", "v168", "v169", "v170", "v171", "v172", "v173", "v174", "v175", "v176", "v177", "v178", "v179", "v180", "v181", "v182", "v183", "v184", "v185", "v186", "v187", "v188", "v189", "v190", "v191", "v192", "v193", "v194", "v195", "v196", "v197", "v198", "v199", "v200", "v201", "v202", "v203", "v204", "v205", "v206", "v207", "v208", "v209", "v210", "v211", "v212", "v213", "v214", "v215", "v216", "v217", "v218", "v219", "v220", "v221", "v222", "v223", "v224", "v225", "v226", "v227", "v228", "v229", "v230", "v231", "v232", "v233", "v234", "v235", "v236", "v237", "v238", "v239", "v240", "v243", "v244", "v245", "v246", "scc", "memory");
#pragma unroll
        for (int i = 0; i < 16; ++i) c[4][i] = c4[i];
        int l2;
        asm volatile("v_mbcnt_lo_u32_b32 %0, -1, 0\n\tv_mbcnt_hi_u32_b32 %0, -1, %0" : "=v"(l2));
        constexpr int ES = RS + 16, E0 = NKT - NSLOT;
        unsigned pv = ctl[4 + (l2 & 7)];
        while (!__all(pv >= (unsigned)(E0 + 5))) {
            __builtin_amdgcn_s_sleep(1);
            pv = ctl[4 + (l2 & 7)];
        }
        const bool bad = __builtin_amdgcn_readfirstlane(pv) == ALL + 1;
        asm volatile("" ::: "memory");
        static_assert(E0 % NSLOT == 0 && 4 * 64 * ES <= 5 * SLOT, "epilogue staging");
        LAS unsigned char* st = lds + L_W + wave * 64 * ES;
        LAS unsigned char* sw = st + (l2 & 15) * ES + (l2 >> 4) * 8;
        LAS unsigned char* srS = st + (l2 >> 4) * ES + (l2 & 15) * 16;
        LAS unsigned char* srH = st + (l2 >> 1) * ES + 256 + (l2 & 1) * 16;
        const __amdgpu_buffer_rsrc_t ro = rsrc(ob, (64 * NL - pn * BN) * 2);
        const unsigned fl = flip * (32 * NL * 2), voS = (l2 >> 4) * (NL * 2) + (l2 & 15) * 16, voH = (l2 >> 1) * (NL * 2) + (l2 & 1) * 16 + oQ;
        const unsigned vo[2][2] = {{voS + fl, voH + fl}, {voS + (32 * NL * 2 - fl), voH + (part ? OOB : 32 * NL * 2)}};
        auto put = [&](int m) {
#pragma unroll
            for (int n = 0; n < NB; ++n) {
                const int e = 4 * (m * NB + n);
                const f32x4 v = (f32x4){c[e >> 5][e & 31], c[e >> 5][(e & 31) + 1], c[e >> 5][(e & 31) + 2], c[e >> 5][(e & 31) + 3]};
                *(LAS u32x2*)(sw + m * 16 * ES + n * 32) = __builtin_bit_cast(u32x2, __builtin_convertvector(v, bf16x4));
            }
        };
        u32x4 o[2][5];
        auto get = [&](int m) {
#pragma unroll
            for (int s = 0; s < 4; ++s) o[m & 1][s] = *(LAS u32x4*)(srS + (m * 16 + s * 4) * ES);
            if (m & 1) o[1][4] = *(LAS u32x4*)(srH + (m - 1) * 16 * ES);
        };
        auto out = [&](int m) {
#pragma unroll
            for (int s = 0; s < 4 + (m & 1); ++s)
                __builtin_amdgcn_raw_buffer_store_b128(o[m & 1][s], ro, s < 4 ? vo[m >> 1][0] + ((m & 1) * 16 + s * 4) * NL * 2 : vo[m >> 1][1],
                                                       0, AG_OAUX);
        };
        if (!bad) {
            put(0); get(0); put(1); SB(); out(0); get(1); put(2); SB(); out(1); get(2); put(3); SB(); out(2); get(3); out(3);
        } else {
#pragma unroll
            for (int s = 0; s < 10; ++s) o[s / 5][s % 5] = (u32x4){0x7fc07fc0u, 0x7fc07fc0u, 0x7fc07fc0u, 0x7fc07fc0u};
            out(0); out(1); out(2); out(3);
        }
        asm volatile("s_mov_b64 exec, -1\n\ts_sub_u32 %0, %0, 1" : "+s"(dry) :: "scc", "memory");
        if (dry < 0) break;
      }
    } else if (wave == 4) {
        __builtin_amdgcn_s_setprio(AG_WPRIO);
        const __amdgpu_buffer_rsrc_t rw = rsrc(a.w + pn * BN * 2, (unsigned)(K * NL - pn * BN) * 2);
        unsigned vo[NPIECE / 2];
#pragma unroll
        for (int i = 0; i < NPIECE / 2; ++i) {
            const unsigned o = i * 1024 + lane * 16, r = o / RS, cb = o % RS;
            const unsigned k = ((r >> 2) & 3) * 8 + (r >> 4) * 4 + (r & 3);
            vo[i] = k * (NL * 2) + cb + (cb < 256 ? 0u : oQ - 256);
        }
        const __amdgpu_buffer_rsrc_t rall = rsrc(a.w, K * NL * 2);
        const unsigned ln = ((xcd >> 1) * (4 * NTN) + tl) * 9 + lane, po = lane < 9 && ln < BK * NL * 2 / 128 ? ln * 128 + AG_PD * (BK * NL * 2) : OOB;
        int pub = 0;
        unsigned so = 0;
        for (int kt = 0; kt < NKT; ++kt) {
            if (kt >= NSLOT) {
                const unsigned nd = kt - NSLOT + 1;
                if (!__all(prog[lane & 3] >= nd)) {
                    if (pub < kt) {
                        asm volatile("s_waitcnt vmcnt(0)" ::: "memory");
                        pub = kt;
                        ctl[lane & 3] = pub;
                    }
                    while (!__all(prog[lane & 3] >= nd)) __builtin_amdgcn_s_sleep(AG_WSLEEP);
                }
                asm volatile("" ::: "memory");
                if ((nd & 7) == 0) STAMP(16 + (nd >> 3));
            }
            LAS unsigned char* dst = lds + L_W + so;
            {
#pragma unroll
                for (int i = 0; i < NPIECE; ++i)
                    __builtin_amdgcn_raw_ptr_buffer_load_lds(rw, (LAS void*)(dst + i * 1024), 16, vo[i % (NPIECE / 2)],
                                                             (kt * 2 + i / (NPIECE / 2)) * (32 * NL * 2), 0, 0);
            }
            if (AG_PD) __builtin_amdgcn_raw_ptr_buffer_load_lds(rall, (LAS void*)(lds + L_SINK), 4, po + kt * (BK * NL * 2), 0, 0, 0);
            so = so + SLOT == NSLOT * SLOT ? 0u : so + SLOT;
            if (kt + 1 - pub > 2) {
                asm volatile("s_waitcnt vmcnt(%0)" :: "n"(2 * NPIECE + (AG_PD ? 3 : 0)) : "memory");
                ++pub;
                ctl[lane & 3] = pub;
            }
        }
        asm volatile("s_waitcnt vmcnt(0)" ::: "memory");
        ctl[lane & 3] = ALL;
    }
    else if (wave == 5) {
        __builtin_amdgcn_s_setprio(AG_PPRIO);
        GAS unsigned* flag = (GAS unsigned*)(myws + W_FLAG) + pm * NKT * 8 * FS;
        const unsigned long long t0 = rt();
        int cur = 0;
        unsigned long long npoll = 0;
        while (cur < NKT) {
            ++npoll;
            const int idx = cur * 8 + lane;
            unsigned v = call;
            if (lane < AG_PW * 8 && idx < NKT * 8) v = __hip_atomic_load(flag + idx * FS, __ATOMIC_RELAXED, __HIP_MEMORY_SCOPE_SYSTEM);
            const unsigned long long miss = __ballot((int)(v - call) < 0);
            const int adv = miss ? (__builtin_ctzll(miss) >> 3) : AG_PW;
            if (adv > 0) {
                if (cur == 0) STAMP(4);
                if (((cur + adv) >> 3) != (cur >> 3)) STAMP(24 + (((cur + adv) >> 3) & 7));
                cur = cur + adv < NKT ? cur + adv : NKT;
                asm volatile("" ::: "memory");
                ctl[4 + (lane & 3)] = cur < NKT ? cur : ALL;
            } else {
                if (give_up(tmo, call, t0)) {
                    __hip_atomic_store(tmo, call, __ATOMIC_RELAXED, __HIP_MEMORY_SCOPE_AGENT);
                    ctl[4 + (lane & 3)] = ALL + 1;
                    break;
                }
                __builtin_amdgcn_s_sleep(AG_PSLEEP);
            }
        }
        STAMP(5);
    }
    } else {
        LAS unsigned char* lx = lds + L_W + wave * 8192;
        LAS unsigned char* ls = lx + 2048;
        LAS unsigned short* lp = (LAS unsigned short*)(lx + 4096);
        if (wave == 0) STAMP(11);
        const size_t xt_off = W_XT + (size_t)(call & 1u) * XT_BYTES, pk_off = W_PK + (size_t)(call & 1u) * PK_BYTES;
        if (wave < SW) {
            const int sid = wave * NSC + cid;
            __amdgpu_buffer_rsrc_t rd[NR];
#pragma unroll
            for (int i = 0; i < NR; ++i) rd[i] = rsrc(a.ws[i], (unsigned)W_END);
            const unsigned xt0 = (unsigned)xt_off + a.rank * NBLK * BLK, pk0 = (unsigned)pk_off + a.rank * NBLK * PBLK;
            const unsigned tbase = (call - 1u) * (unsigned)NBLK - 2u * NSEND;
            const unsigned tag = tag_of(call);
            auto low = [&](const u32x4 (&d)[4], int j, unsigned& e) {
                const unsigned d0 = d[j >> 1][(j & 1) * 2], d1 = d[j >> 1][(j & 1) * 2 + 1];
                const unsigned r0 = __builtin_amdgcn_alignbit(d0, d0, 31), r1 = __builtin_amdgcn_alignbit(d1, d1, 31);
                e = perm(r1, r0, 0x07050301u);
                return perm(r1, r0, 0x06040200u);
            };
            auto pack = [&](const u32x4 (&d)[4], u32x4 (&pk)[3], bool& raw, bool& annex) {
                unsigned e[8], sum = 0;
                *(LAS u32x2*)(ls + 512 + lane * 8) = (u32x2){0u, 0u};
#pragma unroll
                for (int j = 0; j < 8; ++j) {
                    pk[j >> 2][j & 3] = low(d, j, e[j]);
                    sum = __builtin_amdgcn_sad_u8(e[j], 0u, sum);
                }
                const unsigned mean = (unsigned)__builtin_amdgcn_readlane(scan(sum), 63) >> 11;
                const unsigned e0 = mean < 7u ? 7u : mean > 252u ? 252u : mean;
                const unsigned k2 = key2(e0), t2 = table2(e0), t3l = table3(e0, 0), t3h = table3(e0, 1);
                u32x4 x[2];
                u32x2 c = {0u, 0u}, mv[8];
                unsigned mask = 0, i3[8];
#pragma unroll
                for (int j = 0; j < 8; ++j) {
                    const unsigned sel = (e[j] ^ k2) & 3u * B1;
                    const unsigned t = ((perm(t2, t2, sel) ^ e[j]) >> 1) + 0x7f7f7f7fu;
                    const unsigned m = perm(t, t << 8, 0x0b090a08u);
                    const unsigned code = sel | (m & 3u * B1);
                    c[j >> 2] |= __builtin_amdgcn_udot4(code, 0x40100401u, 0u, false) << (8 * (j & 3));
                    mv[j] = *(LAS u32x2*)(lm + __builtin_amdgcn_udot4(code & (code >> 1) & B1, 0x40201008u, 0u, false));
                    i3[j] = (e[j] ^ e0 * B1) & m & 7u * B1;
                    const unsigned y = (perm(t3h, t3l, i3[j]) ^ e[j]) & m;
                    x[j >> 2][j & 3] = y;
                    mask = ((((y & 0x7f7f7f7fu) + 0x7f7f7f7fu) | y) & 0x80808080u) | (mask >> 1);
                }
                *(LAS u32x4*)(lx + lane * 32) = x[0];
                *(LAS u32x4*)(lx + lane * 32 + 16) = x[1];
                *(LAS u32x2*)(ls + lane * 8) = c;
                unsigned cnt = 0;
                for (;;) {
                    const unsigned long long act = __ballot(mask != 0u);
                    if (act == 0ull || cnt > NPATCH) break;
                    const unsigned p = __builtin_ctz(mask | 0x80000000u);
                    const unsigned at = __builtin_amdgcn_mbcnt_hi((unsigned)(act >> 32), __builtin_amdgcn_mbcnt_lo((unsigned)act, cnt));
                    lp[mask != 0u ? at : 127u] = (unsigned short)(lane * 32 + (p & 7u) * 4 + (p >> 3));
                    mask &= mask - 1u;
                    cnt += __builtin_popcountll(act);
                }
                FENCE();
                const unsigned pos = lp[lane] & 0x7ffu;
                unsigned long long acc = 0;
                unsigned nb = 0;
#pragma unroll
                for (int j = 7; j >= 0; --j) {
                    const unsigned g = perm(0u, i3[j], mv[j].x);
                    acc = (acc << mv[j].y) | __builtin_amdgcn_udot4(g, 0x003e0801u, g >> 15, false);
                    nb += mv[j].y;
                }
                const unsigned xb = lx[pos];
                const unsigned in = scan(nb), off = in - nb + cnt * 32u;
                const unsigned nd = cnt + (((unsigned)__builtin_amdgcn_readlane(in, 63) + 31u) >> 5);
                raw = cnt > NPATCH || nd > NLONG || __ballot(nb > 63u) != 0ull;
                annex = !raw && nd > NTAIL;
                LAS unsigned* lt = (LAS unsigned*)(ls + 512) + HW;
                const unsigned long long sa = acc << (off & 31u);
                __hip_atomic_fetch_or(lt + (off >> 5), (unsigned)sa, __ATOMIC_RELAXED, __HIP_MEMORY_SCOPE_WAVEFRONT);
                __hip_atomic_fetch_or(lt + (off >> 5) + 1, (unsigned)(sa >> 32), __ATOMIC_RELAXED, __HIP_MEMORY_SCOPE_WAVEFRONT);
                __hip_atomic_fetch_or(lt + (off >> 5) + 2, (unsigned)(acc >> 33) >> (~off & 31u), __ATOMIC_RELAXED, __HIP_MEMORY_SCOPE_WAVEFRONT);
                lt[(unsigned)lane < cnt ? lane : 300] = pos | (xb << 16);
                FENCE();
                pk[2] = *(LAS u32x4*)(ls + lane * 16);
                FENCE();
                const unsigned long long m0 = __ballot((int)pk[0].x < 0), m1 = __ballot((int)pk[1].x < 0);
                if (lane == 33) pk[2].x = (unsigned)m1, pk[2].y = (unsigned)(m1 >> 32);
                const unsigned long long m2 = __ballot((int)pk[2].x < 0);
                if (lane == 32) pk[2] = (u32x4){e0 | (raw ? 256u : cnt << 10) | (annex ? 512u : 0u) | (unsigned)(m2 >> 33) << 16, (unsigned)m0, (unsigned)(m0 >> 32), (unsigned)m2};
#pragma unroll
                for (int s = 0; s < 3; ++s) pk[s].x = bfi(s < 2 || lane < 48 ? 0x7fffffffu : ~0u, pk[s].x, tag);
            };
            unsigned b0 = sid, b1 = sid + NSEND;
            u32x4 nx[4], pk[3];
            bool raw, annex;
            unsigned skip = 0;
            if (AG_EARLY == 1 || wave < AG_EARLY - 1) {
#pragma unroll
                for (int j = 0; j < 8; ++j) pk[j >> 2][j & 3] = low(d, j, skip);
#pragma unroll
                for (int s = 0; s < 2; ++s) pk[s].x = (pk[s].x & 0x7fffffffu) | tag;
#pragma unroll
                for (int i = 0; i < NR - 1; ++i)
#pragma unroll
                    for (int s = 0; s < 2; ++s)
                        store16(pk[s], rd[i], pk0 + b0 * PBLK + lane * 16 + s * 1024);
                SB();
                skip = OOB;
            }
            pack(d, pk, raw, annex);
            if (wave == 0) STAMP(6);
            unsigned long long nsent = 0;
            unsigned seq = wave;
            while (b0 < (unsigned)NBLK) {
                const unsigned sp = pk0 + b0 * PBLK + lane * 16;
                if (annex) {
#pragma unroll
                    for (int i = 0; i < NR - 1; ++i)
                        store16(pk[2], rd[i], (lane >> 3) == 6 ? sp + 2048 : OOB);
                    wait_vm0();
                }
                while (ctl[0] < seq) __builtin_amdgcn_s_sleep(1);
                FENCE();
                if (raw) {
                    const unsigned so = xt0 + b0 * BLK + lane * 16;
#pragma unroll
                    for (int i = 0; i < NR - 1; ++i)
#pragma unroll
                        for (int s = 0; s < 4; ++s)
                            store16(d[s], rd[i], so + s * 1024);
                    wait_vm0();
                }
                TRACE(a.rank, b0, 0);
                fetch(nx, b1);
                const unsigned tk = __builtin_amdgcn_raw_ptr_buffer_atomic_add_i32(1, rd[NR - 1], lane == 0 ? (unsigned)W_TICKET : OOB, 0, 0);
                const unsigned sp2 = lane < 48 ? sp + 2048 : OOB;
#pragma unroll
                for (int i = 0; i < NR - 1; ++i)
#pragma unroll
                    for (int s = 0; s < 2; ++s)
                        store16(pk[s], rd[i], (sp | skip) + s * 1024);
                skip = 0;
#pragma unroll
                for (int i = 0; i < NR - 1; ++i)
                    store16(pk[2], rd[i], sp2);
                FENCE();
                if (lane == 0) __hip_atomic_fetch_max((LAS unsigned*)(lds + L_CTL), seq + 1, __ATOMIC_RELAXED, __HIP_MEMORY_SCOPE_WORKGROUP);
                seq = AG_TURN == 2 ? 0 : seq + SW;
                TRACE(a.rank, b0, 1);
#pragma unroll
                for (int s = 0; s < 4; ++s) d[s] = nx[s];
                pack(d, pk, raw, annex);
                b0 = b1;
                ++nsent;
                b1 = __builtin_amdgcn_readfirstlane(tk) - tbase;
            }
            ctl[0] = ~0u;
            if (wave == 0) STAMP(7);
        }
    }
}
}

extern "C" void kernel_launch(int rank, int nranks, ncclComm_t comm,
                              hipStream_t stream,
                              void* const* d_in, const int* in_sizes, int n_in,
                              void* d_out, int out_size,
                              const int* in_dtypes, int out_dtype,
                              void* const* d_out_all,
                              void* const* d_ws_all, size_t ws_bytes) {
    if (nranks != ag::NR || (unsigned)rank >= (unsigned)ag::NR || n_in != 2 || ws_bytes < ag::W_END ||
        in_sizes[0] != ag::MS * ag::K || in_sizes[1] != ag::K * ag::NL || out_size != ag::M * ag::NL) {
        fprintf(stderr, "ag_gemm: unexpected problem shape\n");
        return;
    }
    ag::Args a;
    a.x = (const char*)d_in[0];
    a.w = (const char*)d_in[1];
    a.out = (char*)d_out;
    for (int i = 0; i < ag::NR; ++i) a.ws[i] = (char*)d_ws_all[(rank + 1 + i) % ag::NR];
    a.rank = rank;
    a.pad = 0;
    hipLaunchKernelGGL(ag::ag_gemm, dim3(ag::NWG), dim3(ag::NTHR), 0, stream, a);
    HIP_CHECK(hipGetLastError());
}
```

```cpp
#include <hip/hip_runtime.h>
#include <rccl/rccl.h>
#include <cstdio>
#include <cstdint>
#include <type_traits>

#define HIP_CHECK(expr) do { hipError_t _e = (expr); if (_e != hipSuccess) { \
    fprintf(stderr, "HIP error %s at %s:%d\n", hipGetErrorString(_e), __FILE__, __LINE__); \
    return; } } while (0)

#define LAS __attribute__((address_space(3)))
#define GAS __attribute__((address_space(1)))
typedef unsigned u32x4 __attribute__((ext_vector_type(4)));
typedef unsigned u32x2 __attribute__((ext_vector_type(2)));
typedef unsigned short u16x2 __attribute__((ext_vector_type(2)));
typedef short s16x4 __attribute__((ext_vector_type(4)));
typedef short s16x8 __attribute__((ext_vector_type(8)));
typedef __bf16 bf16x4 __attribute__((ext_vector_type(4)));
typedef __bf16 bf16x8 __attribute__((ext_vector_type(8)));
typedef float f32x4 __attribute__((ext_vector_type(4)));
typedef float f32x16 __attribute__((ext_vector_type(16)));
typedef float f32x32 __attribute__((ext_vector_type(32)));

#define AG_SW 4
#define AG_PSLEEP 2
#define AG_TURN 1
#define AG_UPRIO 0
#define AG_PF 2
#define AG_PF0 3
#define AG_PFAUX 0
#define AG_PFD 0
#define AG_CPRIO 0
#define AG_WSLEEP 2
#define AG_WPRIO 0
#define AG_T1 1
#define AG_A32 1
#define AG_PQ 0
#define AG_PPRIO 2
#define AG_DRY 1
#define AG_SPEC 1
#define AG_CPF 1
#define AG_PD 0
#define AG_UHI 0
#define AG_BAL 0
#define AG_APW 1
#define AG_STRIP 1
#define AG_HOLE 0
#define AG_EARLY 3
#define AG_SD 0
#define AG_PP 0
#define AG_MAP 1
#define AG_FAUX 17
#define AG_USLEEP 1
#define AG_HAUX 17
#define AG_UAUX 17
#define AG_FS 32
#define AG_PW 2
#define AG_OAUX 17

namespace ag {
constexpr int NR = 8;
constexpr int M = 4096, K = 4096, NL = 2048;
constexpr int MS = M / NR;
constexpr int NB = 9;
constexpr int BM = 256, BN = 128, BK = 64;
constexpr int NKT = K / BK;
constexpr int NG = MS / 32;
constexpr int BLK = 32 * BK * 2;
constexpr int NBLK = NG * NKT;
constexpr int RS = NB * 32;
constexpr int SLOT = BK * RS, NSLOT = 8, NPIECE = SLOT / 1024;
constexpr int NTN = NL / BN - 1;
constexpr int NWG = 256, CPX = 4;
constexpr int NWAVE = 8, NTHR = NWAVE * 64;
constexpr int NSC = NWG - 16 * NTN, SW = AG_SW, NSEND = NSC * SW;
static_assert(2 * NSEND <= NBLK && SW <= NWAVE, "sender geometry");
constexpr int PBLK = 3072;
constexpr unsigned HW = 6, NPATCH = 32 - HW, NTAIL = 64 - HW, NLONG = NTAIL + 32;
constexpr int NU = 4 * NTN;
constexpr int L_SEL = 0, L_W = 2048;
constexpr int L_CTL = L_W + NSLOT * SLOT;
constexpr unsigned ALL = 1000;
constexpr int L_UNP = L_CTL + 128, UNP = 2048 + 64 + 448;
constexpr int L_SINK = L_UNP + 2 * UNP, LDS_BYTES = L_SINK + 256;
static_assert(LDS_BYTES > 80 * 1024 && LDS_BYTES <= 160 * 1024, "one workgroup per CU");
constexpr size_t W_CNT = 0;
constexpr size_t W_TMO = W_CNT + (size_t)NWG * 128;
constexpr size_t W_TICKET = W_TMO + 1024;
constexpr size_t W_SCNT = W_TICKET + 1024;
constexpr size_t W_DBG = 768 * 1024;
constexpr size_t W_XT = 1u << 20;
constexpr size_t XT_BYTES = (size_t)M * K * 2;
constexpr size_t W_PK = W_XT + 2 * XT_BYTES;
constexpr size_t PK_BYTES = (size_t)NR * NBLK * PBLK;
constexpr size_t W_FLAG = W_PK + 2 * PK_BYTES;
constexpr int FS = AG_FS;
constexpr unsigned FL_BYTES = NR * NBLK * FS * 4;
constexpr size_t W_STRIP = W_FLAG + FL_BYTES;
constexpr size_t W_END = W_STRIP + (size_t)K * BN * 2;
constexpr int NSTRIP = (NKT - NSLOT) * 16;
static_assert(W_SCNT + 1024 <= W_DBG && NSTRIP <= 16 * NTN * 4, "flags");
constexpr unsigned OOB = 0x40000000u;
constexpr unsigned long long DEADLINE = 30000000ull;

struct Args {
    const char* x;
    const char* w;
    char* out;
    char* ws[NR];
    int rank;
    int pad;
};

__device__ __forceinline__ unsigned long long rt() { return __builtin_amdgcn_s_memrealtime(); }
__device__ __forceinline__ __amdgpu_buffer_rsrc_t rsrc(const void* p, unsigned bytes) {
    return __builtin_amdgcn_make_buffer_rsrc(const_cast<void*>(p), 0, bytes, 0x00020000);
}
__device__ __forceinline__ void store16(u32x4 d, __amdgpu_buffer_rsrc_t r, unsigned off) {
    __builtin_amdgcn_raw_buffer_store_b128(d, r, off, 0, 17);
}
__device__ __forceinline__ void wait_vm0() {
    __builtin_amdgcn_s_waitcnt(0x0F70);
    asm volatile("s_waitcnt vmcnt(0)" ::: "memory");
}
__device__ __forceinline__ bool give_up(GAS unsigned* tmo, unsigned call, unsigned long long t0) {
    const unsigned long long dt = rt() - t0;
    if (dt < 50000ull) return false;
    return dt > DEADLINE || __builtin_amdgcn_readfirstlane(__hip_atomic_load(tmo, __ATOMIC_RELAXED, __HIP_MEMORY_SCOPE_AGENT)) == call;
}
constexpr unsigned B1 = 0x01010101u;
__device__ __forceinline__ unsigned tag_of(unsigned call) { return ((call + 1u) << 30) & 0x80000000u; }
__device__ __forceinline__ unsigned bfi(unsigned m, unsigned a, unsigned b) { return (a & m) | (b & ~m); }
__device__ __forceinline__ unsigned perm(unsigned hi, unsigned lo, unsigned sel) { return __builtin_amdgcn_perm(hi, lo, sel); }
__device__ __forceinline__ unsigned key2(unsigned e0) { return (((e0 + 3u) & 3u) ^ 3u) * B1; }
__device__ __forceinline__ unsigned table2(unsigned e0) {
    return e0 * B1 + (((0x03020100u ^ key2(e0)) + (4u - (e0 & 3u)) * B1) & 3u * B1);
}
__device__ __forceinline__ unsigned table3(unsigned e0, unsigned hi) {
    const unsigned r = (e0 & 7u) * B1;
    return e0 * B1 - ((8u * B1 + r - ((0x03020100u + hi * 4u * B1) ^ r)) & 7u * B1) + (hi ? 0u : 3u);
}
__device__ __forceinline__ unsigned scan(unsigned v) {
    v += (unsigned)__builtin_amdgcn_update_dpp(0, (int)v, 0x111, 0xf, 0xf, true);
    v += (unsigned)__builtin_amdgcn_update_dpp(0, (int)v, 0x112, 0xf, 0xf, true);
    v += (unsigned)__builtin_amdgcn_update_dpp(0, (int)v, 0x114, 0xf, 0xf, true);
    v += (unsigned)__builtin_amdgcn_update_dpp(0, (int)v, 0x118, 0xf, 0xf, true);
    v += (unsigned)__builtin_amdgcn_update_dpp(0, (int)v, 0x142, 0xa, 0xf, false);
    v += (unsigned)__builtin_amdgcn_update_dpp(0, (int)v, 0x143, 0xc, 0xf, false);
    return v;
}
__device__ __forceinline__ u32x2 mover(unsigned m, bool gather) {
    unsigned sel = gather ? 12u * B1 : 0u, n = 0;
#pragma unroll
    for (unsigned b = 0; b < 4; ++b) {
        const unsigned c = gather ? ((m >> b) & 1u) * 3u : (m >> (2u * b)) & 3u;
        if (gather) sel ^= c ? (12u ^ b) << (8u * n) : 0u;
        else sel |= (c == 3u ? n : 4u + c) << (8u * b);
        n += c == 3u;
    }
    return (u32x2){sel, 3u * n};
}
#define STAMP(i) do {} while (0)
#define STAMPC(i) do {} while (0)
#undef TRACE
#define TRACE(s, b, i) do {} while (0)

#define SB() __builtin_amdgcn_sched_barrier(0)
#define FENCE() asm volatile("" ::: "memory")
#define AG_T0 ""
#define AG_KLOOP \
    "s_cmp_eq_u32 %[dry], 0\n" \
    "s_cbranch_scc1 GO%=\n" \
    "v_mov_b32 v244, -2\n" \
    "s_mov_b64 exec, 0\n" \
    "s_mov_b32 %[nd], 64\n" \
    "s_mov_b32 %[ps], 0\n" \
    "s_branch R1%=\n" \
    "GO%=:\n" \
    "v_mov_b64 v[0:1], 0\n" \
    "v_mov_b64 v[2:3], 0\n" \
    "v_mov_b64 v[4:5], 0\n" \
    "v_mov_b64 v[6:7], 0\n" \
    "v_mov_b64 v[8:9], 0\n" \
    "v_mov_b64 v[10:11], 0\n" \
    "v_mov_b64 v[12:13], 0\n" \
    "v_mov_b64 v[14:15], 0\n" \
    "v_mov_b64 v[16:17], 0\n" \
    "v_mov_b64 v[18:19], 0\n" \
    "v_mov_b64 v[20:21], 0\n" \
    "v_mov_b64 v[22:23], 0\n" \
    "v_mov_b64 v[24:25], 0\n" \
    "v_mov_b64 v[26:27], 0\n" \
    "v_mov_b64 v[28:29], 0\n" \
    "v_mov_b64 v[30:31], 0\n" \
    "v_mov_b64 v[32:33], 0\n" \
    "v_mov_b64 v[34:35], 0\n" \
    "v_mov_b64 v[36:37], 0\n" \
    "v_mov_b64 v[38:39], 0\n" \
    "v_mov_b64 v[40:41], 0\n" \
    "v_mov_b64 v[42:43], 0\n" \
    "v_mov_b64 v[44:45], 0\n" \
    "v_mov_b64 v[46:47], 0\n" \
    "v_mov_b64 v[48:49], 0\n" \
    "v_mov_b64 v[50:51], 0\n" \
    "v_mov_b64 v[52:53], 0\n" \
    "v_mov_b64 v[54:55], 0\n" \
    "v_mov_b64 v[56:57], 0\n" \
    "v_mov_b64 v[58:59], 0\n" \
    "v_mov_b64 v[60:61], 0\n" \
    "v_mov_b64 v[62:63], 0\n" \
    "v_mov_b64 v[64:65], 0\n" \
    "v_mov_b64 v[66:67], 0\n" \
    "v_mov_b64 v[68:69], 0\n" \
    "v_mov_b64 v[70:71], 0\n" \
    "v_mov_b64 v[72:73], 0\n" \
    "v_mov_b64 v[74:75], 0\n" \
    "v_mov_b64 v[76:77], 0\n" \
    "v_mov_b64 v[78:79], 0\n" \
    "v_mov_b64 v[80:81], 0\n" \
    "v_mov_b64 v[82:83], 0\n" \
    "v_mov_b64 v[84:85], 0\n" \
    "v_mov_b64 v[86:87], 0\n" \
    "v_mov_b64 v[88:89], 0\n" \
    "v_mov_b64 v[90:91], 0\n" \
    "v_mov_b64 v[92:93], 0\n" \
    "v_mov_b64 v[94:95], 0\n" \
    "v_mov_b64 v[96:97], 0\n" \
    "v_mov_b64 v[98:99], 0\n" \
    "v_mov_b64 v[100:101], 0\n" \
    "v_mov_b64 v[102:103], 0\n" \
    "v_mov_b64 v[104:105], 0\n" \
    "v_mov_b64 v[106:107], 0\n" \
    "v_mov_b64 v[108:109], 0\n" \
    "v_mov_b64 v[110:111], 0\n" \
    "v_mov_b64 v[112:113], 0\n" \
    "v_mov_b64 v[114:115], 0\n" \
    "v_mov_b64 v[116:117], 0\n" \
    "v_mov_b64 v[118:119], 0\n" \
    "v_mov_b64 v[120:121], 0\n" \
    "v_mov_b64 v[122:123], 0\n" \
    "v_mov_b64 v[124:125], 0\n" \
    "v_mov_b64 v[126:127], 0\n" \
    "v_mov_b64 v[128:129], 0\n" \
    "v_mov_b64 v[130:131], 0\n" \
    "v_mov_b64 v[132:133], 0\n" \
    "v_mov_b64 v[134:135], 0\n" \
    "v_mov_b64 v[136:137], 0\n" \
    "v_mov_b64 v[138:139], 0\n" \
    "v_mov_b64 v[140:141], 0\n" \
    "v_mov_b64 v[142:143], 0\n" \
    "v_mov_b32 v240, v241\n" \
    "v_xor_b32 v243, 0x1000, v242\n" \
    "v_mov_b32 v246, 0\n" \
    "s_mov_b32 %[nd], 1\n" \
    "s_mov_b32 %[ps], 0\n" \
    "P0%=:\n" \
    "ds_read2_b32 v[244:245], v247 offset1:4\n" \
    "s_waitcnt lgkmcnt(0)\n" \
    "v_min_u32 v244, v244, v245\n" \
    "s_nop 0\n" \
    "v_readfirstlane_b32 %[hv], v244\n" \
    "s_cmp_ge_u32 %[hv], %[nd]\n" \
    "s_cbranch_scc1 R0%=\n" \
    "s_sleep 1\n" \
    "s_branch P0%=\n" \
    "R0%=:\n" \
    "buffer_load_dwordx4 v[144:147], v242, %[rs], 0 offen offset:0 sc0 sc1\n" \
    "buffer_load_dwordx4 v[148:151], v242, %[rs], 0 offen offset:1024 sc0 sc1\n" \
    "buffer_load_dwordx4 v[152:155], v242, %[rs], 0 offen offset:2048 sc0 sc1\n" \
    "buffer_load_dwordx4 v[156:159], v242, %[rs], 0 offen offset:3072 sc0 sc1\n" \
    "buffer_load_dwordx4 v[160:163], v243, %[rs], 0 offen offset:0 sc0 sc1\n" \
    "buffer_load_dwordx4 v[164:167], v243, %[rs], 0 offen offset:1024 sc0 sc1\n" \
    "buffer_load_dwordx4 v[168:171], v243, %[rs], 0 offen offset:2048 sc0 sc1\n" \
    "buffer_load_dwordx4 v[172:175], v243, %[rs], 0 offen offset:3072 sc0 sc1\n" \
    "v_add_u32 v242, 0x10000, v242\n" \
    "v_add_u32 v243, 0x10000, v243\n" \
    "s_mov_b32 %[nd], 2\n" \
    "P1%=:\n" \
    "ds_read2_b32 v[244:245], v247 offset1:4\n" \
    "s_waitcnt lgkmcnt(0)\n" \
    "v_min_u32 v244, v244, v245\n" \
    "s_nop 0\n" \
    "v_readfirstlane_b32 %[hv], v244\n" \
    "s_cmp_ge_u32 %[hv], %[nd]\n" \
    "s_cbranch_scc1 R1%=\n" \
    "s_sleep 1\n" \
    "s_branch P1%=\n" \
    "R1%=:\n" \
    AG_T0 \
    "s_cmp_eq_u32 %[n8], 0\n" \
    "s_cbranch_scc0 E8%=\n" \
    "ds_read_b64_tr_b16 v[208:209], v240 offset:0\n" \
    "ds_read_b64_tr_b16 v[210:211], v240 offset:4608\n" \
    "ds_read_b64_tr_b16 v[212:213], v240 offset:9216\n" \
    "ds_read_b64_tr_b16 v[214:215], v240 offset:13824\n" \
    "ds_read_b64_tr_b16 v[216:217], v240 offset:32\n" \
    "ds_read_b64_tr_b16 v[218:219], v240 offset:4640\n" \
    "ds_read_b64_tr_b16 v[220:221], v240 offset:9248\n" \
    "ds_read_b64_tr_b16 v[222:223], v240 offset:13856\n" \
    "s_waitcnt lgkmcnt(4) vmcnt(0)\n" \
    "L9_%=:\n" \
    "v_mfma_f32_16x16x32_bf16 v[0:3], v[208:211], v[144:147], v[0:3]\n" \
    "ds_read_b64_tr_b16 v[224:225], v240 offset:64\n" \
    "v_mfma_f32_16x16x32_bf16 v[0:3], v[212:215], v[148:151], v[0:3]\n" \
    "ds_read_b64_tr_b16 v[226:227], v240 offset:4672\n" \
    "v_mfma_f32_16x16x32_bf16 v[36:39], v[212:215], v[156:159], v[36:39]\n" \
    "ds_read_b64_tr_b16 v[228:229], v240 offset:9280\n" \
    "v_mfma_f32_16x16x32_bf16 v[36:39], v[208:211], v[152:155], v[36:39]\n" \
    "ds_read_b64_tr_b16 v[230:231], v240 offset:13888\n" \
    "v_mfma_f32_16x16x32_bf16 v[72:75], v[208:211], v[160:163], v[72:75]\n" \
    "buffer_load_dwordx4 v[176:179], v242, %[rs], 0 offen offset:0 sc0 sc1\n" \
    "v_mfma_f32_16x16x32_bf16 v[72:75], v[212:215], v[164:167], v[72:75]\n" \
    "buffer_load_dwordx4 v[180:183], v242, %[rs], 0 offen offset:1024 sc0 sc1\n" \
    "v_mfma_f32_16x16x32_bf16 v[108:111], v[212:215], v[172:175], v[108:111]\n" \
    "buffer_load_dwordx4 v[184:187], v242, %[rs], 0 offen offset:2048 sc0 sc1\n" \
    "v_mfma_f32_16x16x32_bf16 v[108:111], v[208:211], v[168:171], v[108:111]\n" \
    "s_waitcnt lgkmcnt(4)\n" \
    "v_mfma_f32_16x16x32_bf16 v[112:115], v[216:219], v[168:171], v[112:115]\n" \
    "ds_read_b64_tr_b16 v[208:209], v240 offset:96\n" \
    "s_add_u32 %[nd], %[nd], 1\n" \
    "v_mfma_f32_16x16x32_bf16 v[112:115], v[220:223], v[172:175], v[112:115]\n" \
    "ds_read_b64_tr_b16 v[210:211], v240 offset:4704\n" \
    "s_add_u32 %[ps], %[ps], 18432\n" \
    "v_mfma_f32_16x16x32_bf16 v[76:79], v[220:223], v[164:167], v[76:79]\n" \
    "ds_read_b64_tr_b16 v[212:213], v240 offset:9312\n" \
    "s_cmp_eq_u32 %[ps], 147456\n" \
    "v_mfma_f32_16x16x32_bf16 v[76:79], v[216:219], v[160:163], v[76:79]\n" \
    "ds_read_b64_tr_b16 v[214:215], v240 offset:13920\n" \
    "s_cselect_b32 %[ps], 0, %[ps]\n" \
    "v_mfma_f32_16x16x32_bf16 v[40:43], v[216:219], v[152:155], v[40:43]\n" \
    "buffer_load_dwordx4 v[188:191], v242, %[rs], 0 offen offset:3072 sc0 sc1\n" \
    "v_mfma_f32_16x16x32_bf16 v[40:43], v[220:223], v[156:159], v[40:43]\n" \
    "buffer_load_dwordx4 v[192:195], v243, %[rs], 0 offen offset:0 sc0 sc1\n" \
    "v_mfma_f32_16x16x32_bf16 v[4:7], v[220:223], v[148:151], v[4:7]\n" \
    "buffer_load_dwordx4 v[196:199], v243, %[rs], 0 offen offset:1024 sc0 sc1\n" \
    "v_mfma_f32_16x16x32_bf16 v[4:7], v[216:219], v[144:147], v[4:7]\n" \
    "s_waitcnt lgkmcnt(4)\n" \
    "v_mfma_f32_16x16x32_bf16 v[8:11], v[224:227], v[144:147], v[8:11]\n" \
    "ds_read_b64_tr_b16 v[216:217], v240 offset:128\n" \
    "v_mfma_f32_16x16x32_bf16 v[8:11], v[228:231], v[148:151], v[8:11]\n" \
    "ds_read_b64_tr_b16 v[218:219], v240 offset:4736\n" \
    "v_mfma_f32_16x16x32_bf16 v[44:47], v[228:231], v[156:159], v[44:47]\n" \
    "ds_read_b64_tr_b16 v[220:221], v240 offset:9344\n" \
    "v_mfma_f32_16x16x32_bf16 v[44:47], v[224:227], v[152:155], v[44:47]\n" \
    "ds_read_b64_tr_b16 v[222:223], v240 offset:13952\n" \
    "v_mfma_f32_16x16x32_bf16 v[80:83], v[224:227], v[160:163], v[80:83]\n" \
    "buffer_load_dwordx4 v[200:203], v243, %[rs], 0 offen offset:2048 sc0 sc1\n" \
    "v_mfma_f32_16x16x32_bf16 v[80:83], v[228:231], v[164:167], v[80:83]\n" \
    "buffer_load_dwordx4 v[204:207], v243, %[rs], 0 offen offset:3072 sc0 sc1\n" \
    "v_mfma_f32_16x16x32_bf16 v[116:119], v[228:231], v[172:175], v[116:119]\n" \
    "v_add_u32 v242, 0x10000, v242\n" \
    "v_mfma_f32_16x16x32_bf16 v[116:119], v[224:227], v[168:171], v[116:119]\n" \
    "s_waitcnt lgkmcnt(4)\n" \
    "v_mfma_f32_16x16x32_bf16 v[120:123], v[208:211], v[168:171], v[120:123]\n" \
    "ds_read_b64_tr_b16 v[224:225], v240 offset:160\n" \
    "v_mfma_f32_16x16x32_bf16 v[120:123], v[212:215], v[172:175], v[120:123]\n" \
    "ds_read_b64_tr_b16 v[226:227], v240 offset:4768\n" \
    "v_mfma_f32_16x16x32_bf16 v[84:87], v[212:215], v[164:167], v[84:87]\n" \
    "ds_read_b64_tr_b16 v[228:229], v240 offset:9376\n" \
    "v_mfma_f32_16x16x32_bf16 v[84:87], v[208:211], v[160:163], v[84:87]\n" \
    "ds_read_b64_tr_b16 v[230:231], v240 offset:13984\n" \
    "v_mfma_f32_16x16x32_bf16 v[48:51], v[208:211], v[152:155], v[48:51]\n" \
    "v_add_u32 v243, 0x10000, v243\n" \
    "v_mfma_f32_16x16x32_bf16 v[48:51], v[212:215], v[156:159], v[48:51]\n" \
    "v_add_u32 v246, 1, v246\n" \
    "v_mfma_f32_16x16x32_bf16 v[12:15], v[212:215], v[148:151], v[12:15]\n" \
    "v_mfma_f32_16x16x32_bf16 v[12:15], v[208:211], v[144:147], v[12:15]\n" \
    "s_waitcnt lgkmcnt(4)\n" \
    "v_mfma_f32_16x16x32_bf16 v[16:19], v[216:219], v[144:147], v[16:19]\n" \
    "ds_read_b64_tr_b16 v[208:209], v240 offset:192\n" \
    "v_mfma_f32_16x16x32_bf16 v[16:19], v[220:223], v[148:151], v[16:19]\n" \
    "ds_read_b64_tr_b16 v[210:211], v240 offset:4800\n" \
    "v_mfma_f32_16x16x32_bf16 v[52:55], v[220:223], v[156:159], v[52:55]\n" \
    "ds_read_b64_tr_b16 v[212:213], v240 offset:9408\n" \
    "v_mfma_f32_16x16x32_bf16 v[52:55], v[216:219], v[152:155], v[52:55]\n" \
    "ds_read_b64_tr_b16 v[214:215], v240 offset:14016\n" \
    "v_mfma_f32_16x16x32_bf16 v[88:91], v[216:219], v[160:163], v[88:91]\n" \
    "v_mfma_f32_16x16x32_bf16 v[88:91], v[220:223], v[164:167], v[88:91]\n" \
    "v_mfma_f32_16x16x32_bf16 v[124:127], v[220:223], v[172:175], v[124:127]\n" \
    "v_mfma_f32_16x16x32_bf16 v[124:127], v[216:219], v[168:171], v[124:127]\n" \
    "s_waitcnt lgkmcnt(4)\n" \
    "v_mfma_f32_16x16x32_bf16 v[128:131], v[224:227], v[168:171], v[128:131]\n" \
    "ds_read_b64_tr_b16 v[216:217], v240 offset:224\n" \
    "v_mfma_f32_16x16x32_bf16 v[128:131], v[228:231], v[172:175], v[128:131]\n" \
    "ds_read_b64_tr_b16 v[218:219], v240 offset:4832\n" \
    "v_mfma_f32_16x16x32_bf16 v[92:95], v[228:231], v[164:167], v[92:95]\n" \
    "ds_read_b64_tr_b16 v[220:221], v240 offset:9440\n" \
    "v_mfma_f32_16x16x32_bf16 v[92:95], v[224:227], v[160:163], v[92:95]\n" \
    "ds_read_b64_tr_b16 v[222:223], v240 offset:14048\n" \
    "v_mfma_f32_16x16x32_bf16 v[56:59], v[224:227], v[152:155], v[56:59]\n" \
    "v_mfma_f32_16x16x32_bf16 v[56:59], v[228:231], v[156:159], v[56:59]\n" \
    "v_mfma_f32_16x16x32_bf16 v[20:23], v[228:231], v[148:151], v[20:23]\n" \
    "v_mfma_f32_16x16x32_bf16 v[20:23], v[224:227], v[144:147], v[20:23]\n" \
    "s_waitcnt lgkmcnt(4)\n" \
    "v_mfma_f32_16x16x32_bf16 v[24:27], v[208:211], v[144:147], v[24:27]\n" \
    "ds_read_b64_tr_b16 v[224:225], v240 offset:256\n" \
    "v_mfma_f32_16x16x32_bf16 v[24:27], v[212:215], v[148:151], v[24:27]\n" \
    "ds_read_b64_tr_b16 v[226:227], v240 offset:4864\n" \
    "v_mfma_f32_16x16x32_bf16 v[60:63], v[212:215], v[156:159], v[60:63]\n" \
    "ds_read_b64_tr_b16 v[228:229], v240 offset:9472\n" \
    "v_mfma_f32_16x16x32_bf16 v[60:63], v[208:211], v[152:155], v[60:63]\n" \
    "ds_read_b64_tr_b16 v[230:231], v240 offset:14080\n" \
    "v_mfma_f32_16x16x32_bf16 v[96:99], v[208:211], v[160:163], v[96:99]\n" \
    "v_add_u32 v240, %[ps], v241\n" \
    "v_mfma_f32_16x16x32_bf16 v[96:99], v[212:215], v[164:167], v[96:99]\n" \
    "ds_write_b32 v247, v246 offset:32\n" \
    "v_mfma_f32_16x16x32_bf16 v[132:135], v[212:215], v[172:175], v[132:135]\n" \
    "v_mfma_f32_16x16x32_bf16 v[132:135], v[208:211], v[168:171], v[132:135]\n" \
    "s_waitcnt lgkmcnt(5)\n" \
    "v_mfma_f32_16x16x32_bf16 v[136:139], v[216:219], v[168:171], v[136:139]\n" \
    "ds_read2_b32 v[244:245], v247 offset1:4\n" \
    "ds_read_b64_tr_b16 v[208:209], v240 offset:0\n" \
    "v_mfma_f32_16x16x32_bf16 v[136:139], v[220:223], v[172:175], v[136:139]\n" \
    "ds_read_b64_tr_b16 v[210:211], v240 offset:4608\n" \
    "v_mfma_f32_16x16x32_bf16 v[100:103], v[220:223], v[164:167], v[100:103]\n" \
    "ds_read_b64_tr_b16 v[212:213], v240 offset:9216\n" \
    "v_mfma_f32_16x16x32_bf16 v[100:103], v[216:219], v[160:163], v[100:103]\n" \
    "ds_read_b64_tr_b16 v[214:215], v240 offset:13824\n" \
    "v_mfma_f32_16x16x32_bf16 v[64:67], v[216:219], v[152:155], v[64:67]\n" \
    "v_mfma_f32_16x16x32_bf16 v[64:67], v[220:223], v[156:159], v[64:67]\n" \
    "v_mfma_f32_16x16x32_bf16 v[28:31], v[220:223], v[148:151], v[28:31]\n" \
    "v_mfma_f32_16x16x32_bf16 v[28:31], v[216:219], v[144:147], v[28:31]\n" \
    "s_waitcnt lgkmcnt(4)\n" \
    "v_mfma_f32_16x16x32_bf16 v[32:35], v[224:227], v[144:147], v[32:35]\n" \
    "ds_read_b64_tr_b16 v[216:217], v240 offset:32\n" \
    "v_mfma_f32_16x16x32_bf16 v[32:35], v[228:231], v[148:151], v[32:35]\n" \
    "ds_read_b64_tr_b16 v[218:219], v240 offset:4640\n" \
    "v_mfma_f32_16x16x32_bf16 v[68:71], v[228:231], v[156:159], v[68:71]\n" \
    "ds_read_b64_tr_b16 v[220:221], v240 offset:9248\n" \
    "v_mfma_f32_16x16x32_bf16 v[68:71], v[224:227], v[152:155], v[68:71]\n" \
    "ds_read_b64_tr_b16 v[222:223], v240 offset:13856\n" \
    "v_mfma_f32_16x16x32_bf16 v[104:107], v[224:227], v[160:163], v[104:107]\n" \
    "v_min_u32 v244, v244, v245\n" \
    "v_mfma_f32_16x16x32_bf16 v[104:107], v[228:231], v[164:167], v[104:107]\n" \
    "v_readfirstlane_b32 %[hv], v244\n" \
    "v_mfma_f32_16x16x32_bf16 v[140:143], v[228:231], v[172:175], v[140:143]\n" \
    "s_cmp_ge_u32 %[hv], %[nd]\n" \
    "v_mfma_f32_16x16x32_bf16 v[140:143], v[224:227], v[168:171], v[140:143]\n" \
    "s_waitcnt lgkmcnt(4) vmcnt(0)\n" \
    "s_cbranch_scc0 P9_%=\n" \
    "R9_%=:\n" \
    "v_mfma_f32_16x16x32_bf16 v[0:3], v[208:211], v[176:179], v[0:3]\n" \
    "ds_read_b64_tr_b16 v[224:225], v240 offset:64\n" \
    "v_mfma_f32_16x16x32_bf16 v[0:3], v[212:215], v[180:183], v[0:3]\n" \
    "ds_read_b64_tr_b16 v[226:227], v240 offset:4672\n" \
    "v_mfma_f32_16x16x32_bf16 v[36:39], v[212:215], v[188:191], v[36:39]\n" \
    "ds_read_b64_tr_b16 v[228:229], v240 offset:9280\n" \
    "v_mfma_f32_16x16x32_bf16 v[36:39], v[208:211], v[184:187], v[36:39]\n" \
    "ds_read_b64_tr_b16 v[230:231], v240 offset:13888\n" \
    "v_mfma_f32_16x16x32_bf16 v[72:75], v[208:211], v[192:195], v[72:75]\n" \
    "buffer_load_dwordx4 v[144:147], v242, %[rs], 0 offen offset:0 sc0 sc1\n" \
    "v_mfma_f32_16x16x32_bf16 v[72:75], v[212:215], v[196:199], v[72:75]\n" \
    "buffer_load_dwordx4 v[148:151], v242, %[rs], 0 offen offset:1024 sc0 sc1\n" \
    "v_mfma_f32_16x16x32_bf16 v[108:111], v[212:215], v[204:207], v[108:111]\n" \
    "buffer_load_dwordx4 v[152:155], v242, %[rs], 0 offen offset:2048 sc0 sc1\n" \
    "v_mfma_f32_16x16x32_bf16 v[108:111], v[208:211], v[200:203], v[108:111]\n" \
    "s_waitcnt lgkmcnt(4)\n" \
    "v_mfma_f32_16x16x32_bf16 v[112:115], v[216:219], v[200:203], v[112:115]\n" \
    "ds_read_b64_tr_b16 v[208:209], v240 offset:96\n" \
    "s_add_u32 %[nd], %[nd], 1\n" \
    "v_mfma_f32_16x16x32_bf16 v[112:115], v[220:223], v[204:207], v[112:115]\n" \
    "ds_read_b64_tr_b16 v[210:211], v240 offset:4704\n" \
    "s_add_u32 %[ps], %[ps], 18432\n" \
    "v_mfma_f32_16x16x32_bf16 v[76:79], v[220:223], v[196:199], v[76:79]\n" \
    "ds_read_b64_tr_b16 v[212:213], v240 offset:9312\n" \
    "s_cmp_eq_u32 %[ps], 147456\n" \
    "v_mfma_f32_16x16x32_bf16 v[76:79], v[216:219], v[192:195], v[76:79]\n" \
    "ds_read_b64_tr_b16 v[214:215], v240 offset:13920\n" \
    "s_cselect_b32 %[ps], 0, %[ps]\n" \
    "v_mfma_f32_16x16x32_bf16 v[40:43], v[216:219], v[184:187], v[40:43]\n" \
    "buffer_load_dwordx4 v[156:159], v242, %[rs], 0 offen offset:3072 sc0 sc1\n" \
    "v_mfma_f32_16x16x32_bf16 v[40:43], v[220:223], v[188:191], v[40:43]\n" \
    "buffer_load_dwordx4 v[160:163], v243, %[rs], 0 offen offset:0 sc0 sc1\n" \
    "v_mfma_f32_16x16x32_bf16 v[4:7], v[220:223], v[180:183], v[4:7]\n" \
    "buffer_load_dwordx4 v[164:167], v243, %[rs], 0 offen offset:1024 sc0 sc1\n" \
    "v_mfma_f32_16x16x32_bf16 v[4:7], v[216:219], v[176:179], v[4:7]\n" \
    "s_waitcnt lgkmcnt(4)\n" \
    "v_mfma_f32_16x16x32_bf16 v[8:11], v[224:227], v[176:179], v[8:11]\n" \
    "ds_read_b64_tr_b16 v[216:217], v240 offset:128\n" \
    "s_cmp_eq_u32 %[nd], 66\n" \
    "v_mfma_f32_16x16x32_bf16 v[8:11], v[228:231], v[180:183], v[8:11]\n" \
    "ds_read_b64_tr_b16 v[218:219], v240 offset:4736\n" \
    "s_cselect_b32 %[ck], -1, %[nd]\n" \
    "v_mfma_f32_16x16x32_bf16 v[44:47], v[228:231], v[188:191], v[44:47]\n" \
    "ds_read_b64_tr_b16 v[220:221], v240 offset:9344\n" \
    "v_mfma_f32_16x16x32_bf16 v[44:47], v[224:227], v[184:187], v[44:47]\n" \
    "ds_read_b64_tr_b16 v[222:223], v240 offset:13952\n" \
    "v_mfma_f32_16x16x32_bf16 v[80:83], v[224:227], v[192:195], v[80:83]\n" \
    "buffer_load_dwordx4 v[168:171], v243, %[rs], 0 offen offset:2048 sc0 sc1\n" \
    "v_mfma_f32_16x16x32_bf16 v[80:83], v[228:231], v[196:199], v[80:83]\n" \
    "buffer_load_dwordx4 v[172:175], v243, %[rs], 0 offen offset:3072 sc0 sc1\n" \
    "v_mfma_f32_16x16x32_bf16 v[116:119], v[228:231], v[204:207], v[116:119]\n" \
    "v_add_u32 v242, 0x10000, v242\n" \
    "v_mfma_f32_16x16x32_bf16 v[116:119], v[224:227], v[200:203], v[116:119]\n" \
    "s_waitcnt lgkmcnt(4)\n" \
    "v_mfma_f32_16x16x32_bf16 v[120:123], v[208:211], v[200:203], v[120:123]\n" \
    "ds_read_b64_tr_b16 v[224:225], v240 offset:160\n" \
    "v_mfma_f32_16x16x32_bf16 v[120:123], v[212:215], v[204:207], v[120:123]\n" \
    "ds_read_b64_tr_b16 v[226:227], v240 offset:4768\n" \
    "v_mfma_f32_16x16x32_bf16 v[84:87], v[212:215], v[196:199], v[84:87]\n" \
    "ds_read_b64_tr_b16 v[228:229], v240 offset:9376\n" \
    "v_mfma_f32_16x16x32_bf16 v[84:87], v[208:211], v[192:195], v[84:87]\n" \
    "ds_read_b64_tr_b16 v[230:231], v240 offset:13984\n" \
    "v_mfma_f32_16x16x32_bf16 v[48:51], v[208:211], v[184:187], v[48:51]\n" \
    "v_add_u32 v243, 0x10000, v243\n" \
    "v_mfma_f32_16x16x32_bf16 v[48:51], v[212:215], v[188:191], v[48:51]\n" \
    "v_add_u32 v246, 1, v246\n" \
    "v_mfma_f32_16x16x32_bf16 v[12:15], v[212:215], v[180:183], v[12:15]\n" \
    "v_mfma_f32_16x16x32_bf16 v[12:15], v[208:211], v[176:179], v[12:15]\n" \
    "s_waitcnt lgkmcnt(4)\n" \
    "v_mfma_f32_16x16x32_bf16 v[16:19], v[216:219], v[176:179], v[16:19]\n" \
    "ds_read_b64_tr_b16 v[208:209], v240 offset:192\n" \
    "v_mfma_f32_16x16x32_bf16 v[16:19], v[220:223], v[180:183], v[16:19]\n" \
    "ds_read_b64_tr_b16 v[210:211], v240 offset:4800\n" \
    "v_mfma_f32_16x16x32_bf16 v[52:55], v[220:223], v[188:191], v[52:55]\n" \
    "ds_read_b64_tr_b16 v[212:213], v240 offset:9408\n" \
    "v_mfma_f32_16x16x32_bf16 v[52:55], v[216:219], v[184:187], v[52:55]\n" \
    "ds_read_b64_tr_b16 v[214:215], v240 offset:14016\n" \
    "v_mfma_f32_16x16x32_bf16 v[88:91], v[216:219], v[192:195], v[88:91]\n" \
    "v_mfma_f32_16x16x32_bf16 v[88:91], v[220:223], v[196:199], v[88:91]\n" \
    "v_mfma_f32_16x16x32_bf16 v[124:127], v[220:223], v[204:207], v[124:127]\n" \
    "v_mfma_f32_16x16x32_bf16 v[124:127], v[216:219], v[200:203], v[124:127]\n" \
    "s_waitcnt lgkmcnt(4)\n" \
    "v_mfma_f32_16x16x32_bf16 v[128:131], v[224:227], v[200:203], v[128:131]\n" \
    "ds_read_b64_tr_b16 v[216:217], v240 offset:224\n" \
    "v_mfma_f32_16x16x32_bf16 v[128:131], v[228:231], v[204:207], v[128:131]\n" \
    "ds_read_b64_tr_b16 v[218:219], v240 offset:4832\n" \
    "v_mfma_f32_16x16x32_bf16 v[92:95], v[228:231], v[196:199], v[92:95]\n" \
    "ds_read_b64_tr_b16 v[220:221], v240 offset:9440\n" \
    "v_mfma_f32_16x16x32_bf16 v[92:95], v[224:227], v[192:195], v[92:95]\n" \
    "ds_read_b64_tr_b16 v[222:223], v240 offset:14048\n" \
    "v_mfma_f32_16x16x32_bf16 v[56:59], v[224:227], v[184:187], v[56:59]\n" \
    "v_mfma_f32_16x16x32_bf16 v[56:59], v[228:231], v[188:191], v[56:59]\n" \
    "v_mfma_f32_16x16x32_bf16 v[20:23], v[228:231], v[180:183], v[20:23]\n" \
    "v_mfma_f32_16x16x32_bf16 v[20:23], v[224:227], v[176:179], v[20:23]\n" \
    "s_waitcnt lgkmcnt(4)\n" \
    "v_mfma_f32_16x16x32_bf16 v[24:27], v[208:211], v[176:179], v[24:27]\n" \
    "ds_read_b64_tr_b16 v[224:225], v240 offset:256\n" \
    "v_mfma_f32_16x16x32_bf16 v[24:27], v[212:215], v[180:183], v[24:27]\n" \
    "ds_read_b64_tr_b16 v[226:227], v240 offset:4864\n" \
    "v_mfma_f32_16x16x32_bf16 v[60:63], v[212:215], v[188:191], v[60:63]\n" \
    "ds_read_b64_tr_b16 v[228:229], v240 offset:9472\n" \
    "v_mfma_f32_16x16x32_bf16 v[60:63], v[208:211], v[184:187], v[60:63]\n" \
    "ds_read_b64_tr_b16 v[230:231], v240 offset:14080\n" \
    "v_mfma_f32_16x16x32_bf16 v[96:99], v[208:211], v[192:195], v[96:99]\n" \
    "v_add_u32 v240, %[ps], v241\n" \
    "v_mfma_f32_16x16x32_bf16 v[96:99], v[212:215], v[196:199], v[96:99]\n" \
    "ds_write_b32 v247, v246 offset:32\n" \
    "v_mfma_f32_16x16x32_bf16 v[132:135], v[212:215], v[204:207], v[132:135]\n" \
    "v_mfma_f32_16x16x32_bf16 v[132:135], v[208:211], v[200:203], v[132:135]\n" \
    "s_waitcnt lgkmcnt(5)\n" \
    "v_mfma_f32_16x16x32_bf16 v[136:139], v[216:219], v[200:203], v[136:139]\n" \
    "ds_read2_b32 v[244:245], v247 offset1:4\n" \
    "ds_read_b64_tr_b16 v[208:209], v240 offset:0\n" \
    "v_mfma_f32_16x16x32_bf16 v[136:139], v[220:223], v[204:207], v[136:139]\n" \
    "ds_read_b64_tr_b16 v[210:211], v240 offset:4608\n" \
    "v_mfma_f32_16x16x32_bf16 v[100:103], v[220:223], v[196:199], v[100:103]\n" \
    "ds_read_b64_tr_b16 v[212:213], v240 offset:9216\n" \
    "v_mfma_f32_16x16x32_bf16 v[100:103], v[216:219], v[192:195], v[100:103]\n" \
    "ds_read_b64_tr_b16 v[214:215], v240 offset:13824\n" \
    "v_mfma_f32_16x16x32_bf16 v[64:67], v[216:219], v[184:187], v[64:67]\n" \
    "v_mfma_f32_16x16x32_bf16 v[64:67], v[220:223], v[188:191], v[64:67]\n" \
    "v_mfma_f32_16x16x32_bf16 v[28:31], v[220:223], v[180:183], v[28:31]\n" \
    "v_mfma_f32_16x16x32_bf16 v[28:31], v[216:219], v[176:179], v[28:31]\n" \
    "s_waitcnt lgkmcnt(4)\n" \
    "v_mfma_f32_16x16x32_bf16 v[32:35], v[224:227], v[176:179], v[32:35]\n" \
    "ds_read_b64_tr_b16 v[216:217], v240 offset:32\n" \
    "v_mfma_f32_16x16x32_bf16 v[32:35], v[228:231], v[180:183], v[32:35]\n" \
    "ds_read_b64_tr_b16 v[218:219], v240 offset:4640\n" \
    "v_mfma_f32_16x16x32_bf16 v[68:71], v[228:231], v[188:191], v[68:71]\n" \
    "ds_read_b64_tr_b16 v[220:221], v240 offset:9248\n" \
    "v_mfma_f32_16x16x32_bf16 v[68:71], v[224:227], v[184:187], v[68:71]\n" \
    "ds_read_b64_tr_b16 v[222:223], v240 offset:13856\n" \
    "v_mfma_f32_16x16x32_bf16 v[104:107], v[224:227], v[192:195], v[104:107]\n" \
    "v_min_u32 v244, v244, v245\n" \
    "v_mfma_f32_16x16x32_bf16 v[104:107], v[228:231], v[196:199], v[104:107]\n" \
    "v_readfirstlane_b32 %[hv], v244\n" \
    "v_mfma_f32_16x16x32_bf16 v[140:143], v[228:231], v[204:207], v[140:143]\n" \
    "s_cmp_ge_u32 %[hv], %[ck]\n" \
    "v_mfma_f32_16x16x32_bf16 v[140:143], v[224:227], v[200:203], v[140:143]\n" \
    "s_waitcnt lgkmcnt(4) vmcnt(0)\n" \
    "s_cbranch_scc1 L9_%=\n" \
    "s_cmp_eq_u32 %[nd], 66\n" \
    "s_cbranch_scc1 LEND%=\n" \
    "Q9_%=:\n" \
    "ds_read2_b32 v[244:245], v247 offset1:4\n" \
    "s_waitcnt lgkmcnt(0)\n" \
    "v_min_u32 v244, v244, v245\n" \
    "s_nop 0\n" \
    "v_readfirstlane_b32 %[hv], v244\n" \
    "s_cmp_ge_u32 %[hv], %[nd]\n" \
    "s_cbranch_scc1 L9_%=\n" \
    "s_sleep 1\n" \
    "s_branch Q9_%=\n" \
    "E8%=:\n" \
    "ds_read_b64_tr_b16 v[208:209], v240 offset:0\n" \
    "ds_read_b64_tr_b16 v[210:211], v240 offset:4608\n" \
    "ds_read_b64_tr_b16 v[212:213], v240 offset:9216\n" \
    "ds_read_b64_tr_b16 v[214:215], v240 offset:13824\n" \
    "ds_read_b64_tr_b16 v[216:217], v240 offset:32\n" \
    "ds_read_b64_tr_b16 v[218:219], v240 offset:4640\n" \
    "ds_read_b64_tr_b16 v[220:221], v240 offset:9248\n" \
    "ds_read_b64_tr_b16 v[222:223], v240 offset:13856\n" \
    "ds_read_b64_tr_b16 v[224:225], v240 offset:64\n" \
    "ds_read_b64_tr_b16 v[226:227], v240 offset:4672\n" \
    "ds_read_b64_tr_b16 v[228:229], v240 offset:9280\n" \
    "ds_read_b64_tr_b16 v[230:231], v240 offset:13888\n" \
    "s_waitcnt lgkmcnt(8) vmcnt(0)\n" \
    "L8_%=:\n" \
    "v_mfma_f32_16x16x32_bf16 v[0:3], v[208:211], v[144:147], v[0:3]\n" \
    "ds_read_b64_tr_b16 v[232:233], v240 offset:96\n" \
    "v_mfma_f32_16x16x32_bf16 v[0:3], v[212:215], v[148:151], v[0:3]\n" \
    "ds_read_b64_tr_b16 v[234:235], v240 offset:4704\n" \
    "v_mfma_f32_16x16x32_bf16 v[36:39], v[212:215], v[156:159], v[36:39]\n" \
    "ds_read_b64_tr_b16 v[236:237], v240 offset:9312\n" \
    "v_mfma_f32_16x16x32_bf16 v[36:39], v[208:211], v[152:155], v[36:39]\n" \
    "ds_read_b64_tr_b16 v[238:239], v240 offset:13920\n" \
    "v_mfma_f32_16x16x32_bf16 v[72:75], v[208:211], v[160:163], v[72:75]\n" \
    "buffer_load_dwordx4 v[176:179], v242, %[rs], 0 offen offset:0 sc0 sc1\n" \
    "v_mfma_f32_16x16x32_bf16 v[72:75], v[212:215], v[164:167], v[72:75]\n" \
    "buffer_load_dwordx4 v[180:183], v242, %[rs], 0 offen offset:1024 sc0 sc1\n" \
    "v_mfma_f32_16x16x32_bf16 v[108:111], v[212:215], v[172:175], v[108:111]\n" \
    "buffer_load_dwordx4 v[184:187], v242, %[rs], 0 offen offset:2048 sc0 sc1\n" \
    "v_mfma_f32_16x16x32_bf16 v[108:111], v[208:211], v[168:171], v[108:111]\n" \
    "s_waitcnt lgkmcnt(8)\n" \
    "v_mfma_f32_16x16x32_bf16 v[112:115], v[216:219], v[168:171], v[112:115]\n" \
    "ds_read_b64_tr_b16 v[208:209], v240 offset:128\n" \
    "s_add_u32 %[nd], %[nd], 1\n" \
    "v_mfma_f32_16x16x32_bf16 v[112:115], v[220:223], v[172:175], v[112:115]\n" \
    "ds_read_b64_tr_b16 v[210:211], v240 offset:4736\n" \
    "s_add_u32 %[ps], %[ps], 18432\n" \
    "v_mfma_f32_16x16x32_bf16 v[76:79], v[220:223], v[164:167], v[76:79]\n" \
    "ds_read_b64_tr_b16 v[212:213], v240 offset:9344\n" \
    "s_cmp_eq_u32 %[ps], 147456\n" \
    "v_mfma_f32_16x16x32_bf16 v[76:79], v[216:219], v[160:163], v[76:79]\n" \
    "ds_read_b64_tr_b16 v[214:215], v240 offset:13952\n" \
    "s_cselect_b32 %[ps], 0, %[ps]\n" \
    "v_mfma_f32_16x16x32_bf16 v[40:43], v[216:219], v[152:155], v[40:43]\n" \
    "buffer_load_dwordx4 v[188:191], v242, %[rs], 0 offen offset:3072 sc0 sc1\n" \
    "v_mfma_f32_16x16x32_bf16 v[40:43], v[220:223], v[156:159], v[40:43]\n" \
    "buffer_load_dwordx4 v[192:195], v243, %[rs], 0 offen offset:0 sc0 sc1\n" \
    "v_mfma_f32_16x16x32_bf16 v[4:7], v[220:223], v[148:151], v[4:7]\n" \
    "buffer_load_dwordx4 v[196:199], v243, %[rs], 0 offen offset:1024 sc0 sc1\n" \
    "v_mfma_f32_16x16x32_bf16 v[4:7], v[216:219], v[144:147], v[4:7]\n" \
    "s_waitcnt lgkmcnt(8)\n" \
    "v_mfma_f32_16x16x32_bf16 v[8:11], v[224:227], v[144:147], v[8:11]\n" \
    "ds_read_b64_tr_b16 v[216:217], v240 offset:160\n" \
    "v_mfma_f32_16x16x32_bf16 v[8:11], v[228:231], v[148:151], v[8:11]\n" \
    "ds_read_b64_tr_b16 v[218:219], v240 offset:4768\n" \
    "v_mfma_f32_16x16x32_bf16 v[44:47], v[228:231], v[156:159], v[44:47]\n" \
    "ds_read_b64_tr_b16 v[220:221], v240 offset:9376\n" \
    "v_mfma_f32_16x16x32_bf16 v[44:47], v[224:227], v[152:155], v[44:47]\n" \
    "ds_read_b64_tr_b16 v[222:223], v240 offset:13984\n" \
    "v_mfma_f32_16x16x32_bf16 v[80:83], v[224:227], v[160:163], v[80:83]\n" \
    "buffer_load_dwordx4 v[200:203], v243, %[rs], 0 offen offset:2048 sc0 sc1\n" \
    "v_mfma_f32_16x16x32_bf16 v[80:83], v[228:231], v[164:167], v[80:83]\n" \
    "buffer_load_dwordx4 v[204:207], v243, %[rs], 0 offen offset:3072 sc0 sc1\n" \
    "v_mfma_f32_16x16x32_bf16 v[116:119], v[228:231], v[172:175], v[116:119]\n" \
    "v_add_u32 v242, 0x10000, v242\n" \
    "v_mfma_f32_16x16x32_bf16 v[116:119], v[224:227], v[168:171], v[116:119]\n" \
    "s_waitcnt lgkmcnt(8)\n" \
    "v_mfma_f32_16x16x32_bf16 v[120:123], v[232:235], v[168:171], v[120:123]\n" \
    "ds_read_b64_tr_b16 v[224:225], v240 offset:192\n" \
    "v_mfma_f32_16x16x32_bf16 v[120:123], v[236:239], v[172:175], v[120:123]\n" \
    "ds_read_b64_tr_b16 v[226:227], v240 offset:4800\n" \
    "v_mfma_f32_16x16x32_bf16 v[84:87], v[236:239], v[164:167], v[84:87]\n" \
    "ds_read_b64_tr_b16 v[228:229], v240 offset:9408\n" \
    "v_mfma_f32_16x16x32_bf16 v[84:87], v[232:235], v[160:163], v[84:87]\n" \
    "ds_read_b64_tr_b16 v[230:231], v240 offset:14016\n" \
    "v_mfma_f32_16x16x32_bf16 v[48:51], v[232:235], v[152:155], v[48:51]\n" \
    "v_add_u32 v243, 0x10000, v243\n" \
    "v_mfma_f32_16x16x32_bf16 v[48:51], v[236:239], v[156:159], v[48:51]\n" \
    "v_add_u32 v246, 1, v246\n" \
    "v_mfma_f32_16x16x32_bf16 v[12:15], v[236:239], v[148:151], v[12:15]\n" \
    "v_mfma_f32_16x16x32_bf16 v[12:15], v[232:235], v[144:147], v[12:15]\n" \
    "s_waitcnt lgkmcnt(8)\n" \
    "v_mfma_f32_16x16x32_bf16 v[16:19], v[208:211], v[144:147], v[16:19]\n" \
    "ds_read_b64_tr_b16 v[232:233], v240 offset:224\n" \
    "v_mfma_f32_16x16x32_bf16 v[16:19], v[212:215], v[148:151], v[16:19]\n" \
    "ds_read_b64_tr_b16 v[234:235], v240 offset:4832\n" \
    "v_mfma_f32_16x16x32_bf16 v[52:55], v[212:215], v[156:159], v[52:55]\n" \
    "ds_read_b64_tr_b16 v[236:237], v240 offset:9440\n" \
    "v_mfma_f32_16x16x32_bf16 v[52:55], v[208:211], v[152:155], v[52:55]\n" \
    "ds_read_b64_tr_b16 v[238:239], v240 offset:14048\n" \
    "v_mfma_f32_16x16x32_bf16 v[88:91], v[208:211], v[160:163], v[88:91]\n" \
    "v_add_u32 v240, %[ps], v241\n" \
    "v_mfma_f32_16x16x32_bf16 v[88:91], v[212:215], v[164:167], v[88:91]\n" \
    "ds_write_b32 v247, v246 offset:32\n" \
    "v_mfma_f32_16x16x32_bf16 v[124:127], v[212:215], v[172:175], v[124:127]\n" \
    "v_mfma_f32_16x16x32_bf16 v[124:127], v[208:211], v[168:171], v[124:127]\n" \
    "s_waitcnt lgkmcnt(9)\n" \
    "v_mfma_f32_16x16x32_bf16 v[128:131], v[216:219], v[168:171], v[128:131]\n" \
    "ds_read_b64_tr_b16 v[208:209], v240 offset:0\n" \
    "v_mfma_f32_16x16x32_bf16 v[128:131], v[220:223], v[172:175], v[128:131]\n" \
    "ds_read_b64_tr_b16 v[210:211], v240 offset:4608\n" \
    "v_mfma_f32_16x16x32_bf16 v[92:95], v[220:223], v[164:167], v[92:95]\n" \
    "ds_read_b64_tr_b16 v[212:213], v240 offset:9216\n" \
    "v_mfma_f32_16x16x32_bf16 v[92:95], v[216:219], v[160:163], v[92:95]\n" \
    "ds_read_b64_tr_b16 v[214:215], v240 offset:13824\n" \
    "v_mfma_f32_16x16x32_bf16 v[56:59], v[216:219], v[152:155], v[56:59]\n" \
    "v_mfma_f32_16x16x32_bf16 v[56:59], v[220:223], v[156:159], v[56:59]\n" \
    "v_mfma_f32_16x16x32_bf16 v[20:23], v[220:223], v[148:151], v[20:23]\n" \
    "v_mfma_f32_16x16x32_bf16 v[20:23], v[216:219], v[144:147], v[20:23]\n" \
    "s_waitcnt lgkmcnt(9)\n" \
    "v_mfma_f32_16x16x32_bf16 v[24:27], v[224:227], v[144:147], v[24:27]\n" \
    "ds_read2_b32 v[244:245], v247 offset1:4\n" \
    "ds_read_b64_tr_b16 v[216:217], v240 offset:32\n" \
    "v_mfma_f32_16x16x32_bf16 v[24:27], v[228:231], v[148:151], v[24:27]\n" \
    "ds_read_b64_tr_b16 v[218:219], v240 offset:4640\n" \
    "v_mfma_f32_16x16x32_bf16 v[60:63], v[228:231], v[156:159], v[60:63]\n" \
    "ds_read_b64_tr_b16 v[220:221], v240 offset:9248\n" \
    "v_mfma_f32_16x16x32_bf16 v[60:63], v[224:227], v[152:155], v[60:63]\n" \
    "ds_read_b64_tr_b16 v[222:223], v240 offset:13856\n" \
    "v_mfma_f32_16x16x32_bf16 v[96:99], v[224:227], v[160:163], v[96:99]\n" \
    "v_mfma_f32_16x16x32_bf16 v[96:99], v[228:231], v[164:167], v[96:99]\n" \
    "v_mfma_f32_16x16x32_bf16 v[132:135], v[228:231], v[172:175], v[132:135]\n" \
    "v_mfma_f32_16x16x32_bf16 v[132:135], v[224:227], v[168:171], v[132:135]\n" \
    "s_waitcnt lgkmcnt(4)\n" \
    "v_mfma_f32_16x16x32_bf16 v[136:139], v[232:235], v[168:171], v[136:139]\n" \
    "ds_read_b64_tr_b16 v[224:225], v240 offset:64\n" \
    "v_mfma_f32_16x16x32_bf16 v[136:139], v[236:239], v[172:175], v[136:139]\n" \
    "ds_read_b64_tr_b16 v[226:227], v240 offset:4672\n" \
    "v_mfma_f32_16x16x32_bf16 v[100:103], v[236:239], v[164:167], v[100:103]\n" \
    "ds_read_b64_tr_b16 v[228:229], v240 offset:9280\n" \
    "v_mfma_f32_16x16x32_bf16 v[100:103], v[232:235], v[160:163], v[100:103]\n" \
    "ds_read_b64_tr_b16 v[230:231], v240 offset:13888\n" \
    "v_mfma_f32_16x16x32_bf16 v[64:67], v[232:235], v[152:155], v[64:67]\n" \
    "v_min_u32 v244, v244, v245\n" \
    "v_mfma_f32_16x16x32_bf16 v[64:67], v[236:239], v[156:159], v[64:67]\n" \
    "v_readfirstlane_b32 %[hv], v244\n" \
    "v_mfma_f32_16x16x32_bf16 v[28:31], v[236:239], v[148:151], v[28:31]\n" \
    "s_cmp_ge_u32 %[hv], %[nd]\n" \
    "v_mfma_f32_16x16x32_bf16 v[28:31], v[232:235], v[144:147], v[28:31]\n" \
    "s_waitcnt lgkmcnt(8) vmcnt(0)\n" \
    "s_cbranch_scc0 P8_%=\n" \
    "R8_%=:\n" \
    "v_mfma_f32_16x16x32_bf16 v[0:3], v[208:211], v[176:179], v[0:3]\n" \
    "ds_read_b64_tr_b16 v[232:233], v240 offset:96\n" \
    "v_mfma_f32_16x16x32_bf16 v[0:3], v[212:215], v[180:183], v[0:3]\n" \
    "ds_read_b64_tr_b16 v[234:235], v240 offset:4704\n" \
    "v_mfma_f32_16x16x32_bf16 v[36:39], v[212:215], v[188:191], v[36:39]\n" \
    "ds_read_b64_tr_b16 v[236:237], v240 offset:9312\n" \
    "v_mfma_f32_16x16x32_bf16 v[36:39], v[208:211], v[184:187], v[36:39]\n" \
    "ds_read_b64_tr_b16 v[238:239], v240 offset:13920\n" \
    "v_mfma_f32_16x16x32_bf16 v[72:75], v[208:211], v[192:195], v[72:75]\n" \
    "buffer_load_dwordx4 v[144:147], v242, %[rs], 0 offen offset:0 sc0 sc1\n" \
    "v_mfma_f32_16x16x32_bf16 v[72:75], v[212:215], v[196:199], v[72:75]\n" \
    "buffer_load_dwordx4 v[148:151], v242, %[rs], 0 offen offset:1024 sc0 sc1\n" \
    "v_mfma_f32_16x16x32_bf16 v[108:111], v[212:215], v[204:207], v[108:111]\n" \
    "buffer_load_dwordx4 v[152:155], v242, %[rs], 0 offen offset:2048 sc0 sc1\n" \
    "v_mfma_f32_16x16x32_bf16 v[108:111], v[208:211], v[200:203], v[108:111]\n" \
    "s_waitcnt lgkmcnt(8)\n" \
    "v_mfma_f32_16x16x32_bf16 v[112:115], v[216:219], v[200:203], v[112:115]\n" \
    "ds_read_b64_tr_b16 v[208:209], v240 offset:128\n" \
    "s_add_u32 %[nd], %[nd], 1\n" \
    "v_mfma_f32_16x16x32_bf16 v[112:115], v[220:223], v[204:207], v[112:115]\n" \
    "ds_read_b64_tr_b16 v[210:211], v240 offset:4736\n" \
    "s_add_u32 %[ps], %[ps], 18432\n" \
    "v_mfma_f32_16x16x32_bf16 v[76:79], v[220:223], v[196:199], v[76:79]\n" \
    "ds_read_b64_tr_b16 v[212:213], v240 offset:9344\n" \
    "s_cmp_eq_u32 %[ps], 147456\n" \
    "v_mfma_f32_16x16x32_bf16 v[76:79], v[216:219], v[192:195], v[76:79]\n" \
    "ds_read_b64_tr_b16 v[214:215], v240 offset:13952\n" \
    "s_cselect_b32 %[ps], 0, %[ps]\n" \
    "v_mfma_f32_16x16x32_bf16 v[40:43], v[216:219], v[184:187], v[40:43]\n" \
    "buffer_load_dwordx4 v[156:159], v242, %[rs], 0 offen offset:3072 sc0 sc1\n" \
    "v_mfma_f32_16x16x32_bf16 v[40:43], v[220:223], v[188:191], v[40:43]\n" \
    "buffer_load_dwordx4 v[160:163], v243, %[rs], 0 offen offset:0 sc0 sc1\n" \
    "v_mfma_f32_16x16x32_bf16 v[4:7], v[220:223], v[180:183], v[4:7]\n" \
    "buffer_load_dwordx4 v[164:167], v243, %[rs], 0 offen offset:1024 sc0 sc1\n" \
    "v_mfma_f32_16x16x32_bf16 v[4:7], v[216:219], v[176:179], v[4:7]\n" \
    "s_waitcnt lgkmcnt(8)\n" \
    "v_mfma_f32_16x16x32_bf16 v[8:11], v[224:227], v[176:179], v[8:11]\n" \
    "ds_read_b64_tr_b16 v[216:217], v240 offset:160\n" \
    "s_cmp_eq_u32 %[nd], 66\n" \
    "v_mfma_f32_16x16x32_bf16 v[8:11], v[228:231], v[180:183], v[8:11]\n" \
    "ds_read_b64_tr_b16 v[218:219], v240 offset:4768\n" \
    "s_cselect_b32 %[ck], -1, %[nd]\n" \
    "v_mfma_f32_16x16x32_bf16 v[44:47], v[228:231], v[188:191], v[44:47]\n" \
    "ds_read_b64_tr_b16 v[220:221], v240 offset:9376\n" \
    "v_mfma_f32_16x16x32_bf16 v[44:47], v[224:227], v[184:187], v[44:47]\n" \
    "ds_read_b64_tr_b16 v[222:223], v240 offset:13984\n" \
    "v_mfma_f32_16x16x32_bf16 v[80:83], v[224:227], v[192:195], v[80:83]\n" \
    "buffer_load_dwordx4 v[168:171], v243, %[rs], 0 offen offset:2048 sc0 sc1\n" \
    "v_mfma_f32_16x16x32_bf16 v[80:83], v[228:231], v[196:199], v[80:83]\n" \
    "buffer_load_dwordx4 v[172:175], v243, %[rs], 0 offen offset:3072 sc0 sc1\n" \
    "v_mfma_f32_16x16x32_bf16 v[116:119], v[228:231], v[204:207], v[116:119]\n" \
    "v_add_u32 v242, 0x10000, v242\n" \
    "v_mfma_f32_16x16x32_bf16 v[116:119], v[224:227], v[200:203], v[116:119]\n" \
    "s_waitcnt lgkmcnt(8)\n" \
    "v_mfma_f32_16x16x32_bf16 v[120:123], v[232:235], v[200:203], v[120:123]\n" \
    "ds_read_b64_tr_b16 v[224:225], v240 offset:192\n" \
    "v_mfma_f32_16x16x32_bf16 v[120:123], v[236:239], v[204:207], v[120:123]\n" \
    "ds_read_b64_tr_b16 v[226:227], v240 offset:4800\n" \
    "v_mfma_f32_16x16x32_bf16 v[84:87], v[236:239], v[196:199], v[84:87]\n" \
    "ds_read_b64_tr_b16 v[228:229], v240 offset:9408\n" \
    "v_mfma_f32_16x16x32_bf16 v[84:87], v[232:235], v[192:195], v[84:87]\n" \
    "ds_read_b64_tr_b16 v[230:231], v240 offset:14016\n" \
    "v_mfma_f32_16x16x32_bf16 v[48:51], v[232:235], v[184:187], v[48:51]\n" \
    "v_add_u32 v243, 0x10000, v243\n" \
    "v_mfma_f32_16x16x32_bf16 v[48:51], v[236:239], v[188:191], v[48:51]\n" \
    "v_add_u32 v246, 1, v246\n" \
    "v_mfma_f32_16x16x32_bf16 v[12:15], v[236:239], v[180:183], v[12:15]\n" \
    "v_mfma_f32_16x16x32_bf16 v[12:15], v[232:235], v[176:179], v[12:15]\n" \
    "s_waitcnt lgkmcnt(8)\n" \
    "v_mfma_f32_16x16x32_bf16 v[16:19], v[208:211], v[176:179], v[16:19]\n" \
    "ds_read_b64_tr_b16 v[232:233], v240 offset:224\n" \
    "v_mfma_f32_16x16x32_bf16 v[16:19], v[212:215], v[180:183], v[16:19]\n" \
    "ds_read_b64_tr_b16 v[234:235], v240 offset:4832\n" \
    "v_mfma_f32_16x16x32_bf16 v[52:55], v[212:215], v[188:191], v[52:55]\n" \
    "ds_read_b64_tr_b16 v[236:237], v240 offset:9440\n" \
    "v_mfma_f32_16x16x32_bf16 v[52:55], v[208:211], v[184:187], v[52:55]\n" \
    "ds_read_b64_tr_b16 v[238:239], v240 offset:14048\n" \
    "v_mfma_f32_16x16x32_bf16 v[88:91], v[208:211], v[192:195], v[88:91]\n" \
    "v_add_u32 v240, %[ps], v241\n" \
    "v_mfma_f32_16x16x32_bf16 v[88:91], v[212:215], v[196:199], v[88:91]\n" \
    "ds_write_b32 v247, v246 offset:32\n" \
    "v_mfma_f32_16x16x32_bf16 v[124:127], v[212:215], v[204:207], v[124:127]\n" \
    "v_mfma_f32_16x16x32_bf16 v[124:127], v[208:211], v[200:203], v[124:127]\n" \
    "s_waitcnt lgkmcnt(9)\n" \
    "v_mfma_f32_16x16x32_bf16 v[128:131], v[216:219], v[200:203], v[128:131]\n" \
    "ds_read_b64_tr_b16 v[208:209], v240 offset:0\n" \
    "v_mfma_f32_16x16x32_bf16 v[128:131], v[220:223], v[204:207], v[128:131]\n" \
    "ds_read_b64_tr_b16 v[210:211], v240 offset:4608\n" \
    "v_mfma_f32_16x16x32_bf16 v[92:95], v[220:223], v[196:199], v[92:95]\n" \
    "ds_read_b64_tr_b16 v[212:213], v240 offset:9216\n" \
    "v_mfma_f32_16x16x32_bf16 v[92:95], v[216:219], v[192:195], v[92:95]\n" \
    "ds_read_b64_tr_b16 v[214:215], v240 offset:13824\n" \
    "v_mfma_f32_16x16x32_bf16 v[56:59], v[216:219], v[184:187], v[56:59]\n" \
    "v_mfma_f32_16x16x32_bf16 v[56:59], v[220:223], v[188:191], v[56:59]\n" \
    "v_mfma_f32_16x16x32_bf16 v[20:23], v[220:223], v[180:183], v[20:23]\n" \
    "v_mfma_f32_16x16x32_bf16 v[20:23], v[216:219], v[176:179], v[20:23]\n" \
    "s_waitcnt lgkmcnt(9)\n" \
    "v_mfma_f32_16x16x32_bf16 v[24:27], v[224:227], v[176:179], v[24:27]\n" \
    "ds_read2_b32 v[244:245], v247 offset1:4\n" \
    "ds_read_b64_tr_b16 v[216:217], v240 offset:32\n" \
    "v_mfma_f32_16x16x32_bf16 v[24:27], v[228:231], v[180:183], v[24:27]\n" \
    "ds_read_b64_tr_b16 v[218:219], v240 offset:4640\n" \
    "v_mfma_f32_16x16x32_bf16 v[60:63], v[228:231], v[188:191], v[60:63]\n" \
    "ds_read_b64_tr_b16 v[220:221], v240 offset:9248\n" \
    "v_mfma_f32_16x16x32_bf16 v[60:63], v[224:227], v[184:187], v[60:63]\n" \
    "ds_read_b64_tr_b16 v[222:223], v240 offset:13856\n" \
    "v_mfma_f32_16x16x32_bf16 v[96:99], v[224:227], v[192:195], v[96:99]\n" \
    "v_mfma_f32_16x16x32_bf16 v[96:99], v[228:231], v[196:199], v[96:99]\n" \
    "v_mfma_f32_16x16x32_bf16 v[132:135], v[228:231], v[204:207], v[132:135]\n" \
    "v_mfma_f32_16x16x32_bf16 v[132:135], v[224:227], v[200:203], v[132:135]\n" \
    "s_waitcnt lgkmcnt(4)\n" \
    "v_mfma_f32_16x16x32_bf16 v[136:139], v[232:235], v[200:203], v[136:139]\n" \
    "ds_read_b64_tr_b16 v[224:225], v240 offset:64\n" \
    "v_mfma_f32_16x16x32_bf16 v[136:139], v[236:239], v[204:207], v[136:139]\n" \
    "ds_read_b64_tr_b16 v[226:227], v240 offset:4672\n" \
    "v_mfma_f32_16x16x32_bf16 v[100:103], v[236:239], v[196:199], v[100:103]\n" \
    "ds_read_b64_tr_b16 v[228:229], v240 offset:9280\n" \
    "v_mfma_f32_16x16x32_bf16 v[100:103], v[232:235], v[192:195], v[100:103]\n" \
    "ds_read_b64_tr_b16 v[230:231], v240 offset:13888\n" \
    "v_mfma_f32_16x16x32_bf16 v[64:67], v[232:235], v[184:187], v[64:67]\n" \
    "v_min_u32 v244, v244, v245\n" \
    "v_mfma_f32_16x16x32_bf16 v[64:67], v[236:239], v[188:191], v[64:67]\n" \
    "v_readfirstlane_b32 %[hv], v244\n" \
    "v_mfma_f32_16x16x32_bf16 v[28:31], v[236:239], v[180:183], v[28:31]\n" \
    "s_cmp_ge_u32 %[hv], %[ck]\n" \
    "v_mfma_f32_16x16x32_bf16 v[28:31], v[232:235], v[176:179], v[28:31]\n" \
    "s_waitcnt lgkmcnt(8) vmcnt(0)\n" \
    "s_cbranch_scc1 L8_%=\n" \
    "s_cmp_eq_u32 %[nd], 66\n" \
    "s_cbranch_scc1 LEND%=\n" \
    "Q8_%=:\n" \
    "ds_read2_b32 v[244:245], v247 offset1:4\n" \
    "s_waitcnt lgkmcnt(0)\n" \
    "v_min_u32 v244, v244, v245\n" \
    "s_nop 0\n" \
    "v_readfirstlane_b32 %[hv], v244\n" \
    "s_cmp_ge_u32 %[hv], %[nd]\n" \
    "s_cbranch_scc1 L8_%=\n" \
    "s_sleep 1\n" \
    "s_branch Q8_%=\n" \
    "LEND%=:\n" \
    "s_waitcnt vmcnt(0) lgkmcnt(0)\n" \
    "s_nop 15\n" \
    ""

__global__ void __launch_bounds__(NTHR, 2) ag_gemm(const Args a) {
    __shared__ __attribute__((aligned(1024))) unsigned char lds_raw[LDS_BYTES];
    unsigned code;
    {
        unsigned long long pc;
        unsigned len;
        asm volatile("s_getpc_b64 %0\nAG_CODE0:\n\ts_mov_b32 %1, AG_CODE1-AG_CODE0" : "=s"(pc), "=s"(len));
        code = __builtin_amdgcn_raw_buffer_load_b32(rsrc((const void*)pc, len), threadIdx.x * 64, 0, 0);
        SB();
    }
    LAS unsigned char* lds = (LAS unsigned char*)lds_raw;
    volatile LAS unsigned* ctl = (volatile LAS unsigned*)(lds + L_CTL);
    volatile LAS unsigned* prog = ctl + 8;
    const int lane = threadIdx.x & 63;
    const int wave = __builtin_amdgcn_readfirstlane(threadIdx.x >> 6);
    const int bid = blockIdx.x;
    char* const myws = a.ws[NR - 1];
    asm volatile("" :: "s"(a.x), "s"(a.ws[0]), "s"(a.ws[1]), "s"(a.ws[2]), "s"(a.ws[3]), "s"(a.ws[4]), "s"(a.ws[5]), "s"(a.ws[6]), "s"(myws), "s"(a.rank));
    const int xcd = bid & 7, loc = bid >> 3;
    const bool is_comm = loc < 2;
    const int cid = loc * 8 + xcd, tl = (xcd & 1) * (2 * NTN) + loc - 2, pm = (xcd >> 1) * 4 + (tl & 3), pn = tl >> 2;
    const int pq = (pn & 1) ? -1 : pn >> 1, part = pq >> 31, flip = 0;

    if (wave == 0) STAMP(0);
    const __amdgpu_buffer_rsrc_t rin = rsrc(a.x, MS * K * 2);
    auto fetch = [&](u32x4 (&d)[4], unsigned b) {
        const unsigned o = (b < (unsigned)NBLK ? (b % NG) * 32 * (K * 2) + (b / NG) * (BK * 2) : OOB) + (lane & 15) * (K * 2) + (lane >> 4) * 16;
#pragma unroll
        for (int s = 0; s < 4; ++s)
            d[s] = __builtin_bit_cast(u32x4, __builtin_amdgcn_raw_buffer_load_b128(rin, o + (s & 1) * 64, (s >> 1) * 16 * (K * 2), 0));
    };
    u32x4 d[4];
    fetch(d, is_comm && wave < SW ? wave * NSC + cid : NBLK);
    GAS unsigned* cnt = (GAS unsigned*)(myws + W_CNT + (size_t)bid * 128);
    const unsigned cv = __hip_atomic_load(cnt, __ATOMIC_RELAXED, __HIP_MEMORY_SCOPE_AGENT);
    SB();
    if (wave == 0) STAMP(10);
    if (threadIdx.x < 32) ((LAS unsigned*)(lds + L_CTL))[threadIdx.x] = 0u;
    if (threadIdx.x < 256) *(LAS u32x2*)(lds + L_SEL + threadIdx.x * 8) = is_comm ? mover(threadIdx.x, true) : mover(threadIdx.x, false);
    SB();
    asm volatile("" :: "v"(code));
    const unsigned call = __builtin_amdgcn_readfirstlane(cv) + 1u;
    __syncthreads();
    if (threadIdx.x == 0) __hip_atomic_store(cnt, call, __ATOMIC_RELAXED, __HIP_MEMORY_SCOPE_AGENT);
    LAS unsigned char* lm = lds + L_SEL;
    int mul = loc;
    asm volatile("" : "+s"(mul));
    if (mul > 1) {
    GAS unsigned* tmo = (GAS unsigned*)(myws + W_TMO);
    const size_t xt_off = W_XT + (size_t)(call & 1u) * XT_BYTES;
    const unsigned oQ = pq < 0 ? OOB : ((NTN - pn) * BN + pq * 16) * 2;
    if (wave >= 6) {
        __builtin_amdgcn_s_setprio(AG_UPRIO);
        const int src = ((xcd + 2) & 6) + wave - 6;
        const unsigned u = tl;
        const __amdgpu_buffer_rsrc_t rf = rsrc(myws + W_FLAG + (size_t)src * NBLK * FS * 4, NBLK * FS * 4);
        const __amdgpu_buffer_rsrc_t rp = rsrc(myws + W_PK + (size_t)(call & 1u) * PK_BYTES + (size_t)src * NBLK * PBLK, NBLK * PBLK);
        const __amdgpu_buffer_rsrc_t rx = rsrc(myws + xt_off + (size_t)src * NBLK * BLK, NBLK * BLK);
        auto done = [&](unsigned b) {
            const unsigned g = b % NG, kt = b / NG;
            const unsigned fo = lane == 0 ? (((g >> 3) * NKT + kt) * 8 + (g & 7)) * FS * 4 : OOB;
            __builtin_amdgcn_raw_buffer_store_b32(call, rf, fo, 0, 17);
        };
        if (wave == 7) STAMP(8);
        unsigned long long lat_r = 0, lat_w = 0, lat_n = 0;
        if (src == a.rank) {
            fetch(d, u);
            for (unsigned b = u; b < (unsigned)NBLK; b += NU) {
#pragma unroll
                for (int s = 0; s < 4; ++s)
                    __builtin_amdgcn_raw_buffer_store_b128(d[s], rx, b * BLK + lane * 16 + s * 1024, 0, AG_UAUX);
                fetch(d, b + NU);
                wait_vm0();
                done(b);
            }
        } else {
            LAS unsigned char* lx = lds + L_UNP + (wave - 6) * UNP;
            LAS unsigned char* ls = lx + 2112;
            *(LAS u32x4*)(lx + lane * 32) = (u32x4){0u, 0u, 0u, 0u};
            *(LAS u32x4*)(lx + lane * 32 + 16) = (u32x4){0u, 0u, 0u, 0u};
            const unsigned long long want = tag_of(call) ? ~0ull : 0ull;
            const unsigned keep = (lane & 1) ? ~0u : 0x7fffffffu, keept = lane < 16 ? 0x7fffffffu : ~0u, sh = 31 - (lane >> 1), lw = (lane < 24 ? lane : 24) * 16;
            const unsigned long long t0 = rt();
            unsigned b = u, pend = ~0u;
            for (;;) {
                const unsigned so = b * PBLK;
                const unsigned hl = __builtin_amdgcn_raw_buffer_load_b32(rp, lane < 32 && b < (unsigned)NBLK ? so + 2560 + lane * 4 : OOB, 0, AG_HAUX);
                wait_vm0();
                const bool first = pend != ~0u;
                if (first) {
                    done(pend);
                    TRACE(src, pend, 2);
                    pend = ~0u;
                }
                if (b >= (unsigned)NBLK) break;
                if (((unsigned)(__ballot((int)hl < 0) ^ want) & 0x11111111u) != 0u) {
                    if (give_up(tmo, call, t0)) {
                        __hip_atomic_store(tmo, call, __ATOMIC_RELAXED, __HIP_MEMORY_SCOPE_AGENT);
                        break;
                    }
                    __builtin_amdgcn_s_sleep(AG_USLEEP);
                    continue;
                }
                if (first) lat_w += 100;
                ++lat_n;
                u32x4 p[2], t, x[2];
                u32x2 c;
                const unsigned h0 = __builtin_amdgcn_readlane(hl, 0);
                c = __builtin_bit_cast(u32x2, __builtin_amdgcn_raw_buffer_load_b64(rp, so + lane * 8 + 2048, 0, 17));
                t = __builtin_bit_cast(u32x4, __builtin_amdgcn_raw_buffer_load_b128(rp, lane < ((h0 & 512u) ? 24 : 16) ? so + lane * 16 + 2560 : OOB, 0, 17));
#pragma unroll
                for (int s = 0; s < 2; ++s)
                    p[s] = __builtin_bit_cast(u32x4, __builtin_amdgcn_raw_buffer_load_b128(rp, so + lane * 16 + s * 1024, 0, 17));
                TRACE(src, b, 1);
                const unsigned m2l = __builtin_amdgcn_readlane(hl, 3), m1l = bfi(0x7fffffffu, __builtin_amdgcn_readlane(hl, 4), h0 << 15);
                const unsigned long long m0 = (unsigned)__builtin_amdgcn_readlane(hl, 1) | ((unsigned long long)(unsigned)__builtin_amdgcn_readlane(hl, 2) << 32);
                const unsigned long long m1 = m1l | ((unsigned long long)(unsigned)__builtin_amdgcn_readlane(hl, 5) << 32);
                const unsigned e0 = h0 & 255u, cnt = (h0 >> 10) & 31u;
                const unsigned pa = (unsigned)lane - HW < cnt ? hl & 0x7ffu : 2048u;
                lx[pa] = (unsigned char)(hl >> 16);
                FENCE();
                x[0] = *(LAS u32x4*)(lx + lane * 32);
                x[1] = *(LAS u32x4*)(lx + lane * 32 + 16);
                FENCE();
                lx[pa] = 0;
                unsigned t2 = table2(e0), t3l = table3(e0, 0), t3h = table3(e0, 1), top0, top1, topc = m2l << sh, topt = h0 << (16 - (lane & 15));
                unsigned vo = (h0 & 256u) ? OOB : b * BLK + lane * 16, lq = (unsigned)(size_t)ls + (HW + cnt) * 4u;
                asm("v_cndmask_b32 %0, 0, -1, %1" : "=v"(top0) : "s"(m0));
                asm("v_cndmask_b32 %0, 0, -1, %1" : "=v"(top1) : "s"(m1));
                asm volatile("" : "+v"(t2), "+v"(t3l), "+v"(top0), "+v"(top1), "+v"(topc), "+v"(topt), "+v"(vo));
                asm volatile("" : "+s"(t3h), "+s"(lq));
                if (((__ballot((int)p[0].x < 0) ^ want) | (__ballot((int)p[1].x < 0) ^ want) |
                     ((__ballot((int)c.x < 0) ^ want) & 0x5555555555555555ull) | ((__ballot((int)t.x < 0) ^ want) & 0xffffull)) != 0ull) {
                    if (!give_up(tmo, call, t0)) continue;
                    __hip_atomic_store(tmo, call, __ATOMIC_RELAXED, __HIP_MEMORY_SCOPE_AGENT);
                    break;
                }
                t.x = bfi(keept, t.x, topt);
                *(LAS u32x4*)(ls + lw) = t;
                c.x = bfi(keep, c.x, topc);
                const u32x2 f = c & (c >> 1) & 0x55555555u;
                const unsigned nb = 3u * (__builtin_popcount(f.x) + __builtin_popcount(f.y));
                const unsigned off = scan(nb) - nb;
                u32x2 mv[8];
                FENCE();
                LAS unsigned* lt = (LAS unsigned*)(size_t)(((off >> 5) << 2) + lq);
                const unsigned w0 = lt[0], w1 = lt[1], w2 = lt[2];
#pragma unroll
                for (int j = 0; j < 8; ++j)
                    mv[j] = *(LAS u32x2*)(lm + ((c[j >> 2] >> (8 * (j & 3))) & 255u) * 8);
                FENCE();
                unsigned al = __builtin_amdgcn_alignbit(w1, w0, off), ah = __builtin_amdgcn_alignbit(w2, w1, off);
                p[0].x = bfi(0x7fffffffu, p[0].x, top0);
                p[1].x = bfi(0x7fffffffu, p[1].x, top1);
#pragma unroll
                for (int s = 0; s < 4; ++s) {
                    u32x4 d;
#pragma unroll
                    for (int i = 0; i < 2; ++i) {
                        const int j = 2 * s + i;
                        unsigned q = al;
                        al = __builtin_amdgcn_alignbit(ah, al, mv[j].y);
                        ah >>= mv[j].y;
                        asm("v_lshrrev_b32_sdwa %0, %1, %0 dst_sel:WORD_1 dst_unused:UNUSED_PRESERVE src0_sel:DWORD src1_sel:DWORD" : "+v"(q) : "v"(6), "v"(al));
                        const unsigned g = perm(t3h, t3l, (q & 0x00070007u) | ((q << 5) & 0x07000700u));
                        const unsigned e = perm(t2, g, mv[j].x) ^ x[j >> 2][j & 3];
                        const unsigned r0 = perm(e, p[j >> 2][j & 3], 0x05010400u), r1 = perm(e, p[j >> 2][j & 3], 0x07030602u);
                        d[2 * i] = __builtin_amdgcn_alignbit(r0, r0, 1);
                        d[2 * i + 1] = __builtin_amdgcn_alignbit(r1, r1, 1);
                    }
                    __builtin_amdgcn_raw_buffer_store_b128(d, rx, vo + s * 1024, 0, AG_UAUX);
                }
                pend = b;
                b += NU;
            }
        }
        if (wave == 7) STAMP(9);
        asm volatile("AG_CODE1:");
    }
    else if (wave < 4) {
        const unsigned su = ((xcd >> 1) * (4 * NTN) + tl) * 4 + wave, sk = NSLOT * BK + su * 4;
        const u32x4 sv = __builtin_bit_cast(u32x4, __builtin_amdgcn_raw_buffer_load_b128(rsrc(a.w, K * NL * 2),
            (sk + ((lane >> 1) & 3)) * (NL * 2) + NTN * BN * 2 + (lane >> 3) * 32 + (lane & 1) * 16, 0, 0));
        auto strip = [&]() {
            const unsigned r0 = (sk & ~31u) + ((sk >> 2) & 1) * 16 + ((sk >> 3) & 3) * 4;
            store16(sv, rsrc(myws + W_STRIP, K * BN * 2), su < (unsigned)NSTRIP ? (lane >> 3) * (K * 32) + r0 * 32 + (lane & 7) * 16 : OOB);
        };
        if (wave == 3) strip();
        constexpr int PW = AG_DRY ? 3 : 4;
        if (wave < PW) {
            const unsigned cw = ((xcd >> 1) * (4 * NTN) + tl) * PW + wave;
            constexpr int W1 = (AG_PD ? AG_PD : NKT) * BK * NL * 2;
            const __amdgpu_buffer_rsrc_t rw = rsrc(a.w, W1), rs = rsrc(a.x, MS * K * 2);
            constexpr int W0 = AG_PF0 * BK * NL * 2, ST = 16 * NTN * PW * 1024, NW = (W1 - W0 + ST - 1) / ST, NX = (MS * K * 2 + ST - 1) / ST;
            u32x4 pf[NW], px[NX];
#pragma unroll
            for (int i = 0; i < NW; ++i)
                pf[i] = __builtin_bit_cast(u32x4, __builtin_amdgcn_raw_buffer_load_b128(rw, W0 + i * ST + cw * 1024 + lane * 16, 0, AG_PFAUX));
            if (AG_PF == 2) {
#pragma unroll
                for (int i = 0; i < NX; ++i)
                    px[i] = __builtin_bit_cast(u32x4, __builtin_amdgcn_raw_buffer_load_b128(rs, i * ST + cw * 1024 + lane * 16, 0, AG_PFAUX));
                SB();
                strip();
#pragma unroll
                for (int i = 0; i < NX; ++i) asm volatile("" :: "v"(px[i]));
            }
#pragma unroll
            for (int i = 0; i < NW; ++i) asm volatile("" :: "v"(pf[i]));
            if (wave == 0) STAMP(12);
        }
        wait_vm0();
        if (lane == 0 && su < (unsigned)NSTRIP)
            __hip_atomic_fetch_add((GAS unsigned*)(myws + W_SCNT + (su & 7) * 128), 1u, __ATOMIC_RELAXED, __HIP_MEMORY_SCOPE_AGENT);
        const int g0 = (pm & 1) * 8 + wave * 2;
        const unsigned long long xb = (unsigned long long)(myws + xt_off + ((size_t)(pm >> 1) * NBLK + g0) * BLK);
        const u32x4 ra = (u32x4){(unsigned)xb, (unsigned)(xb >> 32), (unsigned)((NBLK - g0) * BLK), 0x00020000u};
        const unsigned lb = (unsigned)(unsigned long long)lds;
        int dry = AG_DRY ? (wave + 1) >> 2 : 0;
      for (;;) {
        int l1;
        asm volatile("v_mbcnt_lo_u32_b32 %0, -1, 0\n\tv_mbcnt_hi_u32_b32 %0, -1, %0" : "=v"(l1));
        unsigned vx = l1 * 16 + flip * BLK, have, need, pos, chk;
        f32x32 c[5];
        f32x16 c4;
        unsigned long long t1 = 0, t2 = 0;
        char* ob = a.out + ((size_t)(pm * BM + wave * 64) * NL + pn * BN) * 2;
        asm volatile(AG_KLOOP
                     : "={v[0:31]}"(c[0]), "={v[32:63]}"(c[1]), "={v[64:95]}"(c[2]), "={v[96:127]}"(c[3]), "={v[128:143]}"(c4),
                       "+{v242}"(vx), [hv] "=&s"(have), [nd] "=&s"(need), [ps] "=&s"(pos), [ck] "=&s"(chk), [t1] "+s"(t1), [t2] "+s"(t2)
                     : "{v241}"(lb + L_W + (l1 >> 2) * RS + (l1 & 3) * 8), "{v247}"(lb + L_CTL + wave * 4 + (l1 >> 6)), [rs] "s"(ra), [n8] "s"(part), "s"(ob),
                       [dry] "s"(dry)
                     : "v144", "v145", "v146", "v147", "v148", "v149", "v150", "v151", "v152", "v153", "v154", "v155", "v156", "v157", "v158", "v159", "v160", "v161", "v162", "v163", "v164", "v165", "v166", "v167", "v168", "v169", "v170", "v171", "v172", "v173", "v174", "v175", "v176", "v177", "v178", "v179", "v180", "v181", "v182", "v183", "v184", "v185", "v186", "v187", "v188", "v189", "v190", "v191", "v192", "v193", "v194", "v195", "v196", "v197", "v198", "v199", "v200", "v201", "v202", "v203", "v204", "v205", "v206", "v207", "v208", "v209", "v210", "v211", "v212", "v213", "v214", "v215", "v216", "v217", "v218", "v219", "v220", "v221", "v222", "v223", "v224", "v225", "v226", "v227", "v228", "v229", "v230", "v231", "v232", "v233", "v234", "v235", "v236", "v237", "v238", "v239", "v240", "v243", "v244", "v245", "v246", "scc", "memory");
#pragma unroll
        for (int i = 0; i < 16; ++i) c[4][i] = c4[i];
        int l2;
        asm volatile("v_mbcnt_lo_u32_b32 %0, -1, 0\n\tv_mbcnt_hi_u32_b32 %0, -1, %0" : "=v"(l2));
        constexpr int ES = RS + 16, E0 = NKT - NSLOT;
        unsigned pv = ctl[4 + (l2 & 7)];
        while (!__all(pv >= (unsigned)(E0 + 5))) {
            __builtin_amdgcn_s_sleep(1);
            pv = ctl[4 + (l2 & 7)];
        }
        const bool bad = __builtin_amdgcn_readfirstlane(pv) == ALL + 1;
        asm volatile("" ::: "memory");
        static_assert(E0 % NSLOT == 0 && 4 * 64 * ES <= 5 * SLOT, "epilogue staging");
        LAS unsigned char* st = lds + L_W + wave * 64 * ES;
        LAS unsigned char* sw = st + (l2 & 15) * ES + (l2 >> 4) * 8;
        LAS unsigned char* srS = st + (l2 >> 4) * ES + (l2 & 15) * 16;
        LAS unsigned char* srH = st + (l2 >> 1) * ES + 256 + (l2 & 1) * 16;
        const __amdgpu_buffer_rsrc_t ro = rsrc(ob, (64 * NL - pn * BN) * 2);
        const unsigned fl = flip * (32 * NL * 2), voS = (l2 >> 4) * (NL * 2) + (l2 & 15) * 16, voH = (l2 >> 1) * (NL * 2) + (l2 & 1) * 16 + oQ;
        const unsigned vo[2][2] = {{voS + fl, voH + fl}, {voS + (32 * NL * 2 - fl), voH + (part ? OOB : 32 * NL * 2)}};
        auto put = [&](int m) {
#pragma unroll
            for (int n = 0; n < NB; ++n) {
                const int e = 4 * (m * NB + n);
                const f32x4 v = (f32x4){c[e >> 5][e & 31], c[e >> 5][(e & 31) + 1], c[e >> 5][(e & 31) + 2], c[e >> 5][(e & 31) + 3]};
                *(LAS u32x2*)(sw + m * 16 * ES + n * 32) = __builtin_bit_cast(u32x2, __builtin_convertvector(v, bf16x4));
            }
        };
        u32x4 o[2][5];
        auto get = [&](int m) {
#pragma unroll
            for (int s = 0; s < 4; ++s) o[m & 1][s] = *(LAS u32x4*)(srS + (m * 16 + s * 4) * ES);
            if (m & 1) o[1][4] = *(LAS u32x4*)(srH + (m - 1) * 16 * ES);
        };
        auto out = [&](int m) {
#pragma unroll
            for (int s = 0; s < 4 + (m & 1); ++s)
                __builtin_amdgcn_raw_buffer_store_b128(o[m & 1][s], ro, s < 4 ? vo[m >> 1][0] + ((m & 1) * 16 + s * 4) * NL * 2 : vo[m >> 1][1],
                                                       0, AG_OAUX);
        };
        if (!bad) {
            put(0); get(0); put(1); SB(); out(0); get(1); put(2); SB(); out(1); get(2); put(3); SB(); out(2); get(3); out(3);
        } else {
#pragma unroll
            for (int s = 0; s < 10; ++s) o[s / 5][s % 5] = (u32x4){0x7fc07fc0u, 0x7fc07fc0u, 0x7fc07fc0u, 0x7fc07fc0u};
            out(0); out(1); out(2); out(3);
        }
        asm volatile("s_mov_b64 exec, -1\n\ts_sub_u32 %0, %0, 1" : "+s"(dry) :: "scc", "memory");
        if (dry < 0) break;
      }
    } else if (wave == 4) {
        __builtin_amdgcn_s_setprio(AG_WPRIO);
        unsigned long long gp[NPIECE / 2], gs[NPIECE / 2];
#pragma unroll
        for (int i = 0; i < NPIECE / 2; ++i) {
            const unsigned o = i * 1024 + lane * 16, r = o / RS, cb = o % RS;
            const unsigned k = ((r >> 2) & 3) * 8 + (r >> 4) * 4 + (r & 3);
            gp[i] = (unsigned long long)a.w + pn * BN * 2 + k * (NL * 2) + (cb < 256 ? cb : pq < 0 ? cb - 32 : cb - 256 + oQ);
            gs[i] = 32 * NL * 2;
        }
        const __amdgpu_buffer_rsrc_t rall = rsrc(a.w, K * NL * 2);
        const unsigned ln = ((xcd >> 1) * (4 * NTN) + tl) * 9 + lane, po = lane < 9 && ln < BK * NL * 2 / 128 ? ln * 128 + AG_PD * (BK * NL * 2) : OOB;
        int pub = 0;
        unsigned so = 0;
        for (int kt = 0; kt < NKT; ++kt) {
            if (kt >= NSLOT) {
                const unsigned nd = kt - NSLOT + 1;
                if (!__all(prog[lane & 3] >= nd)) {
                    if (pub < kt) {
                        asm volatile("s_waitcnt vmcnt(0)" ::: "memory");
                        pub = kt;
                        ctl[lane & 3] = pub;
                    }
                    while (!__all(prog[lane & 3] >= nd)) __builtin_amdgcn_s_sleep(AG_WSLEEP);
                }
                asm volatile("" ::: "memory");
                if ((nd & 7) == 0) STAMP(16 + (nd >> 3));
            }
            if (kt == NSLOT && pq >= 0) {
                const unsigned long long t0 = rt();
                while (!__all((int)(__hip_atomic_load((GAS unsigned*)(myws + W_SCNT + (lane & 7) * 128), __ATOMIC_RELAXED, __HIP_MEMORY_SCOPE_AGENT) - call * (NSTRIP / 8)) >= 0)) {
                    if (give_up(tmo, call, t0)) {
                        __hip_atomic_store(tmo, call, __ATOMIC_RELAXED, __HIP_MEMORY_SCOPE_AGENT);
                        break;
                    }
                    __builtin_amdgcn_s_sleep(4);
                }
                FENCE();
#pragma unroll
                for (int i = 0; i < NPIECE / 2; ++i) {
                    const unsigned o = i * 1024 + lane * 16, r = o / RS, cb = o % RS;
                    if (cb >= 256) gp[i] = (unsigned long long)myws + W_STRIP + pq * (K * 32) + (NSLOT * BK + r) * 32 + cb - 256, gs[i] = 1024;
                }
            }
            LAS unsigned char* dst = lds + L_W + so;
            {
#pragma unroll
                for (int i = 0; i < NPIECE; ++i)
                {
                    __builtin_amdgcn_global_load_lds((const GAS void*)gp[i % (NPIECE / 2)], (LAS void*)(dst + i * 1024), 16, 0, 0);
                    gp[i % (NPIECE / 2)] += gs[i % (NPIECE / 2)];
                }
            }
            if (AG_PD) __builtin_amdgcn_raw_ptr_buffer_load_lds(rall, (LAS void*)(lds + L_SINK), 4, po + kt * (BK * NL * 2), 0, 0, 0);
            so = so + SLOT == NSLOT * SLOT ? 0u : so + SLOT;
            if (kt + 1 - pub > 2) {
                asm volatile("s_waitcnt vmcnt(%0)" :: "n"(2 * NPIECE + (AG_PD ? 3 : 0)) : "memory");
                ++pub;
                ctl[lane & 3] = pub;
            }
        }
        asm volatile("s_waitcnt vmcnt(0)" ::: "memory");
        ctl[lane & 3] = ALL;
    }
    else if (wave == 5) {
        __builtin_amdgcn_s_setprio(AG_PPRIO);
        GAS unsigned* flag = (GAS unsigned*)(myws + W_FLAG) + pm * NKT * 8 * FS;
        const unsigned long long t0 = rt();
        int cur = 0, win = AG_PW, last = 0;
        unsigned long long npoll = 0;
        while (cur < NKT) {
            ++npoll;
            const int idx = cur * 8 + lane;
            unsigned v = call;
            if (lane < win * 8 && idx < NKT * 8) v = __hip_atomic_load(flag + idx * FS, __ATOMIC_RELAXED, __HIP_MEMORY_SCOPE_SYSTEM);
            const unsigned long long miss = __ballot((int)(v - call) < 0);
            const int adv = miss ? (__builtin_ctzll(miss) >> 3) : win;
            if (AG_APW) { win = adv > 0 && last > 0 ? AG_PW : 1; last = adv; }
            if (adv > 0) {
                if (cur == 0) STAMP(4);
                if (((cur + adv) >> 3) != (cur >> 3)) STAMP(24 + (((cur + adv) >> 3) & 7));
                cur = cur + adv < NKT ? cur + adv : NKT;
                asm volatile("" ::: "memory");
                ctl[4 + (lane & 3)] = cur < NKT ? cur : ALL;
            } else {
                if (give_up(tmo, call, t0)) {
                    __hip_atomic_store(tmo, call, __ATOMIC_RELAXED, __HIP_MEMORY_SCOPE_AGENT);
                    ctl[4 + (lane & 3)] = ALL + 1;
                    break;
                }
                __builtin_amdgcn_s_sleep(AG_PSLEEP);
            }
        }
        STAMP(5);
    }
    } else {
        LAS unsigned char* lx = lds + L_W + wave * 8192;
        LAS unsigned char* ls = lx + 2048;
        LAS unsigned short* lp = (LAS unsigned short*)(lx + 4096);
        if (wave == 0) STAMP(11);
        const size_t xt_off = W_XT + (size_t)(call & 1u) * XT_BYTES, pk_off = W_PK + (size_t)(call & 1u) * PK_BYTES;
        if (wave < SW) {
            const int sid = wave * NSC + cid;
            __amdgpu_buffer_rsrc_t rd[NR];
#pragma unroll
            for (int i = 0; i < NR; ++i) rd[i] = rsrc(a.ws[i], (unsigned)W_END);
            const unsigned xt0 = (unsigned)xt_off + a.rank * NBLK * BLK, pk0 = (unsigned)pk_off + a.rank * NBLK * PBLK;
            const unsigned tbase = (call - 1u) * (unsigned)NBLK - 2u * NSEND;
            const unsigned tag = tag_of(call);
            auto low = [&](const u32x4 (&d)[4], int j, unsigned& e) {
                const unsigned d0 = d[j >> 1][(j & 1) * 2], d1 = d[j >> 1][(j & 1) * 2 + 1];
                const unsigned r0 = __builtin_amdgcn_alignbit(d0, d0, 31), r1 = __builtin_amdgcn_alignbit(d1, d1, 31);
                e = perm(r1, r0, 0x07050301u);
                return perm(r1, r0, 0x06040200u);
            };
            auto pack = [&](const u32x4 (&d)[4], u32x4 (&pk)[3], bool& raw, bool& annex) {
                unsigned e[8], sum = 0;
                *(LAS u32x2*)(ls + 512 + lane * 8) = (u32x2){0u, 0u};
#pragma unroll
                for (int j = 0; j < 8; ++j) {
                    pk[j >> 2][j & 3] = low(d, j, e[j]);
                    sum = __builtin_amdgcn_sad_u8(e[j], 0u, sum);
                }
                const unsigned mean = (unsigned)__builtin_amdgcn_readlane(scan(sum), 63) >> 11;
                const unsigned e0 = mean < 7u ? 7u : mean > 252u ? 252u : mean;
                const unsigned k2 = key2(e0), t2 = table2(e0), t3l = table3(e0, 0), t3h = table3(e0, 1);
                u32x4 x[2];
                u32x2 c = {0u, 0u}, mv[8];
                unsigned mask = 0, i3[8];
#pragma unroll
                for (int j = 0; j < 8; ++j) {
                    const unsigned sel = (e[j] ^ k2) & 3u * B1;
                    const unsigned t = ((perm(t2, t2, sel) ^ e[j]) >> 1) + 0x7f7f7f7fu;
                    const unsigned m = perm(t, t << 8, 0x0b090a08u);
                    const unsigned code = sel | (m & 3u * B1);
                    c[j >> 2] |= __builtin_amdgcn_udot4(code, 0x40100401u, 0u, false) << (8 * (j & 3));
                    mv[j] = *(LAS u32x2*)(lm + __builtin_amdgcn_udot4(code & (code >> 1) & B1, 0x40201008u, 0u, false));
                    i3[j] = (e[j] ^ e0 * B1) & m & 7u * B1;
                    const unsigned y = (perm(t3h, t3l, i3[j]) ^ e[j]) & m;
                    x[j >> 2][j & 3] = y;
                    mask = ((((y & 0x7f7f7f7fu) + 0x7f7f7f7fu) | y) & 0x80808080u) | (mask >> 1);
                }
                *(LAS u32x4*)(lx + lane * 32) = x[0];
                *(LAS u32x4*)(lx + lane * 32 + 16) = x[1];
                *(LAS u32x2*)(ls + lane * 8) = c;
                unsigned cnt = 0;
                for (;;) {
                    const unsigned long long act = __ballot(mask != 0u);
                    if (act == 0ull || cnt > NPATCH) break;
                    const unsigned p = __builtin_ctz(mask | 0x80000000u);
                    const unsigned at = __builtin_amdgcn_mbcnt_hi((unsigned)(act >> 32), __builtin_amdgcn_mbcnt_lo((unsigned)act, cnt));
                    lp[mask != 0u ? at : 127u] = (unsigned short)(lane * 32 + (p & 7u) * 4 + (p >> 3));
                    mask &= mask - 1u;
                    cnt += __builtin_popcountll(act);
                }
                FENCE();
                const unsigned pos = lp[lane] & 0x7ffu;
                unsigned long long acc = 0;
                unsigned nb = 0;
#pragma unroll
                for (int j = 7; j >= 0; --j) {
                    const unsigned g = perm(0u, i3[j], mv[j].x);
                    acc = (acc << mv[j].y) | __builtin_amdgcn_udot4(g, 0x003e0801u, g >> 15, false);
                    nb += mv[j].y;
                }
                const unsigned xb = lx[pos];
                const unsigned in = scan(nb), off = in - nb + cnt * 32u;
                const unsigned nd = cnt + (((unsigned)__builtin_amdgcn_readlane(in, 63) + 31u) >> 5);
                raw = cnt > NPATCH || nd > NLONG || __ballot(nb > 63u) != 0ull;
                annex = !raw && nd > NTAIL;
                LAS unsigned* lt = (LAS unsigned*)(ls + 512) + HW;
                const unsigned long long sa = acc << (off & 31u);
                __hip_atomic_fetch_or(lt + (off >> 5), (unsigned)sa, __ATOMIC_RELAXED, __HIP_MEMORY_SCOPE_WAVEFRONT);
                __hip_atomic_fetch_or(lt + (off >> 5) + 1, (unsigned)(sa >> 32), __ATOMIC_RELAXED, __HIP_MEMORY_SCOPE_WAVEFRONT);
                __hip_atomic_fetch_or(lt + (off >> 5) + 2, (unsigned)(acc >> 33) >> (~off & 31u), __ATOMIC_RELAXED, __HIP_MEMORY_SCOPE_WAVEFRONT);
                lt[(unsigned)lane < cnt ? lane : 300] = pos | (xb << 16);
                FENCE();
                pk[2] = *(LAS u32x4*)(ls + lane * 16);
                FENCE();
                const unsigned long long m0 = __ballot((int)pk[0].x < 0), m1 = __ballot((int)pk[1].x < 0);
                if (lane == 33) pk[2].x = (unsigned)m1, pk[2].y = (unsigned)(m1 >> 32);
                const unsigned long long m2 = __ballot((int)pk[2].x < 0);
                if (lane == 32) pk[2] = (u32x4){e0 | (raw ? 256u : cnt << 10) | (annex ? 512u : 0u) | (unsigned)(m2 >> 33) << 16, (unsigned)m0, (unsigned)(m0 >> 32), (unsigned)m2};
#pragma unroll
                for (int s = 0; s < 3; ++s) pk[s].x = bfi(s < 2 || lane < 48 ? 0x7fffffffu : ~0u, pk[s].x, tag);
            };
            unsigned b0 = sid, b1 = sid + NSEND;
            u32x4 nx[4], pk[3];
            bool raw, annex;
            unsigned skip = 0;
            if (AG_EARLY == 1 || wave < AG_EARLY - 1) {
#pragma unroll
                for (int j = 0; j < 8; ++j) pk[j >> 2][j & 3] = low(d, j, skip);
#pragma unroll
                for (int s = 0; s < 2; ++s) pk[s].x = (pk[s].x & 0x7fffffffu) | tag;
#pragma unroll
                for (int i = 0; i < NR - 1; ++i)
#pragma unroll
                    for (int s = 0; s < 2; ++s)
                        store16(pk[s], rd[i], pk0 + b0 * PBLK + lane * 16 + s * 1024);
                SB();
                skip = OOB;
            }
            pack(d, pk, raw, annex);
            if (wave == 0) STAMP(6);
            unsigned long long nsent = 0;
            unsigned seq = wave;
            while (b0 < (unsigned)NBLK) {
                const unsigned sp = pk0 + b0 * PBLK + lane * 16;
                if (annex) {
#pragma unroll
                    for (int i = 0; i < NR - 1; ++i)
                        store16(pk[2], rd[i], (lane >> 3) == 6 ? sp + 2048 : OOB);
                    wait_vm0();
                }
                while (ctl[0] < seq) __builtin_amdgcn_s_sleep(1);
                FENCE();
                if (raw) {
                    const unsigned so = xt0 + b0 * BLK + lane * 16;
#pragma unroll
                    for (int i = 0; i < NR - 1; ++i)
#pragma unroll
                        for (int s = 0; s < 4; ++s)
                            store16(d[s], rd[i], so + s * 1024);
                    wait_vm0();
                }
                TRACE(a.rank, b0, 0);
                fetch(nx, b1);
                const unsigned tk = __builtin_amdgcn_raw_ptr_buffer_atomic_add_i32(1, rd[NR - 1], lane == 0 ? (unsigned)W_TICKET : OOB, 0, 0);
                const unsigned sp2 = lane < 48 ? sp + 2048 : OOB;
#pragma unroll
                for (int i = 0; i < NR - 1; ++i)
#pragma unroll
                    for (int s = 0; s < 2; ++s)
                        store16(pk[s], rd[i], (sp | skip) + s * 1024);
                skip = 0;
#pragma unroll
                for (int i = 0; i < NR - 1; ++i)
                    store16(pk[2], rd[i], sp2);
                FENCE();
                if (lane == 0) __hip_atomic_fetch_max((LAS unsigned*)(lds + L_CTL), seq + 1, __ATOMIC_RELAXED, __HIP_MEMORY_SCOPE_WORKGROUP);
                seq = AG_TURN == 2 ? 0 : seq + SW;
                TRACE(a.rank, b0, 1);
#pragma unroll
                for (int s = 0; s < 4; ++s) d[s] = nx[s];
                pack(d, pk, raw, annex);
                b0 = b1;
                ++nsent;
                b1 = __builtin_amdgcn_readfirstlane(tk) - tbase;
            }
            ctl[0] = ~0u;
            if (wave == 0) STAMP(7);
        }
    }
}
}

extern "C" void kernel_launch(int rank, int nranks, ncclComm_t comm,
                              hipStream_t stream,
                              void* const* d_in, const int* in_sizes, int n_in,
                              void* d_out, int out_size,
                              const int* in_dtypes, int out_dtype,
                              void* const* d_out_all,
                              void* const* d_ws_all, size_t ws_bytes) {
    if (nranks != ag::NR || (unsigned)rank >= (unsigned)ag::NR || n_in != 2 || ws_bytes < ag::W_END ||
        in_sizes[0] != ag::MS * ag::K || in_sizes[1] != ag::K * ag::NL || out_size != ag::M * ag::NL) {
        fprintf(stderr, "ag_gemm: unexpected problem shape\n");
        return;
    }
    ag::Args a;
    a.x = (const char*)d_in[0];
    a.w = (const char*)d_in[1];
    a.out = (char*)d_out;
    for (int i = 0; i < ag::NR; ++i) a.ws[i] = (char*)d_ws_all[(rank + 1 + i) % ag::NR];
    a.rank = rank;
    a.pad = 0;
    hipLaunchKernelGGL(ag::ag_gemm, dim3(ag::NWG), dim3(ag::NTHR), 0, stream, a);
    HIP_CHECK(hipGetLastError());
}
```

```cpp
#include <hip/hip_runtime.h>
#include <rccl/rccl.h>
#include <cstdio>
#include <cstdint>
#include <type_traits>

#define HIP_CHECK(expr) do { hipError_t _e = (expr); if (_e != hipSuccess) { \
    fprintf(stderr, "HIP error %s at %s:%d\n", hipGetErrorString(_e), __FILE__, __LINE__); \
    return; } } while (0)

#define LAS __attribute__((address_space(3)))
#define GAS __attribute__((address_space(1)))
typedef unsigned u32x4 __attribute__((ext_vector_type(4)));
typedef unsigned u32x2 __attribute__((ext_vector_type(2)));
typedef unsigned short u16x2 __attribute__((ext_vector_type(2)));
typedef short s16x4 __attribute__((ext_vector_type(4)));
typedef short s16x8 __attribute__((ext_vector_type(8)));
typedef __bf16 bf16x4 __attribute__((ext_vector_type(4)));
typedef __bf16 bf16x8 __attribute__((ext_vector_type(8)));
typedef float f32x4 __attribute__((ext_vector_type(4)));
typedef float f32x16 __attribute__((ext_vector_type(16)));
typedef float f32x32 __attribute__((ext_vector_type(32)));

#define AG_SW 4
#define AG_PSLEEP 2
#define AG_TURN 1
#define AG_UPRIO 0
#define AG_PF 2
#define AG_PF0 3
#define AG_PFAUX 0
#define AG_PFD 0
#define AG_CPRIO 0
#define AG_WSLEEP 2
#define AG_WPRIO 0
#define AG_T1 1
#define AG_A32 1
#define AG_PQ 0
#define AG_PPRIO 2
#define AG_DRY 1
#define AG_SPEC 1
#define AG_CPF 1
#define AG_PD 0
#define AG_UHI 0
#define AG_BAL 0
#define AG_APW 1
#define AG_TP 0
#define AG_TW 0
#define AG_TS 6
#define AG_STRIP 1
#define AG_HOLE 0
#define AG_EARLY 3
#define AG_SD 0
#define AG_PP 0
#define AG_MAP 1
#define AG_FAUX 17
#define AG_USLEEP 1
#define AG_HAUX 17
#define AG_UAUX 17
#define AG_FS 32
#define AG_PW 2
#define AG_OAUX 17

namespace ag {
constexpr int NR = 8;
constexpr int M = 4096, K = 4096, NL = 2048;
constexpr int MS = M / NR;
constexpr int NB = 9;
constexpr int BM = 256, BN = 128, BK = 64;
constexpr int NKT = K / BK;
constexpr int NG = MS / 32;
constexpr int BLK = 32 * BK * 2;
constexpr int NBLK = NG * NKT;
constexpr int RS = NB * 32;
constexpr int SLOT = BK * RS, NSLOT = 8, NPIECE = SLOT / 1024;
constexpr int NTN = NL / BN - 1;
constexpr int NWG = 256, CPX = 4;
constexpr int NWAVE = 8, NTHR = NWAVE * 64;
constexpr int NSC = NWG - 16 * NTN, SW = AG_SW, NSEND = NSC * SW;
static_assert(2 * NSEND <= NBLK && SW <= NWAVE, "sender geometry");
constexpr int PBLK = 3072;
constexpr unsigned HW = 6, NPATCH = 32 - HW, NTAIL = 64 - HW, NLONG = NTAIL + 32;
constexpr int NU = 4 * NTN;
constexpr int L_SEL = 0, L_W = 2048;
constexpr int L_CTL = L_W + NSLOT * SLOT;
constexpr unsigned ALL = 1000;
constexpr int L_UNP = L_CTL + 128, UNP = 2048 + 64 + 448;
constexpr int L_SINK = L_UNP + 2 * UNP, LDS_BYTES = L_SINK + 256;
static_assert(LDS_BYTES > 80 * 1024 && LDS_BYTES <= 160 * 1024, "one workgroup per CU");
constexpr size_t W_CNT = 0;
constexpr size_t W_TMO = W_CNT + (size_t)NWG * 128;
constexpr size_t W_TICKET = W_TMO + 1024;
constexpr size_t W_SCNT = W_TICKET + 1024;
constexpr size_t W_DBG = 768 * 1024;
constexpr size_t W_XT = 1u << 20;
constexpr size_t XT_BYTES = (size_t)M * K * 2;
constexpr size_t W_PK = W_XT + 2 * XT_BYTES;
constexpr size_t PK_BYTES = (size_t)NR * NBLK * PBLK;
constexpr size_t W_FLAG = W_PK + 2 * PK_BYTES;
constexpr int FS = AG_FS;
constexpr unsigned FL_BYTES = NR * NBLK * FS * 4;
constexpr size_t W_STRIP = W_FLAG + FL_BYTES;
constexpr size_t W_END = W_STRIP + (size_t)K * BN * 2;
constexpr int NSTRIP = (NKT - NSLOT) * 16;
static_assert(W_SCNT + 1024 <= W_DBG && NSTRIP <= 16 * NTN * 4, "flags");
constexpr unsigned OOB = 0x40000000u;
constexpr unsigned long long DEADLINE = 30000000ull;

struct Args {
    const char* x;
    const char* w;
    char* out;
    char* ws[NR];
    int rank;
    int pad;
};

__device__ __forceinline__ unsigned long long rt() { return __builtin_amdgcn_s_memrealtime(); }
__device__ __forceinline__ __amdgpu_buffer_rsrc_t rsrc(const void* p, unsigned bytes) {
    return __builtin_amdgcn_make_buffer_rsrc(const_cast<void*>(p), 0, bytes, 0x00020000);
}
__device__ __forceinline__ void store16(u32x4 d, __amdgpu_buffer_rsrc_t r, unsigned off) {
    __builtin_amdgcn_raw_buffer_store_b128(d, r, off, 0, 17);
}
__device__ __forceinline__ void wait_vm0() {
    __builtin_amdgcn_s_waitcnt(0x0F70);
    asm volatile("s_waitcnt vmcnt(0)" ::: "memory");
}
__device__ __forceinline__ bool give_up(GAS unsigned* tmo, unsigned call, unsigned long long t0) {
    const unsigned long long dt = rt() - t0;
    if (dt < 50000ull) return false;
    return dt > DEADLINE || __builtin_amdgcn_readfirstlane(__hip_atomic_load(tmo, __ATOMIC_RELAXED, __HIP_MEMORY_SCOPE_AGENT)) == call;
}
constexpr unsigned B1 = 0x01010101u;
__device__ __forceinline__ unsigned tag_of(unsigned call) { return ((call + 1u) << 30) & 0x80000000u; }
__device__ __forceinline__ unsigned bfi(unsigned m, unsigned a, unsigned b) { return (a & m) | (b & ~m); }
__device__ __forceinline__ unsigned perm(unsigned hi, unsigned lo, unsigned sel) { return __builtin_amdgcn_perm(hi, lo, sel); }
__device__ __forceinline__ unsigned key2(unsigned e0) { return (((e0 + 3u) & 3u) ^ 3u) * B1; }
__device__ __forceinline__ unsigned table2(unsigned e0) {
    return e0 * B1 + (((0x03020100u ^ key2(e0)) + (4u - (e0 & 3u)) * B1) & 3u * B1);
}
__device__ __forceinline__ unsigned table3(unsigned e0, unsigned hi) {
    const unsigned r = (e0 & 7u) * B1;
    return e0 * B1 - ((8u * B1 + r - ((0x03020100u + hi * 4u * B1) ^ r)) & 7u * B1) + (hi ? 0u : 3u);
}
__device__ __forceinline__ unsigned scan(unsigned v) {
    v += (unsigned)__builtin_amdgcn_update_dpp(0, (int)v, 0x111, 0xf, 0xf, true);
    v += (unsigned)__builtin_amdgcn_update_dpp(0, (int)v, 0x112, 0xf, 0xf, true);
    v += (unsigned)__builtin_amdgcn_update_dpp(0, (int)v, 0x114, 0xf, 0xf, true);
    v += (unsigned)__builtin_amdgcn_update_dpp(0, (int)v, 0x118, 0xf, 0xf, true);
    v += (unsigned)__builtin_amdgcn_update_dpp(0, (int)v, 0x142, 0xa, 0xf, false);
    v += (unsigned)__builtin_amdgcn_update_dpp(0, (int)v, 0x143, 0xc, 0xf, false);
    return v;
}
__device__ __forceinline__ u32x2 mover(unsigned m, bool gather) {
    unsigned sel = gather ? 12u * B1 : 0u, n = 0;
#pragma unroll
    for (unsigned b = 0; b < 4; ++b) {
        const unsigned c = gather ? ((m >> b) & 1u) * 3u : (m >> (2u * b)) & 3u;
        if (gather) sel ^= c ? (12u ^ b) << (8u * n) : 0u;
        else sel |= (c == 3u ? n : 4u + c) << (8u * b);
        n += c == 3u;
    }
    return (u32x2){sel, 3u * n};
}
#define STAMP(i) do {} while (0)
#define STAMPC(i) do {} while (0)
#undef TRACE
#define TRACE(s, b, i) do {} while (0)

#define SB() __builtin_amdgcn_sched_barrier(0)
#define FENCE() asm volatile("" ::: "memory")
#define AG_T0 ""
#define AG_KLOOP \
    "s_cmp_eq_u32 %[dry], 0\n" \
    "s_cbranch_scc1 GO%=\n" \
    "v_mov_b32 v244, -2\n" \
    "s_mov_b64 exec, 0\n" \
    "s_mov_b32 %[nd], 64\n" \
    "s_mov_b32 %[ps], 0\n" \
    "s_branch R1%=\n" \
    "GO%=:\n" \
    "v_mov_b64 v[0:1], 0\n" \
    "v_mov_b64 v[2:3], 0\n" \
    "v_mov_b64 v[4:5], 0\n" \
    "v_mov_b64 v[6:7], 0\n" \
    "v_mov_b64 v[8:9], 0\n" \
    "v_mov_b64 v[10:11], 0\n" \
    "v_mov_b64 v[12:13], 0\n" \
    "v_mov_b64 v[14:15], 0\n" \
    "v_mov_b64 v[16:17], 0\n" \
    "v_mov_b64 v[18:19], 0\n" \
    "v_mov_b64 v[20:21], 0\n" \
    "v_mov_b64 v[22:23], 0\n" \
    "v_mov_b64 v[24:25], 0\n" \
    "v_mov_b64 v[26:27], 0\n" \
    "v_mov_b64 v[28:29], 0\n" \
    "v_mov_b64 v[30:31], 0\n" \
    "v_mov_b64 v[32:33], 0\n" \
    "v_mov_b64 v[34:35], 0\n" \
    "v_mov_b64 v[36:37], 0\n" \
    "v_mov_b64 v[38:39], 0\n" \
    "v_mov_b64 v[40:41], 0\n" \
    "v_mov_b64 v[42:43], 0\n" \
    "v_mov_b64 v[44:45], 0\n" \
    "v_mov_b64 v[46:47], 0\n" \
    "v_mov_b64 v[48:49], 0\n" \
    "v_mov_b64 v[50:51], 0\n" \
    "v_mov_b64 v[52:53], 0\n" \
    "v_mov_b64 v[54:55], 0\n" \
    "v_mov_b64 v[56:57], 0\n" \
    "v_mov_b64 v[58:59], 0\n" \
    "v_mov_b64 v[60:61], 0\n" \
    "v_mov_b64 v[62:63], 0\n" \
    "v_mov_b64 v[64:65], 0\n" \
    "v_mov_b64 v[66:67], 0\n" \
    "v_mov_b64 v[68:69], 0\n" \
    "v_mov_b64 v[70:71], 0\n" \
    "v_mov_b64 v[72:73], 0\n" \
    "v_mov_b64 v[74:75], 0\n" \
    "v_mov_b64 v[76:77], 0\n" \
    "v_mov_b64 v[78:79], 0\n" \
    "v_mov_b64 v[80:81], 0\n" \
    "v_mov_b64 v[82:83], 0\n" \
    "v_mov_b64 v[84:85], 0\n" \
    "v_mov_b64 v[86:87], 0\n" \
    "v_mov_b64 v[88:89], 0\n" \
    "v_mov_b64 v[90:91], 0\n" \
    "v_mov_b64 v[92:93], 0\n" \
    "v_mov_b64 v[94:95], 0\n" \
    "v_mov_b64 v[96:97], 0\n" \
    "v_mov_b64 v[98:99], 0\n" \
    "v_mov_b64 v[100:101], 0\n" \
    "v_mov_b64 v[102:103], 0\n" \
    "v_mov_b64 v[104:105], 0\n" \
    "v_mov_b64 v[106:107], 0\n" \
    "v_mov_b64 v[108:109], 0\n" \
    "v_mov_b64 v[110:111], 0\n" \
    "v_mov_b64 v[112:113], 0\n" \
    "v_mov_b64 v[114:115], 0\n" \
    "v_mov_b64 v[116:117], 0\n" \
    "v_mov_b64 v[118:119], 0\n" \
    "v_mov_b64 v[120:121], 0\n" \
    "v_mov_b64 v[122:123], 0\n" \
    "v_mov_b64 v[124:125], 0\n" \
    "v_mov_b64 v[126:127], 0\n" \
    "v_mov_b64 v[128:129], 0\n" \
    "v_mov_b64 v[130:131], 0\n" \
    "v_mov_b64 v[132:133], 0\n" \
    "v_mov_b64 v[134:135], 0\n" \
    "v_mov_b64 v[136:137], 0\n" \
    "v_mov_b64 v[138:139], 0\n" \
    "v_mov_b64 v[140:141], 0\n" \
    "v_mov_b64 v[142:143], 0\n" \
    "v_mov_b32 v240, v241\n" \
    "v_xor_b32 v243, 0x1000, v242\n" \
    "v_mov_b32 v246, 0\n" \
    "s_mov_b32 %[nd], 1\n" \
    "s_mov_b32 %[ps], 0\n" \
    "P0%=:\n" \
    "ds_read2_b32 v[244:245], v247 offset1:4\n" \
    "s_waitcnt lgkmcnt(0)\n" \
    "v_min_u32 v244, v244, v245\n" \
    "s_nop 0\n" \
    "v_readfirstlane_b32 %[hv], v244\n" \
    "s_cmp_ge_u32 %[hv], %[nd]\n" \
    "s_cbranch_scc1 R0%=\n" \
    "s_sleep 1\n" \
    "s_branch P0%=\n" \
    "R0%=:\n" \
    "buffer_load_dwordx4 v[144:147], v242, %[rs], 0 offen offset:0 sc0 sc1\n" \
    "buffer_load_dwordx4 v[148:151], v242, %[rs], 0 offen offset:1024 sc0 sc1\n" \
    "buffer_load_dwordx4 v[152:155], v242, %[rs], 0 offen offset:2048 sc0 sc1\n" \
    "buffer_load_dwordx4 v[156:159], v242, %[rs], 0 offen offset:3072 sc0 sc1\n" \
    "buffer_load_dwordx4 v[160:163], v243, %[rs], 0 offen offset:0 sc0 sc1\n" \
    "buffer_load_dwordx4 v[164:167], v243, %[rs], 0 offen offset:1024 sc0 sc1\n" \
    "buffer_load_dwordx4 v[168:171], v243, %[rs], 0 offen offset:2048 sc0 sc1\n" \
    "buffer_load_dwordx4 v[172:175], v243, %[rs], 0 offen offset:3072 sc0 sc1\n" \
    "v_add_u32 v242, 0x10000, v242\n" \
    "v_add_u32 v243, 0x10000, v243\n" \
    "s_mov_b32 %[nd], 2\n" \
    "P1%=:\n" \
    "ds_read2_b32 v[244:245], v247 offset1:4\n" \
    "s_waitcnt lgkmcnt(0)\n" \
    "v_min_u32 v244, v244, v245\n" \
    "s_nop 0\n" \
    "v_readfirstlane_b32 %[hv], v244\n" \
    "s_cmp_ge_u32 %[hv], %[nd]\n" \
    "s_cbranch_scc1 R1%=\n" \
    "s_sleep 1\n" \
    "s_branch P1%=\n" \
    "R1%=:\n" \
    AG_T0 \
    "s_cmp_eq_u32 %[n8], 0\n" \
    "s_cbranch_scc0 E8%=\n" \
    "ds_read_b64_tr_b16 v[208:209], v240 offset:0\n" \
    "ds_read_b64_tr_b16 v[210:211], v240 offset:4608\n" \
    "ds_read_b64_tr_b16 v[212:213], v240 offset:9216\n" \
    "ds_read_b64_tr_b16 v[214:215], v240 offset:13824\n" \
    "ds_read_b64_tr_b16 v[216:217], v240 offset:32\n" \
    "ds_read_b64_tr_b16 v[218:219], v240 offset:4640\n" \
    "ds_read_b64_tr_b16 v[220:221], v240 offset:9248\n" \
    "ds_read_b64_tr_b16 v[222:223], v240 offset:13856\n" \
    "s_waitcnt lgkmcnt(4) vmcnt(0)\n" \
    "L9_%=:\n" \
    "v_mfma_f32_16x16x32_bf16 v[0:3], v[208:211], v[144:147], v[0:3]\n" \
    "ds_read_b64_tr_b16 v[224:225], v240 offset:64\n" \
    "v_mfma_f32_16x16x32_bf16 v[0:3], v[212:215], v[148:151], v[0:3]\n" \
    "ds_read_b64_tr_b16 v[226:227], v240 offset:4672\n" \
    "v_mfma_f32_16x16x32_bf16 v[36:39], v[212:215], v[156:159], v[36:39]\n" \
    "ds_read_b64_tr_b16 v[228:229], v240 offset:9280\n" \
    "v_mfma_f32_16x16x32_bf16 v[36:39], v[208:211], v[152:155], v[36:39]\n" \
    "ds_read_b64_tr_b16 v[230:231], v240 offset:13888\n" \
    "v_mfma_f32_16x16x32_bf16 v[72:75], v[208:211], v[160:163], v[72:75]\n" \
    "buffer_load_dwordx4 v[176:179], v242, %[rs], 0 offen offset:0 sc0 sc1\n" \
    "v_mfma_f32_16x16x32_bf16 v[72:75], v[212:215], v[164:167], v[72:75]\n" \
    "buffer_load_dwordx4 v[180:183], v242, %[rs], 0 offen offset:1024 sc0 sc1\n" \
    "v_mfma_f32_16x16x32_bf16 v[108:111], v[212:215], v[172:175], v[108:111]\n" \
    "buffer_load_dwordx4 v[184:187], v242, %[rs], 0 offen offset:2048 sc0 sc1\n" \
    "v_mfma_f32_16x16x32_bf16 v[108:111], v[208:211], v[168:171], v[108:111]\n" \
    "s_waitcnt lgkmcnt(4)\n" \
    "v_mfma_f32_16x16x32_bf16 v[112:115], v[216:219], v[168:171], v[112:115]\n" \
    "ds_read_b64_tr_b16 v[208:209], v240 offset:96\n" \
    "s_add_u32 %[nd], %[nd], 1\n" \
    "v_mfma_f32_16x16x32_bf16 v[112:115], v[220:223], v[172:175], v[112:115]\n" \
    "ds_read_b64_tr_b16 v[210:211], v240 offset:4704\n" \
    "s_add_u32 %[ps], %[ps], 18432\n" \
    "v_mfma_f32_16x16x32_bf16 v[76:79], v[220:223], v[164:167], v[76:79]\n" \
    "ds_read_b64_tr_b16 v[212:213], v240 offset:9312\n" \
    "s_cmp_eq_u32 %[ps], 147456\n" \
    "v_mfma_f32_16x16x32_bf16 v[76:79], v[216:219], v[160:163], v[76:79]\n" \
    "ds_read_b64_tr_b16 v[214:215], v240 offset:13920\n" \
    "s_cselect_b32 %[ps], 0, %[ps]\n" \
    "v_mfma_f32_16x16x32_bf16 v[40:43], v[216:219], v[152:155], v[40:43]\n" \
    "buffer_load_dwordx4 v[188:191], v242, %[rs], 0 offen offset:3072 sc0 sc1\n" \
    "v_mfma_f32_16x16x32_bf16 v[40:43], v[220:223], v[156:159], v[40:43]\n" \
    "buffer_load_dwordx4 v[192:195], v243, %[rs], 0 offen offset:0 sc0 sc1\n" \
    "v_mfma_f32_16x16x32_bf16 v[4:7], v[220:223], v[148:151], v[4:7]\n" \
    "buffer_load_dwordx4 v[196:199], v243, %[rs], 0 offen offset:1024 sc0 sc1\n" \
    "v_mfma_f32_16x16x32_bf16 v[4:7], v[216:219], v[144:147], v[4:7]\n" \
    "s_waitcnt lgkmcnt(4)\n" \
    "v_mfma_f32_16x16x32_bf16 v[8:11], v[224:227], v[144:147], v[8:11]\n" \
    "ds_read_b64_tr_b16 v[216:217], v240 offset:128\n" \
    "v_mfma_f32_16x16x32_bf16 v[8:11], v[228:231], v[148:151], v[8:11]\n" \
    "ds_read_b64_tr_b16 v[218:219], v240 offset:4736\n" \
    "v_mfma_f32_16x16x32_bf16 v[44:47], v[228:231], v[156:159], v[44:47]\n" \
    "ds_read_b64_tr_b16 v[220:221], v240 offset:9344\n" \
    "v_mfma_f32_16x16x32_bf16 v[44:47], v[224:227], v[152:155], v[44:47]\n" \
    "ds_read_b64_tr_b16 v[222:223], v240 offset:13952\n" \
    "v_mfma_f32_16x16x32_bf16 v[80:83], v[224:227], v[160:163], v[80:83]\n" \
    "buffer_load_dwordx4 v[200:203], v243, %[rs], 0 offen offset:2048 sc0 sc1\n" \
    "v_mfma_f32_16x16x32_bf16 v[80:83], v[228:231], v[164:167], v[80:83]\n" \
    "buffer_load_dwordx4 v[204:207], v243, %[rs], 0 offen offset:3072 sc0 sc1\n" \
    "v_mfma_f32_16x16x32_bf16 v[116:119], v[228:231], v[172:175], v[116:119]\n" \
    "v_add_u32 v242, 0x10000, v242\n" \
    "v_mfma_f32_16x16x32_bf16 v[116:119], v[224:227], v[168:171], v[116:119]\n" \
    "s_waitcnt lgkmcnt(4)\n" \
    "v_mfma_f32_16x16x32_bf16 v[120:123], v[208:211], v[168:171], v[120:123]\n" \
    "ds_read_b64_tr_b16 v[224:225], v240 offset:160\n" \
    "v_mfma_f32_16x16x32_bf16 v[120:123], v[212:215], v[172:175], v[120:123]\n" \
    "ds_read_b64_tr_b16 v[226:227], v240 offset:4768\n" \
    "v_mfma_f32_16x16x32_bf16 v[84:87], v[212:215], v[164:167], v[84:87]\n" \
    "ds_read_b64_tr_b16 v[228:229], v240 offset:9376\n" \
    "v_mfma_f32_16x16x32_bf16 v[84:87], v[208:211], v[160:163], v[84:87]\n" \
    "ds_read_b64_tr_b16 v[230:231], v240 offset:13984\n" \
    "v_mfma_f32_16x16x32_bf16 v[48:51], v[208:211], v[152:155], v[48:51]\n" \
    "v_add_u32 v243, 0x10000, v243\n" \
    "v_mfma_f32_16x16x32_bf16 v[48:51], v[212:215], v[156:159], v[48:51]\n" \
    "v_add_u32 v246, 1, v246\n" \
    "v_mfma_f32_16x16x32_bf16 v[12:15], v[212:215], v[148:151], v[12:15]\n" \
    "v_mfma_f32_16x16x32_bf16 v[12:15], v[208:211], v[144:147], v[12:15]\n" \
    "s_waitcnt lgkmcnt(4)\n" \
    "v_mfma_f32_16x16x32_bf16 v[16:19], v[216:219], v[144:147], v[16:19]\n" \
    "ds_read_b64_tr_b16 v[208:209], v240 offset:192\n" \
    "v_mfma_f32_16x16x32_bf16 v[16:19], v[220:223], v[148:151], v[16:19]\n" \
    "ds_read_b64_tr_b16 v[210:211], v240 offset:4800\n" \
    "v_mfma_f32_16x16x32_bf16 v[52:55], v[220:223], v[156:159], v[52:55]\n" \
    "ds_read_b64_tr_b16 v[212:213], v240 offset:9408\n" \
    "v_mfma_f32_16x16x32_bf16 v[52:55], v[216:219], v[152:155], v[52:55]\n" \
    "ds_read_b64_tr_b16 v[214:215], v240 offset:14016\n" \
    "v_mfma_f32_16x16x32_bf16 v[88:91], v[216:219], v[160:163], v[88:91]\n" \
    "v_mfma_f32_16x16x32_bf16 v[88:91], v[220:223], v[164:167], v[88:91]\n" \
    "v_mfma_f32_16x16x32_bf16 v[124:127], v[220:223], v[172:175], v[124:127]\n" \
    "v_mfma_f32_16x16x32_bf16 v[124:127], v[216:219], v[168:171], v[124:127]\n" \
    "s_waitcnt lgkmcnt(4)\n" \
    "v_mfma_f32_16x16x32_bf16 v[128:131], v[224:227], v[168:171], v[128:131]\n" \
    "ds_read_b64_tr_b16 v[216:217], v240 offset:224\n" \
    "v_mfma_f32_16x16x32_bf16 v[128:131], v[228:231], v[172:175], v[128:131]\n" \
    "ds_read_b64_tr_b16 v[218:219], v240 offset:4832\n" \
    "v_mfma_f32_16x16x32_bf16 v[92:95], v[228:231], v[164:167], v[92:95]\n" \
    "ds_read_b64_tr_b16 v[220:221], v240 offset:9440\n" \
    "v_mfma_f32_16x16x32_bf16 v[92:95], v[224:227], v[160:163], v[92:95]\n" \
    "ds_read_b64_tr_b16 v[222:223], v240 offset:14048\n" \
    "v_mfma_f32_16x16x32_bf16 v[56:59], v[224:227], v[152:155], v[56:59]\n" \
    "v_mfma_f32_16x16x32_bf16 v[56:59], v[228:231], v[156:159], v[56:59]\n" \
    "v_mfma_f32_16x16x32_bf16 v[20:23], v[228:231], v[148:151], v[20:23]\n" \
    "v_mfma_f32_16x16x32_bf16 v[20:23], v[224:227], v[144:147], v[20:23]\n" \
    "s_waitcnt lgkmcnt(4)\n" \
    "v_mfma_f32_16x16x32_bf16 v[24:27], v[208:211], v[144:147], v[24:27]\n" \
    "ds_read_b64_tr_b16 v[224:225], v240 offset:256\n" \
    "v_mfma_f32_16x16x32_bf16 v[24:27], v[212:215], v[148:151], v[24:27]\n" \
    "ds_read_b64_tr_b16 v[226:227], v240 offset:4864\n" \
    "v_mfma_f32_16x16x32_bf16 v[60:63], v[212:215], v[156:159], v[60:63]\n" \
    "ds_read_b64_tr_b16 v[228:229], v240 offset:9472\n" \
    "v_mfma_f32_16x16x32_bf16 v[60:63], v[208:211], v[152:155], v[60:63]\n" \
    "ds_read_b64_tr_b16 v[230:231], v240 offset:14080\n" \
    "v_mfma_f32_16x16x32_bf16 v[96:99], v[208:211], v[160:163], v[96:99]\n" \
    "v_add_u32 v240, %[ps], v241\n" \
    "v_mfma_f32_16x16x32_bf16 v[96:99], v[212:215], v[164:167], v[96:99]\n" \
    "ds_write_b32 v247, v246 offset:32\n" \
    "v_mfma_f32_16x16x32_bf16 v[132:135], v[212:215], v[172:175], v[132:135]\n" \
    "v_mfma_f32_16x16x32_bf16 v[132:135], v[208:211], v[168:171], v[132:135]\n" \
    "s_waitcnt lgkmcnt(5)\n" \
    "v_mfma_f32_16x16x32_bf16 v[136:139], v[216:219], v[168:171], v[136:139]\n" \
    "ds_read2_b32 v[244:245], v247 offset1:4\n" \
    "ds_read_b64_tr_b16 v[208:209], v240 offset:0\n" \
    "v_mfma_f32_16x16x32_bf16 v[136:139], v[220:223], v[172:175], v[136:139]\n" \
    "ds_read_b64_tr_b16 v[210:211], v240 offset:4608\n" \
    "v_mfma_f32_16x16x32_bf16 v[100:103], v[220:223], v[164:167], v[100:103]\n" \
    "ds_read_b64_tr_b16 v[212:213], v240 offset:9216\n" \
    "v_mfma_f32_16x16x32_bf16 v[100:103], v[216:219], v[160:163], v[100:103]\n" \
    "ds_read_b64_tr_b16 v[214:215], v240 offset:13824\n" \
    "v_mfma_f32_16x16x32_bf16 v[64:67], v[216:219], v[152:155], v[64:67]\n" \
    "v_mfma_f32_16x16x32_bf16 v[64:67], v[220:223], v[156:159], v[64:67]\n" \
    "v_mfma_f32_16x16x32_bf16 v[28:31], v[220:223], v[148:151], v[28:31]\n" \
    "v_mfma_f32_16x16x32_bf16 v[28:31], v[216:219], v[144:147], v[28:31]\n" \
    "s_waitcnt lgkmcnt(4)\n" \
    "v_mfma_f32_16x16x32_bf16 v[32:35], v[224:227], v[144:147], v[32:35]\n" \
    "ds_read_b64_tr_b16 v[216:217], v240 offset:32\n" \
    "v_mfma_f32_16x16x32_bf16 v[32:35], v[228:231], v[148:151], v[32:35]\n" \
    "ds_read_b64_tr_b16 v[218:219], v240 offset:4640\n" \
    "v_mfma_f32_16x16x32_bf16 v[68:71], v[228:231], v[156:159], v[68:71]\n" \
    "ds_read_b64_tr_b16 v[220:221], v240 offset:9248\n" \
    "v_mfma_f32_16x16x32_bf16 v[68:71], v[224:227], v[152:155], v[68:71]\n" \
    "ds_read_b64_tr_b16 v[222:223], v240 offset:13856\n" \
    "v_mfma_f32_16x16x32_bf16 v[104:107], v[224:227], v[160:163], v[104:107]\n" \
    "v_min_u32 v244, v244, v245\n" \
    "v_mfma_f32_16x16x32_bf16 v[104:107], v[228:231], v[164:167], v[104:107]\n" \
    "v_readfirstlane_b32 %[hv], v244\n" \
    "v_mfma_f32_16x16x32_bf16 v[140:143], v[228:231], v[172:175], v[140:143]\n" \
    "s_cmp_ge_u32 %[hv], %[nd]\n" \
    "v_mfma_f32_16x16x32_bf16 v[140:143], v[224:227], v[168:171], v[140:143]\n" \
    "s_waitcnt lgkmcnt(4) vmcnt(0)\n" \
    "s_cbranch_scc0 P9_%=\n" \
    "R9_%=:\n" \
    "v_mfma_f32_16x16x32_bf16 v[0:3], v[208:211], v[176:179], v[0:3]\n" \
    "ds_read_b64_tr_b16 v[224:225], v240 offset:64\n" \
    "v_mfma_f32_16x16x32_bf16 v[0:3], v[212:215], v[180:183], v[0:3]\n" \
    "ds_read_b64_tr_b16 v[226:227], v240 offset:4672\n" \
    "v_mfma_f32_16x16x32_bf16 v[36:39], v[212:215], v[188:191], v[36:39]\n" \
    "ds_read_b64_tr_b16 v[228:229], v240 offset:9280\n" \
    "v_mfma_f32_16x16x32_bf16 v[36:39], v[208:211], v[184:187], v[36:39]\n" \
    "ds_read_b64_tr_b16 v[230:231], v240 offset:13888\n" \
    "v_mfma_f32_16x16x32_bf16 v[72:75], v[208:211], v[192:195], v[72:75]\n" \
    "buffer_load_dwordx4 v[144:147], v242, %[rs], 0 offen offset:0 sc0 sc1\n" \
    "v_mfma_f32_16x16x32_bf16 v[72:75], v[212:215], v[196:199], v[72:75]\n" \
    "buffer_load_dwordx4 v[148:151], v242, %[rs], 0 offen offset:1024 sc0 sc1\n" \
    "v_mfma_f32_16x16x32_bf16 v[108:111], v[212:215], v[204:207], v[108:111]\n" \
    "buffer_load_dwordx4 v[152:155], v242, %[rs], 0 offen offset:2048 sc0 sc1\n" \
    "v_mfma_f32_16x16x32_bf16 v[108:111], v[208:211], v[200:203], v[108:111]\n" \
    "s_waitcnt lgkmcnt(4)\n" \
    "v_mfma_f32_16x16x32_bf16 v[112:115], v[216:219], v[200:203], v[112:115]\n" \
    "ds_read_b64_tr_b16 v[208:209], v240 offset:96\n" \
    "s_add_u32 %[nd], %[nd], 1\n" \
    "v_mfma_f32_16x16x32_bf16 v[112:115], v[220:223], v[204:207], v[112:115]\n" \
    "ds_read_b64_tr_b16 v[210:211], v240 offset:4704\n" \
    "s_add_u32 %[ps], %[ps], 18432\n" \
    "v_mfma_f32_16x16x32_bf16 v[76:79], v[220:223], v[196:199], v[76:79]\n" \
    "ds_read_b64_tr_b16 v[212:213], v240 offset:9312\n" \
    "s_cmp_eq_u32 %[ps], 147456\n" \
    "v_mfma_f32_16x16x32_bf16 v[76:79], v[216:219], v[192:195], v[76:79]\n" \
    "ds_read_b64_tr_b16 v[214:215], v240 offset:13920\n" \
    "s_cselect_b32 %[ps], 0, %[ps]\n" \
    "v_mfma_f32_16x16x32_bf16 v[40:43], v[216:219], v[184:187], v[40:43]\n" \
    "buffer_load_dwordx4 v[156:159], v242, %[rs], 0 offen offset:3072 sc0 sc1\n" \
    "v_mfma_f32_16x16x32_bf16 v[40:43], v[220:223], v[188:191], v[40:43]\n" \
    "buffer_load_dwordx4 v[160:163], v243, %[rs], 0 offen offset:0 sc0 sc1\n" \
    "v_mfma_f32_16x16x32_bf16 v[4:7], v[220:223], v[180:183], v[4:7]\n" \
    "buffer_load_dwordx4 v[164:167], v243, %[rs], 0 offen offset:1024 sc0 sc1\n" \
    "v_mfma_f32_16x16x32_bf16 v[4:7], v[216:219], v[176:179], v[4:7]\n" \
    "s_waitcnt lgkmcnt(4)\n" \
    "v_mfma_f32_16x16x32_bf16 v[8:11], v[224:227], v[176:179], v[8:11]\n" \
    "ds_read_b64_tr_b16 v[216:217], v240 offset:128\n" \
    "s_cmp_eq_u32 %[nd], 66\n" \
    "v_mfma_f32_16x16x32_bf16 v[8:11], v[228:231], v[180:183], v[8:11]\n" \
    "ds_read_b64_tr_b16 v[218:219], v240 offset:4736\n" \
    "s_cselect_b32 %[ck], -1, %[nd]\n" \
    "v_mfma_f32_16x16x32_bf16 v[44:47], v[228:231], v[188:191], v[44:47]\n" \
    "ds_read_b64_tr_b16 v[220:221], v240 offset:9344\n" \
    "v_mfma_f32_16x16x32_bf16 v[44:47], v[224:227], v[184:187], v[44:47]\n" \
    "ds_read_b64_tr_b16 v[222:223], v240 offset:13952\n" \
    "v_mfma_f32_16x16x32_bf16 v[80:83], v[224:227], v[192:195], v[80:83]\n" \
    "buffer_load_dwordx4 v[168:171], v243, %[rs], 0 offen offset:2048 sc0 sc1\n" \
    "v_mfma_f32_16x16x32_bf16 v[80:83], v[228:231], v[196:199], v[80:83]\n" \
    "buffer_load_dwordx4 v[172:175], v243, %[rs], 0 offen offset:3072 sc0 sc1\n" \
    "v_mfma_f32_16x16x32_bf16 v[116:119], v[228:231], v[204:207], v[116:119]\n" \
    "v_add_u32 v242, 0x10000, v242\n" \
    "v_mfma_f32_16x16x32_bf16 v[116:119], v[224:227], v[200:203], v[116:119]\n" \
    "s_waitcnt lgkmcnt(4)\n" \
    "v_mfma_f32_16x16x32_bf16 v[120:123], v[208:211], v[200:203], v[120:123]\n" \
    "ds_read_b64_tr_b16 v[224:225], v240 offset:160\n" \
    "v_mfma_f32_16x16x32_bf16 v[120:123], v[212:215], v[204:207], v[120:123]\n" \
    "ds_read_b64_tr_b16 v[226:227], v240 offset:4768\n" \
    "v_mfma_f32_16x16x32_bf16 v[84:87], v[212:215], v[196:199], v[84:87]\n" \
    "ds_read_b64_tr_b16 v[228:229], v240 offset:9376\n" \
    "v_mfma_f32_16x16x32_bf16 v[84:87], v[208:211], v[192:195], v[84:87]\n" \
    "ds_read_b64_tr_b16 v[230:231], v240 offset:13984\n" \
    "v_mfma_f32_16x16x32_bf16 v[48:51], v[208:211], v[184:187], v[48:51]\n" \
    "v_add_u32 v243, 0x10000, v243\n" \
    "v_mfma_f32_16x16x32_bf16 v[48:51], v[212:215], v[188:191], v[48:51]\n" \
    "v_add_u32 v246, 1, v246\n" \
    "v_mfma_f32_16x16x32_bf16 v[12:15], v[212:215], v[180:183], v[12:15]\n" \
    "v_mfma_f32_16x16x32_bf16 v[12:15], v[208:211], v[176:179], v[12:15]\n" \
    "s_waitcnt lgkmcnt(4)\n" \
    "v_mfma_f32_16x16x32_bf16 v[16:19], v[216:219], v[176:179], v[16:19]\n" \
    "ds_read_b64_tr_b16 v[208:209], v240 offset:192\n" \
    "v_mfma_f32_16x16x32_bf16 v[16:19], v[220:223], v[180:183], v[16:19]\n" \
    "ds_read_b64_tr_b16 v[210:211], v240 offset:4800\n" \
    "v_mfma_f32_16x16x32_bf16 v[52:55], v[220:223], v[188:191], v[52:55]\n" \
    "ds_read_b64_tr_b16 v[212:213], v240 offset:9408\n" \
    "v_mfma_f32_16x16x32_bf16 v[52:55], v[216:219], v[184:187], v[52:55]\n" \
    "ds_read_b64_tr_b16 v[214:215], v240 offset:14016\n" \
    "v_mfma_f32_16x16x32_bf16 v[88:91], v[216:219], v[192:195], v[88:91]\n" \
    "v_mfma_f32_16x16x32_bf16 v[88:91], v[220:223], v[196:199], v[88:91]\n" \
    "v_mfma_f32_16x16x32_bf16 v[124:127], v[220:223], v[204:207], v[124:127]\n" \
    "v_mfma_f32_16x16x32_bf16 v[124:127], v[216:219], v[200:203], v[124:127]\n" \
    "s_waitcnt lgkmcnt(4)\n" \
    "v_mfma_f32_16x16x32_bf16 v[128:131], v[224:227], v[200:203], v[128:131]\n" \
    "ds_read_b64_tr_b16 v[216:217], v240 offset:224\n" \
    "v_mfma_f32_16x16x32_bf16 v[128:131], v[228:231], v[204:207], v[128:131]\n" \
    "ds_read_b64_tr_b16 v[218:219], v240 offset:4832\n" \
    "v_mfma_f32_16x16x32_bf16 v[92:95], v[228:231], v[196:199], v[92:95]\n" \
    "ds_read_b64_tr_b16 v[220:221], v240 offset:9440\n" \
    "v_mfma_f32_16x16x32_bf16 v[92:95], v[224:227], v[192:195], v[92:95]\n" \
    "ds_read_b64_tr_b16 v[222:223], v240 offset:14048\n" \
    "v_mfma_f32_16x16x32_bf16 v[56:59], v[224:227], v[184:187], v[56:59]\n" \
    "v_mfma_f32_16x16x32_bf16 v[56:59], v[228:231], v[188:191], v[56:59]\n" \
    "v_mfma_f32_16x16x32_bf16 v[20:23], v[228:231], v[180:183], v[20:23]\n" \
    "v_mfma_f32_16x16x32_bf16 v[20:23], v[224:227], v[176:179], v[20:23]\n" \
    "s_waitcnt lgkmcnt(4)\n" \
    "v_mfma_f32_16x16x32_bf16 v[24:27], v[208:211], v[176:179], v[24:27]\n" \
    "ds_read_b64_tr_b16 v[224:225], v240 offset:256\n" \
    "v_mfma_f32_16x16x32_bf16 v[24:27], v[212:215], v[180:183], v[24:27]\n" \
    "ds_read_b64_tr_b16 v[226:227], v240 offset:4864\n" \
    "v_mfma_f32_16x16x32_bf16 v[60:63], v[212:215], v[188:191], v[60:63]\n" \
    "ds_read_b64_tr_b16 v[228:229], v240 offset:9472\n" \
    "v_mfma_f32_16x16x32_bf16 v[60:63], v[208:211], v[184:187], v[60:63]\n" \
    "ds_read_b64_tr_b16 v[230:231], v240 offset:14080\n" \
    "v_mfma_f32_16x16x32_bf16 v[96:99], v[208:211], v[192:195], v[96:99]\n" \
    "v_add_u32 v240, %[ps], v241\n" \
    "v_mfma_f32_16x16x32_bf16 v[96:99], v[212:215], v[196:199], v[96:99]\n" \
    "ds_write_b32 v247, v246 offset:32\n" \
    "v_mfma_f32_16x16x32_bf16 v[132:135], v[212:215], v[204:207], v[132:135]\n" \
    "v_mfma_f32_16x16x32_bf16 v[132:135], v[208:211], v[200:203], v[132:135]\n" \
    "s_waitcnt lgkmcnt(5)\n" \
    "v_mfma_f32_16x16x32_bf16 v[136:139], v[216:219], v[200:203], v[136:139]\n" \
    "ds_read2_b32 v[244:245], v247 offset1:4\n" \
    "ds_read_b64_tr_b16 v[208:209], v240 offset:0\n" \
    "v_mfma_f32_16x16x32_bf16 v[136:139], v[220:223], v[204:207], v[136:139]\n" \
    "ds_read_b64_tr_b16 v[210:211], v240 offset:4608\n" \
    "v_mfma_f32_16x16x32_bf16 v[100:103], v[220:223], v[196:199], v[100:103]\n" \
    "ds_read_b64_tr_b16 v[212:213], v240 offset:9216\n" \
    "v_mfma_f32_16x16x32_bf16 v[100:103], v[216:219], v[192:195], v[100:103]\n" \
    "ds_read_b64_tr_b16 v[214:215], v240 offset:13824\n" \
    "v_mfma_f32_16x16x32_bf16 v[64:67], v[216:219], v[184:187], v[64:67]\n" \
    "v_mfma_f32_16x16x32_bf16 v[64:67], v[220:223], v[188:191], v[64:67]\n" \
    "v_mfma_f32_16x16x32_bf16 v[28:31], v[220:223], v[180:183], v[28:31]\n" \
    "v_mfma_f32_16x16x32_bf16 v[28:31], v[216:219], v[176:179], v[28:31]\n" \
    "s_waitcnt lgkmcnt(4)\n" \
    "v_mfma_f32_16x16x32_bf16 v[32:35], v[224:227], v[176:179], v[32:35]\n" \
    "ds_read_b64_tr_b16 v[216:217], v240 offset:32\n" \
    "v_mfma_f32_16x16x32_bf16 v[32:35], v[228:231], v[180:183], v[32:35]\n" \
    "ds_read_b64_tr_b16 v[218:219], v240 offset:4640\n" \
    "v_mfma_f32_16x16x32_bf16 v[68:71], v[228:231], v[188:191], v[68:71]\n" \
    "ds_read_b64_tr_b16 v[220:221], v240 offset:9248\n" \
    "v_mfma_f32_16x16x32_bf16 v[68:71], v[224:227], v[184:187], v[68:71]\n" \
    "ds_read_b64_tr_b16 v[222:223], v240 offset:13856\n" \
    "v_mfma_f32_16x16x32_bf16 v[104:107], v[224:227], v[192:195], v[104:107]\n" \
    "v_min_u32 v244, v244, v245\n" \
    "v_mfma_f32_16x16x32_bf16 v[104:107], v[228:231], v[196:199], v[104:107]\n" \
    "v_readfirstlane_b32 %[hv], v244\n" \
    "v_mfma_f32_16x16x32_bf16 v[140:143], v[228:231], v[204:207], v[140:143]\n" \
    "s_cmp_ge_u32 %[hv], %[ck]\n" \
    "v_mfma_f32_16x16x32_bf16 v[140:143], v[224:227], v[200:203], v[140:143]\n" \
    "s_waitcnt lgkmcnt(4) vmcnt(0)\n" \
    "s_cbranch_scc1 L9_%=\n" \
    "s_cmp_eq_u32 %[nd], 66\n" \
    "s_cbranch_scc1 LEND%=\n" \
    "Q9_%=:\n" \
    "ds_read2_b32 v[244:245], v247 offset1:4\n" \
    "s_waitcnt lgkmcnt(0)\n" \
    "v_min_u32 v244, v244, v245\n" \
    "s_nop 0\n" \
    "v_readfirstlane_b32 %[hv], v244\n" \
    "s_cmp_ge_u32 %[hv], %[nd]\n" \
    "s_cbranch_scc1 L9_%=\n" \
    "s_sleep 1\n" \
    "s_branch Q9_%=\n" \
    "E8%=:\n" \
    "ds_read_b64_tr_b16 v[208:209], v240 offset:0\n" \
    "ds_read_b64_tr_b16 v[210:211], v240 offset:4608\n" \
    "ds_read_b64_tr_b16 v[212:213], v240 offset:9216\n" \
    "ds_read_b64_tr_b16 v[214:215], v240 offset:13824\n" \
    "ds_read_b64_tr_b16 v[216:217], v240 offset:32\n" \
    "ds_read_b64_tr_b16 v[218:219], v240 offset:4640\n" \
    "ds_read_b64_tr_b16 v[220:221], v240 offset:9248\n" \
    "ds_read_b64_tr_b16 v[222:223], v240 offset:13856\n" \
    "ds_read_b64_tr_b16 v[224:225], v240 offset:64\n" \
    "ds_read_b64_tr_b16 v[226:227], v240 offset:4672\n" \
    "ds_read_b64_tr_b16 v[228:229], v240 offset:9280\n" \
    "ds_read_b64_tr_b16 v[230:231], v240 offset:13888\n" \
    "s_waitcnt lgkmcnt(8) vmcnt(0)\n" \
    "L8_%=:\n" \
    "v_mfma_f32_16x16x32_bf16 v[0:3], v[208:211], v[144:147], v[0:3]\n" \
    "ds_read_b64_tr_b16 v[232:233], v240 offset:96\n" \
    "v_mfma_f32_16x16x32_bf16 v[0:3], v[212:215], v[148:151], v[0:3]\n" \
    "ds_read_b64_tr_b16 v[234:235], v240 offset:4704\n" \
    "v_mfma_f32_16x16x32_bf16 v[36:39], v[212:215], v[156:159], v[36:39]\n" \
    "ds_read_b64_tr_b16 v[236:237], v240 offset:9312\n" \
    "v_mfma_f32_16x16x32_bf16 v[36:39], v[208:211], v[152:155], v[36:39]\n" \
    "ds_read_b64_tr_b16 v[238:239], v240 offset:13920\n" \
    "v_mfma_f32_16x16x32_bf16 v[72:75], v[208:211], v[160:163], v[72:75]\n" \
    "buffer_load_dwordx4 v[176:179], v242, %[rs], 0 offen offset:0 sc0 sc1\n" \
    "v_mfma_f32_16x16x32_bf16 v[72:75], v[212:215], v[164:167], v[72:75]\n" \
    "buffer_load_dwordx4 v[180:183], v242, %[rs], 0 offen offset:1024 sc0 sc1\n" \
    "v_mfma_f32_16x16x32_bf16 v[108:111], v[212:215], v[172:175], v[108:111]\n" \
    "buffer_load_dwordx4 v[184:187], v242, %[rs], 0 offen offset:2048 sc0 sc1\n" \
    "v_mfma_f32_16x16x32_bf16 v[108:111], v[208:211], v[168:171], v[108:111]\n" \
    "s_waitcnt lgkmcnt(8)\n" \
    "v_mfma_f32_16x16x32_bf16 v[112:115], v[216:219], v[168:171], v[112:115]\n" \
    "ds_read_b64_tr_b16 v[208:209], v240 offset:128\n" \
    "s_add_u32 %[nd], %[nd], 1\n" \
    "v_mfma_f32_16x16x32_bf16 v[112:115], v[220:223], v[172:175], v[112:115]\n" \
    "ds_read_b64_tr_b16 v[210:211], v240 offset:4736\n" \
    "s_add_u32 %[ps], %[ps], 18432\n" \
    "v_mfma_f32_16x16x32_bf16 v[76:79], v[220:223], v[164:167], v[76:79]\n" \
    "ds_read_b64_tr_b16 v[212:213], v240 offset:9344\n" \
    "s_cmp_eq_u32 %[ps], 147456\n" \
    "v_mfma_f32_16x16x32_bf16 v[76:79], v[216:219], v[160:163], v[76:79]\n" \
    "ds_read_b64_tr_b16 v[214:215], v240 offset:13952\n" \
    "s_cselect_b32 %[ps], 0, %[ps]\n" \
    "v_mfma_f32_16x16x32_bf16 v[40:43], v[216:219], v[152:155], v[40:43]\n" \
    "buffer_load_dwordx4 v[188:191], v242, %[rs], 0 offen offset:3072 sc0 sc1\n" \
    "v_mfma_f32_16x16x32_bf16 v[40:43], v[220:223], v[156:159], v[40:43]\n" \
    "buffer_load_dwordx4 v[192:195], v243, %[rs], 0 offen offset:0 sc0 sc1\n" \
    "v_mfma_f32_16x16x32_bf16 v[4:7], v[220:223], v[148:151], v[4:7]\n" \
    "buffer_load_dwordx4 v[196:199], v243, %[rs], 0 offen offset:1024 sc0 sc1\n" \
    "v_mfma_f32_16x16x32_bf16 v[4:7], v[216:219], v[144:147], v[4:7]\n" \
    "s_waitcnt lgkmcnt(8)\n" \
    "v_mfma_f32_16x16x32_bf16 v[8:11], v[224:227], v[144:147], v[8:11]\n" \
    "ds_read_b64_tr_b16 v[216:217], v240 offset:160\n" \
    "v_mfma_f32_16x16x32_bf16 v[8:11], v[228:231], v[148:151], v[8:11]\n" \
    "ds_read_b64_tr_b16 v[218:219], v240 offset:4768\n" \
    "v_mfma_f32_16x16x32_bf16 v[44:47], v[228:231], v[156:159], v[44:47]\n" \
    "ds_read_b64_tr_b16 v[220:221], v240 offset:9376\n" \
    "v_mfma_f32_16x16x32_bf16 v[44:47], v[224:227], v[152:155], v[44:47]\n" \
    "ds_read_b64_tr_b16 v[222:223], v240 offset:13984\n" \
    "v_mfma_f32_16x16x32_bf16 v[80:83], v[224:227], v[160:163], v[80:83]\n" \
    "buffer_load_dwordx4 v[200:203], v243, %[rs], 0 offen offset:2048 sc0 sc1\n" \
    "v_mfma_f32_16x16x32_bf16 v[80:83], v[228:231], v[164:167], v[80:83]\n" \
    "buffer_load_dwordx4 v[204:207], v243, %[rs], 0 offen offset:3072 sc0 sc1\n" \
    "v_mfma_f32_16x16x32_bf16 v[116:119], v[228:231], v[172:175], v[116:119]\n" \
    "v_add_u32 v242, 0x10000, v242\n" \
    "v_mfma_f32_16x16x32_bf16 v[116:119], v[224:227], v[168:171], v[116:119]\n" \
    "s_waitcnt lgkmcnt(8)\n" \
    "v_mfma_f32_16x16x32_bf16 v[120:123], v[232:235], v[168:171], v[120:123]\n" \
    "ds_read_b64_tr_b16 v[224:225], v240 offset:192\n" \
    "v_mfma_f32_16x16x32_bf16 v[120:123], v[236:239], v[172:175], v[120:123]\n" \
    "ds_read_b64_tr_b16 v[226:227], v240 offset:4800\n" \
    "v_mfma_f32_16x16x32_bf16 v[84:87], v[236:239], v[164:167], v[84:87]\n" \
    "ds_read_b64_tr_b16 v[228:229], v240 offset:9408\n" \
    "v_mfma_f32_16x16x32_bf16 v[84:87], v[232:235], v[160:163], v[84:87]\n" \
    "ds_read_b64_tr_b16 v[230:231], v240 offset:14016\n" \
    "v_mfma_f32_16x16x32_bf16 v[48:51], v[232:235], v[152:155], v[48:51]\n" \
    "v_add_u32 v243, 0x10000, v243\n" \
    "v_mfma_f32_16x16x32_bf16 v[48:51], v[236:239], v[156:159], v[48:51]\n" \
    "v_add_u32 v246, 1, v246\n" \
    "v_mfma_f32_16x16x32_bf16 v[12:15], v[236:239], v[148:151], v[12:15]\n" \
    "v_mfma_f32_16x16x32_bf16 v[12:15], v[232:235], v[144:147], v[12:15]\n" \
    "s_waitcnt lgkmcnt(8)\n" \
    "v_mfma_f32_16x16x32_bf16 v[16:19], v[208:211], v[144:147], v[16:19]\n" \
    "ds_read_b64_tr_b16 v[232:233], v240 offset:224\n" \
    "v_mfma_f32_16x16x32_bf16 v[16:19], v[212:215], v[148:151], v[16:19]\n" \
    "ds_read_b64_tr_b16 v[234:235], v240 offset:4832\n" \
    "v_mfma_f32_16x16x32_bf16 v[52:55], v[212:215], v[156:159], v[52:55]\n" \
    "ds_read_b64_tr_b16 v[236:237], v240 offset:9440\n" \
    "v_mfma_f32_16x16x32_bf16 v[52:55], v[208:211], v[152:155], v[52:55]\n" \
    "ds_read_b64_tr_b16 v[238:239], v240 offset:14048\n" \
    "v_mfma_f32_16x16x32_bf16 v[88:91], v[208:211], v[160:163], v[88:91]\n" \
    "v_add_u32 v240, %[ps], v241\n" \
    "v_mfma_f32_16x16x32_bf16 v[88:91], v[212:215], v[164:167], v[88:91]\n" \
    "ds_write_b32 v247, v246 offset:32\n" \
    "v_mfma_f32_16x16x32_bf16 v[124:127], v[212:215], v[172:175], v[124:127]\n" \
    "v_mfma_f32_16x16x32_bf16 v[124:127], v[208:211], v[168:171], v[124:127]\n" \
    "s_waitcnt lgkmcnt(9)\n" \
    "v_mfma_f32_16x16x32_bf16 v[128:131], v[216:219], v[168:171], v[128:131]\n" \
    "ds_read_b64_tr_b16 v[208:209], v240 offset:0\n" \
    "v_mfma_f32_16x16x32_bf16 v[128:131], v[220:223], v[172:175], v[128:131]\n" \
    "ds_read_b64_tr_b16 v[210:211], v240 offset:4608\n" \
    "v_mfma_f32_16x16x32_bf16 v[92:95], v[220:223], v[164:167], v[92:95]\n" \
    "ds_read_b64_tr_b16 v[212:213], v240 offset:9216\n" \
    "v_mfma_f32_16x16x32_bf16 v[92:95], v[216:219], v[160:163], v[92:95]\n" \
    "ds_read_b64_tr_b16 v[214:215], v240 offset:13824\n" \
    "v_mfma_f32_16x16x32_bf16 v[56:59], v[216:219], v[152:155], v[56:59]\n" \
    "v_mfma_f32_16x16x32_bf16 v[56:59], v[220:223], v[156:159], v[56:59]\n" \
    "v_mfma_f32_16x16x32_bf16 v[20:23], v[220:223], v[148:151], v[20:23]\n" \
    "v_mfma_f32_16x16x32_bf16 v[20:23], v[216:219], v[144:147], v[20:23]\n" \
    "s_waitcnt lgkmcnt(9)\n" \
    "v_mfma_f32_16x16x32_bf16 v[24:27], v[224:227], v[144:147], v[24:27]\n" \
    "ds_read2_b32 v[244:245], v247 offset1:4\n" \
    "ds_read_b64_tr_b16 v[216:217], v240 offset:32\n" \
    "v_mfma_f32_16x16x32_bf16 v[24:27], v[228:231], v[148:151], v[24:27]\n" \
    "ds_read_b64_tr_b16 v[218:219], v240 offset:4640\n" \
    "v_mfma_f32_16x16x32_bf16 v[60:63], v[228:231], v[156:159], v[60:63]\n" \
    "ds_read_b64_tr_b16 v[220:221], v240 offset:9248\n" \
    "v_mfma_f32_16x16x32_bf16 v[60:63], v[224:227], v[152:155], v[60:63]\n" \
    "ds_read_b64_tr_b16 v[222:223], v240 offset:13856\n" \
    "v_mfma_f32_16x16x32_bf16 v[96:99], v[224:227], v[160:163], v[96:99]\n" \
    "v_mfma_f32_16x16x32_bf16 v[96:99], v[228:231], v[164:167], v[96:99]\n" \
    "v_mfma_f32_16x16x32_bf16 v[132:135], v[228:231], v[172:175], v[132:135]\n" \
    "v_mfma_f32_16x16x32_bf16 v[132:135], v[224:227], v[168:171], v[132:135]\n" \
    "s_waitcnt lgkmcnt(4)\n" \
    "v_mfma_f32_16x16x32_bf16 v[136:139], v[232:235], v[168:171], v[136:139]\n" \
    "ds_read_b64_tr_b16 v[224:225], v240 offset:64\n" \
    "v_mfma_f32_16x16x32_bf16 v[136:139], v[236:239], v[172:175], v[136:139]\n" \
    "ds_read_b64_tr_b16 v[226:227], v240 offset:4672\n" \
    "v_mfma_f32_16x16x32_bf16 v[100:103], v[236:239], v[164:167], v[100:103]\n" \
    "ds_read_b64_tr_b16 v[228:229], v240 offset:9280\n" \
    "v_mfma_f32_16x16x32_bf16 v[100:103], v[232:235], v[160:163], v[100:103]\n" \
    "ds_read_b64_tr_b16 v[230:231], v240 offset:13888\n" \
    "v_mfma_f32_16x16x32_bf16 v[64:67], v[232:235], v[152:155], v[64:67]\n" \
    "v_min_u32 v244, v244, v245\n" \
    "v_mfma_f32_16x16x32_bf16 v[64:67], v[236:239], v[156:159], v[64:67]\n" \
    "v_readfirstlane_b32 %[hv], v244\n" \
    "v_mfma_f32_16x16x32_bf16 v[28:31], v[236:239], v[148:151], v[28:31]\n" \
    "s_cmp_ge_u32 %[hv], %[nd]\n" \
    "v_mfma_f32_16x16x32_bf16 v[28:31], v[232:235], v[144:147], v[28:31]\n" \
    "s_waitcnt lgkmcnt(8) vmcnt(0)\n" \
    "s_cbranch_scc0 P8_%=\n" \
    "R8_%=:\n" \
    "v_mfma_f32_16x16x32_bf16 v[0:3], v[208:211], v[176:179], v[0:3]\n" \
    "ds_read_b64_tr_b16 v[232:233], v240 offset:96\n" \
    "v_mfma_f32_16x16x32_bf16 v[0:3], v[212:215], v[180:183], v[0:3]\n" \
    "ds_read_b64_tr_b16 v[234:235], v240 offset:4704\n" \
    "v_mfma_f32_16x16x32_bf16 v[36:39], v[212:215], v[188:191], v[36:39]\n" \
    "ds_read_b64_tr_b16 v[236:237], v240 offset:9312\n" \
    "v_mfma_f32_16x16x32_bf16 v[36:39], v[208:211], v[184:187], v[36:39]\n" \
    "ds_read_b64_tr_b16 v[238:239], v240 offset:13920\n" \
    "v_mfma_f32_16x16x32_bf16 v[72:75], v[208:211], v[192:195], v[72:75]\n" \
    "buffer_load_dwordx4 v[144:147], v242, %[rs], 0 offen offset:0 sc0 sc1\n" \
    "v_mfma_f32_16x16x32_bf16 v[72:75], v[212:215], v[196:199], v[72:75]\n" \
    "buffer_load_dwordx4 v[148:151], v242, %[rs], 0 offen offset:1024 sc0 sc1\n" \
    "v_mfma_f32_16x16x32_bf16 v[108:111], v[212:215], v[204:207], v[108:111]\n" \
    "buffer_load_dwordx4 v[152:155], v242, %[rs], 0 offen offset:2048 sc0 sc1\n" \
    "v_mfma_f32_16x16x32_bf16 v[108:111], v[208:211], v[200:203], v[108:111]\n" \
    "s_waitcnt lgkmcnt(8)\n" \
    "v_mfma_f32_16x16x32_bf16 v[112:115], v[216:219], v[200:203], v[112:115]\n" \
    "ds_read_b64_tr_b16 v[208:209], v240 offset:128\n" \
    "s_add_u32 %[nd], %[nd], 1\n" \
    "v_mfma_f32_16x16x32_bf16 v[112:115], v[220:223], v[204:207], v[112:115]\n" \
    "ds_read_b64_tr_b16 v[210:211], v240 offset:4736\n" \
    "s_add_u32 %[ps], %[ps], 18432\n" \
    "v_mfma_f32_16x16x32_bf16 v[76:79], v[220:223], v[196:199], v[76:79]\n" \
    "ds_read_b64_tr_b16 v[212:213], v240 offset:9344\n" \
    "s_cmp_eq_u32 %[ps], 147456\n" \
    "v_mfma_f32_16x16x32_bf16 v[76:79], v[216:219], v[192:195], v[76:79]\n" \
    "ds_read_b64_tr_b16 v[214:215], v240 offset:13952\n" \
    "s_cselect_b32 %[ps], 0, %[ps]\n" \
    "v_mfma_f32_16x16x32_bf16 v[40:43], v[216:219], v[184:187], v[40:43]\n" \
    "buffer_load_dwordx4 v[156:159], v242, %[rs], 0 offen offset:3072 sc0 sc1\n" \
    "v_mfma_f32_16x16x32_bf16 v[40:43], v[220:223], v[188:191], v[40:43]\n" \
    "buffer_load_dwordx4 v[160:163], v243, %[rs], 0 offen offset:0 sc0 sc1\n" \
    "v_mfma_f32_16x16x32_bf16 v[4:7], v[220:223], v[180:183], v[4:7]\n" \
    "buffer_load_dwordx4 v[164:167], v243, %[rs], 0 offen offset:1024 sc0 sc1\n" \
    "v_mfma_f32_16x16x32_bf16 v[4:7], v[216:219], v[176:179], v[4:7]\n" \
    "s_waitcnt lgkmcnt(8)\n" \
    "v_mfma_f32_16x16x32_bf16 v[8:11], v[224:227], v[176:179], v[8:11]\n" \
    "ds_read_b64_tr_b16 v[216:217], v240 offset:160\n" \
    "s_cmp_eq_u32 %[nd], 66\n" \
    "v_mfma_f32_16x16x32_bf16 v[8:11], v[228:231], v[180:183], v[8:11]\n" \
    "ds_read_b64_tr_b16 v[218:219], v240 offset:4768\n" \
    "s_cselect_b32 %[ck], -1, %[nd]\n" \
    "v_mfma_f32_16x16x32_bf16 v[44:47], v[228:231], v[188:191], v[44:47]\n" \
    "ds_read_b64_tr_b16 v[220:221], v240 offset:9376\n" \
    "v_mfma_f32_16x16x32_bf16 v[44:47], v[224:227], v[184:187], v[44:47]\n" \
    "ds_read_b64_tr_b16 v[222:223], v240 offset:13984\n" \
    "v_mfma_f32_16x16x32_bf16 v[80:83], v[224:227], v[192:195], v[80:83]\n" \
    "buffer_load_dwordx4 v[168:171], v243, %[rs], 0 offen offset:2048 sc0 sc1\n" \
    "v_mfma_f32_16x16x32_bf16 v[80:83], v[228:231], v[196:199], v[80:83]\n" \
    "buffer_load_dwordx4 v[172:175], v243, %[rs], 0 offen offset:3072 sc0 sc1\n" \
    "v_mfma_f32_16x16x32_bf16 v[116:119], v[228:231], v[204:207], v[116:119]\n" \
    "v_add_u32 v242, 0x10000, v242\n" \
    "v_mfma_f32_16x16x32_bf16 v[116:119], v[224:227], v[200:203], v[116:119]\n" \
    "s_waitcnt lgkmcnt(8)\n" \
    "v_mfma_f32_16x16x32_bf16 v[120:123], v[232:235], v[200:203], v[120:123]\n" \
    "ds_read_b64_tr_b16 v[224:225], v240 offset:192\n" \
    "v_mfma_f32_16x16x32_bf16 v[120:123], v[236:239], v[204:207], v[120:123]\n" \
    "ds_read_b64_tr_b16 v[226:227], v240 offset:4800\n" \
    "v_mfma_f32_16x16x32_bf16 v[84:87], v[236:239], v[196:199], v[84:87]\n" \
    "ds_read_b64_tr_b16 v[228:229], v240 offset:9408\n" \
    "v_mfma_f32_16x16x32_bf16 v[84:87], v[232:235], v[192:195], v[84:87]\n" \
    "ds_read_b64_tr_b16 v[230:231], v240 offset:14016\n" \
    "v_mfma_f32_16x16x32_bf16 v[48:51], v[232:235], v[184:187], v[48:51]\n" \
    "v_add_u32 v243, 0x10000, v243\n" \
    "v_mfma_f32_16x16x32_bf16 v[48:51], v[236:239], v[188:191], v[48:51]\n" \
    "v_add_u32 v246, 1, v246\n" \
    "v_mfma_f32_16x16x32_bf16 v[12:15], v[236:239], v[180:183], v[12:15]\n" \
    "v_mfma_f32_16x16x32_bf16 v[12:15], v[232:235], v[176:179], v[12:15]\n" \
    "s_waitcnt lgkmcnt(8)\n" \
    "v_mfma_f32_16x16x32_bf16 v[16:19], v[208:211], v[176:179], v[16:19]\n" \
    "ds_read_b64_tr_b16 v[232:233], v240 offset:224\n" \
    "v_mfma_f32_16x16x32_bf16 v[16:19], v[212:215], v[180:183], v[16:19]\n" \
    "ds_read_b64_tr_b16 v[234:235], v240 offset:4832\n" \
    "v_mfma_f32_16x16x32_bf16 v[52:55], v[212:215], v[188:191], v[52:55]\n" \
    "ds_read_b64_tr_b16 v[236:237], v240 offset:9440\n" \
    "v_mfma_f32_16x16x32_bf16 v[52:55], v[208:211], v[184:187], v[52:55]\n" \
    "ds_read_b64_tr_b16 v[238:239], v240 offset:14048\n" \
    "v_mfma_f32_16x16x32_bf16 v[88:91], v[208:211], v[192:195], v[88:91]\n" \
    "v_add_u32 v240, %[ps], v241\n" \
    "v_mfma_f32_16x16x32_bf16 v[88:91], v[212:215], v[196:199], v[88:91]\n" \
    "ds_write_b32 v247, v246 offset:32\n" \
    "v_mfma_f32_16x16x32_bf16 v[124:127], v[212:215], v[204:207], v[124:127]\n" \
    "v_mfma_f32_16x16x32_bf16 v[124:127], v[208:211], v[200:203], v[124:127]\n" \
    "s_waitcnt lgkmcnt(9)\n" \
    "v_mfma_f32_16x16x32_bf16 v[128:131], v[216:219], v[200:203], v[128:131]\n" \
    "ds_read_b64_tr_b16 v[208:209], v240 offset:0\n" \
    "v_mfma_f32_16x16x32_bf16 v[128:131], v[220:223], v[204:207], v[128:131]\n" \
    "ds_read_b64_tr_b16 v[210:211], v240 offset:4608\n" \
    "v_mfma_f32_16x16x32_bf16 v[92:95], v[220:223], v[196:199], v[92:95]\n" \
    "ds_read_b64_tr_b16 v[212:213], v240 offset:9216\n" \
    "v_mfma_f32_16x16x32_bf16 v[92:95], v[216:219], v[192:195], v[92:95]\n" \
    "ds_read_b64_tr_b16 v[214:215], v240 offset:13824\n" \
    "v_mfma_f32_16x16x32_bf16 v[56:59], v[216:219], v[184:187], v[56:59]\n" \
    "v_mfma_f32_16x16x32_bf16 v[56:59], v[220:223], v[188:191], v[56:59]\n" \
    "v_mfma_f32_16x16x32_bf16 v[20:23], v[220:223], v[180:183], v[20:23]\n" \
    "v_mfma_f32_16x16x32_bf16 v[20:23], v[216:219], v[176:179], v[20:23]\n" \
    "s_waitcnt lgkmcnt(9)\n" \
    "v_mfma_f32_16x16x32_bf16 v[24:27], v[224:227], v[176:179], v[24:27]\n" \
    "ds_read2_b32 v[244:245], v247 offset1:4\n" \
    "ds_read_b64_tr_b16 v[216:217], v240 offset:32\n" \
    "v_mfma_f32_16x16x32_bf16 v[24:27], v[228:231], v[180:183], v[24:27]\n" \
    "ds_read_b64_tr_b16 v[218:219], v240 offset:4640\n" \
    "v_mfma_f32_16x16x32_bf16 v[60:63], v[228:231], v[188:191], v[60:63]\n" \
    "ds_read_b64_tr_b16 v[220:221], v240 offset:9248\n" \
    "v_mfma_f32_16x16x32_bf16 v[60:63], v[224:227], v[184:187], v[60:63]\n" \
    "ds_read_b64_tr_b16 v[222:223], v240 offset:13856\n" \
    "v_mfma_f32_16x16x32_bf16 v[96:99], v[224:227], v[192:195], v[96:99]\n" \
    "v_mfma_f32_16x16x32_bf16 v[96:99], v[228:231], v[196:199], v[96:99]\n" \
    "v_mfma_f32_16x16x32_bf16 v[132:135], v[228:231], v[204:207], v[132:135]\n" \
    "v_mfma_f32_16x16x32_bf16 v[132:135], v[224:227], v[200:203], v[132:135]\n" \
    "s_waitcnt lgkmcnt(4)\n" \
    "v_mfma_f32_16x16x32_bf16 v[136:139], v[232:235], v[200:203], v[136:139]\n" \
    "ds_read_b64_tr_b16 v[224:225], v240 offset:64\n" \
    "v_mfma_f32_16x16x32_bf16 v[136:139], v[236:239], v[204:207], v[136:139]\n" \
    "ds_read_b64_tr_b16 v[226:227], v240 offset:4672\n" \
    "v_mfma_f32_16x16x32_bf16 v[100:103], v[236:239], v[196:199], v[100:103]\n" \
    "ds_read_b64_tr_b16 v[228:229], v240 offset:9280\n" \
    "v_mfma_f32_16x16x32_bf16 v[100:103], v[232:235], v[192:195], v[100:103]\n" \
    "ds_read_b64_tr_b16 v[230:231], v240 offset:13888\n" \
    "v_mfma_f32_16x16x32_bf16 v[64:67], v[232:235], v[184:187], v[64:67]\n" \
    "v_min_u32 v244, v244, v245\n" \
    "v_mfma_f32_16x16x32_bf16 v[64:67], v[236:239], v[188:191], v[64:67]\n" \
    "v_readfirstlane_b32 %[hv], v244\n" \
    "v_mfma_f32_16x16x32_bf16 v[28:31], v[236:239], v[180:183], v[28:31]\n" \
    "s_cmp_ge_u32 %[hv], %[ck]\n" \
    "v_mfma_f32_16x16x32_bf16 v[28:31], v[232:235], v[176:179], v[28:31]\n" \
    "s_waitcnt lgkmcnt(8) vmcnt(0)\n" \
    "s_cbranch_scc1 L8_%=\n" \
    "s_cmp_eq_u32 %[nd], 66\n" \
    "s_cbranch_scc1 LEND%=\n" \
    "Q8_%=:\n" \
    "ds_read2_b32 v[244:245], v247 offset1:4\n" \
    "s_waitcnt lgkmcnt(0)\n" \
    "v_min_u32 v244, v244, v245\n" \
    "s_nop 0\n" \
    "v_readfirstlane_b32 %[hv], v244\n" \
    "s_cmp_ge_u32 %[hv], %[nd]\n" \
    "s_cbranch_scc1 L8_%=\n" \
    "s_sleep 1\n" \
    "s_branch Q8_%=\n" \
    "LEND%=:\n" \
    "s_waitcnt vmcnt(0) lgkmcnt(0)\n" \
    "s_nop 15\n" \
    ""

__global__ void __launch_bounds__(NTHR, 2) ag_gemm(const Args a) {
    __shared__ __attribute__((aligned(1024))) unsigned char lds_raw[LDS_BYTES];
    unsigned code;
    {
        unsigned long long pc;
        unsigned len;
        asm volatile("s_getpc_b64 %0\nAG_CODE0:\n\ts_mov_b32 %1, AG_CODE1-AG_CODE0" : "=s"(pc), "=s"(len));
        code = __builtin_amdgcn_raw_buffer_load_b32(rsrc((const void*)pc, len), threadIdx.x * 64, 0, 0);
        SB();
    }
    LAS unsigned char* lds = (LAS unsigned char*)lds_raw;
    volatile LAS unsigned* ctl = (volatile LAS unsigned*)(lds + L_CTL);
    volatile LAS unsigned* prog = ctl + 8;
    const int lane = threadIdx.x & 63;
    const int wave = __builtin_amdgcn_readfirstlane(threadIdx.x >> 6);
    const int bid = blockIdx.x;
    char* const myws = a.ws[NR - 1];
    asm volatile("" :: "s"(a.x), "s"(a.ws[0]), "s"(a.ws[1]), "s"(a.ws[2]), "s"(a.ws[3]), "s"(a.ws[4]), "s"(a.ws[5]), "s"(a.ws[6]), "s"(myws), "s"(a.rank));
    const int xcd = bid & 7, loc = bid >> 3;
    const bool is_comm = loc < 2;
    const int cid = loc * 8 + xcd, tl = (xcd & 1) * (2 * NTN) + loc - 2, pm = (xcd >> 1) * 4 + (tl & 3), pn = tl >> 2;
    const int pq = (pn & 1) ? -1 : pn >> 1, part = pq >> 31, flip = 0;

    if (wave == 0) STAMP(0);
    const __amdgpu_buffer_rsrc_t rin = rsrc(a.x, MS * K * 2);
    auto fetch = [&](u32x4 (&d)[4], unsigned b) {
        const unsigned o = (b < (unsigned)NBLK ? (b % NG) * 32 * (K * 2) + (b / NG) * (BK * 2) : OOB) + (lane & 15) * (K * 2) + (lane >> 4) * 16;
#pragma unroll
        for (int s = 0; s < 4; ++s)
            d[s] = __builtin_bit_cast(u32x4, __builtin_amdgcn_raw_buffer_load_b128(rin, o + (s & 1) * 64, (s >> 1) * 16 * (K * 2), 0));
    };
    u32x4 d[4];
    fetch(d, is_comm && wave < SW ? wave * NSC + cid : NBLK);
    GAS unsigned* cnt = (GAS unsigned*)(myws + W_CNT + (size_t)bid * 128);
    const unsigned cv = __hip_atomic_load(cnt, __ATOMIC_RELAXED, __HIP_MEMORY_SCOPE_AGENT);
    SB();
    if (wave == 0) STAMP(10);
    if (threadIdx.x < 32) ((LAS unsigned*)(lds + L_CTL))[threadIdx.x] = 0u;
    if (threadIdx.x < 256) *(LAS u32x2*)(lds + L_SEL + threadIdx.x * 8) = is_comm ? mover(threadIdx.x, true) : mover(threadIdx.x, false);
    SB();
    asm volatile("" :: "v"(code));
    const unsigned call = __builtin_amdgcn_readfirstlane(cv) + 1u;
    __syncthreads();
    if (threadIdx.x == 0) __hip_atomic_store(cnt, call, __ATOMIC_RELAXED, __HIP_MEMORY_SCOPE_AGENT);
    LAS unsigned char* lm = lds + L_SEL;
    int mul = loc;
    asm volatile("" : "+s"(mul));
    if (mul > 1) {
    GAS unsigned* tmo = (GAS unsigned*)(myws + W_TMO);
    const size_t xt_off = W_XT + (size_t)(call & 1u) * XT_BYTES;
    const unsigned oQ = pq < 0 ? OOB : ((NTN - pn) * BN + pq * 16) * 2;
    if (wave >= 6) {
        __builtin_amdgcn_s_setprio(AG_UPRIO);
        const int src = ((xcd + 2) & 6) + wave - 6;
        const unsigned u = tl;
        const __amdgpu_buffer_rsrc_t rf = rsrc(myws + W_FLAG + (size_t)src * NBLK * FS * 4, NBLK * FS * 4);
        const __amdgpu_buffer_rsrc_t rp = rsrc(myws + W_PK + (size_t)(call & 1u) * PK_BYTES + (size_t)src * NBLK * PBLK, NBLK * PBLK);
        const __amdgpu_buffer_rsrc_t rx = rsrc(myws + xt_off + (size_t)src * NBLK * BLK, NBLK * BLK);
        auto done = [&](unsigned b) {
            const unsigned g = b % NG, kt = b / NG;
            const unsigned fo = lane == 0 ? (((g >> 3) * NKT + kt) * 8 + (g & 7)) * FS * 4 : OOB;
            __builtin_amdgcn_raw_buffer_store_b32(call, rf, fo, 0, 17);
        };
        if (wave == 7) STAMP(8);
        unsigned long long lat_r = 0, lat_w = 0, lat_n = 0;
        if (src == a.rank) {
            fetch(d, u);
            for (unsigned b = u; b < (unsigned)NBLK; b += NU) {
#pragma unroll
                for (int s = 0; s < 4; ++s)
                    __builtin_amdgcn_raw_buffer_store_b128(d[s], rx, b * BLK + lane * 16 + s * 1024, 0, AG_UAUX);
                fetch(d, b + NU);
                wait_vm0();
                done(b);
            }
        } else {
            LAS unsigned char* lx = lds + L_UNP + (wave - 6) * UNP;
            LAS unsigned char* ls = lx + 2112;
            *(LAS u32x4*)(lx + lane * 32) = (u32x4){0u, 0u, 0u, 0u};
            *(LAS u32x4*)(lx + lane * 32 + 16) = (u32x4){0u, 0u, 0u, 0u};
            const unsigned long long want = tag_of(call) ? ~0ull : 0ull;
            const unsigned keep = (lane & 1) ? ~0u : 0x7fffffffu, keept = lane < 16 ? 0x7fffffffu : ~0u, sh = 31 - (lane >> 1), lw = (lane < 24 ? lane : 24) * 16;
            const unsigned long long t0 = rt();
            unsigned b = u, pend = ~0u;
            for (;;) {
                const unsigned so = b * PBLK;
                const unsigned hl = __builtin_amdgcn_raw_buffer_load_b32(rp, lane < 32 && b < (unsigned)NBLK ? so + 2560 + lane * 4 : OOB, 0, AG_HAUX);
                wait_vm0();
                const bool first = pend != ~0u;
                if (first) {
                    done(pend);
                    TRACE(src, pend, 2);
                    pend = ~0u;
                }
                if (b >= (unsigned)NBLK) break;
                if (((unsigned)(__ballot((int)hl < 0) ^ want) & 0x11111111u) != 0u) {
                    if (give_up(tmo, call, t0)) {
                        __hip_atomic_store(tmo, call, __ATOMIC_RELAXED, __HIP_MEMORY_SCOPE_AGENT);
                        break;
                    }
                    __builtin_amdgcn_s_sleep(AG_USLEEP);
                    continue;
                }
                if (first) lat_w += 100;
                ++lat_n;
                u32x4 p[2], t, x[2];
                u32x2 c;
                const unsigned h0 = __builtin_amdgcn_readlane(hl, 0);
                c = __builtin_bit_cast(u32x2, __builtin_amdgcn_raw_buffer_load_b64(rp, so + lane * 8 + 2048, 0, 17));
                t = __builtin_bit_cast(u32x4, __builtin_amdgcn_raw_buffer_load_b128(rp, lane < ((h0 & 512u) ? 24 : 16) ? so + lane * 16 + 2560 : OOB, 0, 17));
#pragma unroll
                for (int s = 0; s < 2; ++s)
                    p[s] = __builtin_bit_cast(u32x4, __builtin_amdgcn_raw_buffer_load_b128(rp, so + lane * 16 + s * 1024, 0, 17));
                TRACE(src, b, 1);
                const unsigned m2l = __builtin_amdgcn_readlane(hl, 3), m1l = bfi(0x7fffffffu, __builtin_amdgcn_readlane(hl, 4), h0 << 15);
                const unsigned long long m0 = (unsigned)__builtin_amdgcn_readlane(hl, 1) | ((unsigned long long)(unsigned)__builtin_amdgcn_readlane(hl, 2) << 32);
                const unsigned long long m1 = m1l | ((unsigned long long)(unsigned)__builtin_amdgcn_readlane(hl, 5) << 32);
                const unsigned e0 = h0 & 255u, cnt = (h0 >> 10) & 31u;
                const unsigned pa = (unsigned)lane - HW < cnt ? hl & 0x7ffu : 2048u;
                lx[pa] = (unsigned char)(hl >> 16);
                FENCE();
                x[0] = *(LAS u32x4*)(lx + lane * 32);
                x[1] = *(LAS u32x4*)(lx + lane * 32 + 16);
                FENCE();
                lx[pa] = 0;
                unsigned t2 = table2(e0), t3l = table3(e0, 0), t3h = table3(e0, 1), top0, top1, topc = m2l << sh, topt = h0 << (16 - (lane & 15));
                unsigned vo = (h0 & 256u) ? OOB : b * BLK + lane * 16, lq = (unsigned)(size_t)ls + (HW + cnt) * 4u;
                asm("v_cndmask_b32 %0, 0, -1, %1" : "=v"(top0) : "s"(m0));
                asm("v_cndmask_b32 %0, 0, -1, %1" : "=v"(top1) : "s"(m1));
                asm volatile("" : "+v"(t2), "+v"(t3l), "+v"(top0), "+v"(top1), "+v"(topc), "+v"(topt), "+v"(vo));
                asm volatile("" : "+s"(t3h), "+s"(lq));
                if (((__ballot((int)p[0].x < 0) ^ want) | (__ballot((int)p[1].x < 0) ^ want) |
                     ((__ballot((int)c.x < 0) ^ want) & 0x5555555555555555ull) | ((__ballot((int)t.x < 0) ^ want) & 0xffffull)) != 0ull) {
                    if (!give_up(tmo, call, t0)) continue;
                    __hip_atomic_store(tmo, call, __ATOMIC_RELAXED, __HIP_MEMORY_SCOPE_AGENT);
                    break;
                }
                t.x = bfi(keept, t.x, topt);
                *(LAS u32x4*)(ls + lw) = t;
                c.x = bfi(keep, c.x, topc);
                const u32x2 f = c & (c >> 1) & 0x55555555u;
                const unsigned nb = 3u * (__builtin_popcount(f.x) + __builtin_popcount(f.y));
                const unsigned off = scan(nb) - nb;
                u32x2 mv[8];
                FENCE();
                LAS unsigned* lt = (LAS unsigned*)(size_t)(((off >> 5) << 2) + lq);
                const unsigned w0 = lt[0], w1 = lt[1], w2 = lt[2];
#pragma unroll
                for (int j = 0; j < 8; ++j)
                    mv[j] = *(LAS u32x2*)(lm + ((c[j >> 2] >> (8 * (j & 3))) & 255u) * 8);
                FENCE();
                unsigned al = __builtin_amdgcn_alignbit(w1, w0, off), ah = __builtin_amdgcn_alignbit(w2, w1, off);
                p[0].x = bfi(0x7fffffffu, p[0].x, top0);
                p[1].x = bfi(0x7fffffffu, p[1].x, top1);
#pragma unroll
                for (int s = 0; s < 4; ++s) {
                    u32x4 d;
#pragma unroll
                    for (int i = 0; i < 2; ++i) {
                        const int j = 2 * s + i;
                        unsigned q = al;
                        al = __builtin_amdgcn_alignbit(ah, al, mv[j].y);
                        ah >>= mv[j].y;
                        asm("v_lshrrev_b32_sdwa %0, %1, %0 dst_sel:WORD_1 dst_unused:UNUSED_PRESERVE src0_sel:DWORD src1_sel:DWORD" : "+v"(q) : "v"(6), "v"(al));
                        const unsigned g = perm(t3h, t3l, (q & 0x00070007u) | ((q << 5) & 0x07000700u));
                        const unsigned e = perm(t2, g, mv[j].x) ^ x[j >> 2][j & 3];
                        const unsigned r0 = perm(e, p[j >> 2][j & 3], 0x05010400u), r1 = perm(e, p[j >> 2][j & 3], 0x07030602u);
                        d[2 * i] = __builtin_amdgcn_alignbit(r0, r0, 1);
                        d[2 * i + 1] = __builtin_amdgcn_alignbit(r1, r1, 1);
                    }
                    __builtin_amdgcn_raw_buffer_store_b128(d, rx, vo + s * 1024, 0, AG_UAUX);
                }
                pend = b;
                b += NU;
            }
        }
        if (wave == 7) STAMP(9);
        asm volatile("AG_CODE1:");
    }
    else if (wave < 4) {
        const unsigned su = ((xcd >> 1) * (4 * NTN) + tl) * 4 + wave, sk = NSLOT * BK + su * 4;
        const u32x4 sv = __builtin_bit_cast(u32x4, __builtin_amdgcn_raw_buffer_load_b128(rsrc(a.w, K * NL * 2),
            (sk + ((lane >> 1) & 3)) * (NL * 2) + NTN * BN * 2 + (lane >> 3) * 32 + (lane & 1) * 16, 0, 0));
        auto strip = [&]() {
            const unsigned r0 = (sk & ~31u) + ((sk >> 2) & 1) * 16 + ((sk >> 3) & 3) * 4;
            store16(sv, rsrc(myws + W_STRIP, K * BN * 2), su < (unsigned)NSTRIP ? (lane >> 3) * (K * 32) + r0 * 32 + (lane & 7) * 16 : OOB);
        };
        if (wave == 3) strip();
        constexpr int PW = AG_DRY ? 3 : 4;
        if (wave < PW) {
            const unsigned cw = ((xcd >> 1) * (4 * NTN) + tl) * PW + wave;
            constexpr int W1 = (AG_PD ? AG_PD : NKT) * BK * NL * 2;
            const __amdgpu_buffer_rsrc_t rw = rsrc(a.w, W1), rs = rsrc(a.x, MS * K * 2);
            constexpr int W0 = AG_PF0 * BK * NL * 2, ST = 16 * NTN * PW * 1024, NW = (W1 - W0 + ST - 1) / ST, NX = (MS * K * 2 + ST - 1) / ST;
            u32x4 pf[NW], px[NX];
#pragma unroll
            for (int i = 0; i < NW; ++i)
                pf[i] = __builtin_bit_cast(u32x4, __builtin_amdgcn_raw_buffer_load_b128(rw, W0 + i * ST + cw * 1024 + lane * 16, 0, AG_PFAUX));
            if (AG_PF == 2) {
#pragma unroll
                for (int i = 0; i < NX; ++i)
                    px[i] = __builtin_bit_cast(u32x4, __builtin_amdgcn_raw_buffer_load_b128(rs, i * ST + cw * 1024 + lane * 16, 0, AG_PFAUX));
                SB();
                strip();
#pragma unroll
                for (int i = 0; i < NX; ++i) asm volatile("" :: "v"(px[i]));
            }
#pragma unroll
            for (int i = 0; i < NW; ++i) asm volatile("" :: "v"(pf[i]));
            if (wave == 0) STAMP(12);
        }
        wait_vm0();
        if (lane == 0 && su < (unsigned)NSTRIP)
            __hip_atomic_fetch_add((GAS unsigned*)(myws + W_SCNT + (su & 7) * 128), 1u, __ATOMIC_RELAXED, __HIP_MEMORY_SCOPE_AGENT);
        const int g0 = (pm & 1) * 8 + wave * 2;
        const unsigned long long xb = (unsigned long long)(myws + xt_off + ((size_t)(pm >> 1) * NBLK + g0) * BLK);
        const u32x4 ra = (u32x4){(unsigned)xb, (unsigned)(xb >> 32), (unsigned)((NBLK - g0) * BLK), 0x00020000u};
        const unsigned lb = (unsigned)(unsigned long long)lds;
        int dry = AG_DRY ? (wave + 1) >> 2 : 0;
      for (;;) {
        int l1;
        asm volatile("v_mbcnt_lo_u32_b32 %0, -1, 0\n\tv_mbcnt_hi_u32_b32 %0, -1, %0" : "=v"(l1));
        unsigned vx = l1 * 16 + flip * BLK, have, need, pos, chk;
        f32x32 c[5];
        f32x16 c4;
        unsigned long long t1 = 0, t2 = 0;
        char* ob = a.out + ((size_t)(pm * BM + wave * 64) * NL + pn * BN) * 2;
        asm volatile(AG_KLOOP
                     : "={v[0:31]}"(c[0]), "={v[32:63]}"(c[1]), "={v[64:95]}"(c[2]), "={v[96:127]}"(c[3]), "={v[128:143]}"(c4),
                       "+{v242}"(vx), [hv] "=&s"(have), [nd] "=&s"(need), [ps] "=&s"(pos), [ck] "=&s"(chk), [t1] "+s"(t1), [t2] "+s"(t2)
                     : "{v241}"(lb + L_W + (l1 >> 2) * RS + (l1 & 3) * 8), "{v247}"(lb + L_CTL + wave * 4 + (l1 >> 6)), [rs] "s"(ra), [n8] "s"(part), "s"(ob),
                       [dry] "s"(dry)
                     : "v144", "v145", "v146", "v147", "v148", "v149", "v150", "v151", "v152", "v153", "v154", "v155", "v156", "v157", "v158", "v159", "v160", "v161", "v162", "v163", "v164", "v165", "v166", "v167", "v168", "v169", "v170", "v171", "v172", "v173", "v174", "v175", "v176", "v177", "v178", "v179", "v180", "v181", "v182", "v183", "v184", "v185", "v186", "v187", "v188", "v189", "v190", "v191", "v192", "v193", "v194", "v195", "v196", "v197", "v198", "v199", "v200", "v201", "v202", "v203", "v204", "v205", "v206", "v207", "v208", "v209", "v210", "v211", "v212", "v213", "v214", "v215", "v216", "v217", "v218", "v219", "v220", "v221", "v222", "v223", "v224", "v225", "v226", "v227", "v228", "v229", "v230", "v231", "v232", "v233", "v234", "v235", "v236", "v237", "v238", "v239", "v240", "v243", "v244", "v245", "v246", "scc", "memory");
#pragma unroll
        for (int i = 0; i < 16; ++i) c[4][i] = c4[i];
        int l2;
        asm volatile("v_mbcnt_lo_u32_b32 %0, -1, 0\n\tv_mbcnt_hi_u32_b32 %0, -1, %0" : "=v"(l2));
        constexpr int ES = RS + 16, E0 = NKT - NSLOT;
        unsigned pv = ctl[4 + (l2 & 7)];
        while (!__all(pv >= (unsigned)(E0 + 5))) {
            __builtin_amdgcn_s_sleep(1);
            pv = ctl[4 + (l2 & 7)];
        }
        const bool bad = __builtin_amdgcn_readfirstlane(pv) == ALL + 1;
        asm volatile("" ::: "memory");
        static_assert(E0 % NSLOT == 0 && 4 * 64 * ES <= 5 * SLOT, "epilogue staging");
        LAS unsigned char* st = lds + L_W + wave * 64 * ES;
        LAS unsigned char* sw = st + (l2 & 15) * ES + (l2 >> 4) * 8;
        LAS unsigned char* srS = st + (l2 >> 4) * ES + (l2 & 15) * 16;
        LAS unsigned char* srH = st + (l2 >> 1) * ES + 256 + (l2 & 1) * 16;
        const __amdgpu_buffer_rsrc_t ro = rsrc(ob, (64 * NL - pn * BN) * 2);
        const unsigned fl = flip * (32 * NL * 2), voS = (l2 >> 4) * (NL * 2) + (l2 & 15) * 16, voH = (l2 >> 1) * (NL * 2) + (l2 & 1) * 16 + oQ;
        const unsigned vo[2][2] = {{voS + fl, voH + fl}, {voS + (32 * NL * 2 - fl), voH + (part ? OOB : 32 * NL * 2)}};
        auto put = [&](int m) {
#pragma unroll
            for (int n = 0; n < NB; ++n) {
                const int e = 4 * (m * NB + n);
                const f32x4 v = (f32x4){c[e >> 5][e & 31], c[e >> 5][(e & 31) + 1], c[e >> 5][(e & 31) + 2], c[e >> 5][(e & 31) + 3]};
                *(LAS u32x2*)(sw + m * 16 * ES + n * 32) = __builtin_bit_cast(u32x2, __builtin_convertvector(v, bf16x4));
            }
        };
        u32x4 o[2][5];
        auto get = [&](int m) {
#pragma unroll
            for (int s = 0; s < 4; ++s) o[m & 1][s] = *(LAS u32x4*)(srS + (m * 16 + s * 4) * ES);
            if (m & 1) o[1][4] = *(LAS u32x4*)(srH + (m - 1) * 16 * ES);
        };
        auto out = [&](int m) {
#pragma unroll
            for (int s = 0; s < 4 + (m & 1); ++s)
                __builtin_amdgcn_raw_buffer_store_b128(o[m & 1][s], ro, s < 4 ? vo[m >> 1][0] + ((m & 1) * 16 + s * 4) * NL * 2 : vo[m >> 1][1],
                                                       0, AG_OAUX);
        };
        if (!bad) {
            put(0); get(0); put(1); SB(); out(0); get(1); put(2); SB(); out(1); get(2); put(3); SB(); out(2); get(3); out(3);
        } else {
#pragma unroll
            for (int s = 0; s < 10; ++s) o[s / 5][s % 5] = (u32x4){0x7fc07fc0u, 0x7fc07fc0u, 0x7fc07fc0u, 0x7fc07fc0u};
            out(0); out(1); out(2); out(3);
        }
        asm volatile("s_mov_b64 exec, -1\n\ts_sub_u32 %0, %0, 1" : "+s"(dry) :: "scc", "memory");
        if (dry < 0) break;
      }
    } else if (wave == 4) {
        __builtin_amdgcn_s_setprio(AG_WPRIO);
        unsigned long long gp[NPIECE / 2], gs[NPIECE / 2];
#pragma unroll
        for (int i = 0; i < NPIECE / 2; ++i) {
            const unsigned o = i * 1024 + lane * 16, r = o / RS, cb = o % RS;
            const unsigned k = ((r >> 2) & 3) * 8 + (r >> 4) * 4 + (r & 3);
            gp[i] = (unsigned long long)a.w + pn * BN * 2 + k * (NL * 2) + (cb < 256 ? cb : pq < 0 ? cb - 32 : cb - 256 + oQ);
            gs[i] = 32 * NL * 2;
        }
        const __amdgpu_buffer_rsrc_t rall = rsrc(a.w, K * NL * 2);
        const unsigned ln = ((xcd >> 1) * (4 * NTN) + tl) * 9 + lane, po = lane < 9 && ln < BK * NL * 2 / 128 ? ln * 128 + AG_PD * (BK * NL * 2) : OOB;
        int pub = 0;
        unsigned so = 0;
        for (int kt = 0; kt < NKT; ++kt) {
            if (kt >= NSLOT) {
                const unsigned nd = kt - NSLOT + 1;
                if (!__all(prog[lane & 3] >= nd)) {
                    if (pub < kt) {
                        asm volatile("s_waitcnt vmcnt(0)" ::: "memory");
                        pub = kt;
                        ctl[lane & 3] = pub;
                    }
                    while (!__all(prog[lane & 3] >= nd)) __builtin_amdgcn_s_sleep(AG_WSLEEP);
                }
                asm volatile("" ::: "memory");
                if ((nd & 7) == 0) STAMP(16 + (nd >> 3));
            }
            if (kt == NSLOT && pq >= 0) {
                const unsigned long long t0 = rt();
                while (!__all((int)(__hip_atomic_load((GAS unsigned*)(myws + W_SCNT + (lane & 7) * 128), __ATOMIC_RELAXED, __HIP_MEMORY_SCOPE_AGENT) - call * (NSTRIP / 8)) >= 0)) {
                    if (give_up(tmo, call, t0)) {
                        __hip_atomic_store(tmo, call, __ATOMIC_RELAXED, __HIP_MEMORY_SCOPE_AGENT);
                        break;
                    }
                    __builtin_amdgcn_s_sleep(4);
                }
                FENCE();
#pragma unroll
                for (int i = 0; i < NPIECE / 2; ++i) {
                    const unsigned o = i * 1024 + lane * 16, r = o / RS, cb = o % RS;
                    if (cb >= 256) gp[i] = (unsigned long long)myws + W_STRIP + pq * (K * 32) + (NSLOT * BK + r) * 32 + cb - 256, gs[i] = 1024;
                }
            }
            LAS unsigned char* dst = lds + L_W + so;
            {
#pragma unroll
                for (int i = 0; i < NPIECE; ++i)
                {
                    __builtin_amdgcn_global_load_lds((const GAS void*)gp[i % (NPIECE / 2)], (LAS void*)(dst + i * 1024), 16, 0, 0);
                    gp[i % (NPIECE / 2)] += gs[i % (NPIECE / 2)];
                }
            }
            if (AG_PD) __builtin_amdgcn_raw_ptr_buffer_load_lds(rall, (LAS void*)(lds + L_SINK), 4, po + kt * (BK * NL * 2), 0, 0, 0);
            so = so + SLOT == NSLOT * SLOT ? 0u : so + SLOT;
            if (kt + 1 - pub > 2) {
                asm volatile("s_waitcnt vmcnt(%0)" :: "n"(2 * NPIECE + (AG_PD ? 3 : 0)) : "memory");
                ++pub;
                ctl[lane & 3] = pub;
            }
        }
        asm volatile("s_waitcnt vmcnt(0)" ::: "memory");
        ctl[lane & 3] = ALL;
    }
    else if (wave == 5) {
        __builtin_amdgcn_s_setprio(AG_PPRIO);
        GAS unsigned* flag = (GAS unsigned*)(myws + W_FLAG) + pm * NKT * 8 * FS;
        const unsigned long long t0 = rt();
        int cur = 0, win = AG_PW, last = 0;
        unsigned long long npoll = 0;
        while (cur < NKT - AG_TP) {
            ++npoll;
            const int idx = cur * 8 + lane;
            unsigned v = call;
            if (lane < win * 8 && idx < NKT * 8) v = __hip_atomic_load(flag + idx * FS, __ATOMIC_RELAXED, __HIP_MEMORY_SCOPE_SYSTEM);
            const unsigned long long miss = __ballot((int)(v - call) < 0);
            const int adv = miss ? (__builtin_ctzll(miss) >> 3) : win;
            if (AG_APW) { win = adv > 0 && last > 0 ? AG_PW : 1; last = adv; }
            if (adv > 0) {
                if (cur == 0) STAMP(4);
                if (((cur + adv) >> 3) != (cur >> 3)) STAMP(24 + (((cur + adv) >> 3) & 7));
                cur = cur + adv < NKT ? cur + adv : NKT;
                asm volatile("" ::: "memory");
                ctl[4 + (lane & 3)] = cur < NKT ? cur : ALL;
            } else {
                if (give_up(tmo, call, t0)) {
                    __hip_atomic_store(tmo, call, __ATOMIC_RELAXED, __HIP_MEMORY_SCOPE_AGENT);
                    ctl[4 + (lane & 3)] = ALL + 1;
                    cur = NKT;
                    break;
                }
                __builtin_amdgcn_s_sleep(AG_PSLEEP);
            }
        }
        STAMP(5);
    }
    } else {
        LAS unsigned char* lx = lds + L_W + wave * 8192;
        LAS unsigned char* ls = lx + 2048;
        LAS unsigned short* lp = (LAS unsigned short*)(lx + 4096);
        if (wave == 0) STAMP(11);
        const size_t xt_off = W_XT + (size_t)(call & 1u) * XT_BYTES, pk_off = W_PK + (size_t)(call & 1u) * PK_BYTES;
        if (wave < SW) {
            const int sid = wave * NSC + cid;
            __amdgpu_buffer_rsrc_t rd[NR];
#pragma unroll
            for (int i = 0; i < NR; ++i) rd[i] = rsrc(a.ws[i], (unsigned)W_END);
            const unsigned xt0 = (unsigned)xt_off + a.rank * NBLK * BLK, pk0 = (unsigned)pk_off + a.rank * NBLK * PBLK;
            const unsigned tbase = (call - 1u) * (unsigned)NBLK - 2u * NSEND;
            const unsigned tag = tag_of(call);
            auto low = [&](const u32x4 (&d)[4], int j, unsigned& e) {
                const unsigned d0 = d[j >> 1][(j & 1) * 2], d1 = d[j >> 1][(j & 1) * 2 + 1];
                const unsigned r0 = __builtin_amdgcn_alignbit(d0, d0, 31), r1 = __builtin_amdgcn_alignbit(d1, d1, 31);
                e = perm(r1, r0, 0x07050301u);
                return perm(r1, r0, 0x06040200u);
            };
            auto pack = [&](const u32x4 (&d)[4], u32x4 (&pk)[3], bool& raw, bool& annex) {
                unsigned e[8], sum = 0;
                *(LAS u32x2*)(ls + 512 + lane * 8) = (u32x2){0u, 0u};
#pragma unroll
                for (int j = 0; j < 8; ++j) {
                    pk[j >> 2][j & 3] = low(d, j, e[j]);
                    sum = __builtin_amdgcn_sad_u8(e[j], 0u, sum);
                }
                const unsigned mean = (unsigned)__builtin_amdgcn_readlane(scan(sum), 63) >> 11;
                const unsigned e0 = mean < 7u ? 7u : mean > 252u ? 252u : mean;
                const unsigned k2 = key2(e0), t2 = table2(e0), t3l = table3(e0, 0), t3h = table3(e0, 1);
                u32x4 x[2];
                u32x2 c = {0u, 0u}, mv[8];
                unsigned mask = 0, i3[8];
#pragma unroll
                for (int j = 0; j < 8; ++j) {
                    const unsigned sel = (e[j] ^ k2) & 3u * B1;
                    const unsigned t = ((perm(t2, t2, sel) ^ e[j]) >> 1) + 0x7f7f7f7fu;
                    const unsigned m = perm(t, t << 8, 0x0b090a08u);
                    const unsigned code = sel | (m & 3u * B1);
                    c[j >> 2] |= __builtin_amdgcn_udot4(code, 0x40100401u, 0u, false) << (8 * (j & 3));
                    mv[j] = *(LAS u32x2*)(lm + __builtin_amdgcn_udot4(code & (code >> 1) & B1, 0x40201008u, 0u, false));
                    i3[j] = (e[j] ^ e0 * B1) & m & 7u * B1;
                    const unsigned y = (perm(t3h, t3l, i3[j]) ^ e[j]) & m;
                    x[j >> 2][j & 3] = y;
                    mask = ((((y & 0x7f7f7f7fu) + 0x7f7f7f7fu) | y) & 0x80808080u) | (mask >> 1);
                }
                *(LAS u32x4*)(lx + lane * 32) = x[0];
                *(LAS u32x4*)(lx + lane * 32 + 16) = x[1];
                *(LAS u32x2*)(ls + lane * 8) = c;
                unsigned cnt = 0;
                for (;;) {
                    const unsigned long long act = __ballot(mask != 0u);
                    if (act == 0ull || cnt > NPATCH) break;
                    const unsigned p = __builtin_ctz(mask | 0x80000000u);
                    const unsigned at = __builtin_amdgcn_mbcnt_hi((unsigned)(act >> 32), __builtin_amdgcn_mbcnt_lo((unsigned)act, cnt));
                    lp[mask != 0u ? at : 127u] = (unsigned short)(lane * 32 + (p & 7u) * 4 + (p >> 3));
                    mask &= mask - 1u;
                    cnt += __builtin_popcountll(act);
                }
                FENCE();
                const unsigned pos = lp[lane] & 0x7ffu;
                unsigned long long acc = 0;
                unsigned nb = 0;
#pragma unroll
                for (int j = 7; j >= 0; --j) {
                    const unsigned g = perm(0u, i3[j], mv[j].x);
                    acc = (acc << mv[j].y) | __builtin_amdgcn_udot4(g, 0x003e0801u, g >> 15, false);
                    nb += mv[j].y;
                }
                const unsigned xb = lx[pos];
                const unsigned in = scan(nb), off = in - nb + cnt * 32u;
                const unsigned nd = cnt + (((unsigned)__builtin_amdgcn_readlane(in, 63) + 31u) >> 5);
                raw = cnt > NPATCH || nd > NLONG || __ballot(nb > 63u) != 0ull;
                annex = !raw && nd > NTAIL;
                LAS unsigned* lt = (LAS unsigned*)(ls + 512) + HW;
                const unsigned long long sa = acc << (off & 31u);
                __hip_atomic_fetch_or(lt + (off >> 5), (unsigned)sa, __ATOMIC_RELAXED, __HIP_MEMORY_SCOPE_WAVEFRONT);
                __hip_atomic_fetch_or(lt + (off >> 5) + 1, (unsigned)(sa >> 32), __ATOMIC_RELAXED, __HIP_MEMORY_SCOPE_WAVEFRONT);
                __hip_atomic_fetch_or(lt + (off >> 5) + 2, (unsigned)(acc >> 33) >> (~off & 31u), __ATOMIC_RELAXED, __HIP_MEMORY_SCOPE_WAVEFRONT);
                lt[(unsigned)lane < cnt ? lane : 300] = pos | (xb << 16);
                FENCE();
                pk[2] = *(LAS u32x4*)(ls + lane * 16);
                FENCE();
                const unsigned long long m0 = __ballot((int)pk[0].x < 0), m1 = __ballot((int)pk[1].x < 0);
                if (lane == 33) pk[2].x = (unsigned)m1, pk[2].y = (unsigned)(m1 >> 32);
                const unsigned long long m2 = __ballot((int)pk[2].x < 0);
                if (lane == 32) pk[2] = (u32x4){e0 | (raw ? 256u : cnt << 10) | (annex ? 512u : 0u) | (unsigned)(m2 >> 33) << 16, (unsigned)m0, (unsigned)(m0 >> 32), (unsigned)m2};
#pragma unroll
                for (int s = 0; s < 3; ++s) pk[s].x = bfi(s < 2 || lane < 48 ? 0x7fffffffu : ~0u, pk[s].x, tag);
            };
            unsigned b0 = sid, b1 = sid + NSEND;
            u32x4 nx[4], pk[3];
            bool raw, annex;
            unsigned skip = 0;
            if (AG_EARLY == 1 || wave < AG_EARLY - 1) {
#pragma unroll
                for (int j = 0; j < 8; ++j) pk[j >> 2][j & 3] = low(d, j, skip);
#pragma unroll
                for (int s = 0; s < 2; ++s) pk[s].x = (pk[s].x & 0x7fffffffu) | tag;
#pragma unroll
                for (int i = 0; i < NR - 1; ++i)
#pragma unroll
                    for (int s = 0; s < 2; ++s)
                        store16(pk[s], rd[i], pk0 + b0 * PBLK + lane * 16 + s * 1024);
                SB();
                skip = OOB;
            }
            pack(d, pk, raw, annex);
            if (wave == 0) STAMP(6);
            unsigned long long nsent = 0;
            unsigned seq = wave;
            while (b0 < (unsigned)NBLK) {
                const unsigned sp = pk0 + b0 * PBLK + lane * 16;
                if (annex) {
#pragma unroll
                    for (int i = 0; i < NR - 1; ++i)
                        store16(pk[2], rd[i], (lane >> 3) == 6 ? sp + 2048 : OOB);
                    wait_vm0();
                }
                while (ctl[0] < seq) __builtin_amdgcn_s_sleep(1);
                FENCE();
                if (raw) {
                    const unsigned so = xt0 + b0 * BLK + lane * 16;
#pragma unroll
                    for (int i = 0; i < NR - 1; ++i)
#pragma unroll
                        for (int s = 0; s < 4; ++s)
                            store16(d[s], rd[i], so + s * 1024);
                    wait_vm0();
                }
                TRACE(a.rank, b0, 0);
                fetch(nx, b1);
                const unsigned tk = __builtin_amdgcn_raw_ptr_buffer_atomic_add_i32(1, rd[NR - 1], lane == 0 ? (unsigned)W_TICKET : OOB, 0, 0);
                const unsigned sp2 = lane < 48 ? sp + 2048 : OOB;
#pragma unroll
                for (int i = 0; i < NR - 1; ++i)
#pragma unroll
                    for (int s = 0; s < 2; ++s)
                        store16(pk[s], rd[i], (sp | skip) + s * 1024);
                skip = 0;
#pragma unroll
                for (int i = 0; i < NR - 1; ++i)
                    store16(pk[2], rd[i], sp2);
                FENCE();
                if (lane == 0) __hip_atomic_fetch_max((LAS unsigned*)(lds + L_CTL), seq + 1, __ATOMIC_RELAXED, __HIP_MEMORY_SCOPE_WORKGROUP);
                seq = AG_TURN == 2 ? 0 : seq + SW;
                TRACE(a.rank, b0, 1);
#pragma unroll
                for (int s = 0; s < 4; ++s) d[s] = nx[s];
                pack(d, pk, raw, annex);
                b0 = b1;
                ++nsent;
                b1 = __builtin_amdgcn_readfirstlane(tk) - tbase;
            }
            ctl[0] = ~0u;
            if (wave == 0) STAMP(7);
        }
    }
}
}

extern "C" void kernel_launch(int rank, int nranks, ncclComm_t comm,
                              hipStream_t stream,
                              void* const* d_in, const int* in_sizes, int n_in,
                              void* d_out, int out_size,
                              const int* in_dtypes, int out_dtype,
                              void* const* d_out_all,
                              void* const* d_ws_all, size_t ws_bytes) {
    if (nranks != ag::NR || (unsigned)rank >= (unsigned)ag::NR || n_in != 2 || ws_bytes < ag::W_END ||
        in_sizes[0] != ag::MS * ag::K || in_sizes[1] != ag::K * ag::NL || out_size != ag::M * ag::NL) {
        fprintf(stderr, "ag_gemm: unexpected problem shape\n");
        return;
    }
    ag::Args a;
    a.x = (const char*)d_in[0];
    a.w = (const char*)d_in[1];
    a.out = (char*)d_out;
    for (int i = 0; i < ag::NR; ++i) a.ws[i] = (char*)d_ws_all[(rank + 1 + i) % ag::NR];
    a.rank = rank;
    a.pad = 0;
    hipLaunchKernelGGL(ag::ag_gemm, dim3(ag::NWG), dim3(ag::NTHR), 0, stream, a);
    HIP_CHECK(hipGetLastError());
}
```

```cpp
#include <hip/hip_runtime.h>
#include <rccl/rccl.h>
#include <cstdio>
#include <cstdint>
#include <type_traits>

#define HIP_CHECK(expr) do { hipError_t _e = (expr); if (_e != hipSuccess) { \
    fprintf(stderr, "HIP error %s at %s:%d\n", hipGetErrorString(_e), __FILE__, __LINE__); \
    return; } } while (0)

#define LAS __attribute__((address_space(3)))
#define GAS __attribute__((address_space(1)))
typedef unsigned u32x4 __attribute__((ext_vector_type(4)));
typedef unsigned u32x2 __attribute__((ext_vector_type(2)));
typedef unsigned short u16x2 __attribute__((ext_vector_type(2)));
typedef short s16x4 __attribute__((ext_vector_type(4)));
typedef short s16x8 __attribute__((ext_vector_type(8)));
typedef __bf16 bf16x4 __attribute__((ext_vector_type(4)));
typedef __bf16 bf16x8 __attribute__((ext_vector_type(8)));
typedef float f32x4 __attribute__((ext_vector_type(4)));
typedef float f32x16 __attribute__((ext_vector_type(16)));
typedef float f32x32 __attribute__((ext_vector_type(32)));

#define AG_SW 4
#define AG_PSLEEP 2
#define AG_TURN 1
#define AG_UPRIO 0
#define AG_PF 2
#define AG_PF0 3
#define AG_PFAUX 0
#define AG_PFD 0
#define AG_CPRIO 0
#define AG_WSLEEP 2
#define AG_WPRIO 0
#define AG_T1 1
#define AG_A32 1
#define AG_PQ 0
#define AG_PPRIO 2
#define AG_DRY 1
#define AG_SPEC 1
#define AG_CPF 1
#define AG_PD 0
#define AG_UHI 0
#define AG_BAL 0
#define AG_APW 1
#define AG_TP 2
#define AG_TW 0
#define AG_TB 0
#define AG_TBP 1
#define AG_XP 0
#define AG_XAUX 17
#define AG_WAUX 0
#define AG_BAUX 17
#define AG_TS 6
#define AG_STRIP 1
#define AG_HOLE 0
#define AG_EARLY 3
#define AG_SD 0
#define AG_PP 0
#define AG_MAP 1
#define AG_FAUX 17
#define AG_USLEEP 1
#define AG_HAUX 17
#define AG_UAUX 17
#define AG_FS 32
#define AG_PW 2
#define AG_OAUX 2

namespace ag {
constexpr int NR = 8;
constexpr int M = 4096, K = 4096, NL = 2048;
constexpr int MS = M / NR;
constexpr int NB = 9;
constexpr int BM = 256, BN = 128, BK = 64;
constexpr int NKT = K / BK;
constexpr int NG = MS / 32;
constexpr int BLK = 32 * BK * 2;
constexpr int NBLK = NG * NKT;
constexpr int RS = NB * 32;
constexpr int SLOT = BK * RS, NSLOT = 8, NPIECE = SLOT / 1024;
constexpr int NTN = NL / BN - 1;
constexpr int NWG = 256, CPX = 4;
constexpr int NWAVE = 8, NTHR = NWAVE * 64;
constexpr int NSC = NWG - 16 * NTN, SW = AG_SW, NSEND = NSC * SW;
static_assert(2 * NSEND <= NBLK && SW <= NWAVE, "sender geometry");
constexpr int PBLK = 3072;
constexpr unsigned HW = 6, NPATCH = 32 - HW, NTAIL = 64 - HW, NLONG = NTAIL + 32;
constexpr int NU = 4 * NTN;
constexpr int L_SEL = 0, L_W = 2048;
constexpr int L_CTL = L_W + NSLOT * SLOT;
constexpr unsigned ALL = 1000;
constexpr int L_UNP = L_CTL + 128, UNP = 2048 + 64 + 448;
constexpr int L_SINK = L_UNP + 2 * UNP, LDS_BYTES = L_SINK + 256;
static_assert(LDS_BYTES > 80 * 1024 && LDS_BYTES <= 160 * 1024, "one workgroup per CU");
constexpr size_t W_CNT = 0;
constexpr size_t W_TMO = W_CNT + (size_t)NWG * 128;
constexpr size_t W_TICKET = W_TMO + 1024;
constexpr size_t W_SCNT = W_TICKET + 1024;
constexpr size_t W_DBG = 768 * 1024;
constexpr size_t W_XT = 1u << 20;
constexpr size_t XT_BYTES = (size_t)M * K * 2;
constexpr size_t W_PK = W_XT + 2 * XT_BYTES;
constexpr size_t PK_BYTES = (size_t)NR * NBLK * PBLK;
constexpr size_t W_FLAG = W_PK + 2 * PK_BYTES;
constexpr int FS = AG_FS;
constexpr unsigned FL_BYTES = NR * NBLK * FS * 4;
constexpr size_t W_STRIP = W_FLAG + FL_BYTES;
constexpr size_t W_END = W_STRIP + (size_t)K * BN * 2;
constexpr int NSTRIP = (NKT - NSLOT) * 16;
static_assert(W_SCNT + 1024 <= W_DBG && NSTRIP <= 16 * NTN * 4, "flags");
constexpr unsigned OOB = 0x40000000u;
constexpr unsigned long long DEADLINE = 30000000ull;

struct Args {
    const char* x;
    const char* w;
    char* out;
    char* ws[NR];
    int rank;
    int pad;
};

__device__ __forceinline__ unsigned long long rt() { return __builtin_amdgcn_s_memrealtime(); }
__device__ __forceinline__ __amdgpu_buffer_rsrc_t rsrc(const void* p, unsigned bytes) {
    return __builtin_amdgcn_make_buffer_rsrc(const_cast<void*>(p), 0, bytes, 0x00020000);
}
__device__ __forceinline__ void store16(u32x4 d, __amdgpu_buffer_rsrc_t r, unsigned off) {
    __builtin_amdgcn_raw_buffer_store_b128(d, r, off, 0, 17);
}
__device__ __forceinline__ void wait_vm0() {
    __builtin_amdgcn_s_waitcnt(0x0F70);
    asm volatile("s_waitcnt vmcnt(0)" ::: "memory");
}
__device__ __forceinline__ bool give_up(GAS unsigned* tmo, unsigned call, unsigned long long t0) {
    const unsigned long long dt = rt() - t0;
    if (dt < 50000ull) return false;
    return dt > DEADLINE || __builtin_amdgcn_readfirstlane(__hip_atomic_load(tmo, __ATOMIC_RELAXED, __HIP_MEMORY_SCOPE_AGENT)) == call;
}
constexpr unsigned B1 = 0x01010101u;
__device__ __forceinline__ unsigned tag_of(unsigned call) { return ((call + 1u) << 30) & 0x80000000u; }
__device__ __forceinline__ unsigned bfi(unsigned m, unsigned a, unsigned b) { return (a & m) | (b & ~m); }
__device__ __forceinline__ unsigned perm(unsigned hi, unsigned lo, unsigned sel) { return __builtin_amdgcn_perm(hi, lo, sel); }
__device__ __forceinline__ unsigned key2(unsigned e0) { return (((e0 + 3u) & 3u) ^ 3u) * B1; }
__device__ __forceinline__ unsigned table2(unsigned e0) {
    return e0 * B1 + (((0x03020100u ^ key2(e0)) + (4u - (e0 & 3u)) * B1) & 3u * B1);
}
__device__ __forceinline__ unsigned table3(unsigned e0, unsigned hi) {
    const unsigned r = (e0 & 7u) * B1;
    return e0 * B1 - ((8u * B1 + r - ((0x03020100u + hi * 4u * B1) ^ r)) & 7u * B1) + (hi ? 0u : 3u);
}
__device__ __forceinline__ unsigned scan(unsigned v) {
    v += (unsigned)__builtin_amdgcn_update_dpp(0, (int)v, 0x111, 0xf, 0xf, true);
    v += (unsigned)__builtin_amdgcn_update_dpp(0, (int)v, 0x112, 0xf, 0xf, true);
    v += (unsigned)__builtin_amdgcn_update_dpp(0, (int)v, 0x114, 0xf, 0xf, true);
    v += (unsigned)__builtin_amdgcn_update_dpp(0, (int)v, 0x118, 0xf, 0xf, true);
    v += (unsigned)__builtin_amdgcn_update_dpp(0, (int)v, 0x142, 0xa, 0xf, false);
    v += (unsigned)__builtin_amdgcn_update_dpp(0, (int)v, 0x143, 0xc, 0xf, false);
    return v;
}
__device__ __forceinline__ u32x2 mover(unsigned m, bool gather) {
    unsigned sel = gather ? 12u * B1 : 0u, n = 0;
#pragma unroll
    for (unsigned b = 0; b < 4; ++b) {
        const unsigned c = gather ? ((m >> b) & 1u) * 3u : (m >> (2u * b)) & 3u;
        if (gather) sel ^= c ? (12u ^ b) << (8u * n) : 0u;
        else sel |= (c == 3u ? n : 4u + c) << (8u * b);
        n += c == 3u;
    }
    return (u32x2){sel, 3u * n};
}
#define STAMP(i) do {} while (0)
#define STAMPC(i) do {} while (0)
#undef TRACE
#define TRACE(s, b, i) do {} while (0)

#define SB() __builtin_amdgcn_sched_barrier(0)
#define FENCE() asm volatile("" ::: "memory")
#define AG_T0 ""
#define AG_KLOOP \
    "s_cmp_eq_u32 %[dry], 0\n" \
    "s_cbranch_scc1 GO%=\n" \
    "v_mov_b32 v244, -2\n" \
    "s_mov_b64 exec, 0\n" \
    "s_mov_b32 %[nd], 62\n" \
    "s_mov_b32 %[ps], 0\n" \
    "s_branch R1%=\n" \
    "GO%=:\n" \
    "v_mov_b64 v[0:1], 0\n" \
    "v_mov_b64 v[2:3], 0\n" \
    "v_mov_b64 v[4:5], 0\n" \
    "v_mov_b64 v[6:7], 0\n" \
    "v_mov_b64 v[8:9], 0\n" \
    "v_mov_b64 v[10:11], 0\n" \
    "v_mov_b64 v[12:13], 0\n" \
    "v_mov_b64 v[14:15], 0\n" \
    "v_mov_b64 v[16:17], 0\n" \
    "v_mov_b64 v[18:19], 0\n" \
    "v_mov_b64 v[20:21], 0\n" \
    "v_mov_b64 v[22:23], 0\n" \
    "v_mov_b64 v[24:25], 0\n" \
    "v_mov_b64 v[26:27], 0\n" \
    "v_mov_b64 v[28:29], 0\n" \
    "v_mov_b64 v[30:31], 0\n" \
    "v_mov_b64 v[32:33], 0\n" \
    "v_mov_b64 v[34:35], 0\n" \
    "v_mov_b64 v[36:37], 0\n" \
    "v_mov_b64 v[38:39], 0\n" \
    "v_mov_b64 v[40:41], 0\n" \
    "v_mov_b64 v[42:43], 0\n" \
    "v_mov_b64 v[44:45], 0\n" \
    "v_mov_b64 v[46:47], 0\n" \
    "v_mov_b64 v[48:49], 0\n" \
    "v_mov_b64 v[50:51], 0\n" \
    "v_mov_b64 v[52:53], 0\n" \
    "v_mov_b64 v[54:55], 0\n" \
    "v_mov_b64 v[56:57], 0\n" \
    "v_mov_b64 v[58:59], 0\n" \
    "v_mov_b64 v[60:61], 0\n" \
    "v_mov_b64 v[62:63], 0\n" \
    "v_mov_b64 v[64:65], 0\n" \
    "v_mov_b64 v[66:67], 0\n" \
    "v_mov_b64 v[68:69], 0\n" \
    "v_mov_b64 v[70:71], 0\n" \
    "v_mov_b64 v[72:73], 0\n" \
    "v_mov_b64 v[74:75], 0\n" \
    "v_mov_b64 v[76:77], 0\n" \
    "v_mov_b64 v[78:79], 0\n" \
    "v_mov_b64 v[80:81], 0\n" \
    "v_mov_b64 v[82:83], 0\n" \
    "v_mov_b64 v[84:85], 0\n" \
    "v_mov_b64 v[86:87], 0\n" \
    "v_mov_b64 v[88:89], 0\n" \
    "v_mov_b64 v[90:91], 0\n" \
    "v_mov_b64 v[92:93], 0\n" \
    "v_mov_b64 v[94:95], 0\n" \
    "v_mov_b64 v[96:97], 0\n" \
    "v_mov_b64 v[98:99], 0\n" \
    "v_mov_b64 v[100:101], 0\n" \
    "v_mov_b64 v[102:103], 0\n" \
    "v_mov_b64 v[104:105], 0\n" \
    "v_mov_b64 v[106:107], 0\n" \
    "v_mov_b64 v[108:109], 0\n" \
    "v_mov_b64 v[110:111], 0\n" \
    "v_mov_b64 v[112:113], 0\n" \
    "v_mov_b64 v[114:115], 0\n" \
    "v_mov_b64 v[116:117], 0\n" \
    "v_mov_b64 v[118:119], 0\n" \
    "v_mov_b64 v[120:121], 0\n" \
    "v_mov_b64 v[122:123], 0\n" \
    "v_mov_b64 v[124:125], 0\n" \
    "v_mov_b64 v[126:127], 0\n" \
    "v_mov_b64 v[128:129], 0\n" \
    "v_mov_b64 v[130:131], 0\n" \
    "v_mov_b64 v[132:133], 0\n" \
    "v_mov_b64 v[134:135], 0\n" \
    "v_mov_b64 v[136:137], 0\n" \
    "v_mov_b64 v[138:139], 0\n" \
    "v_mov_b64 v[140:141], 0\n" \
    "v_mov_b64 v[142:143], 0\n" \
    "v_mov_b32 v240, v241\n" \
    "v_xor_b32 v243, 0x1000, v242\n" \
    "v_mov_b32 v246, 0\n" \
    "s_mov_b32 %[nd], 1\n" \
    "s_mov_b32 %[ps], 0\n" \
    "P0%=:\n" \
    "ds_read2_b32 v[244:245], v247 offset1:4\n" \
    "s_waitcnt lgkmcnt(0)\n" \
    "v_min_u32 v244, v244, v245\n" \
    "s_nop 0\n" \
    "v_readfirstlane_b32 %[hv], v244\n" \
    "s_cmp_ge_u32 %[hv], %[nd]\n" \
    "s_cbranch_scc1 R0%=\n" \
    "s_sleep 1\n" \
    "s_branch P0%=\n" \
    "R0%=:\n" \
    "buffer_load_dwordx4 v[144:147], v242, %[rs], 0 offen offset:0 sc0 sc1\n" \
    "buffer_load_dwordx4 v[148:151], v242, %[rs], 0 offen offset:1024 sc0 sc1\n" \
    "buffer_load_dwordx4 v[152:155], v242, %[rs], 0 offen offset:2048 sc0 sc1\n" \
    "buffer_load_dwordx4 v[156:159], v242, %[rs], 0 offen offset:3072 sc0 sc1\n" \
    "buffer_load_dwordx4 v[160:163], v243, %[rs], 0 offen offset:0 sc0 sc1\n" \
    "buffer_load_dwordx4 v[164:167], v243, %[rs], 0 offen offset:1024 sc0 sc1\n" \
    "buffer_load_dwordx4 v[168:171], v243, %[rs], 0 offen offset:2048 sc0 sc1\n" \
    "buffer_load_dwordx4 v[172:175], v243, %[rs], 0 offen offset:3072 sc0 sc1\n" \
    "v_add_u32 v242, 0x10000, v242\n" \
    "v_add_u32 v243, 0x10000, v243\n" \
    "s_mov_b32 %[nd], 2\n" \
    "P1%=:\n" \
    "ds_read2_b32 v[244:245], v247 offset1:4\n" \
    "s_waitcnt lgkmcnt(0)\n" \
    "v_min_u32 v244, v244, v245\n" \
    "s_nop 0\n" \
    "v_readfirstlane_b32 %[hv], v244\n" \
    "s_cmp_ge_u32 %[hv], %[nd]\n" \
    "s_cbranch_scc1 R1%=\n" \
    "s_sleep 1\n" \
    "s_branch P1%=\n" \
    "R1%=:\n" \
    AG_T0 \
    "s_cmp_eq_u32 %[n8], 0\n" \
    "s_cbranch_scc0 E8%=\n" \
    "ds_read_b64_tr_b16 v[208:209], v240 offset:0\n" \
    "ds_read_b64_tr_b16 v[210:211], v240 offset:4608\n" \
    "ds_read_b64_tr_b16 v[212:213], v240 offset:9216\n" \
    "ds_read_b64_tr_b16 v[214:215], v240 offset:13824\n" \
    "ds_read_b64_tr_b16 v[216:217], v240 offset:32\n" \
    "ds_read_b64_tr_b16 v[218:219], v240 offset:4640\n" \
    "ds_read_b64_tr_b16 v[220:221], v240 offset:9248\n" \
    "ds_read_b64_tr_b16 v[222:223], v240 offset:13856\n" \
    "s_waitcnt lgkmcnt(4) vmcnt(0)\n" \
    "L9_%=:\n" \
    "v_mfma_f32_16x16x32_bf16 v[0:3], v[208:211], v[144:147], v[0:3]\n" \
    "ds_read_b64_tr_b16 v[224:225], v240 offset:64\n" \
    "v_mfma_f32_16x16x32_bf16 v[0:3], v[212:215], v[148:151], v[0:3]\n" \
    "ds_read_b64_tr_b16 v[226:227], v240 offset:4672\n" \
    "s_waitcnt vmcnt(4)\n" \
    "v_mfma_f32_16x16x32_bf16 v[36:39], v[212:215], v[156:159], v[36:39]\n" \
    "ds_read_b64_tr_b16 v[228:229], v240 offset:9280\n" \
    "v_mfma_f32_16x16x32_bf16 v[36:39], v[208:211], v[152:155], v[36:39]\n" \
    "ds_read_b64_tr_b16 v[230:231], v240 offset:13888\n" \
    "s_waitcnt vmcnt(2)\n" \
    "v_mfma_f32_16x16x32_bf16 v[72:75], v[208:211], v[160:163], v[72:75]\n" \
    "buffer_load_dwordx4 v[176:179], v242, %[rs], 0 offen offset:0 sc0 sc1\n" \
    "v_mfma_f32_16x16x32_bf16 v[72:75], v[212:215], v[164:167], v[72:75]\n" \
    "buffer_load_dwordx4 v[180:183], v242, %[rs], 0 offen offset:1024 sc0 sc1\n" \
    "s_waitcnt vmcnt(2)\n" \
    "v_mfma_f32_16x16x32_bf16 v[108:111], v[212:215], v[172:175], v[108:111]\n" \
    "buffer_load_dwordx4 v[184:187], v242, %[rs], 0 offen offset:2048 sc0 sc1\n" \
    "v_mfma_f32_16x16x32_bf16 v[108:111], v[208:211], v[168:171], v[108:111]\n" \
    "s_waitcnt lgkmcnt(4)\n" \
    "v_mfma_f32_16x16x32_bf16 v[112:115], v[216:219], v[168:171], v[112:115]\n" \
    "ds_read_b64_tr_b16 v[208:209], v240 offset:96\n" \
    "s_add_u32 %[nd], %[nd], 1\n" \
    "v_mfma_f32_16x16x32_bf16 v[112:115], v[220:223], v[172:175], v[112:115]\n" \
    "ds_read_b64_tr_b16 v[210:211], v240 offset:4704\n" \
    "s_add_u32 %[ps], %[ps], 18432\n" \
    "v_mfma_f32_16x16x32_bf16 v[76:79], v[220:223], v[164:167], v[76:79]\n" \
    "ds_read_b64_tr_b16 v[212:213], v240 offset:9312\n" \
    "s_cmp_eq_u32 %[ps], 147456\n" \
    "v_mfma_f32_16x16x32_bf16 v[76:79], v[216:219], v[160:163], v[76:79]\n" \
    "ds_read_b64_tr_b16 v[214:215], v240 offset:13920\n" \
    "s_cselect_b32 %[ps], 0, %[ps]\n" \
    "v_mfma_f32_16x16x32_bf16 v[40:43], v[216:219], v[152:155], v[40:43]\n" \
    "buffer_load_dwordx4 v[188:191], v242, %[rs], 0 offen offset:3072 sc0 sc1\n" \
    "v_mfma_f32_16x16x32_bf16 v[40:43], v[220:223], v[156:159], v[40:43]\n" \
    "buffer_load_dwordx4 v[192:195], v243, %[rs], 0 offen offset:0 sc0 sc1\n" \
    "v_mfma_f32_16x16x32_bf16 v[4:7], v[220:223], v[148:151], v[4:7]\n" \
    "buffer_load_dwordx4 v[196:199], v243, %[rs], 0 offen offset:1024 sc0 sc1\n" \
    "v_mfma_f32_16x16x32_bf16 v[4:7], v[216:219], v[144:147], v[4:7]\n" \
    "s_waitcnt lgkmcnt(4)\n" \
    "v_mfma_f32_16x16x32_bf16 v[8:11], v[224:227], v[144:147], v[8:11]\n" \
    "ds_read_b64_tr_b16 v[216:217], v240 offset:128\n" \
    "s_cmp_eq_u32 %[nd], 65\n" \
    "v_mfma_f32_16x16x32_bf16 v[8:11], v[228:231], v[148:151], v[8:11]\n" \
    "ds_read_b64_tr_b16 v[218:219], v240 offset:4736\n" \
    "s_cselect_b32 %[ck], -1, %[nd]\n" \
    "v_mfma_f32_16x16x32_bf16 v[44:47], v[228:231], v[156:159], v[44:47]\n" \
    "ds_read_b64_tr_b16 v[220:221], v240 offset:9344\n" \
    "v_mfma_f32_16x16x32_bf16 v[44:47], v[224:227], v[152:155], v[44:47]\n" \
    "ds_read_b64_tr_b16 v[222:223], v240 offset:13952\n" \
    "v_mfma_f32_16x16x32_bf16 v[80:83], v[224:227], v[160:163], v[80:83]\n" \
    "buffer_load_dwordx4 v[200:203], v243, %[rs], 0 offen offset:2048 sc0 sc1\n" \
    "v_mfma_f32_16x16x32_bf16 v[80:83], v[228:231], v[164:167], v[80:83]\n" \
    "buffer_load_dwordx4 v[204:207], v243, %[rs], 0 offen offset:3072 sc0 sc1\n" \
    "v_mfma_f32_16x16x32_bf16 v[116:119], v[228:231], v[172:175], v[116:119]\n" \
    "v_add_u32 v242, 0x10000, v242\n" \
    "v_mfma_f32_16x16x32_bf16 v[116:119], v[224:227], v[168:171], v[116:119]\n" \
    "s_waitcnt lgkmcnt(4)\n" \
    "v_mfma_f32_16x16x32_bf16 v[120:123], v[208:211], v[168:171], v[120:123]\n" \
    "ds_read_b64_tr_b16 v[224:225], v240 offset:160\n" \
    "v_mfma_f32_16x16x32_bf16 v[120:123], v[212:215], v[172:175], v[120:123]\n" \
    "ds_read_b64_tr_b16 v[226:227], v240 offset:4768\n" \
    "v_mfma_f32_16x16x32_bf16 v[84:87], v[212:215], v[164:167], v[84:87]\n" \
    "ds_read_b64_tr_b16 v[228:229], v240 offset:9376\n" \
    "v_mfma_f32_16x16x32_bf16 v[84:87], v[208:211], v[160:163], v[84:87]\n" \
    "ds_read_b64_tr_b16 v[230:231], v240 offset:13984\n" \
    "v_mfma_f32_16x16x32_bf16 v[48:51], v[208:211], v[152:155], v[48:51]\n" \
    "v_add_u32 v243, 0x10000, v243\n" \
    "v_mfma_f32_16x16x32_bf16 v[48:51], v[212:215], v[156:159], v[48:51]\n" \
    "v_add_u32 v246, 1, v246\n" \
    "v_mfma_f32_16x16x32_bf16 v[12:15], v[212:215], v[148:151], v[12:15]\n" \
    "v_mfma_f32_16x16x32_bf16 v[12:15], v[208:211], v[144:147], v[12:15]\n" \
    "s_waitcnt lgkmcnt(4)\n" \
    "v_mfma_f32_16x16x32_bf16 v[16:19], v[216:219], v[144:147], v[16:19]\n" \
    "ds_read_b64_tr_b16 v[208:209], v240 offset:192\n" \
    "v_mfma_f32_16x16x32_bf16 v[16:19], v[220:223], v[148:151], v[16:19]\n" \
    "ds_read_b64_tr_b16 v[210:211], v240 offset:4800\n" \
    "v_mfma_f32_16x16x32_bf16 v[52:55], v[220:223], v[156:159], v[52:55]\n" \
    "ds_read_b64_tr_b16 v[212:213], v240 offset:9408\n" \
    "v_mfma_f32_16x16x32_bf16 v[52:55], v[216:219], v[152:155], v[52:55]\n" \
    "ds_read_b64_tr_b16 v[214:215], v240 offset:14016\n" \
    "v_mfma_f32_16x16x32_bf16 v[88:91], v[216:219], v[160:163], v[88:91]\n" \
    "v_mfma_f32_16x16x32_bf16 v[88:91], v[220:223], v[164:167], v[88:91]\n" \
    "v_mfma_f32_16x16x32_bf16 v[124:127], v[220:223], v[172:175], v[124:127]\n" \
    "v_mfma_f32_16x16x32_bf16 v[124:127], v[216:219], v[168:171], v[124:127]\n" \
    "s_waitcnt lgkmcnt(4)\n" \
    "v_mfma_f32_16x16x32_bf16 v[128:131], v[224:227], v[168:171], v[128:131]\n" \
    "ds_read_b64_tr_b16 v[216:217], v240 offset:224\n" \
    "v_mfma_f32_16x16x32_bf16 v[128:131], v[228:231], v[172:175], v[128:131]\n" \
    "ds_read_b64_tr_b16 v[218:219], v240 offset:4832\n" \
    "v_mfma_f32_16x16x32_bf16 v[92:95], v[228:231], v[164:167], v[92:95]\n" \
    "ds_read_b64_tr_b16 v[220:221], v240 offset:9440\n" \
    "v_mfma_f32_16x16x32_bf16 v[92:95], v[224:227], v[160:163], v[92:95]\n" \
    "ds_read_b64_tr_b16 v[222:223], v240 offset:14048\n" \
    "v_mfma_f32_16x16x32_bf16 v[56:59], v[224:227], v[152:155], v[56:59]\n" \
    "v_mfma_f32_16x16x32_bf16 v[56:59], v[228:231], v[156:159], v[56:59]\n" \
    "v_mfma_f32_16x16x32_bf16 v[20:23], v[228:231], v[148:151], v[20:23]\n" \
    "v_mfma_f32_16x16x32_bf16 v[20:23], v[224:227], v[144:147], v[20:23]\n" \
    "s_waitcnt lgkmcnt(4)\n" \
    "v_mfma_f32_16x16x32_bf16 v[24:27], v[208:211], v[144:147], v[24:27]\n" \
    "ds_read_b64_tr_b16 v[224:225], v240 offset:256\n" \
    "v_mfma_f32_16x16x32_bf16 v[24:27], v[212:215], v[148:151], v[24:27]\n" \
    "ds_read_b64_tr_b16 v[226:227], v240 offset:4864\n" \
    "v_mfma_f32_16x16x32_bf16 v[60:63], v[212:215], v[156:159], v[60:63]\n" \
    "ds_read_b64_tr_b16 v[228:229], v240 offset:9472\n" \
    "v_mfma_f32_16x16x32_bf16 v[60:63], v[208:211], v[152:155], v[60:63]\n" \
    "ds_read_b64_tr_b16 v[230:231], v240 offset:14080\n" \
    "v_mfma_f32_16x16x32_bf16 v[96:99], v[208:211], v[160:163], v[96:99]\n" \
    "v_add_u32 v240, %[ps], v241\n" \
    "v_mfma_f32_16x16x32_bf16 v[96:99], v[212:215], v[164:167], v[96:99]\n" \
    "ds_write_b32 v247, v246 offset:32\n" \
    "v_mfma_f32_16x16x32_bf16 v[132:135], v[212:215], v[172:175], v[132:135]\n" \
    "v_mfma_f32_16x16x32_bf16 v[132:135], v[208:211], v[168:171], v[132:135]\n" \
    "s_waitcnt lgkmcnt(5)\n" \
    "v_mfma_f32_16x16x32_bf16 v[136:139], v[216:219], v[168:171], v[136:139]\n" \
    "ds_read2_b32 v[244:245], v247 offset1:4\n" \
    "ds_read_b64_tr_b16 v[208:209], v240 offset:0\n" \
    "v_mfma_f32_16x16x32_bf16 v[136:139], v[220:223], v[172:175], v[136:139]\n" \
    "ds_read_b64_tr_b16 v[210:211], v240 offset:4608\n" \
    "v_mfma_f32_16x16x32_bf16 v[100:103], v[220:223], v[164:167], v[100:103]\n" \
    "ds_read_b64_tr_b16 v[212:213], v240 offset:9216\n" \
    "v_mfma_f32_16x16x32_bf16 v[100:103], v[216:219], v[160:163], v[100:103]\n" \
    "ds_read_b64_tr_b16 v[214:215], v240 offset:13824\n" \
    "v_mfma_f32_16x16x32_bf16 v[64:67], v[216:219], v[152:155], v[64:67]\n" \
    "v_mfma_f32_16x16x32_bf16 v[64:67], v[220:223], v[156:159], v[64:67]\n" \
    "v_mfma_f32_16x16x32_bf16 v[28:31], v[220:223], v[148:151], v[28:31]\n" \
    "v_mfma_f32_16x16x32_bf16 v[28:31], v[216:219], v[144:147], v[28:31]\n" \
    "s_waitcnt lgkmcnt(4)\n" \
    "v_mfma_f32_16x16x32_bf16 v[32:35], v[224:227], v[144:147], v[32:35]\n" \
    "ds_read_b64_tr_b16 v[216:217], v240 offset:32\n" \
    "v_mfma_f32_16x16x32_bf16 v[32:35], v[228:231], v[148:151], v[32:35]\n" \
    "ds_read_b64_tr_b16 v[218:219], v240 offset:4640\n" \
    "v_mfma_f32_16x16x32_bf16 v[68:71], v[228:231], v[156:159], v[68:71]\n" \
    "ds_read_b64_tr_b16 v[220:221], v240 offset:9248\n" \
    "v_mfma_f32_16x16x32_bf16 v[68:71], v[224:227], v[152:155], v[68:71]\n" \
    "ds_read_b64_tr_b16 v[222:223], v240 offset:13856\n" \
    "v_mfma_f32_16x16x32_bf16 v[104:107], v[224:227], v[160:163], v[104:107]\n" \
    "v_min_u32 v244, v244, v245\n" \
    "v_mfma_f32_16x16x32_bf16 v[104:107], v[228:231], v[164:167], v[104:107]\n" \
    "v_readfirstlane_b32 %[hv], v244\n" \
    "v_mfma_f32_16x16x32_bf16 v[140:143], v[228:231], v[172:175], v[140:143]\n" \
    "s_cmp_ge_u32 %[hv], %[ck]\n" \
    "v_mfma_f32_16x16x32_bf16 v[140:143], v[224:227], v[168:171], v[140:143]\n" \
    "s_waitcnt lgkmcnt(4) vmcnt(6)\n" \
    "s_cbranch_scc0 S9_%=\n" \
    "R9_%=:\n" \
    "v_mfma_f32_16x16x32_bf16 v[0:3], v[208:211], v[176:179], v[0:3]\n" \
    "ds_read_b64_tr_b16 v[224:225], v240 offset:64\n" \
    "v_mfma_f32_16x16x32_bf16 v[0:3], v[212:215], v[180:183], v[0:3]\n" \
    "ds_read_b64_tr_b16 v[226:227], v240 offset:4672\n" \
    "s_waitcnt vmcnt(4)\n" \
    "v_mfma_f32_16x16x32_bf16 v[36:39], v[212:215], v[188:191], v[36:39]\n" \
    "ds_read_b64_tr_b16 v[228:229], v240 offset:9280\n" \
    "v_mfma_f32_16x16x32_bf16 v[36:39], v[208:211], v[184:187], v[36:39]\n" \
    "ds_read_b64_tr_b16 v[230:231], v240 offset:13888\n" \
    "s_waitcnt vmcnt(2)\n" \
    "v_mfma_f32_16x16x32_bf16 v[72:75], v[208:211], v[192:195], v[72:75]\n" \
    "buffer_load_dwordx4 v[144:147], v242, %[rs], 0 offen offset:0 sc0 sc1\n" \
    "v_mfma_f32_16x16x32_bf16 v[72:75], v[212:215], v[196:199], v[72:75]\n" \
    "buffer_load_dwordx4 v[148:151], v242, %[rs], 0 offen offset:1024 sc0 sc1\n" \
    "s_waitcnt vmcnt(2)\n" \
    "v_mfma_f32_16x16x32_bf16 v[108:111], v[212:215], v[204:207], v[108:111]\n" \
    "buffer_load_dwordx4 v[152:155], v242, %[rs], 0 offen offset:2048 sc0 sc1\n" \
    "v_mfma_f32_16x16x32_bf16 v[108:111], v[208:211], v[200:203], v[108:111]\n" \
    "s_waitcnt lgkmcnt(4)\n" \
    "v_mfma_f32_16x16x32_bf16 v[112:115], v[216:219], v[200:203], v[112:115]\n" \
    "ds_read_b64_tr_b16 v[208:209], v240 offset:96\n" \
    "s_add_u32 %[nd], %[nd], 1\n" \
    "v_mfma_f32_16x16x32_bf16 v[112:115], v[220:223], v[204:207], v[112:115]\n" \
    "ds_read_b64_tr_b16 v[210:211], v240 offset:4704\n" \
    "s_add_u32 %[ps], %[ps], 18432\n" \
    "v_mfma_f32_16x16x32_bf16 v[76:79], v[220:223], v[196:199], v[76:79]\n" \
    "ds_read_b64_tr_b16 v[212:213], v240 offset:9312\n" \
    "s_cmp_eq_u32 %[ps], 147456\n" \
    "v_mfma_f32_16x16x32_bf16 v[76:79], v[216:219], v[192:195], v[76:79]\n" \
    "ds_read_b64_tr_b16 v[214:215], v240 offset:13920\n" \
    "s_cselect_b32 %[ps], 0, %[ps]\n" \
    "v_mfma_f32_16x16x32_bf16 v[40:43], v[216:219], v[184:187], v[40:43]\n" \
    "buffer_load_dwordx4 v[156:159], v242, %[rs], 0 offen offset:3072 sc0 sc1\n" \
    "v_mfma_f32_16x16x32_bf16 v[40:43], v[220:223], v[188:191], v[40:43]\n" \
    "buffer_load_dwordx4 v[160:163], v243, %[rs], 0 offen offset:0 sc0 sc1\n" \
    "v_mfma_f32_16x16x32_bf16 v[4:7], v[220:223], v[180:183], v[4:7]\n" \
    "buffer_load_dwordx4 v[164:167], v243, %[rs], 0 offen offset:1024 sc0 sc1\n" \
    "v_mfma_f32_16x16x32_bf16 v[4:7], v[216:219], v[176:179], v[4:7]\n" \
    "s_waitcnt lgkmcnt(4)\n" \
    "v_mfma_f32_16x16x32_bf16 v[8:11], v[224:227], v[176:179], v[8:11]\n" \
    "ds_read_b64_tr_b16 v[216:217], v240 offset:128\n" \
    "v_mfma_f32_16x16x32_bf16 v[8:11], v[228:231], v[180:183], v[8:11]\n" \
    "ds_read_b64_tr_b16 v[218:219], v240 offset:4736\n" \
    "v_mfma_f32_16x16x32_bf16 v[44:47], v[228:231], v[188:191], v[44:47]\n" \
    "ds_read_b64_tr_b16 v[220:221], v240 offset:9344\n" \
    "v_mfma_f32_16x16x32_bf16 v[44:47], v[224:227], v[184:187], v[44:47]\n" \
    "ds_read_b64_tr_b16 v[222:223], v240 offset:13952\n" \
    "v_mfma_f32_16x16x32_bf16 v[80:83], v[224:227], v[192:195], v[80:83]\n" \
    "buffer_load_dwordx4 v[168:171], v243, %[rs], 0 offen offset:2048 sc0 sc1\n" \
    "v_mfma_f32_16x16x32_bf16 v[80:83], v[228:231], v[196:199], v[80:83]\n" \
    "buffer_load_dwordx4 v[172:175], v243, %[rs], 0 offen offset:3072 sc0 sc1\n" \
    "v_mfma_f32_16x16x32_bf16 v[116:119], v[228:231], v[204:207], v[116:119]\n" \
    "v_add_u32 v242, 0x10000, v242\n" \
    "v_mfma_f32_16x16x32_bf16 v[116:119], v[224:227], v[200:203], v[116:119]\n" \
    "s_waitcnt lgkmcnt(4)\n" \
    "v_mfma_f32_16x16x32_bf16 v[120:123], v[208:211], v[200:203], v[120:123]\n" \
    "ds_read_b64_tr_b16 v[224:225], v240 offset:160\n" \
    "v_mfma_f32_16x16x32_bf16 v[120:123], v[212:215], v[204:207], v[120:123]\n" \
    "ds_read_b64_tr_b16 v[226:227], v240 offset:4768\n" \
    "v_mfma_f32_16x16x32_bf16 v[84:87], v[212:215], v[196:199], v[84:87]\n" \
    "ds_read_b64_tr_b16 v[228:229], v240 offset:9376\n" \
    "v_mfma_f32_16x16x32_bf16 v[84:87], v[208:211], v[192:195], v[84:87]\n" \
    "ds_read_b64_tr_b16 v[230:231], v240 offset:13984\n" \
    "v_mfma_f32_16x16x32_bf16 v[48:51], v[208:211], v[184:187], v[48:51]\n" \
    "v_add_u32 v243, 0x10000, v243\n" \
    "v_mfma_f32_16x16x32_bf16 v[48:51], v[212:215], v[188:191], v[48:51]\n" \
    "v_add_u32 v246, 1, v246\n" \
    "v_mfma_f32_16x16x32_bf16 v[12:15], v[212:215], v[180:183], v[12:15]\n" \
    "v_mfma_f32_16x16x32_bf16 v[12:15], v[208:211], v[176:179], v[12:15]\n" \
    "s_waitcnt lgkmcnt(4)\n" \
    "v_mfma_f32_16x16x32_bf16 v[16:19], v[216:219], v[176:179], v[16:19]\n" \
    "ds_read_b64_tr_b16 v[208:209], v240 offset:192\n" \
    "v_mfma_f32_16x16x32_bf16 v[16:19], v[220:223], v[180:183], v[16:19]\n" \
    "ds_read_b64_tr_b16 v[210:211], v240 offset:4800\n" \
    "v_mfma_f32_16x16x32_bf16 v[52:55], v[220:223], v[188:191], v[52:55]\n" \
    "ds_read_b64_tr_b16 v[212:213], v240 offset:9408\n" \
    "v_mfma_f32_16x16x32_bf16 v[52:55], v[216:219], v[184:187], v[52:55]\n" \
    "ds_read_b64_tr_b16 v[214:215], v240 offset:14016\n" \
    "v_mfma_f32_16x16x32_bf16 v[88:91], v[216:219], v[192:195], v[88:91]\n" \
    "v_mfma_f32_16x16x32_bf16 v[88:91], v[220:223], v[196:199], v[88:91]\n" \
    "v_mfma_f32_16x16x32_bf16 v[124:127], v[220:223], v[204:207], v[124:127]\n" \
    "v_mfma_f32_16x16x32_bf16 v[124:127], v[216:219], v[200:203], v[124:127]\n" \
    "s_waitcnt lgkmcnt(4)\n" \
    "v_mfma_f32_16x16x32_bf16 v[128:131], v[224:227], v[200:203], v[128:131]\n" \
    "ds_read_b64_tr_b16 v[216:217], v240 offset:224\n" \
    "v_mfma_f32_16x16x32_bf16 v[128:131], v[228:231], v[204:207], v[128:131]\n" \
    "ds_read_b64_tr_b16 v[218:219], v240 offset:4832\n" \
    "v_mfma_f32_16x16x32_bf16 v[92:95], v[228:231], v[196:199], v[92:95]\n" \
    "ds_read_b64_tr_b16 v[220:221], v240 offset:9440\n" \
    "v_mfma_f32_16x16x32_bf16 v[92:95], v[224:227], v[192:195], v[92:95]\n" \
    "ds_read_b64_tr_b16 v[222:223], v240 offset:14048\n" \
    "v_mfma_f32_16x16x32_bf16 v[56:59], v[224:227], v[184:187], v[56:59]\n" \
    "v_mfma_f32_16x16x32_bf16 v[56:59], v[228:231], v[188:191], v[56:59]\n" \
    "v_mfma_f32_16x16x32_bf16 v[20:23], v[228:231], v[180:183], v[20:23]\n" \
    "v_mfma_f32_16x16x32_bf16 v[20:23], v[224:227], v[176:179], v[20:23]\n" \
    "s_waitcnt lgkmcnt(4)\n" \
    "v_mfma_f32_16x16x32_bf16 v[24:27], v[208:211], v[176:179], v[24:27]\n" \
    "ds_read_b64_tr_b16 v[224:225], v240 offset:256\n" \
    "v_mfma_f32_16x16x32_bf16 v[24:27], v[212:215], v[180:183], v[24:27]\n" \
    "ds_read_b64_tr_b16 v[226:227], v240 offset:4864\n" \
    "v_mfma_f32_16x16x32_bf16 v[60:63], v[212:215], v[188:191], v[60:63]\n" \
    "ds_read_b64_tr_b16 v[228:229], v240 offset:9472\n" \
    "v_mfma_f32_16x16x32_bf16 v[60:63], v[208:211], v[184:187], v[60:63]\n" \
    "ds_read_b64_tr_b16 v[230:231], v240 offset:14080\n" \
    "v_mfma_f32_16x16x32_bf16 v[96:99], v[208:211], v[192:195], v[96:99]\n" \
    "v_add_u32 v240, %[ps], v241\n" \
    "v_mfma_f32_16x16x32_bf16 v[96:99], v[212:215], v[196:199], v[96:99]\n" \
    "ds_write_b32 v247, v246 offset:32\n" \
    "v_mfma_f32_16x16x32_bf16 v[132:135], v[212:215], v[204:207], v[132:135]\n" \
    "v_mfma_f32_16x16x32_bf16 v[132:135], v[208:211], v[200:203], v[132:135]\n" \
    "s_waitcnt lgkmcnt(5)\n" \
    "v_mfma_f32_16x16x32_bf16 v[136:139], v[216:219], v[200:203], v[136:139]\n" \
    "ds_read2_b32 v[244:245], v247 offset1:4\n" \
    "ds_read_b64_tr_b16 v[208:209], v240 offset:0\n" \
    "v_mfma_f32_16x16x32_bf16 v[136:139], v[220:223], v[204:207], v[136:139]\n" \
    "ds_read_b64_tr_b16 v[210:211], v240 offset:4608\n" \
    "v_mfma_f32_16x16x32_bf16 v[100:103], v[220:223], v[196:199], v[100:103]\n" \
    "ds_read_b64_tr_b16 v[212:213], v240 offset:9216\n" \
    "v_mfma_f32_16x16x32_bf16 v[100:103], v[216:219], v[192:195], v[100:103]\n" \
    "ds_read_b64_tr_b16 v[214:215], v240 offset:13824\n" \
    "v_mfma_f32_16x16x32_bf16 v[64:67], v[216:219], v[184:187], v[64:67]\n" \
    "v_mfma_f32_16x16x32_bf16 v[64:67], v[220:223], v[188:191], v[64:67]\n" \
    "v_mfma_f32_16x16x32_bf16 v[28:31], v[220:223], v[180:183], v[28:31]\n" \
    "v_mfma_f32_16x16x32_bf16 v[28:31], v[216:219], v[176:179], v[28:31]\n" \
    "s_waitcnt lgkmcnt(4)\n" \
    "v_mfma_f32_16x16x32_bf16 v[32:35], v[224:227], v[176:179], v[32:35]\n" \
    "ds_read_b64_tr_b16 v[216:217], v240 offset:32\n" \
    "v_mfma_f32_16x16x32_bf16 v[32:35], v[228:231], v[180:183], v[32:35]\n" \
    "ds_read_b64_tr_b16 v[218:219], v240 offset:4640\n" \
    "v_mfma_f32_16x16x32_bf16 v[68:71], v[228:231], v[188:191], v[68:71]\n" \
    "ds_read_b64_tr_b16 v[220:221], v240 offset:9248\n" \
    "v_mfma_f32_16x16x32_bf16 v[68:71], v[224:227], v[184:187], v[68:71]\n" \
    "ds_read_b64_tr_b16 v[222:223], v240 offset:13856\n" \
    "v_mfma_f32_16x16x32_bf16 v[104:107], v[224:227], v[192:195], v[104:107]\n" \
    "v_min_u32 v244, v244, v245\n" \
    "v_mfma_f32_16x16x32_bf16 v[104:107], v[228:231], v[196:199], v[104:107]\n" \
    "v_readfirstlane_b32 %[hv], v244\n" \
    "v_mfma_f32_16x16x32_bf16 v[140:143], v[228:231], v[204:207], v[140:143]\n" \
    "s_cmp_ge_u32 %[hv], %[nd]\n" \
    "v_mfma_f32_16x16x32_bf16 v[140:143], v[224:227], v[200:203], v[140:143]\n" \
    "s_waitcnt lgkmcnt(4) vmcnt(6)\n" \
    "s_cbranch_scc1 L9_%=\n" \
    "Q9_%=:\n" \
    "ds_read2_b32 v[244:245], v247 offset1:4\n" \
    "s_waitcnt lgkmcnt(0)\n" \
    "v_min_u32 v244, v244, v245\n" \
    "s_nop 0\n" \
    "v_readfirstlane_b32 %[hv], v244\n" \
    "s_cmp_ge_u32 %[hv], %[nd]\n" \
    "s_cbranch_scc1 L9_%=\n" \
    "s_sleep 1\n" \
    "s_branch Q9_%=\n" \
    "P9_%=:\n" \
    "ds_read2_b32 v[244:245], v247 offset1:4\n" \
    "s_waitcnt lgkmcnt(0)\n" \
    "v_min_u32 v244, v244, v245\n" \
    "s_nop 0\n" \
    "v_readfirstlane_b32 %[hv], v244\n" \
    "s_cmp_ge_u32 %[hv], %[nd]\n" \
    "s_cbranch_scc1 R9_%=\n" \
    "s_sleep 1\n" \
    "s_branch P9_%=\n" \
    "S9_%=:\n" \
    "s_cmp_eq_u32 %[nd], 65\n" \
    "s_cbranch_scc0 P9_%=\n" \
    "v_mfma_f32_16x16x32_bf16 v[0:3], v[208:211], v[176:179], v[0:3]\n" \
    "v_and_b32 v171, -16, v247\n" \
    "ds_read_b128 v[144:147], v171 offset:32\n" \
    "ds_read_b64_tr_b16 v[224:225], v240 offset:64\n" \
    "v_mfma_f32_16x16x32_bf16 v[0:3], v[212:215], v[180:183], v[0:3]\n" \
    "s_waitcnt vmcnt(4)\n" \
    "v_mbcnt_lo_u32_b32 v175, -1, 0\n" \
    "v_mbcnt_hi_u32_b32 v175, -1, v175\n" \
    "v_mfma_f32_16x16x32_bf16 v[36:39], v[212:215], v[188:191], v[36:39]\n" \
    "ds_read_b64_tr_b16 v[226:227], v240 offset:4672\n" \
    "v_and_b32 v242, 15, v175\n" \
    "v_mul_u32_u24 v242, 304, v242\n" \
    "v_mfma_f32_16x16x32_bf16 v[36:39], v[208:211], v[184:187], v[36:39]\n" \
    "s_waitcnt vmcnt(2)\n" \
    "v_lshrrev_b32 v174, 4, v175\n" \
    "v_lshl_add_u32 v242, v174, 3, v242\n" \
    "v_mfma_f32_16x16x32_bf16 v[72:75], v[208:211], v[192:195], v[72:75]\n" \
    "ds_read_b64_tr_b16 v[228:229], v240 offset:9280\n" \
    "v_add_u32 v242, %[st], v242\n" \
    "v_lshrrev_b32 v174, 3, v175\n" \
    "v_mfma_f32_16x16x32_bf16 v[72:75], v[212:215], v[196:199], v[72:75]\n" \
    "s_waitcnt vmcnt(0)\n" \
    "v_and_b32 v246, 7, v175\n" \
    "v_lshlrev_b32 v246, 4, v246\n" \
    "v_mfma_f32_16x16x32_bf16 v[108:111], v[212:215], v[204:207], v[108:111]\n" \
    "ds_read_b64_tr_b16 v[230:231], v240 offset:13888\n" \
    "v_mul_u32_u24 v243, 304, v174\n" \
    "v_add3_u32 v243, v243, v246, %[st]\n" \
    "v_mfma_f32_16x16x32_bf16 v[108:111], v[208:211], v[200:203], v[108:111]\n" \
    "v_lshl_add_u32 v246, v174, 12, v246\n" \
    "v_lshrrev_b32 v174, 1, v175\n" \
    "v_and_b32 v172, 1, v175\n" \
    "s_waitcnt lgkmcnt(4)\n" \
    "v_mfma_f32_16x16x32_bf16 v[112:115], v[216:219], v[200:203], v[112:115]\n" \
    "ds_read_b64_tr_b16 v[208:209], v240 offset:96\n" \
    "v_cvt_pk_bf16_f32 v0, v0, v1\n" \
    "v_cvt_pk_bf16_f32 v1, v2, v3\n" \
    "v_mfma_f32_16x16x32_bf16 v[112:115], v[220:223], v[204:207], v[112:115]\n" \
    "v_min3_u32 v144, v144, v145, v146\n" \
    "v_min_u32 v244, v144, v147\n" \
    "s_nop 0\n" \
    "v_readfirstlane_b32 %[hv], v244\n" \
    "s_cmp_ge_u32 %[hv], 61\n" \
    "s_cbranch_scc0 W9_%=\n" \
    "V9_%=:\n" \
    "ds_write_b64 v242, v[0:1] offset:0\n" \
    "v_mfma_f32_16x16x32_bf16 v[76:79], v[220:223], v[196:199], v[76:79]\n" \
    "ds_read_b64_tr_b16 v[210:211], v240 offset:4704\n" \
    "v_cvt_pk_bf16_f32 v36, v36, v37\n" \
    "v_cvt_pk_bf16_f32 v37, v38, v39\n" \
    "v_mfma_f32_16x16x32_bf16 v[76:79], v[216:219], v[192:195], v[76:79]\n" \
    "ds_write_b64 v242, v[36:37] offset:4864\n" \
    "v_lshlrev_b32 v172, 4, v172\n" \
    "v_mfma_f32_16x16x32_bf16 v[40:43], v[216:219], v[184:187], v[40:43]\n" \
    "ds_read_b64_tr_b16 v[212:213], v240 offset:9312\n" \
    "v_cvt_pk_bf16_f32 v72, v72, v73\n" \
    "v_cvt_pk_bf16_f32 v73, v74, v75\n" \
    "v_mfma_f32_16x16x32_bf16 v[40:43], v[220:223], v[188:191], v[40:43]\n" \
    "ds_write_b64 v242, v[72:73] offset:9728\n" \
    "v_mul_u32_u24 v173, 304, v174\n" \
    "v_mfma_f32_16x16x32_bf16 v[4:7], v[220:223], v[180:183], v[4:7]\n" \
    "ds_read_b64_tr_b16 v[214:215], v240 offset:13920\n" \
    "v_cvt_pk_bf16_f32 v108, v108, v109\n" \
    "v_cvt_pk_bf16_f32 v109, v110, v111\n" \
    "v_mfma_f32_16x16x32_bf16 v[4:7], v[216:219], v[176:179], v[4:7]\n" \
    "ds_write_b64 v242, v[108:109] offset:14592\n" \
    "v_add3_u32 v173, v173, v172, %[st]\n" \
    "s_waitcnt lgkmcnt(8)\n" \
    "v_mfma_f32_16x16x32_bf16 v[8:11], v[224:227], v[176:179], v[8:11]\n" \
    "ds_read_b64_tr_b16 v[216:217], v240 offset:256\n" \
    "v_cvt_pk_bf16_f32 v112, v112, v113\n" \
    "v_cvt_pk_bf16_f32 v113, v114, v115\n" \
    "v_mfma_f32_16x16x32_bf16 v[8:11], v[228:231], v[180:183], v[8:11]\n" \
    "ds_write_b64 v242, v[112:113] offset:14624\n" \
    "v_lshl_add_u32 v172, v174, 12, v172\n" \
    "v_mfma_f32_16x16x32_bf16 v[44:47], v[228:231], v[188:191], v[44:47]\n" \
    "ds_read_b64_tr_b16 v[218:219], v240 offset:4864\n" \
    "v_cvt_pk_bf16_f32 v76, v76, v77\n" \
    "v_cvt_pk_bf16_f32 v77, v78, v79\n" \
    "v_mfma_f32_16x16x32_bf16 v[44:47], v[224:227], v[184:187], v[44:47]\n" \
    "ds_write_b64 v242, v[76:77] offset:9760\n" \
    "v_add_u32 v172, %[oq], v172\n" \
    "v_mfma_f32_16x16x32_bf16 v[80:83], v[224:227], v[192:195], v[80:83]\n" \
    "ds_read_b64_tr_b16 v[220:221], v240 offset:9472\n" \
    "v_cvt_pk_bf16_f32 v40, v40, v41\n" \
    "v_cvt_pk_bf16_f32 v41, v42, v43\n" \
    "v_mfma_f32_16x16x32_bf16 v[80:83], v[228:231], v[196:199], v[80:83]\n" \
    "ds_write_b64 v242, v[40:41] offset:4896\n" \
    "v_mfma_f32_16x16x32_bf16 v[116:119], v[228:231], v[204:207], v[116:119]\n" \
    "ds_read_b64_tr_b16 v[222:223], v240 offset:14080\n" \
    "v_cvt_pk_bf16_f32 v4, v4, v5\n" \
    "v_cvt_pk_bf16_f32 v5, v6, v7\n" \
    "v_mfma_f32_16x16x32_bf16 v[116:119], v[224:227], v[200:203], v[116:119]\n" \
    "ds_write_b64 v242, v[4:5] offset:32\n" \
    "s_waitcnt lgkmcnt(9)\n" \
    "v_mfma_f32_16x16x32_bf16 v[120:123], v[208:211], v[200:203], v[120:123]\n" \
    "ds_read_b64_tr_b16 v[224:225], v240 offset:128\n" \
    "v_cvt_pk_bf16_f32 v8, v8, v9\n" \
    "v_cvt_pk_bf16_f32 v9, v10, v11\n" \
    "v_mfma_f32_16x16x32_bf16 v[120:123], v[212:215], v[204:207], v[120:123]\n" \
    "ds_write_b64 v242, v[8:9] offset:64\n" \
    "v_mfma_f32_16x16x32_bf16 v[84:87], v[212:215], v[196:199], v[84:87]\n" \
    "ds_read_b64_tr_b16 v[226:227], v240 offset:4736\n" \
    "v_cvt_pk_bf16_f32 v44, v44, v45\n" \
    "v_cvt_pk_bf16_f32 v45, v46, v47\n" \
    "v_mfma_f32_16x16x32_bf16 v[84:87], v[208:211], v[192:195], v[84:87]\n" \
    "ds_write_b64 v242, v[44:45] offset:4928\n" \
    "v_mfma_f32_16x16x32_bf16 v[48:51], v[208:211], v[184:187], v[48:51]\n" \
    "ds_read_b64_tr_b16 v[228:229], v240 offset:9344\n" \
    "v_cvt_pk_bf16_f32 v80, v80, v81\n" \
    "v_cvt_pk_bf16_f32 v81, v82, v83\n" \
    "v_mfma_f32_16x16x32_bf16 v[48:51], v[212:215], v[188:191], v[48:51]\n" \
    "ds_write_b64 v242, v[80:81] offset:9792\n" \
    "v_mfma_f32_16x16x32_bf16 v[12:15], v[212:215], v[180:183], v[12:15]\n" \
    "ds_read_b64_tr_b16 v[230:231], v240 offset:13952\n" \
    "v_cvt_pk_bf16_f32 v116, v116, v117\n" \
    "v_cvt_pk_bf16_f32 v117, v118, v119\n" \
    "v_mfma_f32_16x16x32_bf16 v[12:15], v[208:211], v[176:179], v[12:15]\n" \
    "ds_write_b64 v242, v[116:117] offset:14656\n" \
    "s_waitcnt lgkmcnt(9)\n" \
    "v_mfma_f32_16x16x32_bf16 v[32:35], v[216:219], v[176:179], v[32:35]\n" \
    "ds_read_b64_tr_b16 v[208:209], v240 offset:160\n" \
    "v_cvt_pk_bf16_f32 v120, v120, v121\n" \
    "v_cvt_pk_bf16_f32 v121, v122, v123\n" \
    "v_mfma_f32_16x16x32_bf16 v[32:35], v[220:223], v[180:183], v[32:35]\n" \
    "ds_write_b64 v242, v[120:121] offset:14688\n" \
    "ds_read_b128 v[144:147], v243 offset:14592\n" \
    "ds_read_b128 v[148:151], v243 offset:17024\n" \
    "v_mfma_f32_16x16x32_bf16 v[68:71], v[220:223], v[188:191], v[68:71]\n" \
    "ds_read_b64_tr_b16 v[210:211], v240 offset:4768\n" \
    "v_cvt_pk_bf16_f32 v84, v84, v85\n" \
    "v_cvt_pk_bf16_f32 v85, v86, v87\n" \
    "v_mfma_f32_16x16x32_bf16 v[68:71], v[216:219], v[184:187], v[68:71]\n" \
    "ds_write_b64 v242, v[84:85] offset:9824\n" \
    "ds_read_b128 v[152:155], v243 offset:9728\n" \
    "ds_read_b128 v[156:159], v243 offset:12160\n" \
    "v_mfma_f32_16x16x32_bf16 v[104:107], v[216:219], v[192:195], v[104:107]\n" \
    "ds_read_b64_tr_b16 v[212:213], v240 offset:9376\n" \
    "v_cvt_pk_bf16_f32 v48, v48, v49\n" \
    "v_cvt_pk_bf16_f32 v49, v50, v51\n" \
    "v_mfma_f32_16x16x32_bf16 v[104:107], v[220:223], v[196:199], v[104:107]\n" \
    "ds_write_b64 v242, v[48:49] offset:4960\n" \
    "ds_read_b128 v[160:163], v243 offset:4864\n" \
    "ds_read_b128 v[164:167], v243 offset:7296\n" \
    "v_mfma_f32_16x16x32_bf16 v[140:143], v[220:223], v[204:207], v[140:143]\n" \
    "ds_read_b64_tr_b16 v[214:215], v240 offset:13984\n" \
    "v_cvt_pk_bf16_f32 v12, v12, v13\n" \
    "v_cvt_pk_bf16_f32 v13, v14, v15\n" \
    "v_mfma_f32_16x16x32_bf16 v[140:143], v[216:219], v[200:203], v[140:143]\n" \
    "ds_write_b64 v242, v[12:13] offset:96\n" \
    "ds_read_b128 v[0:3], v243 offset:0\n" \
    "ds_read_b128 v[36:39], v243 offset:2432\n" \
    "s_waitcnt lgkmcnt(15)\n" \
    "v_mfma_f32_16x16x32_bf16 v[124:127], v[224:227], v[200:203], v[124:127]\n" \
    "ds_read_b64_tr_b16 v[216:217], v240 offset:192\n" \
    "v_cvt_pk_bf16_f32 v32, v32, v33\n" \
    "v_cvt_pk_bf16_f32 v33, v34, v35\n" \
    "s_mov_b32 %[ps], 196608\n" \
    "s_waitcnt lgkmcnt(14)\n" \
    "buffer_store_dwordx4 v[144:147], v246, %[ro], %[ps] offen nt\n" \
    "v_mfma_f32_16x16x32_bf16 v[124:127], v[228:231], v[204:207], v[124:127]\n" \
    "ds_write_b64 v242, v[32:33] offset:256\n" \
    "v_mfma_f32_16x16x32_bf16 v[88:91], v[228:231], v[196:199], v[88:91]\n" \
    "ds_read_b64_tr_b16 v[218:219], v240 offset:4800\n" \
    "v_cvt_pk_bf16_f32 v68, v68, v69\n" \
    "v_cvt_pk_bf16_f32 v69, v70, v71\n" \
    "s_mov_b32 %[ps], 229376\n" \
    "s_waitcnt lgkmcnt(15)\n" \
    "buffer_store_dwordx4 v[148:151], v246, %[ro], %[ps] offen nt\n" \
    "v_mfma_f32_16x16x32_bf16 v[88:91], v[224:227], v[192:195], v[88:91]\n" \
    "ds_write_b64 v242, v[68:69] offset:5120\n" \
    "ds_read_b128 v[72:75], v173 offset:256\n" \
    "v_mfma_f32_16x16x32_bf16 v[52:55], v[224:227], v[184:187], v[52:55]\n" \
    "ds_read_b64_tr_b16 v[220:221], v240 offset:9408\n" \
    "v_cvt_pk_bf16_f32 v104, v104, v105\n" \
    "v_cvt_pk_bf16_f32 v105, v106, v107\n" \
    "s_mov_b32 %[ps], 131072\n" \
    "s_waitcnt lgkmcnt(15)\n" \
    "buffer_store_dwordx4 v[152:155], v246, %[ro], %[ps] offen nt\n" \
    "v_mfma_f32_16x16x32_bf16 v[52:55], v[228:231], v[188:191], v[52:55]\n" \
    "ds_write_b64 v242, v[104:105] offset:9984\n" \
    "v_mfma_f32_16x16x32_bf16 v[16:19], v[228:231], v[180:183], v[16:19]\n" \
    "ds_read_b64_tr_b16 v[222:223], v240 offset:14016\n" \
    "v_cvt_pk_bf16_f32 v140, v140, v141\n" \
    "v_cvt_pk_bf16_f32 v141, v142, v143\n" \
    "s_mov_b32 %[ps], 163840\n" \
    "s_waitcnt lgkmcnt(15)\n" \
    "buffer_store_dwordx4 v[156:159], v246, %[ro], %[ps] offen nt\n" \
    "v_mfma_f32_16x16x32_bf16 v[16:19], v[224:227], v[176:179], v[16:19]\n" \
    "ds_write_b64 v242, v[140:141] offset:14848\n" \
    "ds_read_b128 v[108:111], v173 offset:9984\n" \
    "s_waitcnt lgkmcnt(13)\n" \
    "v_mfma_f32_16x16x32_bf16 v[20:23], v[208:211], v[176:179], v[20:23]\n" \
    "ds_read_b64_tr_b16 v[224:225], v240 offset:224\n" \
    "v_cvt_pk_bf16_f32 v124, v124, v125\n" \
    "v_cvt_pk_bf16_f32 v125, v126, v127\n" \
    "s_mov_b32 %[ps], 65536\n" \
    "buffer_store_dwordx4 v[160:163], v246, %[ro], %[ps] offen nt\n" \
    "v_mfma_f32_16x16x32_bf16 v[20:23], v[212:215], v[180:183], v[20:23]\n" \
    "ds_write_b64 v242, v[124:125] offset:14720\n" \
    "v_mfma_f32_16x16x32_bf16 v[56:59], v[212:215], v[188:191], v[56:59]\n" \
    "ds_read_b64_tr_b16 v[226:227], v240 offset:4832\n" \
    "v_cvt_pk_bf16_f32 v88, v88, v89\n" \
    "v_cvt_pk_bf16_f32 v89, v90, v91\n" \
    "s_mov_b32 %[ps], 98304\n" \
    "buffer_store_dwordx4 v[164:167], v246, %[ro], %[ps] offen nt\n" \
    "v_mfma_f32_16x16x32_bf16 v[56:59], v[208:211], v[184:187], v[56:59]\n" \
    "ds_write_b64 v242, v[88:89] offset:9856\n" \
    "v_mfma_f32_16x16x32_bf16 v[92:95], v[208:211], v[192:195], v[92:95]\n" \
    "ds_read_b64_tr_b16 v[228:229], v240 offset:9440\n" \
    "v_cvt_pk_bf16_f32 v52, v52, v53\n" \
    "v_cvt_pk_bf16_f32 v53, v54, v55\n" \
    "s_mov_b32 %[ps], 0\n" \
    "s_waitcnt lgkmcnt(15)\n" \
    "buffer_store_dwordx4 v[0:3], v246, %[ro], %[ps] offen nt\n" \
    "v_mfma_f32_16x16x32_bf16 v[92:95], v[212:215], v[196:199], v[92:95]\n" \
    "ds_write_b64 v242, v[52:53] offset:4992\n" \
    "v_mfma_f32_16x16x32_bf16 v[128:131], v[212:215], v[204:207], v[128:131]\n" \
    "ds_read_b64_tr_b16 v[230:231], v240 offset:14048\n" \
    "v_cvt_pk_bf16_f32 v16, v16, v17\n" \
    "v_cvt_pk_bf16_f32 v17, v18, v19\n" \
    "s_mov_b32 %[ps], 32768\n" \
    "s_waitcnt lgkmcnt(15)\n" \
    "buffer_store_dwordx4 v[36:39], v246, %[ro], %[ps] offen nt\n" \
    "v_mfma_f32_16x16x32_bf16 v[128:131], v[208:211], v[200:203], v[128:131]\n" \
    "ds_write_b64 v242, v[16:17] offset:128\n" \
    "s_waitcnt lgkmcnt(10)\n" \
    "v_mfma_f32_16x16x32_bf16 v[132:135], v[216:219], v[200:203], v[132:135]\n" \
    "v_cvt_pk_bf16_f32 v20, v20, v21\n" \
    "v_cvt_pk_bf16_f32 v21, v22, v23\n" \
    "s_mov_b32 %[ps], 0\n" \
    "buffer_store_dwordx4 v[72:75], v172, %[ro], %[ps] offen nt\n" \
    "v_mfma_f32_16x16x32_bf16 v[132:135], v[220:223], v[204:207], v[132:135]\n" \
    "ds_write_b64 v242, v[20:21] offset:160\n" \
    "v_mfma_f32_16x16x32_bf16 v[96:99], v[220:223], v[196:199], v[96:99]\n" \
    "v_cvt_pk_bf16_f32 v56, v56, v57\n" \
    "v_cvt_pk_bf16_f32 v57, v58, v59\n" \
    "s_mov_b32 %[ps], 131072\n" \
    "s_waitcnt lgkmcnt(9)\n" \
    "buffer_store_dwordx4 v[108:111], v172, %[ro], %[ps] offen nt\n" \
    "v_mfma_f32_16x16x32_bf16 v[96:99], v[216:219], v[192:195], v[96:99]\n" \
    "ds_write_b64 v242, v[56:57] offset:5024\n" \
    "v_mfma_f32_16x16x32_bf16 v[60:63], v[216:219], v[184:187], v[60:63]\n" \
    "v_cvt_pk_bf16_f32 v92, v92, v93\n" \
    "v_cvt_pk_bf16_f32 v93, v94, v95\n" \
    "v_mfma_f32_16x16x32_bf16 v[60:63], v[220:223], v[188:191], v[60:63]\n" \
    "ds_write_b64 v242, v[92:93] offset:9888\n" \
    "v_mfma_f32_16x16x32_bf16 v[24:27], v[220:223], v[180:183], v[24:27]\n" \
    "v_cvt_pk_bf16_f32 v128, v128, v129\n" \
    "v_cvt_pk_bf16_f32 v129, v130, v131\n" \
    "v_mfma_f32_16x16x32_bf16 v[24:27], v[216:219], v[176:179], v[24:27]\n" \
    "ds_write_b64 v242, v[128:129] offset:14752\n" \
    "s_waitcnt lgkmcnt(5)\n" \
    "v_mfma_f32_16x16x32_bf16 v[28:31], v[224:227], v[176:179], v[28:31]\n" \
    "v_cvt_pk_bf16_f32 v132, v132, v133\n" \
    "v_cvt_pk_bf16_f32 v133, v134, v135\n" \
    "v_mfma_f32_16x16x32_bf16 v[28:31], v[228:231], v[180:183], v[28:31]\n" \
    "ds_write_b64 v242, v[132:133] offset:14784\n" \
    "v_mfma_f32_16x16x32_bf16 v[64:67], v[228:231], v[188:191], v[64:67]\n" \
    "v_cvt_pk_bf16_f32 v96, v96, v97\n" \
    "v_cvt_pk_bf16_f32 v97, v98, v99\n" \
    "v_mfma_f32_16x16x32_bf16 v[64:67], v[224:227], v[184:187], v[64:67]\n" \
    "ds_write_b64 v242, v[96:97] offset:9920\n" \
    "v_mfma_f32_16x16x32_bf16 v[100:103], v[224:227], v[192:195], v[100:103]\n" \
    "v_cvt_pk_bf16_f32 v60, v60, v61\n" \
    "v_cvt_pk_bf16_f32 v61, v62, v63\n" \
    "v_mfma_f32_16x16x32_bf16 v[100:103], v[228:231], v[196:199], v[100:103]\n" \
    "ds_write_b64 v242, v[60:61] offset:5056\n" \
    "v_mfma_f32_16x16x32_bf16 v[136:139], v[228:231], v[204:207], v[136:139]\n" \
    "v_cvt_pk_bf16_f32 v24, v24, v25\n" \
    "v_cvt_pk_bf16_f32 v25, v26, v27\n" \
    "v_mfma_f32_16x16x32_bf16 v[136:139], v[224:227], v[200:203], v[136:139]\n" \
    "ds_write_b64 v242, v[24:25] offset:192\n" \
    "v_cvt_pk_bf16_f32 v28, v28, v29\n" \
    "v_cvt_pk_bf16_f32 v29, v30, v31\n" \
    "ds_write_b64 v242, v[28:29] offset:224\n" \
    "v_cvt_pk_bf16_f32 v64, v64, v65\n" \
    "v_cvt_pk_bf16_f32 v65, v66, v67\n" \
    "ds_write_b64 v242, v[64:65] offset:5088\n" \
    "v_cvt_pk_bf16_f32 v100, v100, v101\n" \
    "v_cvt_pk_bf16_f32 v101, v102, v103\n" \
    "ds_write_b64 v242, v[100:101] offset:9952\n" \
    "s_nop 5\n" \
    "v_cvt_pk_bf16_f32 v136, v136, v137\n" \
    "v_cvt_pk_bf16_f32 v137, v138, v139\n" \
    "ds_write_b64 v242, v[136:137] offset:14816\n" \
    "s_branch LEND%=\n" \
    "W9_%=:\n" \
    "s_sleep 1\n" \
    "ds_read_b128 v[144:147], v171 offset:32\n" \
    "s_waitcnt lgkmcnt(0)\n" \
    "v_min3_u32 v144, v144, v145, v146\n" \
    "v_min_u32 v244, v144, v147\n" \
    "s_nop 0\n" \
    "v_readfirstlane_b32 %[hv], v244\n" \
    "s_cmp_ge_u32 %[hv], 61\n" \
    "s_cbranch_scc1 V9_%=\n" \
    "s_branch W9_%=\n" \
    "E8%=:\n" \
    "ds_read_b64_tr_b16 v[208:209], v240 offset:0\n" \
    "ds_read_b64_tr_b16 v[210:211], v240 offset:4608\n" \
    "ds_read_b64_tr_b16 v[212:213], v240 offset:9216\n" \
    "ds_read_b64_tr_b16 v[214:215], v240 offset:13824\n" \
    "ds_read_b64_tr_b16 v[216:217], v240 offset:32\n" \
    "ds_read_b64_tr_b16 v[218:219], v240 offset:4640\n" \
    "ds_read_b64_tr_b16 v[220:221], v240 offset:9248\n" \
    "ds_read_b64_tr_b16 v[222:223], v240 offset:13856\n" \
    "ds_read_b64_tr_b16 v[224:225], v240 offset:64\n" \
    "ds_read_b64_tr_b16 v[226:227], v240 offset:4672\n" \
    "ds_read_b64_tr_b16 v[228:229], v240 offset:9280\n" \
    "ds_read_b64_tr_b16 v[230:231], v240 offset:13888\n" \
    "s_waitcnt lgkmcnt(8) vmcnt(0)\n" \
    "L8_%=:\n" \
    "v_mfma_f32_16x16x32_bf16 v[0:3], v[208:211], v[144:147], v[0:3]\n" \
    "ds_read_b64_tr_b16 v[232:233], v240 offset:96\n" \
    "v_mfma_f32_16x16x32_bf16 v[0:3], v[212:215], v[148:151], v[0:3]\n" \
    "ds_read_b64_tr_b16 v[234:235], v240 offset:4704\n" \
    "s_waitcnt vmcnt(4)\n" \
    "v_mfma_f32_16x16x32_bf16 v[36:39], v[212:215], v[156:159], v[36:39]\n" \
    "ds_read_b64_tr_b16 v[236:237], v240 offset:9312\n" \
    "v_mfma_f32_16x16x32_bf16 v[36:39], v[208:211], v[152:155], v[36:39]\n" \
    "ds_read_b64_tr_b16 v[238:239], v240 offset:13920\n" \
    "s_waitcnt vmcnt(2)\n" \
    "v_mfma_f32_16x16x32_bf16 v[72:75], v[208:211], v[160:163], v[72:75]\n" \
    "buffer_load_dwordx4 v[176:179], v242, %[rs], 0 offen offset:0 sc0 sc1\n" \
    "v_mfma_f32_16x16x32_bf16 v[72:75], v[212:215], v[164:167], v[72:75]\n" \
    "buffer_load_dwordx4 v[180:183], v242, %[rs], 0 offen offset:1024 sc0 sc1\n" \
    "s_waitcnt vmcnt(2)\n" \
    "v_mfma_f32_16x16x32_bf16 v[108:111], v[212:215], v[172:175], v[108:111]\n" \
    "buffer_load_dwordx4 v[184:187], v242, %[rs], 0 offen offset:2048 sc0 sc1\n" \
    "v_mfma_f32_16x16x32_bf16 v[108:111], v[208:211], v[168:171], v[108:111]\n" \
    "s_waitcnt lgkmcnt(8)\n" \
    "v_mfma_f32_16x16x32_bf16 v[112:115], v[216:219], v[168:171], v[112:115]\n" \
    "ds_read_b64_tr_b16 v[208:209], v240 offset:128\n" \
    "s_add_u32 %[nd], %[nd], 1\n" \
    "v_mfma_f32_16x16x32_bf16 v[112:115], v[220:223], v[172:175], v[112:115]\n" \
    "ds_read_b64_tr_b16 v[210:211], v240 offset:4736\n" \
    "s_add_u32 %[ps], %[ps], 18432\n" \
    "v_mfma_f32_16x16x32_bf16 v[76:79], v[220:223], v[164:167], v[76:79]\n" \
    "ds_read_b64_tr_b16 v[212:213], v240 offset:9344\n" \
    "s_cmp_eq_u32 %[ps], 147456\n" \
    "v_mfma_f32_16x16x32_bf16 v[76:79], v[216:219], v[160:163], v[76:79]\n" \
    "ds_read_b64_tr_b16 v[214:215], v240 offset:13952\n" \
    "s_cselect_b32 %[ps], 0, %[ps]\n" \
    "v_mfma_f32_16x16x32_bf16 v[40:43], v[216:219], v[152:155], v[40:43]\n" \
    "buffer_load_dwordx4 v[188:191], v242, %[rs], 0 offen offset:3072 sc0 sc1\n" \
    "v_mfma_f32_16x16x32_bf16 v[40:43], v[220:223], v[156:159], v[40:43]\n" \
    "buffer_load_dwordx4 v[192:195], v243, %[rs], 0 offen offset:0 sc0 sc1\n" \
    "v_mfma_f32_16x16x32_bf16 v[4:7], v[220:223], v[148:151], v[4:7]\n" \
    "buffer_load_dwordx4 v[196:199], v243, %[rs], 0 offen offset:1024 sc0 sc1\n" \
    "v_mfma_f32_16x16x32_bf16 v[4:7], v[216:219], v[144:147], v[4:7]\n" \
    "s_waitcnt lgkmcnt(8)\n" \
    "v_mfma_f32_16x16x32_bf16 v[8:11], v[224:227], v[144:147], v[8:11]\n" \
    "ds_read_b64_tr_b16 v[216:217], v240 offset:160\n" \
    "s_cmp_eq_u32 %[nd], 65\n" \
    "v_mfma_f32_16x16x32_bf16 v[8:11], v[228:231], v[148:151], v[8:11]\n" \
    "ds_read_b64_tr_b16 v[218:219], v240 offset:4768\n" \
    "s_cselect_b32 %[ck], -1, %[nd]\n" \
    "v_mfma_f32_16x16x32_bf16 v[44:47], v[228:231], v[156:159], v[44:47]\n" \
    "ds_read_b64_tr_b16 v[220:221], v240 offset:9376\n" \
    "v_mfma_f32_16x16x32_bf16 v[44:47], v[224:227], v[152:155], v[44:47]\n" \
    "ds_read_b64_tr_b16 v[222:223], v240 offset:13984\n" \
    "v_mfma_f32_16x16x32_bf16 v[80:83], v[224:227], v[160:163], v[80:83]\n" \
    "buffer_load_dwordx4 v[200:203], v243, %[rs], 0 offen offset:2048 sc0 sc1\n" \
    "v_mfma_f32_16x16x32_bf16 v[80:83], v[228:231], v[164:167], v[80:83]\n" \
    "buffer_load_dwordx4 v[204:207], v243, %[rs], 0 offen offset:3072 sc0 sc1\n" \
    "v_mfma_f32_16x16x32_bf16 v[116:119], v[228:231], v[172:175], v[116:119]\n" \
    "v_add_u32 v242, 0x10000, v242\n" \
    "v_mfma_f32_16x16x32_bf16 v[116:119], v[224:227], v[168:171], v[116:119]\n" \
    "s_waitcnt lgkmcnt(8)\n" \
    "v_mfma_f32_16x16x32_bf16 v[120:123], v[232:235], v[168:171], v[120:123]\n" \
    "ds_read_b64_tr_b16 v[224:225], v240 offset:192\n" \
    "v_mfma_f32_16x16x32_bf16 v[120:123], v[236:239], v[172:175], v[120:123]\n" \
    "ds_read_b64_tr_b16 v[226:227], v240 offset:4800\n" \
    "v_mfma_f32_16x16x32_bf16 v[84:87], v[236:239], v[164:167], v[84:87]\n" \
    "ds_read_b64_tr_b16 v[228:229], v240 offset:9408\n" \
    "v_mfma_f32_16x16x32_bf16 v[84:87], v[232:235], v[160:163], v[84:87]\n" \
    "ds_read_b64_tr_b16 v[230:231], v240 offset:14016\n" \
    "v_mfma_f32_16x16x32_bf16 v[48:51], v[232:235], v[152:155], v[48:51]\n" \
    "v_add_u32 v243, 0x10000, v243\n" \
    "v_mfma_f32_16x16x32_bf16 v[48:51], v[236:239], v[156:159], v[48:51]\n" \
    "v_add_u32 v246, 1, v246\n" \
    "v_mfma_f32_16x16x32_bf16 v[12:15], v[236:239], v[148:151], v[12:15]\n" \
    "v_mfma_f32_16x16x32_bf16 v[12:15], v[232:235], v[144:147], v[12:15]\n" \
    "s_waitcnt lgkmcnt(8)\n" \
    "v_mfma_f32_16x16x32_bf16 v[16:19], v[208:211], v[144:147], v[16:19]\n" \
    "ds_read_b64_tr_b16 v[232:233], v240 offset:224\n" \
    "v_mfma_f32_16x16x32_bf16 v[16:19], v[212:215], v[148:151], v[16:19]\n" \
    "ds_read_b64_tr_b16 v[234:235], v240 offset:4832\n" \
    "v_mfma_f32_16x16x32_bf16 v[52:55], v[212:215], v[156:159], v[52:55]\n" \
    "ds_read_b64_tr_b16 v[236:237], v240 offset:9440\n" \
    "v_mfma_f32_16x16x32_bf16 v[52:55], v[208:211], v[152:155], v[52:55]\n" \
    "ds_read_b64_tr_b16 v[238:239], v240 offset:14048\n" \
    "v_mfma_f32_16x16x32_bf16 v[88:91], v[208:211], v[160:163], v[88:91]\n" \
    "v_add_u32 v240, %[ps], v241\n" \
    "v_mfma_f32_16x16x32_bf16 v[88:91], v[212:215], v[164:167], v[88:91]\n" \
    "ds_write_b32 v247, v246 offset:32\n" \
    "v_mfma_f32_16x16x32_bf16 v[124:127], v[212:215], v[172:175], v[124:127]\n" \
    "v_mfma_f32_16x16x32_bf16 v[124:127], v[208:211], v[168:171], v[124:127]\n" \
    "s_waitcnt lgkmcnt(9)\n" \
    "v_mfma_f32_16x16x32_bf16 v[128:131], v[216:219], v[168:171], v[128:131]\n" \
    "ds_read_b64_tr_b16 v[208:209], v240 offset:0\n" \
    "v_mfma_f32_16x16x32_bf16 v[128:131], v[220:223], v[172:175], v[128:131]\n" \
    "ds_read_b64_tr_b16 v[210:211], v240 offset:4608\n" \
    "v_mfma_f32_16x16x32_bf16 v[92:95], v[220:223], v[164:167], v[92:95]\n" \
    "ds_read_b64_tr_b16 v[212:213], v240 offset:9216\n" \
    "v_mfma_f32_16x16x32_bf16 v[92:95], v[216:219], v[160:163], v[92:95]\n" \
    "ds_read_b64_tr_b16 v[214:215], v240 offset:13824\n" \
    "v_mfma_f32_16x16x32_bf16 v[56:59], v[216:219], v[152:155], v[56:59]\n" \
    "v_mfma_f32_16x16x32_bf16 v[56:59], v[220:223], v[156:159], v[56:59]\n" \
    "v_mfma_f32_16x16x32_bf16 v[20:23], v[220:223], v[148:151], v[20:23]\n" \
    "v_mfma_f32_16x16x32_bf16 v[20:23], v[216:219], v[144:147], v[20:23]\n" \
    "s_waitcnt lgkmcnt(9)\n" \
    "v_mfma_f32_16x16x32_bf16 v[24:27], v[224:227], v[144:147], v[24:27]\n" \
    "ds_read2_b32 v[244:245], v247 offset1:4\n" \
    "ds_read_b64_tr_b16 v[216:217], v240 offset:32\n" \
    "v_mfma_f32_16x16x32_bf16 v[24:27], v[228:231], v[148:151], v[24:27]\n" \
    "ds_read_b64_tr_b16 v[218:219], v240 offset:4640\n" \
    "v_mfma_f32_16x16x32_bf16 v[60:63], v[228:231], v[156:159], v[60:63]\n" \
    "ds_read_b64_tr_b16 v[220:221], v240 offset:9248\n" \
    "v_mfma_f32_16x16x32_bf16 v[60:63], v[224:227], v[152:155], v[60:63]\n" \
    "ds_read_b64_tr_b16 v[222:223], v240 offset:13856\n" \
    "v_mfma_f32_16x16x32_bf16 v[96:99], v[224:227], v[160:163], v[96:99]\n" \
    "v_mfma_f32_16x16x32_bf16 v[96:99], v[228:231], v[164:167], v[96:99]\n" \
    "v_mfma_f32_16x16x32_bf16 v[132:135], v[228:231], v[172:175], v[132:135]\n" \
    "v_mfma_f32_16x16x32_bf16 v[132:135], v[224:227], v[168:171], v[132:135]\n" \
    "s_waitcnt lgkmcnt(4)\n" \
    "v_mfma_f32_16x16x32_bf16 v[136:139], v[232:235], v[168:171], v[136:139]\n" \
    "ds_read_b64_tr_b16 v[224:225], v240 offset:64\n" \
    "v_mfma_f32_16x16x32_bf16 v[136:139], v[236:239], v[172:175], v[136:139]\n" \
    "ds_read_b64_tr_b16 v[226:227], v240 offset:4672\n" \
    "v_mfma_f32_16x16x32_bf16 v[100:103], v[236:239], v[164:167], v[100:103]\n" \
    "ds_read_b64_tr_b16 v[228:229], v240 offset:9280\n" \
    "v_mfma_f32_16x16x32_bf16 v[100:103], v[232:235], v[160:163], v[100:103]\n" \
    "ds_read_b64_tr_b16 v[230:231], v240 offset:13888\n" \
    "v_mfma_f32_16x16x32_bf16 v[64:67], v[232:235], v[152:155], v[64:67]\n" \
    "v_min_u32 v244, v244, v245\n" \
    "v_mfma_f32_16x16x32_bf16 v[64:67], v[236:239], v[156:159], v[64:67]\n" \
    "v_readfirstlane_b32 %[hv], v244\n" \
    "v_mfma_f32_16x16x32_bf16 v[28:31], v[236:239], v[148:151], v[28:31]\n" \
    "s_cmp_ge_u32 %[hv], %[ck]\n" \
    "v_mfma_f32_16x16x32_bf16 v[28:31], v[232:235], v[144:147], v[28:31]\n" \
    "s_waitcnt lgkmcnt(8) vmcnt(6)\n" \
    "s_cbranch_scc0 S8_%=\n" \
    "R8_%=:\n" \
    "v_mfma_f32_16x16x32_bf16 v[0:3], v[208:211], v[176:179], v[0:3]\n" \
    "ds_read_b64_tr_b16 v[232:233], v240 offset:96\n" \
    "v_mfma_f32_16x16x32_bf16 v[0:3], v[212:215], v[180:183], v[0:3]\n" \
    "ds_read_b64_tr_b16 v[234:235], v240 offset:4704\n" \
    "s_waitcnt vmcnt(4)\n" \
    "v_mfma_f32_16x16x32_bf16 v[36:39], v[212:215], v[188:191], v[36:39]\n" \
    "ds_read_b64_tr_b16 v[236:237], v240 offset:9312\n" \
    "v_mfma_f32_16x16x32_bf16 v[36:39], v[208:211], v[184:187], v[36:39]\n" \
    "ds_read_b64_tr_b16 v[238:239], v240 offset:13920\n" \
    "s_waitcnt vmcnt(2)\n" \
    "v_mfma_f32_16x16x32_bf16 v[72:75], v[208:211], v[192:195], v[72:75]\n" \
    "buffer_load_dwordx4 v[144:147], v242, %[rs], 0 offen offset:0 sc0 sc1\n" \
    "v_mfma_f32_16x16x32_bf16 v[72:75], v[212:215], v[196:199], v[72:75]\n" \
    "buffer_load_dwordx4 v[148:151], v242, %[rs], 0 offen offset:1024 sc0 sc1\n" \
    "s_waitcnt vmcnt(2)\n" \
    "v_mfma_f32_16x16x32_bf16 v[108:111], v[212:215], v[204:207], v[108:111]\n" \
    "buffer_load_dwordx4 v[152:155], v242, %[rs], 0 offen offset:2048 sc0 sc1\n" \
    "v_mfma_f32_16x16x32_bf16 v[108:111], v[208:211], v[200:203], v[108:111]\n" \
    "s_waitcnt lgkmcnt(8)\n" \
    "v_mfma_f32_16x16x32_bf16 v[112:115], v[216:219], v[200:203], v[112:115]\n" \
    "ds_read_b64_tr_b16 v[208:209], v240 offset:128\n" \
    "s_add_u32 %[nd], %[nd], 1\n" \
    "v_mfma_f32_16x16x32_bf16 v[112:115], v[220:223], v[204:207], v[112:115]\n" \
    "ds_read_b64_tr_b16 v[210:211], v240 offset:4736\n" \
    "s_add_u32 %[ps], %[ps], 18432\n" \
    "v_mfma_f32_16x16x32_bf16 v[76:79], v[220:223], v[196:199], v[76:79]\n" \
    "ds_read_b64_tr_b16 v[212:213], v240 offset:9344\n" \
    "s_cmp_eq_u32 %[ps], 147456\n" \
    "v_mfma_f32_16x16x32_bf16 v[76:79], v[216:219], v[192:195], v[76:79]\n" \
    "ds_read_b64_tr_b16 v[214:215], v240 offset:13952\n" \
    "s_cselect_b32 %[ps], 0, %[ps]\n" \
    "v_mfma_f32_16x16x32_bf16 v[40:43], v[216:219], v[184:187], v[40:43]\n" \
    "buffer_load_dwordx4 v[156:159], v242, %[rs], 0 offen offset:3072 sc0 sc1\n" \
    "v_mfma_f32_16x16x32_bf16 v[40:43], v[220:223], v[188:191], v[40:43]\n" \
    "buffer_load_dwordx4 v[160:163], v243, %[rs], 0 offen offset:0 sc0 sc1\n" \
    "v_mfma_f32_16x16x32_bf16 v[4:7], v[220:223], v[180:183], v[4:7]\n" \
    "buffer_load_dwordx4 v[164:167], v243, %[rs], 0 offen offset:1024 sc0 sc1\n" \
    "v_mfma_f32_16x16x32_bf16 v[4:7], v[216:219], v[176:179], v[4:7]\n" \
    "s_waitcnt lgkmcnt(8)\n" \
    "v_mfma_f32_16x16x32_bf16 v[8:11], v[224:227], v[176:179], v[8:11]\n" \
    "ds_read_b64_tr_b16 v[216:217], v240 offset:160\n" \
    "v_mfma_f32_16x16x32_bf16 v[8:11], v[228:231], v[180:183], v[8:11]\n" \
    "ds_read_b64_tr_b16 v[218:219], v240 offset:4768\n" \
    "v_mfma_f32_16x16x32_bf16 v[44:47], v[228:231], v[188:191], v[44:47]\n" \
    "ds_read_b64_tr_b16 v[220:221], v240 offset:9376\n" \
    "v_mfma_f32_16x16x32_bf16 v[44:47], v[224:227], v[184:187], v[44:47]\n" \
    "ds_read_b64_tr_b16 v[222:223], v240 offset:13984\n" \
    "v_mfma_f32_16x16x32_bf16 v[80:83], v[224:227], v[192:195], v[80:83]\n" \
    "buffer_load_dwordx4 v[168:171], v243, %[rs], 0 offen offset:2048 sc0 sc1\n" \
    "v_mfma_f32_16x16x32_bf16 v[80:83], v[228:231], v[196:199], v[80:83]\n" \
    "buffer_load_dwordx4 v[172:175], v243, %[rs], 0 offen offset:3072 sc0 sc1\n" \
    "v_mfma_f32_16x16x32_bf16 v[116:119], v[228:231], v[204:207], v[116:119]\n" \
    "v_add_u32 v242, 0x10000, v242\n" \
    "v_mfma_f32_16x16x32_bf16 v[116:119], v[224:227], v[200:203], v[116:119]\n" \
    "s_waitcnt lgkmcnt(8)\n" \
    "v_mfma_f32_16x16x32_bf16 v[120:123], v[232:235], v[200:203], v[120:123]\n" \
    "ds_read_b64_tr_b16 v[224:225], v240 offset:192\n" \
    "v_mfma_f32_16x16x32_bf16 v[120:123], v[236:239], v[204:207], v[120:123]\n" \
    "ds_read_b64_tr_b16 v[226:227], v240 offset:4800\n" \
    "v_mfma_f32_16x16x32_bf16 v[84:87], v[236:239], v[196:199], v[84:87]\n" \
    "ds_read_b64_tr_b16 v[228:229], v240 offset:9408\n" \
    "v_mfma_f32_16x16x32_bf16 v[84:87], v[232:235], v[192:195], v[84:87]\n" \
    "ds_read_b64_tr_b16 v[230:231], v240 offset:14016\n" \
    "v_mfma_f32_16x16x32_bf16 v[48:51], v[232:235], v[184:187], v[48:51]\n" \
    "v_add_u32 v243, 0x10000, v243\n" \
    "v_mfma_f32_16x16x32_bf16 v[48:51], v[236:239], v[188:191], v[48:51]\n" \
    "v_add_u32 v246, 1, v246\n" \
    "v_mfma_f32_16x16x32_bf16 v[12:15], v[236:239], v[180:183], v[12:15]\n" \
    "v_mfma_f32_16x16x32_bf16 v[12:15], v[232:235], v[176:179], v[12:15]\n" \
    "s_waitcnt lgkmcnt(8)\n" \
    "v_mfma_f32_16x16x32_bf16 v[16:19], v[208:211], v[176:179], v[16:19]\n" \
    "ds_read_b64_tr_b16 v[232:233], v240 offset:224\n" \
    "v_mfma_f32_16x16x32_bf16 v[16:19], v[212:215], v[180:183], v[16:19]\n" \
    "ds_read_b64_tr_b16 v[234:235], v240 offset:4832\n" \
    "v_mfma_f32_16x16x32_bf16 v[52:55], v[212:215], v[188:191], v[52:55]\n" \
    "ds_read_b64_tr_b16 v[236:237], v240 offset:9440\n" \
    "v_mfma_f32_16x16x32_bf16 v[52:55], v[208:211], v[184:187], v[52:55]\n" \
    "ds_read_b64_tr_b16 v[238:239], v240 offset:14048\n" \
    "v_mfma_f32_16x16x32_bf16 v[88:91], v[208:211], v[192:195], v[88:91]\n" \
    "v_add_u32 v240, %[ps], v241\n" \
    "v_mfma_f32_16x16x32_bf16 v[88:91], v[212:215], v[196:199], v[88:91]\n" \
    "ds_write_b32 v247, v246 offset:32\n" \
    "v_mfma_f32_16x16x32_bf16 v[124:127], v[212:215], v[204:207], v[124:127]\n" \
    "v_mfma_f32_16x16x32_bf16 v[124:127], v[208:211], v[200:203], v[124:127]\n" \
    "s_waitcnt lgkmcnt(9)\n" \
    "v_mfma_f32_16x16x32_bf16 v[128:131], v[216:219], v[200:203], v[128:131]\n" \
    "ds_read_b64_tr_b16 v[208:209], v240 offset:0\n" \
    "v_mfma_f32_16x16x32_bf16 v[128:131], v[220:223], v[204:207], v[128:131]\n" \
    "ds_read_b64_tr_b16 v[210:211], v240 offset:4608\n" \
    "v_mfma_f32_16x16x32_bf16 v[92:95], v[220:223], v[196:199], v[92:95]\n" \
    "ds_read_b64_tr_b16 v[212:213], v240 offset:9216\n" \
    "v_mfma_f32_16x16x32_bf16 v[92:95], v[216:219], v[192:195], v[92:95]\n" \
    "ds_read_b64_tr_b16 v[214:215], v240 offset:13824\n" \
    "v_mfma_f32_16x16x32_bf16 v[56:59], v[216:219], v[184:187], v[56:59]\n" \
    "v_mfma_f32_16x16x32_bf16 v[56:59], v[220:223], v[188:191], v[56:59]\n" \
    "v_mfma_f32_16x16x32_bf16 v[20:23], v[220:223], v[180:183], v[20:23]\n" \
    "v_mfma_f32_16x16x32_bf16 v[20:23], v[216:219], v[176:179], v[20:23]\n" \
    "s_waitcnt lgkmcnt(9)\n" \
    "v_mfma_f32_16x16x32_bf16 v[24:27], v[224:227], v[176:179], v[24:27]\n" \
    "ds_read2_b32 v[244:245], v247 offset1:4\n" \
    "ds_read_b64_tr_b16 v[216:217], v240 offset:32\n" \
    "v_mfma_f32_16x16x32_bf16 v[24:27], v[228:231], v[180:183], v[24:27]\n" \
    "ds_read_b64_tr_b16 v[218:219], v240 offset:4640\n" \
    "v_mfma_f32_16x16x32_bf16 v[60:63], v[228:231], v[188:191], v[60:63]\n" \
    "ds_read_b64_tr_b16 v[220:221], v240 offset:9248\n" \
    "v_mfma_f32_16x16x32_bf16 v[60:63], v[224:227], v[184:187], v[60:63]\n" \
    "ds_read_b64_tr_b16 v[222:223], v240 offset:13856\n" \
    "v_mfma_f32_16x16x32_bf16 v[96:99], v[224:227], v[192:195], v[96:99]\n" \
    "v_mfma_f32_16x16x32_bf16 v[96:99], v[228:231], v[196:199], v[96:99]\n" \
    "v_mfma_f32_16x16x32_bf16 v[132:135], v[228:231], v[204:207], v[132:135]\n" \
    "v_mfma_f32_16x16x32_bf16 v[132:135], v[224:227], v[200:203], v[132:135]\n" \
    "s_waitcnt lgkmcnt(4)\n" \
    "v_mfma_f32_16x16x32_bf16 v[136:139], v[232:235], v[200:203], v[136:139]\n" \
    "ds_read_b64_tr_b16 v[224:225], v240 offset:64\n" \
    "v_mfma_f32_16x16x32_bf16 v[136:139], v[236:239], v[204:207], v[136:139]\n" \
    "ds_read_b64_tr_b16 v[226:227], v240 offset:4672\n" \
    "v_mfma_f32_16x16x32_bf16 v[100:103], v[236:239], v[196:199], v[100:103]\n" \
    "ds_read_b64_tr_b16 v[228:229], v240 offset:9280\n" \
    "v_mfma_f32_16x16x32_bf16 v[100:103], v[232:235], v[192:195], v[100:103]\n" \
    "ds_read_b64_tr_b16 v[230:231], v240 offset:13888\n" \
    "v_mfma_f32_16x16x32_bf16 v[64:67], v[232:235], v[184:187], v[64:67]\n" \
    "v_min_u32 v244, v244, v245\n" \
    "v_mfma_f32_16x16x32_bf16 v[64:67], v[236:239], v[188:191], v[64:67]\n" \
    "v_readfirstlane_b32 %[hv], v244\n" \
    "v_mfma_f32_16x16x32_bf16 v[28:31], v[236:239], v[180:183], v[28:31]\n" \
    "s_cmp_ge_u32 %[hv], %[nd]\n" \
    "v_mfma_f32_16x16x32_bf16 v[28:31], v[232:235], v[176:179], v[28:31]\n" \
    "s_waitcnt lgkmcnt(8) vmcnt(6)\n" \
    "s_cbranch_scc1 L8_%=\n" \
    "Q8_%=:\n" \
    "ds_read2_b32 v[244:245], v247 offset1:4\n" \
    "s_waitcnt lgkmcnt(0)\n" \
    "v_min_u32 v244, v244, v245\n" \
    "s_nop 0\n" \
    "v_readfirstlane_b32 %[hv], v244\n" \
    "s_cmp_ge_u32 %[hv], %[nd]\n" \
    "s_cbranch_scc1 L8_%=\n" \
    "s_sleep 1\n" \
    "s_branch Q8_%=\n" \
    "P8_%=:\n" \
    "ds_read2_b32 v[244:245], v247 offset1:4\n" \
    "s_waitcnt lgkmcnt(0)\n" \
    "v_min_u32 v244, v244, v245\n" \
    "s_nop 0\n" \
    "v_readfirstlane_b32 %[hv], v244\n" \
    "s_cmp_ge_u32 %[hv], %[nd]\n" \
    "s_cbranch_scc1 R8_%=\n" \
    "s_sleep 1\n" \
    "s_branch P8_%=\n" \
    "S8_%=:\n" \
    "s_cmp_eq_u32 %[nd], 65\n" \
    "s_cbranch_scc0 P8_%=\n" \
    "v_mfma_f32_16x16x32_bf16 v[0:3], v[208:211], v[176:179], v[0:3]\n" \
    "v_and_b32 v171, -16, v247\n" \
    "ds_read_b128 v[144:147], v171 offset:32\n" \
    "ds_read_b64_tr_b16 v[232:233], v240 offset:96\n" \
    "v_mfma_f32_16x16x32_bf16 v[0:3], v[212:215], v[180:183], v[0:3]\n" \
    "s_waitcnt vmcnt(4)\n" \
    "v_mbcnt_lo_u32_b32 v175, -1, 0\n" \
    "v_mbcnt_hi_u32_b32 v175, -1, v175\n" \
    "v_mfma_f32_16x16x32_bf16 v[36:39], v[212:215], v[188:191], v[36:39]\n" \
    "ds_read_b64_tr_b16 v[234:235], v240 offset:4704\n" \
    "v_and_b32 v242, 15, v175\n" \
    "v_mul_u32_u24 v242, 304, v242\n" \
    "v_mfma_f32_16x16x32_bf16 v[36:39], v[208:211], v[184:187], v[36:39]\n" \
    "s_waitcnt vmcnt(2)\n" \
    "v_lshrrev_b32 v174, 4, v175\n" \
    "v_lshl_add_u32 v242, v174, 3, v242\n" \
    "v_mfma_f32_16x16x32_bf16 v[72:75], v[208:211], v[192:195], v[72:75]\n" \
    "ds_read_b64_tr_b16 v[236:237], v240 offset:9312\n" \
    "v_add_u32 v242, %[st], v242\n" \
    "v_lshrrev_b32 v174, 3, v175\n" \
    "v_mfma_f32_16x16x32_bf16 v[72:75], v[212:215], v[196:199], v[72:75]\n" \
    "s_waitcnt vmcnt(0)\n" \
    "v_and_b32 v246, 7, v175\n" \
    "v_lshlrev_b32 v246, 4, v246\n" \
    "v_mfma_f32_16x16x32_bf16 v[108:111], v[212:215], v[204:207], v[108:111]\n" \
    "ds_read_b64_tr_b16 v[238:239], v240 offset:13920\n" \
    "v_mul_u32_u24 v243, 304, v174\n" \
    "v_add3_u32 v243, v243, v246, %[st]\n" \
    "v_mfma_f32_16x16x32_bf16 v[108:111], v[208:211], v[200:203], v[108:111]\n" \
    "v_lshl_add_u32 v246, v174, 12, v246\n" \
    "s_waitcnt lgkmcnt(4)\n" \
    "v_mfma_f32_16x16x32_bf16 v[112:115], v[216:219], v[200:203], v[112:115]\n" \
    "ds_read_b64_tr_b16 v[208:209], v240 offset:128\n" \
    "v_cvt_pk_bf16_f32 v0, v0, v1\n" \
    "v_cvt_pk_bf16_f32 v1, v2, v3\n" \
    "v_mfma_f32_16x16x32_bf16 v[112:115], v[220:223], v[204:207], v[112:115]\n" \
    "v_min3_u32 v144, v144, v145, v146\n" \
    "v_min_u32 v244, v144, v147\n" \
    "s_nop 0\n" \
    "v_readfirstlane_b32 %[hv], v244\n" \
    "s_cmp_ge_u32 %[hv], 61\n" \
    "s_cbranch_scc0 W8_%=\n" \
    "V8_%=:\n" \
    "ds_write_b64 v242, v[0:1] offset:0\n" \
    "v_mfma_f32_16x16x32_bf16 v[76:79], v[220:223], v[196:199], v[76:79]\n" \
    "ds_read_b64_tr_b16 v[210:211], v240 offset:4736\n" \
    "v_cvt_pk_bf16_f32 v36, v36, v37\n" \
    "v_cvt_pk_bf16_f32 v37, v38, v39\n" \
    "v_mfma_f32_16x16x32_bf16 v[76:79], v[216:219], v[192:195], v[76:79]\n" \
    "ds_write_b64 v242, v[36:37] offset:4864\n" \
    "v_mfma_f32_16x16x32_bf16 v[40:43], v[216:219], v[184:187], v[40:43]\n" \
    "ds_read_b64_tr_b16 v[212:213], v240 offset:9344\n" \
    "v_cvt_pk_bf16_f32 v72, v72, v73\n" \
    "v_cvt_pk_bf16_f32 v73, v74, v75\n" \
    "v_mfma_f32_16x16x32_bf16 v[40:43], v[220:223], v[188:191], v[40:43]\n" \
    "ds_write_b64 v242, v[72:73] offset:9728\n" \
    "v_mfma_f32_16x16x32_bf16 v[4:7], v[220:223], v[180:183], v[4:7]\n" \
    "ds_read_b64_tr_b16 v[214:215], v240 offset:13952\n" \
    "v_cvt_pk_bf16_f32 v108, v108, v109\n" \
    "v_cvt_pk_bf16_f32 v109, v110, v111\n" \
    "v_mfma_f32_16x16x32_bf16 v[4:7], v[216:219], v[176:179], v[4:7]\n" \
    "ds_write_b64 v242, v[108:109] offset:14592\n" \
    "v_mfma_f32_16x16x32_bf16 v[8:11], v[224:227], v[176:179], v[8:11]\n" \
    "ds_read_b64_tr_b16 v[216:217], v240 offset:160\n" \
    "v_cvt_pk_bf16_f32 v112, v112, v113\n" \
    "v_cvt_pk_bf16_f32 v113, v114, v115\n" \
    "v_mfma_f32_16x16x32_bf16 v[8:11], v[228:231], v[180:183], v[8:11]\n" \
    "ds_write_b64 v242, v[112:113] offset:14624\n" \
    "v_mfma_f32_16x16x32_bf16 v[44:47], v[228:231], v[188:191], v[44:47]\n" \
    "ds_read_b64_tr_b16 v[218:219], v240 offset:4768\n" \
    "v_cvt_pk_bf16_f32 v76, v76, v77\n" \
    "v_cvt_pk_bf16_f32 v77, v78, v79\n" \
    "v_mfma_f32_16x16x32_bf16 v[44:47], v[224:227], v[184:187], v[44:47]\n" \
    "ds_write_b64 v242, v[76:77] offset:9760\n" \
    "v_mfma_f32_16x16x32_bf16 v[80:83], v[224:227], v[192:195], v[80:83]\n" \
    "ds_read_b64_tr_b16 v[220:221], v240 offset:9376\n" \
    "v_cvt_pk_bf16_f32 v40, v40, v41\n" \
    "v_cvt_pk_bf16_f32 v41, v42, v43\n" \
    "v_mfma_f32_16x16x32_bf16 v[80:83], v[228:231], v[196:199], v[80:83]\n" \
    "ds_write_b64 v242, v[40:41] offset:4896\n" \
    "v_mfma_f32_16x16x32_bf16 v[116:119], v[228:231], v[204:207], v[116:119]\n" \
    "ds_read_b64_tr_b16 v[222:223], v240 offset:13984\n" \
    "v_cvt_pk_bf16_f32 v4, v4, v5\n" \
    "v_cvt_pk_bf16_f32 v5, v6, v7\n" \
    "v_mfma_f32_16x16x32_bf16 v[116:119], v[224:227], v[200:203], v[116:119]\n" \
    "ds_write_b64 v242, v[4:5] offset:32\n" \
    "s_waitcnt lgkmcnt(15)\n" \
    "v_mfma_f32_16x16x32_bf16 v[120:123], v[232:235], v[200:203], v[120:123]\n" \
    "ds_read_b64_tr_b16 v[224:225], v240 offset:192\n" \
    "v_cvt_pk_bf16_f32 v8, v8, v9\n" \
    "v_cvt_pk_bf16_f32 v9, v10, v11\n" \
    "v_mfma_f32_16x16x32_bf16 v[120:123], v[236:239], v[204:207], v[120:123]\n" \
    "ds_write_b64 v242, v[8:9] offset:64\n" \
    "v_mfma_f32_16x16x32_bf16 v[84:87], v[236:239], v[196:199], v[84:87]\n" \
    "ds_read_b64_tr_b16 v[226:227], v240 offset:4800\n" \
    "v_cvt_pk_bf16_f32 v44, v44, v45\n" \
    "v_cvt_pk_bf16_f32 v45, v46, v47\n" \
    "v_mfma_f32_16x16x32_bf16 v[84:87], v[232:235], v[192:195], v[84:87]\n" \
    "ds_write_b64 v242, v[44:45] offset:4928\n" \
    "v_mfma_f32_16x16x32_bf16 v[48:51], v[232:235], v[184:187], v[48:51]\n" \
    "ds_read_b64_tr_b16 v[228:229], v240 offset:9408\n" \
    "v_cvt_pk_bf16_f32 v80, v80, v81\n" \
    "v_cvt_pk_bf16_f32 v81, v82, v83\n" \
    "v_mfma_f32_16x16x32_bf16 v[48:51], v[236:239], v[188:191], v[48:51]\n" \
    "ds_write_b64 v242, v[80:81] offset:9792\n" \
    "v_mfma_f32_16x16x32_bf16 v[12:15], v[236:239], v[180:183], v[12:15]\n" \
    "ds_read_b64_tr_b16 v[230:231], v240 offset:14016\n" \
    "v_cvt_pk_bf16_f32 v116, v116, v117\n" \
    "v_cvt_pk_bf16_f32 v117, v118, v119\n" \
    "v_mfma_f32_16x16x32_bf16 v[12:15], v[232:235], v[176:179], v[12:15]\n" \
    "ds_write_b64 v242, v[116:117] offset:14656\n" \
    "s_waitcnt lgkmcnt(15)\n" \
    "v_mfma_f32_16x16x32_bf16 v[16:19], v[208:211], v[176:179], v[16:19]\n" \
    "ds_read_b64_tr_b16 v[232:233], v240 offset:224\n" \
    "v_cvt_pk_bf16_f32 v120, v120, v121\n" \
    "v_cvt_pk_bf16_f32 v121, v122, v123\n" \
    "v_mfma_f32_16x16x32_bf16 v[16:19], v[212:215], v[180:183], v[16:19]\n" \
    "ds_write_b64 v242, v[120:121] offset:14688\n" \
    "ds_read_b128 v[144:147], v243 offset:14592\n" \
    "ds_read_b128 v[148:151], v243 offset:17024\n" \
    "v_mfma_f32_16x16x32_bf16 v[52:55], v[212:215], v[188:191], v[52:55]\n" \
    "ds_read_b64_tr_b16 v[234:235], v240 offset:4832\n" \
    "v_cvt_pk_bf16_f32 v84, v84, v85\n" \
    "v_cvt_pk_bf16_f32 v85, v86, v87\n" \
    "v_mfma_f32_16x16x32_bf16 v[52:55], v[208:211], v[184:187], v[52:55]\n" \
    "ds_write_b64 v242, v[84:85] offset:9824\n" \
    "ds_read_b128 v[152:155], v243 offset:9728\n" \
    "ds_read_b128 v[156:159], v243 offset:12160\n" \
    "v_mfma_f32_16x16x32_bf16 v[88:91], v[208:211], v[192:195], v[88:91]\n" \
    "ds_read_b64_tr_b16 v[236:237], v240 offset:9440\n" \
    "v_cvt_pk_bf16_f32 v48, v48, v49\n" \
    "v_cvt_pk_bf16_f32 v49, v50, v51\n" \
    "v_mfma_f32_16x16x32_bf16 v[88:91], v[212:215], v[196:199], v[88:91]\n" \
    "ds_write_b64 v242, v[48:49] offset:4960\n" \
    "ds_read_b128 v[160:163], v243 offset:4864\n" \
    "ds_read_b128 v[164:167], v243 offset:7296\n" \
    "v_mfma_f32_16x16x32_bf16 v[124:127], v[212:215], v[204:207], v[124:127]\n" \
    "ds_read_b64_tr_b16 v[238:239], v240 offset:14048\n" \
    "v_cvt_pk_bf16_f32 v12, v12, v13\n" \
    "v_cvt_pk_bf16_f32 v13, v14, v15\n" \
    "v_mfma_f32_16x16x32_bf16 v[124:127], v[208:211], v[200:203], v[124:127]\n" \
    "ds_write_b64 v242, v[12:13] offset:96\n" \
    "ds_read_b128 v[0:3], v243 offset:0\n" \
    "ds_read_b128 v[36:39], v243 offset:2432\n" \
    "s_waitcnt lgkmcnt(15)\n" \
    "v_mfma_f32_16x16x32_bf16 v[128:131], v[216:219], v[200:203], v[128:131]\n" \
    "v_cvt_pk_bf16_f32 v16, v16, v17\n" \
    "v_cvt_pk_bf16_f32 v17, v18, v19\n" \
    "s_mov_b32 %[ps], 196608\n" \
    "s_waitcnt lgkmcnt(13)\n" \
    "buffer_store_dwordx4 v[144:147], v246, %[ro], %[ps] offen nt\n" \
    "v_mfma_f32_16x16x32_bf16 v[128:131], v[220:223], v[204:207], v[128:131]\n" \
    "ds_write_b64 v242, v[16:17] offset:128\n" \
    "v_mfma_f32_16x16x32_bf16 v[92:95], v[220:223], v[196:199], v[92:95]\n" \
    "v_cvt_pk_bf16_f32 v52, v52, v53\n" \
    "v_cvt_pk_bf16_f32 v53, v54, v55\n" \
    "s_mov_b32 %[ps], 229376\n" \
    "s_waitcnt lgkmcnt(13)\n" \
    "buffer_store_dwordx4 v[148:151], v246, %[ro], %[ps] offen nt\n" \
    "v_mfma_f32_16x16x32_bf16 v[92:95], v[216:219], v[192:195], v[92:95]\n" \
    "ds_write_b64 v242, v[52:53] offset:4992\n" \
    "v_mfma_f32_16x16x32_bf16 v[56:59], v[216:219], v[184:187], v[56:59]\n" \
    "v_cvt_pk_bf16_f32 v88, v88, v89\n" \
    "v_cvt_pk_bf16_f32 v89, v90, v91\n" \
    "s_mov_b32 %[ps], 131072\n" \
    "s_waitcnt lgkmcnt(11)\n" \
    "buffer_store_dwordx4 v[152:155], v246, %[ro], %[ps] offen nt\n" \
    "v_mfma_f32_16x16x32_bf16 v[56:59], v[220:223], v[188:191], v[56:59]\n" \
    "ds_write_b64 v242, v[88:89] offset:9856\n" \
    "v_mfma_f32_16x16x32_bf16 v[20:23], v[220:223], v[180:183], v[20:23]\n" \
    "v_cvt_pk_bf16_f32 v124, v124, v125\n" \
    "v_cvt_pk_bf16_f32 v125, v126, v127\n" \
    "s_mov_b32 %[ps], 163840\n" \
    "s_waitcnt lgkmcnt(11)\n" \
    "buffer_store_dwordx4 v[156:159], v246, %[ro], %[ps] offen nt\n" \
    "v_mfma_f32_16x16x32_bf16 v[20:23], v[216:219], v[176:179], v[20:23]\n" \
    "ds_write_b64 v242, v[124:125] offset:14720\n" \
    "v_mfma_f32_16x16x32_bf16 v[24:27], v[224:227], v[176:179], v[24:27]\n" \
    "v_cvt_pk_bf16_f32 v128, v128, v129\n" \
    "v_cvt_pk_bf16_f32 v129, v130, v131\n" \
    "s_mov_b32 %[ps], 65536\n" \
    "s_waitcnt lgkmcnt(9)\n" \
    "buffer_store_dwordx4 v[160:163], v246, %[ro], %[ps] offen nt\n" \
    "v_mfma_f32_16x16x32_bf16 v[24:27], v[228:231], v[180:183], v[24:27]\n" \
    "ds_write_b64 v242, v[128:129] offset:14752\n" \
    "v_mfma_f32_16x16x32_bf16 v[60:63], v[228:231], v[188:191], v[60:63]\n" \
    "v_cvt_pk_bf16_f32 v92, v92, v93\n" \
    "v_cvt_pk_bf16_f32 v93, v94, v95\n" \
    "s_mov_b32 %[ps], 98304\n" \
    "s_waitcnt lgkmcnt(9)\n" \
    "buffer_store_dwordx4 v[164:167], v246, %[ro], %[ps] offen nt\n" \
    "v_mfma_f32_16x16x32_bf16 v[60:63], v[224:227], v[184:187], v[60:63]\n" \
    "ds_write_b64 v242, v[92:93] offset:9888\n" \
    "v_mfma_f32_16x16x32_bf16 v[96:99], v[224:227], v[192:195], v[96:99]\n" \
    "v_cvt_pk_bf16_f32 v56, v56, v57\n" \
    "v_cvt_pk_bf16_f32 v57, v58, v59\n" \
    "s_mov_b32 %[ps], 0\n" \
    "s_waitcnt lgkmcnt(7)\n" \
    "buffer_store_dwordx4 v[0:3], v246, %[ro], %[ps] offen nt\n" \
    "v_mfma_f32_16x16x32_bf16 v[96:99], v[228:231], v[196:199], v[96:99]\n" \
    "ds_write_b64 v242, v[56:57] offset:5024\n" \
    "v_mfma_f32_16x16x32_bf16 v[132:135], v[228:231], v[204:207], v[132:135]\n" \
    "v_cvt_pk_bf16_f32 v20, v20, v21\n" \
    "v_cvt_pk_bf16_f32 v21, v22, v23\n" \
    "s_mov_b32 %[ps], 32768\n" \
    "s_waitcnt lgkmcnt(7)\n" \
    "buffer_store_dwordx4 v[36:39], v246, %[ro], %[ps] offen nt\n" \
    "v_mfma_f32_16x16x32_bf16 v[132:135], v[224:227], v[200:203], v[132:135]\n" \
    "ds_write_b64 v242, v[20:21] offset:160\n" \
    "v_mfma_f32_16x16x32_bf16 v[136:139], v[232:235], v[200:203], v[136:139]\n" \
    "v_cvt_pk_bf16_f32 v24, v24, v25\n" \
    "v_cvt_pk_bf16_f32 v25, v26, v27\n" \
    "v_mfma_f32_16x16x32_bf16 v[136:139], v[236:239], v[204:207], v[136:139]\n" \
    "ds_write_b64 v242, v[24:25] offset:192\n" \
    "v_mfma_f32_16x16x32_bf16 v[100:103], v[236:239], v[196:199], v[100:103]\n" \
    "v_cvt_pk_bf16_f32 v60, v60, v61\n" \
    "v_cvt_pk_bf16_f32 v61, v62, v63\n" \
    "v_mfma_f32_16x16x32_bf16 v[100:103], v[232:235], v[192:195], v[100:103]\n" \
    "ds_write_b64 v242, v[60:61] offset:5056\n" \
    "v_mfma_f32_16x16x32_bf16 v[64:67], v[232:235], v[184:187], v[64:67]\n" \
    "v_cvt_pk_bf16_f32 v96, v96, v97\n" \
    "v_cvt_pk_bf16_f32 v97, v98, v99\n" \
    "v_mfma_f32_16x16x32_bf16 v[64:67], v[236:239], v[188:191], v[64:67]\n" \
    "ds_write_b64 v242, v[96:97] offset:9920\n" \
    "v_mfma_f32_16x16x32_bf16 v[28:31], v[236:239], v[180:183], v[28:31]\n" \
    "v_cvt_pk_bf16_f32 v132, v132, v133\n" \
    "v_cvt_pk_bf16_f32 v133, v134, v135\n" \
    "v_mfma_f32_16x16x32_bf16 v[28:31], v[232:235], v[176:179], v[28:31]\n" \
    "ds_write_b64 v242, v[132:133] offset:14784\n" \
    "v_cvt_pk_bf16_f32 v136, v136, v137\n" \
    "v_cvt_pk_bf16_f32 v137, v138, v139\n" \
    "ds_write_b64 v242, v[136:137] offset:14816\n" \
    "v_cvt_pk_bf16_f32 v100, v100, v101\n" \
    "v_cvt_pk_bf16_f32 v101, v102, v103\n" \
    "ds_write_b64 v242, v[100:101] offset:9952\n" \
    "v_cvt_pk_bf16_f32 v64, v64, v65\n" \
    "v_cvt_pk_bf16_f32 v65, v66, v67\n" \
    "ds_write_b64 v242, v[64:65] offset:5088\n" \
    "s_nop 5\n" \
    "v_cvt_pk_bf16_f32 v28, v28, v29\n" \
    "v_cvt_pk_bf16_f32 v29, v30, v31\n" \
    "ds_write_b64 v242, v[28:29] offset:224\n" \
    "s_branch LEND%=\n" \
    "W8_%=:\n" \
    "s_sleep 1\n" \
    "ds_read_b128 v[144:147], v171 offset:32\n" \
    "s_waitcnt lgkmcnt(0)\n" \
    "v_min3_u32 v144, v144, v145, v146\n" \
    "v_min_u32 v244, v144, v147\n" \
    "s_nop 0\n" \
    "v_readfirstlane_b32 %[hv], v244\n" \
    "s_cmp_ge_u32 %[hv], 61\n" \
    "s_cbranch_scc1 V8_%=\n" \
    "s_branch W8_%=\n" \
    "LEND%=:\n" \
    ""

__global__ void __launch_bounds__(NTHR, 2) ag_gemm(const Args a) {
    __shared__ __attribute__((aligned(1024))) unsigned char lds_raw[LDS_BYTES];
    unsigned code;
    {
        unsigned long long pc;
        unsigned len;
        asm volatile("s_getpc_b64 %0\nAG_CODE0:\n\ts_mov_b32 %1, AG_CODE1-AG_CODE0" : "=s"(pc), "=s"(len));
        code = __builtin_amdgcn_raw_buffer_load_b32(rsrc((const void*)pc, len), threadIdx.x * 64, 0, 0);
        SB();
    }
    LAS unsigned char* lds = (LAS unsigned char*)lds_raw;
    volatile LAS unsigned* ctl = (volatile LAS unsigned*)(lds + L_CTL);
    volatile LAS unsigned* prog = ctl + 8;
    const int lane = threadIdx.x & 63;
    const int wave = __builtin_amdgcn_readfirstlane(threadIdx.x >> 6);
    const int bid = blockIdx.x;
    char* const myws = a.ws[NR - 1];
    asm volatile("" :: "s"(a.x), "s"(a.ws[0]), "s"(a.ws[1]), "s"(a.ws[2]), "s"(a.ws[3]), "s"(a.ws[4]), "s"(a.ws[5]), "s"(a.ws[6]), "s"(myws), "s"(a.rank));
    const int xcd = bid & 7, loc = bid >> 3;
    const bool is_comm = loc < 2;
    const int cid = loc * 8 + xcd, tl = (xcd & 1) * (2 * NTN) + loc - 2, pm = (xcd >> 1) * 4 + (tl & 3), pn = tl >> 2;
    const int pq = (pn & 1) ? -1 : pn >> 1, part = pq >> 31, flip = 0;

    if (wave == 0) STAMP(0);
    const __amdgpu_buffer_rsrc_t rin = rsrc(a.x, MS * K * 2);
    auto fetch = [&](u32x4 (&d)[4], unsigned b) {
        const unsigned o = (b < (unsigned)NBLK ? (b % NG) * 32 * (K * 2) + (b / NG) * (BK * 2) : OOB) + (lane & 15) * (K * 2) + (lane >> 4) * 16;
#pragma unroll
        for (int s = 0; s < 4; ++s)
            d[s] = __builtin_bit_cast(u32x4, __builtin_amdgcn_raw_buffer_load_b128(rin, o + (s & 1) * 64, (s >> 1) * 16 * (K * 2), 0));
    };
    u32x4 d[4];
    fetch(d, is_comm && wave < SW ? wave * NSC + cid : NBLK);
    GAS unsigned* cnt = (GAS unsigned*)(myws + W_CNT + (size_t)bid * 128);
    const unsigned cv = __hip_atomic_load(cnt, __ATOMIC_RELAXED, __HIP_MEMORY_SCOPE_AGENT);
    SB();
    if (wave == 0) STAMP(10);
    if (threadIdx.x < 32) ((LAS unsigned*)(lds + L_CTL))[threadIdx.x] = 0u;
    if (threadIdx.x < 256) *(LAS u32x2*)(lds + L_SEL + threadIdx.x * 8) = is_comm ? mover(threadIdx.x, true) : mover(threadIdx.x, false);
    SB();
    asm volatile("" :: "v"(code));
    const unsigned call = __builtin_amdgcn_readfirstlane(cv) + 1u;
    __syncthreads();
    if (threadIdx.x == 0) __hip_atomic_store(cnt, call, __ATOMIC_RELAXED, __HIP_MEMORY_SCOPE_AGENT);
    LAS unsigned char* lm = lds + L_SEL;
    int mul = loc;
    asm volatile("" : "+s"(mul));
    if (mul > 1) {
    GAS unsigned* tmo = (GAS unsigned*)(myws + W_TMO);
    const size_t xt_off = W_XT + (size_t)(call & 1u) * XT_BYTES;
    const unsigned oQ = pq < 0 ? OOB : ((NTN - pn) * BN + pq * 16) * 2;
    if (wave >= 6) {
        __builtin_amdgcn_s_setprio(AG_UPRIO);
        const int src = ((xcd + 2) & 6) + wave - 6;
        const unsigned u = tl;
        const __amdgpu_buffer_rsrc_t rf = rsrc(myws + W_FLAG + (size_t)src * NBLK * FS * 4, NBLK * FS * 4);
        const __amdgpu_buffer_rsrc_t rp = rsrc(myws + W_PK + (size_t)(call & 1u) * PK_BYTES + (size_t)src * NBLK * PBLK, NBLK * PBLK);
        const __amdgpu_buffer_rsrc_t rx = rsrc(myws + xt_off + (size_t)src * NBLK * BLK, NBLK * BLK);
        auto done = [&](unsigned b) {
            const unsigned g = b % NG, kt = b / NG;
            const unsigned fo = lane == 0 ? (((g >> 3) * NKT + kt) * 8 + (g & 7)) * FS * 4 : OOB;
            __builtin_amdgcn_raw_buffer_store_b32(call, rf, fo, 0, 17);
        };
        if (wave == 7) STAMP(8);
        unsigned long long lat_r = 0, lat_w = 0, lat_n = 0;
        if (src == a.rank) {
            fetch(d, u);
            for (unsigned b = u; b < (unsigned)NBLK; b += NU) {
#pragma unroll
                for (int s = 0; s < 4; ++s)
                    __builtin_amdgcn_raw_buffer_store_b128(d[s], rx, b * BLK + lane * 16 + s * 1024, 0, AG_UAUX);
                fetch(d, b + NU);
                wait_vm0();
                done(b);
            }
        } else {
            LAS unsigned char* lx = lds + L_UNP + (wave - 6) * UNP;
            LAS unsigned char* ls = lx + 2112;
            *(LAS u32x4*)(lx + lane * 32) = (u32x4){0u, 0u, 0u, 0u};
            *(LAS u32x4*)(lx + lane * 32 + 16) = (u32x4){0u, 0u, 0u, 0u};
            const unsigned long long want = tag_of(call) ? ~0ull : 0ull;
            const unsigned keep = (lane & 1) ? ~0u : 0x7fffffffu, keept = lane < 16 ? 0x7fffffffu : ~0u, sh = 31 - (lane >> 1), lw = (lane < 24 ? lane : 24) * 16;
            const unsigned long long t0 = rt();
            unsigned b = u, pend = ~0u;
            for (;;) {
                const unsigned so = b * PBLK;
                const unsigned hl = __builtin_amdgcn_raw_buffer_load_b32(rp, lane < 32 && b < (unsigned)NBLK ? so + 2560 + lane * 4 : OOB, 0, AG_HAUX);
                u32x4 p[2], t, x[2];
                u32x2 c;
                auto body = [&](int nt) {
                    c = __builtin_bit_cast(u32x2, __builtin_amdgcn_raw_buffer_load_b64(rp, so + lane * 8 + 2048, 0, AG_BAUX));
                    t = __builtin_bit_cast(u32x4, __builtin_amdgcn_raw_buffer_load_b128(rp, lane < nt ? so + lane * 16 + 2560 : OOB, 0, AG_BAUX));
#pragma unroll
                    for (int s = 0; s < 2; ++s)
                        p[s] = __builtin_bit_cast(u32x4, __builtin_amdgcn_raw_buffer_load_b128(rp, so + lane * 16 + s * 1024, 0, AG_BAUX));
                };
                const bool whole = AG_TB && b - (unsigned)(NBLK - AG_TB) < (unsigned)AG_TB && (AG_TBP || pend == ~0u);
                if (whole) body(16);
                wait_vm0();
                const bool first = pend != ~0u;
                if (first) {
                    done(pend);
                    TRACE(src, pend, 2);
                    pend = ~0u;
                }
                if (b >= (unsigned)NBLK) break;
                if (((unsigned)(__ballot((int)hl < 0) ^ want) & 0x11111111u) != 0u) {
                    if (give_up(tmo, call, t0)) {
                        __hip_atomic_store(tmo, call, __ATOMIC_RELAXED, __HIP_MEMORY_SCOPE_AGENT);
                        break;
                    }
                    __builtin_amdgcn_s_sleep(AG_USLEEP);
                    continue;
                }
                if (first) lat_w += 100;
                ++lat_n;
                const unsigned h0 = __builtin_amdgcn_readlane(hl, 0);
                if (!whole || (h0 & 512u)) body((h0 & 512u) ? 24 : 16);
                TRACE(src, b, 1);
                const unsigned m2l = __builtin_amdgcn_readlane(hl, 3), m1l = bfi(0x7fffffffu, __builtin_amdgcn_readlane(hl, 4), h0 << 15);
                const unsigned long long m0 = (unsigned)__builtin_amdgcn_readlane(hl, 1) | ((unsigned long long)(unsigned)__builtin_amdgcn_readlane(hl, 2) << 32);
                const unsigned long long m1 = m1l | ((unsigned long long)(unsigned)__builtin_amdgcn_readlane(hl, 5) << 32);
                const unsigned e0 = h0 & 255u, cnt = (h0 >> 10) & 31u;
                const unsigned pa = (unsigned)lane - HW < cnt ? hl & 0x7ffu : 2048u;
                lx[pa] = (unsigned char)(hl >> 16);
                FENCE();
                x[0] = *(LAS u32x4*)(lx + lane * 32);
                x[1] = *(LAS u32x4*)(lx + lane * 32 + 16);
                FENCE();
                lx[pa] = 0;
                unsigned t2 = table2(e0), t3l = table3(e0, 0), t3h = table3(e0, 1), top0, top1, topc = m2l << sh, topt = h0 << (16 - (lane & 15));
                unsigned vo = (h0 & 256u) ? OOB : b * BLK + lane * 16, lq = (unsigned)(size_t)ls + (HW + cnt) * 4u;
                asm("v_cndmask_b32 %0, 0, -1, %1" : "=v"(top0) : "s"(m0));
                asm("v_cndmask_b32 %0, 0, -1, %1" : "=v"(top1) : "s"(m1));
                asm volatile("" : "+v"(t2), "+v"(t3l), "+v"(top0), "+v"(top1), "+v"(topc), "+v"(topt), "+v"(vo));
                asm volatile("" : "+s"(t3h), "+s"(lq));
                if (((__ballot((int)p[0].x < 0) ^ want) | (__ballot((int)p[1].x < 0) ^ want) |
                     ((__ballot((int)c.x < 0) ^ want) & 0x5555555555555555ull) | ((__ballot((int)t.x < 0) ^ want) & 0xffffull)) != 0ull) {
                    if (!give_up(tmo, call, t0)) continue;
                    __hip_atomic_store(tmo, call, __ATOMIC_RELAXED, __HIP_MEMORY_SCOPE_AGENT);
                    break;
                }
                t.x = bfi(keept, t.x, topt);
                *(LAS u32x4*)(ls + lw) = t;
                c.x = bfi(keep, c.x, topc);
                const u32x2 f = c & (c >> 1) & 0x55555555u;
                const unsigned nb = 3u * (__builtin_popcount(f.x) + __builtin_popcount(f.y));
                const unsigned off = scan(nb) - nb;
                u32x2 mv[8];
                FENCE();
                LAS unsigned* lt = (LAS unsigned*)(size_t)(((off >> 5) << 2) + lq);
                const unsigned w0 = lt[0], w1 = lt[1], w2 = lt[2];
#pragma unroll
                for (int j = 0; j < 8; ++j)
                    mv[j] = *(LAS u32x2*)(lm + ((c[j >> 2] >> (8 * (j & 3))) & 255u) * 8);
                FENCE();
                unsigned al = __builtin_amdgcn_alignbit(w1, w0, off), ah = __builtin_amdgcn_alignbit(w2, w1, off);
                p[0].x = bfi(0x7fffffffu, p[0].x, top0);
                p[1].x = bfi(0x7fffffffu, p[1].x, top1);
#pragma unroll
                for (int s = 0; s < 4; ++s) {
                    u32x4 d;
#pragma unroll
                    for (int i = 0; i < 2; ++i) {
                        const int j = 2 * s + i;
                        unsigned q = al;
                        al = __builtin_amdgcn_alignbit(ah, al, mv[j].y);
                        ah >>= mv[j].y;
                        asm("v_lshrrev_b32_sdwa %0, %1, %0 dst_sel:WORD_1 dst_unused:UNUSED_PRESERVE src0_sel:DWORD src1_sel:DWORD" : "+v"(q) : "v"(6), "v"(al));
                        const unsigned g = perm(t3h, t3l, (q & 0x00070007u) | ((q << 5) & 0x07000700u));
                        const unsigned e = perm(t2, g, mv[j].x) ^ x[j >> 2][j & 3];
                        const unsigned r0 = perm(e, p[j >> 2][j & 3], 0x05010400u), r1 = perm(e, p[j >> 2][j & 3], 0x07030602u);
                        d[2 * i] = __builtin_amdgcn_alignbit(r0, r0, 1);
                        d[2 * i + 1] = __builtin_amdgcn_alignbit(r1, r1, 1);
                    }
                    __builtin_amdgcn_raw_buffer_store_b128(d, rx, vo + s * 1024, 0, AG_UAUX);
                }
                pend = b;
                b += NU;
            }
        }
        if (wave == 7) STAMP(9);
        asm volatile("AG_CODE1:");
    }
    else if (wave < 4) {
        const unsigned su = ((xcd >> 1) * (4 * NTN) + tl) * 4 + wave, sk = NSLOT * BK + su * 4;
        const u32x4 sv = __builtin_bit_cast(u32x4, __builtin_amdgcn_raw_buffer_load_b128(rsrc(a.w, K * NL * 2),
            (sk + ((lane >> 1) & 3)) * (NL * 2) + NTN * BN * 2 + (lane >> 3) * 32 + (lane & 1) * 16, 0, 0));
        auto strip = [&]() {
            const unsigned r0 = (sk & ~31u) + ((sk >> 2) & 1) * 16 + ((sk >> 3) & 3) * 4;
            store16(sv, rsrc(myws + W_STRIP, K * BN * 2), su < (unsigned)NSTRIP ? (lane >> 3) * (K * 32) + r0 * 32 + (lane & 7) * 16 : OOB);
        };
        if (wave == 3) strip();
        constexpr int PW = AG_DRY ? 3 : 4;
        if (wave < PW) {
            const unsigned cw = ((xcd >> 1) * (4 * NTN) + tl) * PW + wave;
            constexpr int W1 = (AG_PD ? AG_PD : NKT) * BK * NL * 2;
            const __amdgpu_buffer_rsrc_t rw = rsrc(a.w, W1), rs = rsrc(a.x, MS * K * 2);
            constexpr int W0 = AG_PF0 * BK * NL * 2, ST = 16 * NTN * PW * 1024, NW = (W1 - W0 + ST - 1) / ST, NX = (MS * K * 2 + ST - 1) / ST;
            u32x4 pf[NW], px[NX];
#pragma unroll
            for (int i = 0; i < NW; ++i)
                pf[i] = __builtin_bit_cast(u32x4, __builtin_amdgcn_raw_buffer_load_b128(rw, W0 + i * ST + cw * 1024 + lane * 16, 0, AG_PFAUX));
            if (AG_PF == 2) {
#pragma unroll
                for (int i = 0; i < NX; ++i)
                    px[i] = __builtin_bit_cast(u32x4, __builtin_amdgcn_raw_buffer_load_b128(rs, i * ST + cw * 1024 + lane * 16, 0, AG_PFAUX));
                SB();
                strip();
#pragma unroll
                for (int i = 0; i < NX; ++i) asm volatile("" :: "v"(px[i]));
            }
#pragma unroll
            for (int i = 0; i < NW; ++i) asm volatile("" :: "v"(pf[i]));
            if (wave == 0) STAMP(12);
        }
        wait_vm0();
        if (lane == 0 && su < (unsigned)NSTRIP)
            __hip_atomic_fetch_add((GAS unsigned*)(myws + W_SCNT + (su & 7) * 128), 1u, __ATOMIC_RELAXED, __HIP_MEMORY_SCOPE_AGENT);
        const int g0 = (pm & 1) * 8 + wave * 2;
        const unsigned long long xb = (unsigned long long)(myws + xt_off + ((size_t)(pm >> 1) * NBLK + g0) * BLK);
        const u32x4 ra = (u32x4){(unsigned)xb, (unsigned)(xb >> 32), (unsigned)((NBLK - g0) * BLK), 0x00020000u};
        const unsigned lb = (unsigned)(unsigned long long)lds;
        int dry = AG_DRY ? (wave + 1) >> 2 : 0;
      for (;;) {
        int l1;
        asm volatile("v_mbcnt_lo_u32_b32 %0, -1, 0\n\tv_mbcnt_hi_u32_b32 %0, -1, %0" : "=v"(l1));
        unsigned vx = l1 * 16 + flip * BLK, have, need, pos, chk;
        unsigned long long t1 = 0, t2 = 0;
        char* ob = a.out + ((size_t)(pm * BM + wave * 64) * NL + pn * BN) * 2;
        constexpr int ES = 304, E0 = NKT - NSLOT;
        static_assert(E0 % NSLOT == 0 && 4 * 64 * ES <= 5 * SLOT && E0 + 5 == 61 && NL == 2048 && L_CTL % 16 == 0, "epilogue staging");
        const u32x4 rq = (u32x4){(unsigned)(size_t)ob, (unsigned)((size_t)ob >> 32), (unsigned)((64 * NL - pn * BN) * 2), 0x00020000u};
        asm volatile(AG_KLOOP
                     : "+{v242}"(vx), [hv] "=&s"(have), [nd] "=&s"(need), [ps] "=&s"(pos), [ck] "=&s"(chk), [t1] "+s"(t1), [t2] "+s"(t2)
                     : "{v241}"(lb + L_W + (l1 >> 2) * RS + (l1 & 3) * 8), "{v247}"(lb + L_CTL + wave * 4 + (l1 >> 6)), [rs] "s"(ra), [n8] "s"(part),
                       [ro] "s"(rq), [st] "s"(lb + L_W + wave * 64 * ES), [oq] "s"(oQ), [dry] "s"(dry)
                     : "v0", "v1", "v2", "v3", "v4", "v5", "v6", "v7", "v8", "v9", "v10", "v11", "v12", "v13", "v14", "v15", "v16", "v17", "v18", "v19", "v20", "v21", "v22", "v23", "v24", "v25", "v26", "v27", "v28", "v29", "v30", "v31", "v32", "v33", "v34", "v35", "v36", "v37", "v38", "v39", "v40", "v41", "v42", "v43", "v44", "v45", "v46", "v47", "v48", "v49", "v50", "v51", "v52", "v53", "v54", "v55", "v56", "v57", "v58", "v59", "v60", "v61", "v62", "v63", "v64", "v65", "v66", "v67", "v68", "v69", "v70", "v71", "v72", "v73", "v74", "v75", "v76", "v77", "v78", "v79", "v80", "v81", "v82", "v83", "v84", "v85", "v86", "v87", "v88", "v89", "v90", "v91", "v92", "v93", "v94", "v95", "v96", "v97", "v98", "v99", "v100", "v101", "v102", "v103", "v104", "v105", "v106", "v107", "v108", "v109", "v110", "v111", "v112", "v113", "v114", "v115", "v116", "v117", "v118", "v119", "v120", "v121", "v122", "v123", "v124", "v125", "v126", "v127", "v128", "v129", "v130", "v131", "v132", "v133", "v134", "v135", "v136", "v137", "v138", "v139", "v140", "v141", "v142", "v143", "v144", "v145", "v146", "v147", "v148", "v149", "v150", "v151", "v152", "v153", "v154", "v155", "v156", "v157", "v158", "v159", "v160", "v161", "v162", "v163", "v164", "v165", "v166", "v167", "v168", "v169", "v170", "v171", "v172", "v173", "v174", "v175", "v176", "v177", "v178", "v179", "v180", "v181", "v182", "v183", "v184", "v185", "v186", "v187", "v188", "v189", "v190", "v191", "v192", "v193", "v194", "v195", "v196", "v197", "v198", "v199", "v200", "v201", "v202", "v203", "v204", "v205", "v206", "v207", "v208", "v209", "v210", "v211", "v212", "v213", "v214", "v215", "v216", "v217", "v218", "v219", "v220", "v221", "v222", "v223", "v224", "v225", "v226", "v227", "v228", "v229", "v230", "v231", "v232", "v233", "v234", "v235", "v236", "v237", "v238", "v239", "v240", "v243", "v244", "v245", "v246", "scc", "memory");
        int l2;
        asm volatile("v_mbcnt_lo_u32_b32 %0, -1, 0\n\tv_mbcnt_hi_u32_b32 %0, -1, %0" : "=v"(l2));
        {
            LAS unsigned char* sr = lds + L_W + wave * 64 * ES + (l2 >> 3) * ES + 128 + (l2 & 7) * 16;
            const __amdgpu_buffer_rsrc_t ro = rsrc(ob, (64 * NL - pn * BN) * 2);
            const unsigned vo = (l2 >> 3) * (NL * 2) + 128 + (l2 & 7) * 16;
            const bool bad = ctl[4] == ALL + 1;
            u32x4 o[8];
#pragma unroll
            for (int s = 0; s < 8; ++s) o[s] = *(LAS u32x4*)(sr + s * 8 * ES);
#pragma unroll
            for (int s = 0; s < 8; ++s) {
                if (s == 0 && bad) o[s] = (u32x4){0x7fc07fc0u, 0x7fc07fc0u, 0x7fc07fc0u, 0x7fc07fc0u};
                __builtin_amdgcn_raw_buffer_store_b128(o[s], ro, vo, s * 8 * NL * 2, AG_OAUX);
            }
        }
        asm volatile("s_mov_b64 exec, -1\n\ts_sub_u32 %0, %0, 1" : "+s"(dry) :: "scc", "memory");
        if (dry < 0) break;
      }
    } else if (wave == 4) {
        __builtin_amdgcn_s_setprio(AG_WPRIO);
        unsigned long long gp[NPIECE / 2], gs[NPIECE / 2];
#pragma unroll
        for (int i = 0; i < NPIECE / 2; ++i) {
            const unsigned o = i * 1024 + lane * 16, r = o / RS, cb = o % RS;
            const unsigned k = ((r >> 2) & 3) * 8 + (r >> 4) * 4 + (r & 3);
            gp[i] = (unsigned long long)a.w + pn * BN * 2 + k * (NL * 2) + (cb < 256 ? cb : pq < 0 ? cb - 32 : cb - 256 + oQ);
            gs[i] = 32 * NL * 2;
        }
        const __amdgpu_buffer_rsrc_t rall = rsrc(a.w, K * NL * 2);
        const unsigned ln = ((xcd >> 1) * (4 * NTN) + tl) * 9 + lane, po = lane < 9 && ln < BK * NL * 2 / 128 ? ln * 128 + AG_PD * (BK * NL * 2) : OOB;
        int pub = 0;
        unsigned so = 0;
        for (int kt = 0; kt < NKT; ++kt) {
            if (kt >= NSLOT) {
                const unsigned nd = kt - NSLOT + 1;
                if (!__all(prog[lane & 3] >= nd)) {
                    if (pub < kt) {
                        asm volatile("s_waitcnt vmcnt(0)" ::: "memory");
                        pub = kt;
                        ctl[lane & 3] = pub;
                    }
                    while (!__all(prog[lane & 3] >= nd)) __builtin_amdgcn_s_sleep(AG_WSLEEP);
                }
                asm volatile("" ::: "memory");
                if ((nd & 7) == 0) STAMP(16 + (nd >> 3));
            }
            if (kt == NSLOT && pq >= 0) {
                const unsigned long long t0 = rt();
                while (!__all((int)(__hip_atomic_load((GAS unsigned*)(myws + W_SCNT + (lane & 7) * 128), __ATOMIC_RELAXED, __HIP_MEMORY_SCOPE_AGENT) - call * (NSTRIP / 8)) >= 0)) {
                    if (give_up(tmo, call, t0)) {
                        __hip_atomic_store(tmo, call, __ATOMIC_RELAXED, __HIP_MEMORY_SCOPE_AGENT);
                        break;
                    }
                    __builtin_amdgcn_s_sleep(4);
                }
                FENCE();
#pragma unroll
                for (int i = 0; i < NPIECE / 2; ++i) {
                    const unsigned o = i * 1024 + lane * 16, r = o / RS, cb = o % RS;
                    if (cb >= 256) gp[i] = (unsigned long long)myws + W_STRIP + pq * (K * 32) + (NSLOT * BK + r) * 32 + cb - 256, gs[i] = 1024;
                }
            }
            LAS unsigned char* dst = lds + L_W + so;
            {
#pragma unroll
                for (int i = 0; i < NPIECE; ++i)
                {
                    __builtin_amdgcn_global_load_lds((const GAS void*)gp[i % (NPIECE / 2)], (LAS void*)(dst + i * 1024), 16, 0, AG_WAUX);
                    gp[i % (NPIECE / 2)] += gs[i % (NPIECE / 2)];
                }
            }
            if (AG_PD) __builtin_amdgcn_raw_ptr_buffer_load_lds(rall, (LAS void*)(lds + L_SINK), 4, po + kt * (BK * NL * 2), 0, 0, 0);
            so = so + SLOT == NSLOT * SLOT ? 0u : so + SLOT;
            if (kt + 1 - pub > 2) {
                asm volatile("s_waitcnt vmcnt(%0)" :: "n"(2 * NPIECE + (AG_PD ? 3 : 0)) : "memory");
                ++pub;
                ctl[lane & 3] = pub;
            }
        }
        asm volatile("s_waitcnt vmcnt(0)" ::: "memory");
        ctl[lane & 3] = ALL;
    }
    else if (wave == 5) {
        __builtin_amdgcn_s_setprio(AG_PPRIO);
        GAS unsigned* flag = (GAS unsigned*)(myws + W_FLAG) + pm * NKT * 8 * FS;
        const unsigned long long t0 = rt();
        int cur = 0, win = AG_PW, last = 0;
        const __amdgpu_buffer_rsrc_t rt_ = rsrc(myws + xt_off + ((size_t)(pm >> 1) * NBLK + (pm & 1) * 8) * BLK, (NBLK - (pm & 1) * 8) * BLK);
        auto touch = [&](int k0, int k1) {
            for (int k = k0; k < k1; ++k)
                __builtin_amdgcn_raw_ptr_buffer_load_lds(rt_, (LAS void*)(lds + L_SINK), 4,
                    AG_XP == 1 ? (pn & 3) * 8192 + lane * 128 : (pn & 7) * 4096 + lane * 64, k * (NG * BLK), 0, AG_XAUX);
        };
        unsigned long long npoll = 0;
        while (cur < NKT - AG_TP) {
            ++npoll;
            const int idx = cur * 8 + lane;
            unsigned v = call;
            if (lane < win * 8 && idx < NKT * 8) v = __hip_atomic_load(flag + idx * FS, __ATOMIC_RELAXED, __HIP_MEMORY_SCOPE_SYSTEM);
            const unsigned long long miss = __ballot((int)(v - call) < 0);
            const int adv = miss ? (__builtin_ctzll(miss) >> 3) : win;
            if (AG_APW) { win = adv > 0 && last > 0 ? AG_PW : 1; last = adv; }
            if (adv > 0) {
                if (cur == 0) STAMP(4);
                if (((cur + adv) >> 3) != (cur >> 3)) STAMP(24 + (((cur + adv) >> 3) & 7));
                const int was = cur;
                cur = cur + adv < NKT ? cur + adv : NKT;
                asm volatile("" ::: "memory");
                ctl[4 + (lane & 3)] = cur < NKT ? cur : ALL;
                if (AG_XP) touch(was, cur);
            } else {
                if (give_up(tmo, call, t0)) {
                    __hip_atomic_store(tmo, call, __ATOMIC_RELAXED, __HIP_MEMORY_SCOPE_AGENT);
                    ctl[4 + (lane & 3)] = ALL + 1;
                    cur = NKT;
                    break;
                }
                __builtin_amdgcn_s_sleep(AG_PSLEEP);
            }
        }
        if (cur < NKT) {
            const __amdgpu_buffer_rsrc_t rf = rsrc((const void*)(flag + (NKT - 8) * 8 * FS), 64 * FS * 4);
            unsigned long long seen = __ballot((lane >> 3) + NKT - 8 < cur);
            auto ask = [&]() {
                unsigned o;
                asm("v_cndmask_b32 %0, %1, %2, %3" : "=v"(o) : "v"((lane >> 3) + NKT - 8 < cur + AG_PW ? lane * (FS * 4) : OOB), "v"(OOB), "s"(seen));
                return __builtin_amdgcn_raw_buffer_load_b32(rf, o, 0, 17);
            };
            auto take = [&](unsigned v) {
                seen |= __ballot(v == call);
                const int now = ~seen ? NKT - 8 + (__builtin_ctzll(~seen) >> 3) : NKT;
                if (now > cur) {
                    asm volatile("" ::: "memory");
                    ctl[4 + (lane & 3)] = now < NKT ? now : ALL;
                    if (AG_XP) touch(cur, now);
                    cur = now;
                } else if (give_up(tmo, call, t0)) {
                    __hip_atomic_store(tmo, call, __ATOMIC_RELAXED, __HIP_MEMORY_SCOPE_AGENT);
                    ctl[4 + (lane & 3)] = ALL + 1;
                    cur = NKT;
                }
            };
            unsigned v0 = ask(), v1, v2;
            SB();
            __builtin_amdgcn_s_sleep(AG_TS);
            SB();
            v1 = ask();
            SB();
            __builtin_amdgcn_s_sleep(AG_TS);
            SB();
            for (;;) {
                v2 = ask();
                SB();
                take(v0);
                if (cur >= NKT) break;
                v0 = ask();
                SB();
                take(v1);
                if (cur >= NKT) break;
                v1 = ask();
                SB();
                take(v2);
                if (cur >= NKT) break;
            }
        }
        STAMP(5);
    }
    } else {
        LAS unsigned char* lx = lds + L_W + wave * 8192;
        LAS unsigned char* ls = lx + 2048;
        LAS unsigned short* lp = (LAS unsigned short*)(lx + 4096);
        if (wave == 0) STAMP(11);
        const size_t xt_off = W_XT + (size_t)(call & 1u) * XT_BYTES, pk_off = W_PK + (size_t)(call & 1u) * PK_BYTES;
        if (wave < SW) {
            const int sid = wave * NSC + cid;
            __amdgpu_buffer_rsrc_t rd[NR];
#pragma unroll
            for (int i = 0; i < NR; ++i) rd[i] = rsrc(a.ws[i], (unsigned)W_END);
            const unsigned xt0 = (unsigned)xt_off + a.rank * NBLK * BLK, pk0 = (unsigned)pk_off + a.rank * NBLK * PBLK;
            const unsigned tbase = (call - 1u) * (unsigned)NBLK - 2u * NSEND;
            const unsigned tag = tag_of(call);
            auto low = [&](const u32x4 (&d)[4], int j, unsigned& e) {
                const unsigned d0 = d[j >> 1][(j & 1) * 2], d1 = d[j >> 1][(j & 1) * 2 + 1];
                const unsigned r0 = __builtin_amdgcn_alignbit(d0, d0, 31), r1 = __builtin_amdgcn_alignbit(d1, d1, 31);
                e = perm(r1, r0, 0x07050301u);
                return perm(r1, r0, 0x06040200u);
            };
            auto pack = [&](const u32x4 (&d)[4], u32x4 (&pk)[3], bool& raw, bool& annex) {
                unsigned e[8], sum = 0;
                *(LAS u32x2*)(ls + 512 + lane * 8) = (u32x2){0u, 0u};
#pragma unroll
                for (int j = 0; j < 8; ++j) {
                    pk[j >> 2][j & 3] = low(d, j, e[j]);
                    sum = __builtin_amdgcn_sad_u8(e[j], 0u, sum);
                }
                const unsigned mean = (unsigned)__builtin_amdgcn_readlane(scan(sum), 63) >> 11;
                const unsigned e0 = mean < 7u ? 7u : mean > 252u ? 252u : mean;
                const unsigned k2 = key2(e0), t2 = table2(e0), t3l = table3(e0, 0), t3h = table3(e0, 1);
                u32x4 x[2];
                u32x2 c = {0u, 0u}, mv[8];
                unsigned mask = 0, i3[8];
#pragma unroll
                for (int j = 0; j < 8; ++j) {
                    const unsigned sel = (e[j] ^ k2) & 3u * B1;
                    const unsigned t = ((perm(t2, t2, sel) ^ e[j]) >> 1) + 0x7f7f7f7fu;
                    const unsigned m = perm(t, t << 8, 0x0b090a08u);
                    const unsigned code = sel | (m & 3u * B1);
                    c[j >> 2] |= __builtin_amdgcn_udot4(code, 0x40100401u, 0u, false) << (8 * (j & 3));
                    mv[j] = *(LAS u32x2*)(lm + __builtin_amdgcn_udot4(code & (code >> 1) & B1, 0x40201008u, 0u, false));
                    i3[j] = (e[j] ^ e0 * B1) & m & 7u * B1;
                    const unsigned y = (perm(t3h, t3l, i3[j]) ^ e[j]) & m;
                    x[j >> 2][j & 3] = y;
                    mask = ((((y & 0x7f7f7f7fu) + 0x7f7f7f7fu) | y) & 0x80808080u) | (mask >> 1);
                }
                *(LAS u32x4*)(lx + lane * 32) = x[0];
                *(LAS u32x4*)(lx + lane * 32 + 16) = x[1];
                *(LAS u32x2*)(ls + lane * 8) = c;
                unsigned cnt = 0;
                for (;;) {
                    const unsigned long long act = __ballot(mask != 0u);
                    if (act == 0ull || cnt > NPATCH) break;
                    const unsigned p = __builtin_ctz(mask | 0x80000000u);
                    const unsigned at = __builtin_amdgcn_mbcnt_hi((unsigned)(act >> 32), __builtin_amdgcn_mbcnt_lo((unsigned)act, cnt));
                    lp[mask != 0u ? at : 127u] = (unsigned short)(lane * 32 + (p & 7u) * 4 + (p >> 3));
                    mask &= mask - 1u;
                    cnt += __builtin_popcountll(act);
                }
                FENCE();
                const unsigned pos = lp[lane] & 0x7ffu;
                unsigned long long acc = 0;
                unsigned nb = 0;
#pragma unroll
                for (int j = 7; j >= 0; --j) {
                    const unsigned g = perm(0u, i3[j], mv[j].x);
                    acc = (acc << mv[j].y) | __builtin_amdgcn_udot4(g, 0x003e0801u, g >> 15, false);
                    nb += mv[j].y;
                }
                const unsigned xb = lx[pos];
                const unsigned in = scan(nb), off = in - nb + cnt * 32u;
                const unsigned nd = cnt + (((unsigned)__builtin_amdgcn_readlane(in, 63) + 31u) >> 5);
                raw = cnt > NPATCH || nd > NLONG || __ballot(nb > 63u) != 0ull;
                annex = !raw && nd > NTAIL;
                LAS unsigned* lt = (LAS unsigned*)(ls + 512) + HW;
                const unsigned long long sa = acc << (off & 31u);
                __hip_atomic_fetch_or(lt + (off >> 5), (unsigned)sa, __ATOMIC_RELAXED, __HIP_MEMORY_SCOPE_WAVEFRONT);
                __hip_atomic_fetch_or(lt + (off >> 5) + 1, (unsigned)(sa >> 32), __ATOMIC_RELAXED, __HIP_MEMORY_SCOPE_WAVEFRONT);
                __hip_atomic_fetch_or(lt + (off >> 5) + 2, (unsigned)(acc >> 33) >> (~off & 31u), __ATOMIC_RELAXED, __HIP_MEMORY_SCOPE_WAVEFRONT);
                lt[(unsigned)lane < cnt ? lane : 300] = pos | (xb << 16);
                FENCE();
                pk[2] = *(LAS u32x4*)(ls + lane * 16);
                FENCE();
                const unsigned long long m0 = __ballot((int)pk[0].x < 0), m1 = __ballot((int)pk[1].x < 0);
                if (lane == 33) pk[2].x = (unsigned)m1, pk[2].y = (unsigned)(m1 >> 32);
                const unsigned long long m2 = __ballot((int)pk[2].x < 0);
                if (lane == 32) pk[2] = (u32x4){e0 | (raw ? 256u : cnt << 10) | (annex ? 512u : 0u) | (unsigned)(m2 >> 33) << 16, (unsigned)m0, (unsigned)(m0 >> 32), (unsigned)m2};
#pragma unroll
                for (int s = 0; s < 3; ++s) pk[s].x = bfi(s < 2 || lane < 48 ? 0x7fffffffu : ~0u, pk[s].x, tag);
            };
            unsigned b0 = sid, b1 = sid + NSEND;
            u32x4 nx[4], pk[3];
            bool raw, annex;
            unsigned skip = 0;
            if (AG_EARLY == 1 || wave < AG_EARLY - 1) {
#pragma unroll
                for (int j = 0; j < 8; ++j) pk[j >> 2][j & 3] = low(d, j, skip);
#pragma unroll
                for (int s = 0; s < 2; ++s) pk[s].x = (pk[s].x & 0x7fffffffu) | tag;
#pragma unroll
                for (int i = 0; i < NR - 1; ++i)
#pragma unroll
                    for (int s = 0; s < 2; ++s)
                        store16(pk[s], rd[i], pk0 + b0 * PBLK + lane * 16 + s * 1024);
                SB();
                skip = OOB;
            }
            pack(d, pk, raw, annex);
            if (wave == 0) STAMP(6);
            unsigned long long nsent = 0;
            unsigned seq = wave;
            while (b0 < (unsigned)NBLK) {
                const unsigned sp = pk0 + b0 * PBLK + lane * 16;
                if (annex) {
#pragma unroll
                    for (int i = 0; i < NR - 1; ++i)
                        store16(pk[2], rd[i], (lane >> 3) == 6 ? sp + 2048 : OOB);
                    wait_vm0();
                }
                while (ctl[0] < seq) __builtin_amdgcn_s_sleep(1);
                FENCE();
                if (raw) {
                    const unsigned so = xt0 + b0 * BLK + lane * 16;
#pragma unroll
                    for (int i = 0; i < NR - 1; ++i)
#pragma unroll
                        for (int s = 0; s < 4; ++s)
                            store16(d[s], rd[i], so + s * 1024);
                    wait_vm0();
                }
                TRACE(a.rank, b0, 0);
                fetch(nx, b1);
                const unsigned tk = __builtin_amdgcn_raw_ptr_buffer_atomic_add_i32(1, rd[NR - 1], lane == 0 ? (unsigned)W_TICKET : OOB, 0, 0);
                const unsigned sp2 = lane < 48 ? sp + 2048 : OOB;
#pragma unroll
                for (int i = 0; i < NR - 1; ++i)
#pragma unroll
                    for (int s = 0; s < 2; ++s)
                        store16(pk[s], rd[i], (sp | skip) + s * 1024);
                skip = 0;
#pragma unroll
                for (int i = 0; i < NR - 1; ++i)
                    store16(pk[2], rd[i], sp2);
                FENCE();
                if (lane == 0) __hip_atomic_fetch_max((LAS unsigned*)(lds + L_CTL), seq + 1, __ATOMIC_RELAXED, __HIP_MEMORY_SCOPE_WORKGROUP);
                seq = AG_TURN == 2 ? 0 : seq + SW;
                TRACE(a.rank, b0, 1);
#pragma unroll
                for (int s = 0; s < 4; ++s) d[s] = nx[s];
                pack(d, pk, raw, annex);
                b0 = b1;
                ++nsent;
                b1 = __builtin_amdgcn_readfirstlane(tk) - tbase;
            }
            ctl[0] = ~0u;
            if (wave == 0) STAMP(7);
        }
    }
}
}

extern "C" void kernel_launch(int rank, int nranks, ncclComm_t comm,
                              hipStream_t stream,
                              void* const* d_in, const int* in_sizes, int n_in,
                              void* d_out, int out_size,
                              const int* in_dtypes, int out_dtype,
                              void* const* d_out_all,
                              void* const* d_ws_all, size_t ws_bytes) {
    if (nranks != ag::NR || (unsigned)rank >= (unsigned)ag::NR || n_in != 2 || ws_bytes < ag::W_END ||
        in_sizes[0] != ag::MS * ag::K || in_sizes[1] != ag::K * ag::NL || out_size != ag::M * ag::NL) {
        fprintf(stderr, "ag_gemm: unexpected problem shape\n");
        return;
    }
    ag::Args a;
    a.x = (const char*)d_in[0];
    a.w = (const char*)d_in[1];
    a.out = (char*)d_out;
    for (int i = 0; i < ag::NR; ++i) a.ws[i] = (char*)d_ws_all[(rank + 1 + i) % ag::NR];
    a.rank = rank;
    a.pad = 0;
    hipLaunchKernelGGL(ag::ag_gemm, dim3(ag::NWG), dim3(ag::NTHR), 0, stream, a);
    HIP_CHECK(hipGetLastError());
}
```
